```python
import numpy as np

kernel_name = "gpu_ag_gemm_m8192_k8192_n8192_f32_silu"


ABSMAX_RTOL = 0.001

COMPUTE_DTYPE = "float32"
EPILOGUE = "silu"
IN_QUANT = None
OUT_QUANT = None
IN_DTYPES = {"x": "float32", "w_mat": "float32"}
WORLD_SIZE = 8

def setup_inputs():

    x = np.random.randn(8192, 8192).astype(np.float32)
    w_mat = (np.random.randn(8192, 8192) / np.sqrt(8192)).astype(np.float32)
    return {"x": x, "w_mat": w_mat}

def _epilogue(y):
    return (y / (1.0 + np.exp(-np.clip(y, -60.0, 60.0)))).astype(np.float32)

def reference(x, w_mat):
    y = (x @ w_mat).astype(np.float32)
    return _epilogue(y)
```

```cpp
#include <hip/hip_runtime.h>
#include <rccl/rccl.h>
#include <cstdio>

#define HIP_CHECK(expr) do { hipError_t _e = (expr); if (_e != hipSuccess) { \
    fprintf(stderr, "HIP error %s at %s:%d\n", hipGetErrorString(_e), __FILE__, __LINE__); \
    return; } } while (0)

namespace {

typedef _Float16 h8 __attribute__((ext_vector_type(8)));
typedef float f4 __attribute__((ext_vector_type(4)));
typedef unsigned u4 __attribute__((ext_vector_type(4)));
#define LAS __attribute__((address_space(3)))
#define GAS __attribute__((address_space(1)))
#define CAS __attribute__((address_space(4)))

constexpr int KDIM = 8192, NLOC = 1024;
constexpr int NKT = KDIM / 64;
constexpr int NCH = NKT / 2;
constexpr size_t PANEL_B = 4u << 20;
constexpr size_t CG_B = 2u << 20;
constexpr size_t OFF_SEQ = 0;
constexpr size_t OFF_TMO = 64u << 10;
constexpr size_t OFF_CTR = OFF_TMO + 256;
constexpr size_t OFF_FO = 68u << 10;
constexpr size_t OFF_FA = 128u << 10;
constexpr size_t OFF_FB = OFF_FA + 16 * 64 * 32;
constexpr size_t OFF_DBG = 640u << 10;
constexpr size_t OFF_DATA = 1u << 20;
constexpr size_t A_B = 16 * PANEL_B, B_B = 16 * CG_B, SLOT_B = A_B + B_B;
constexpr size_t WS_NEED = OFF_DATA + 2 * SLOT_B;
static_assert(OFF_FB + 16 * 64 * 16 <= OFF_DBG, "flag area");
constexpr unsigned long long DEADLINE = 30000000ull;

#define NP 64
#define NP2 16
#define SKEWD 14
#define OTHR 0
#define AUX 17
constexpr int N1 = 64 * 56, N2 = 512;
static_assert(NP2 <= (NP / 64) * 32 + (NP % 64 < 32 ? NP % 64 : 32), "stage-2 pushers are stage-1 pushers");

struct Params {
    const float* x;
    const float* w;
    float* out[8];
    char* ws[8];
    int rank;
    int pad;
};

#define STAMP(i) do { } while (0)

#define GLD4(dst, voff, sbase) \
    asm volatile("global_load_dwordx4 %0, %1, %2" : "=v"(dst) : "v"(voff), "s"(sbase) : "memory")
#define GST4(voff, data, sbase, imm) \
    asm volatile("global_store_dwordx4 %0, %1, %2 offset:" #imm " sc0 sc1\n\ts_nop 1" :: "v"(voff), "v"(data), "s"(sbase) : "memory")
#define WAIT17(n, v, t) asm volatile("s_waitcnt vmcnt(" #n ")" \
    : "+v"(v[0]), "+v"(v[1]), "+v"(v[2]), "+v"(v[3]), "+v"(v[4]), "+v"(v[5]), "+v"(v[6]), "+v"(v[7]), \
      "+v"(v[8]), "+v"(v[9]), "+v"(v[10]), "+v"(v[11]), "+v"(v[12]), "+v"(v[13]), "+v"(v[14]), "+v"(v[15]), "+v"(t) :: "memory")

__device__ __forceinline__ u4 pack8(float a0, float a1, float a2, float a3, float a4, float a5, float a6, float a7) {
    h8 h = {(_Float16)a0, (_Float16)a1, (_Float16)a2, (_Float16)a3, (_Float16)a4, (_Float16)a5, (_Float16)a6, (_Float16)a7};
    return __builtin_bit_cast(u4, h);
}
__device__ __forceinline__ void flag_store(char* p, unsigned call) {
    __hip_atomic_store((GAS unsigned*)p, call, __ATOMIC_RELAXED, __HIP_MEMORY_SCOPE_SYSTEM);
}
__device__ __forceinline__ void store8(const u4 (&h)[8], char* d0, unsigned half, unsigned vl16) {
    char* d1 = d0 + half;
    GST4(vl16, h[0], d0, 0); GST4(vl16, h[1], d0, 1024); GST4(vl16, h[2], d0, 2048); GST4(vl16, h[3], d0, 3072);
    GST4(vl16, h[4], d1, 0); GST4(vl16, h[5], d1, 1024); GST4(vl16, h[6], d1, 2048); GST4(vl16, h[7], d1, 3072);
}

__device__ __forceinline__ bool unit_decode(int u, int& a, int& b) {
    if (u < N1) {
        const int ch = u / 56, r = u - ch * 56, g = r / 7, pos = r - g * 7;
        if (pos == 0 || pos == 3 || pos == 6) { a = pos == 6 ? 768 + 32 * g : 64 * g + (pos == 3 ? 32 : 0); b = ch; return true; }
        a = 2 * ch + (g >> 2); b = (pos < 3 ? pos - 1 : pos - 2) * 4 + (g & 3);
        return false;
    }
    const int i = (u - N1) >> 2, j = i - SKEWD;
    const int half = i < SKEWD ? 0 : i >= 128 - SKEWD ? 1 : (~j & 1);
    b = i < SKEWD ? i : i >= 128 - SKEWD ? i - 64 : (j >> 1) + ((j & 1) ? SKEWD : 0);
    a = 512 + 128 * half + 32 * (u & 3);
    return true;
}

__device__ __forceinline__ void pusher(const CAS Params& P, char* myws, int pw, int idx2, unsigned call, int lane) {
    const int rank = P.rank, q = rank & 3;
    const unsigned slot = (unsigned)OFF_DATA + (call & 1) * (unsigned)SLOT_B;
    const unsigned long long wsl = (unsigned long long)P.ws[lane & 7];
    unsigned wlo = (unsigned)wsl, whi = (unsigned)(wsl >> 32);
    asm volatile("" : "+v"(wlo), "+v"(whi));
    auto wsp = [&](int r) {
        return (char*)((unsigned long long)(unsigned)__builtin_amdgcn_readlane(wlo, r) | ((unsigned long long)(unsigned)__builtin_amdgcn_readlane(whi, r) << 32));
    };
    const unsigned voffx = (unsigned)(lane & 15) * (KDIM * 4) + (unsigned)(lane >> 4) * 16;
    const unsigned voffw = (unsigned)(lane >> 4) * (4 * NLOC * 4) + (unsigned)(lane & 15) * 16;
    const unsigned vl16 = (unsigned)lane * 16;
    const char* px = (const char*)P.x; const char* pwm = (const char*)P.w;
    f4 v[16]; u4 h[8];
    auto load = [&](int u) {
        int a, b;
        const bool isx = unit_decode(u, a, b);
        const char* src = isx ? px + (size_t)((unsigned)a * (KDIM * 4) + (unsigned)b * 512) : pwm + (size_t)((unsigned)a * (64 * NLOC * 4) + (unsigned)b * 256);
        const unsigned s0 = isx ? 64 : NLOC * 4, s1 = isx ? 16 * KDIM * 4 : 16 * NLOC * 4, s2 = isx ? 256 : 32 * NLOC * 4;
        const unsigned voff = isx ? voffx : voffw;
#pragma unroll
        for (int i = 0; i < 16; ++i) { const char* s = src + ((i & 3) * s0 + ((i >> 2) & 1) * s1 + (i >> 3) * s2); GLD4(v[i], voff, s); }
    };
    const unsigned base1 = (call - 1u) * (unsigned)(N1 - NP) - 2u * NP, base2 = (call - 1u) * (unsigned)N2 - (unsigned)NP2;
    int cur = pw, nxt = NP + pw, mode = 0;
    unsigned tk = 0, pmask = 0, pfs = 0, pfo = 0;
    load(cur);
    WAIT17(0, v, tk);
    for (;;) {
        int a, b;
        const bool isx = unit_decode(cur, a, b);
        if (isx) {
#pragma unroll
            for (int i = 0; i < 8; ++i)
                h[i] = pack8(v[2 * i][0], v[2 * i][1], v[2 * i][2], v[2 * i][3], v[2 * i + 1][0], v[2 * i + 1][1], v[2 * i + 1][2], v[2 * i + 1][3]);
        } else {
#pragma unroll
            for (int i = 0; i < 8; ++i) {
                const int f = i >> 1, k = (i & 1) * 8;
                h[i] = pack8(v[k][f], v[k + 1][f], v[k + 2][f], v[k + 3][f], v[k + 4][f], v[k + 5][f], v[k + 6][f], v[k + 7][f]);
            }
        }
        asm volatile("" : "+v"(h[0]), "+v"(h[1]), "+v"(h[2]), "+v"(h[3]), "+v"(h[4]), "+v"(h[5]), "+v"(h[6]), "+v"(h[7]));
        load(nxt >= 0 ? nxt : cur);
        {
            const unsigned aoff = (unsigned)OFF_CTR + (unsigned)mode * 128u, inc = nxt >= 0 ? 1u : 0u;
            asm volatile("s_mov_b64 exec, 1\n\tglobal_atomic_add %0, %1, %2, %3 sc0\n\ts_mov_b64 exec, -1"
                         : "=v"(tk) : "v"(aoff), "v"(inc), "s"(myws) : "memory");
        }
        const int gd = (rank & 4) ^ (isx && a < 768 ? 0 : 4);
        unsigned offa, offb, fs, fo, mask; int ra, rb;
        if (isx) {
            const int panel = a < 768 ? 3 * q + (a >> 8) : 12 + q;
            offa = offb = slot + (unsigned)panel * (unsigned)PANEL_B + (unsigned)b * 65536u + (unsigned)((a & 255) >> 4) * 2048u;
            fs = fo = (unsigned)OFF_FA + (unsigned)(panel * 2048 + b * 32 + ((a & 255) >> 5) * 4);
            ra = gd + ((q + 1) & 3); rb = gd + ((q + 2) & 3); mask = 15u << gd;
        } else {
            const int cgl = b >> 1, cgr = 8 + 2 * q + (cgl & 1);
            const unsigned o = slot + (unsigned)A_B + (unsigned)a * 16384u + (unsigned)(b & 1) * 8192u;
            offa = o + (unsigned)cgr * (unsigned)CG_B; offb = o + (unsigned)cgl * (unsigned)CG_B;
            const unsigned f = (unsigned)OFF_FB + (unsigned)(a * 8 + (b & 1) * 4);
            fo = f + cgr * 1024; fs = f + cgl * 1024;
            ra = gd + (b >> 2); rb = rank; mask = (1u << ra) | (1u << rank);
        }
        const unsigned half = isx ? 32768u : 4096u;
        store8(h, wsp(ra) + offa, half, vl16);
        store8(h, wsp(rb) + offb, half, vl16);
        WAIT17(16, v, tk);
        if ((pmask >> (lane & 31)) & (lane < 8))
            flag_store((char*)(((unsigned long long)whi << 32) | wlo) + (lane == rank ? pfs : pfo), call);
        if (isx) {
            store8(h, wsp(gd + ((q + 3) & 3)) + offa, half, vl16);
            store8(h, wsp(gd + q) + offa, half, vl16);
        }
        pmask = mask; pfs = fs; pfo = fo;
        if (nxt < 0) break;
        cur = nxt;
        const unsigned t = __builtin_amdgcn_readfirstlane(tk);
        if (mode == 0) {
            const unsigned id = t - base1;
            if (id < (unsigned)N1) nxt = (int)id;
            else { nxt = idx2 >= 0 ? N1 + idx2 : -1; mode = 1; }
        } else {
            const unsigned id = t - base2;
            nxt = id < (unsigned)N2 ? N1 + (int)id : -1;
        }
    }
    asm volatile("s_waitcnt vmcnt(0)" ::: "memory");
    if ((pmask >> (lane & 31)) & (lane < 8))
        flag_store((char*)(((unsigned long long)whi << 32) | wlo) + (lane == rank ? pfs : pfo), call);
}

__device__ __forceinline__ float silu(float y) {
    const float c = __builtin_amdgcn_fmed3f(y, -60.0f, 60.0f);
    const float e = __builtin_amdgcn_exp2f(c * -1.4426950408889634f);
    return y * __builtin_amdgcn_rcpf(1.0f + e);
}

#define WAIT_V(n) asm volatile("s_waitcnt vmcnt(%0)" :: "n"(n) : "memory")
#define WAIT_L0() asm volatile("s_waitcnt lgkmcnt(0)" ::: "memory")
#define BAR() do { __builtin_amdgcn_sched_barrier(0); __builtin_amdgcn_s_barrier(); asm volatile("" ::: "memory"); __builtin_amdgcn_sched_barrier(0); } while (0)
template <int V> struct IC { static constexpr int value = V; };

template <int MF>
__device__ __forceinline__ void gemm_tile(const CAS Params& P, LAS char* lds, char* myws, int hp0, int cg, int oflag,
                                          unsigned call, int wid, int lane, int stamp0) {
    constexpr int ABYTES = MF * 8192, STAGE = ABYTES + 16384, NST = MF == 4 ? 3 : 5;
    const int rank = P.rank, q = rank & 3;
    const size_t slot = OFF_DATA + (size_t)(call & 1) * SLOT_B;
    const char* ga = myws + slot + (size_t)(hp0 >> 1) * PANEL_B + (hp0 & 1) * 16384 + wid * (MF * 1024) + lane * 16;
    const char* gb = myws + slot + A_B + (size_t)cg * CG_B + wid * 2048 + lane * 16;
    const int wm = wid >> 1, wn = wid & 1;
    const unsigned frow = (lane & 2) ? (unsigned)OFF_FB + (unsigned)cg * 1024u
                                     : (unsigned)OFF_FA + (unsigned)(hp0 >> 1) * 2048u + (unsigned)(MF == 4 ? (lane & 1) : (hp0 & 1)) * 16u;
    const unsigned fstep = (lane & 2) ? 16u : 32u;
    GAS unsigned* tmo = (GAS unsigned*)(myws + OFF_TMO);
    int ready = 0;

    auto ensure = [&](int c) {
        if (c < ready) return;
        const unsigned long long t0 = __builtin_amdgcn_s_memrealtime();
        int lim = 15;
        for (;;) {
            int cc = c + ((lane >> 2) < lim ? (lane >> 2) : lim); cc = cc < NCH ? cc : NCH - 1;
            u4 f;
            asm volatile("global_load_dwordx4 %0, %1, %2 sc0 sc1\n\ts_waitcnt vmcnt(0)"
                         : "=&v"(f) : "v"(frow + (unsigned)cc * fstep), "s"(myws) : "memory");
            const bool bad = f[0] < call || f[1] < call || f[2] < call || f[3] < call;
            const unsigned long long m = __builtin_amdgcn_ballot_w64(bad);
            const int n = m ? (__builtin_ctzll(m) >> 2) : lim + 1;
            if (n > 0) { ready = c + n; return; }
            if (__builtin_amdgcn_readfirstlane(__hip_atomic_load(tmo, __ATOMIC_RELAXED, __HIP_MEMORY_SCOPE_AGENT)) == call ||
                __builtin_amdgcn_s_memrealtime() - t0 > DEADLINE) {
                __hip_atomic_store(tmo, call, __ATOMIC_RELAXED, __HIP_MEMORY_SCOPE_AGENT);
                ready = NCH; return;
            }
            lim = 0;
            __builtin_amdgcn_s_sleep(6);
        }
    };
#define GLDS(g, l, imm) __builtin_amdgcn_global_load_lds((const GAS void*)(g), (LAS void*)(l), 16, imm, AUX)
    auto stage = [&](int t, int kf, int bufoff) {
        const char* a = ga + (size_t)t * 32768 + kf * 1024; const char* b = gb + (size_t)t * 16384 + kf * 1024;
        LAS char* la = lds + bufoff + wid * (MF * 1024) + kf * 1024; LAS char* lb = lds + bufoff + ABYTES + wid * 2048 + kf * 1024;
        if (MF == 4) { GLDS(a, la, 0); GLDS(a, la, 2048); GLDS(b, lb, 0); }
        else { GLDS(a, la, 0); GLDS(a, la, 1024); GLDS(b, lb, 0); GLDS(b, lb, 1024); }
    };

    f4 acc[MF][4];
#pragma unroll
    for (int m = 0; m < MF; ++m)
#pragma unroll
        for (int n = 0; n < 4; ++n) acc[m][n] = (f4){0.f, 0.f, 0.f, 0.f};

    const int aoff = wm * (MF * 2048) + lane * 16, boff = ABYTES + wn * 8192 + lane * 16;
    auto phase = [&](auto waitn, int rbuf, int kf, int lt, int lkf, int lbuf, int pc) {
        const LAS char* pa = lds + rbuf + aoff + kf * 1024; const LAS char* pb = lds + rbuf + boff + kf * 1024;
        h8 a[4], b[MF == 4 ? 4 : 8];
        if (MF == 4) {
#pragma unroll
            for (int n = 0; n < 4; ++n) b[n] = *(const LAS h8*)(pb + n * 2048);
#pragma unroll
            for (int m = 0; m < 4; ++m) a[m] = *(const LAS h8*)(pa + m * 2048);
        } else {
#pragma unroll
            for (int n = 0; n < 8; ++n) b[n] = *(const LAS h8*)(pb + n * 1024);
#pragma unroll
            for (int m = 0; m < 4; ++m) a[m] = *(const LAS h8*)(pa + m * 1024);
        }
        __builtin_amdgcn_sched_barrier(0);
        if (lt >= 0) stage(lt, lkf, lbuf);
        if (decltype(waitn)::value >= 0) WAIT_V(decltype(waitn)::value < 0 ? 0 : decltype(waitn)::value);
        WAIT_L0();
        BAR();
        __builtin_amdgcn_s_setprio(1);
        if (MF == 4) {
#pragma unroll
            for (int m = 0; m < 4; ++m)
#pragma unroll
                for (int n = 0; n < 4; ++n) acc[m][n] = __builtin_amdgcn_mfma_f32_16x16x32_f16(a[m], b[n], acc[m][n], 0, 0, 0);
        } else {
#pragma unroll
            for (int k = 0; k < 2; ++k)
#pragma unroll
                for (int m = 0; m < 2; ++m)
#pragma unroll
                    for (int n = 0; n < 4; ++n) acc[m][n] = __builtin_amdgcn_mfma_f32_16x16x32_f16(a[m * 2 + k], b[n * 2 + k], acc[m][n], 0, 0, 0);
        }
        __builtin_amdgcn_s_setprio(0);
        __builtin_amdgcn_sched_barrier(0);
        if (wid == 0 && pc >= 0) ensure(pc);
        BAR();
    };

    if (wid == 0) { ensure(0); ensure(1); if (MF == 2) ensure(2); }
    BAR();
    STAMP(stamp0);
    if (MF == 4) {
        stage(0, 0, 0); stage(0, 1, 0); stage(1, 0, STAGE); stage(1, 1, STAGE); stage(2, 0, 2 * STAGE);
        WAIT_V(12);
        BAR();
        if (wid >= 4) BAR();
        int rb = 0, pb = 2 * STAGE;
        for (int t = 0; t < NKT - 3; ++t) {
            phase(IC<12>{}, rb, 0, t + 2, 1, pb, (t & 1) ? (t + 3) >> 1 : -1);
            phase(IC<12>{}, rb, 1, t + 3, 0, rb, -1);
            pb = rb; rb = rb == 2 * STAGE ? 0 : rb + STAGE;
        }
        static_assert((NKT - 3) % 3 == 2, "tail buffers");
        phase(IC<12>{}, 2 * STAGE, 0, NKT - 1, 1, STAGE, -1);
        phase(IC<9>{}, 2 * STAGE, 1, -1, 0, 0, -1);
        phase(IC<6>{}, 0, 0, -1, 0, 0, -1);
        phase(IC<3>{}, 0, 1, -1, 0, 0, -1);
        phase(IC<0>{}, STAGE, 0, -1, 0, 0, -1);
        phase(IC<-1>{}, STAGE, 1, -1, 0, 0, -1);
    } else {
        stage(0, 0, 0); stage(1, 0, STAGE); stage(2, 0, 2 * STAGE); stage(3, 0, 3 * STAGE);
        WAIT_V(12);
        BAR();
        if (wid >= 4) BAR();
        int rb = 0, pb = 4 * STAGE;
        for (int t = 0; t < NKT - 4; ++t) {
            phase(IC<12>{}, rb, 0, t + 4, 0, pb, (t & 1) && t + 5 < NKT ? (t + 5) >> 1 : -1);
            pb = rb; rb = rb == 4 * STAGE ? 0 : rb + STAGE;
        }
        static_assert((NKT - 4) % 5 == 4, "tail buffers");
        phase(IC<8>{}, 4 * STAGE, 0, -1, 0, 0, -1);
        phase(IC<4>{}, 0, 0, -1, 0, 0, -1);
        phase(IC<0>{}, STAGE, 0, -1, 0, 0, -1);
        phase(IC<-1>{}, 2 * STAGE, 0, -1, 0, 0, -1);
    }
    if (wid < 4) BAR();
    STAMP(stamp0 + 1);

    const bool dead = __builtin_amdgcn_readfirstlane(__hip_atomic_load(tmo, __ATOMIC_RELAXED, __HIP_MEMORY_SCOPE_AGENT)) == call;
    const int panel = hp0 >> 1;
    const int srank = panel < 12 ? (rank & 4) + panel / 3 : ((rank & 4) ^ 4) + (panel - 12);
    const int row0 = srank * 1024 + (panel < 12 ? (panel % 3) * 256 : 768) + (hp0 & 1) * 128 + wm * (MF * 16);
    const bool remote = cg >= 8;
    const int drank = remote ? ((rank & 4) ^ 4) + ((cg - 8) >> 1) : rank;
    const int col0 = (remote ? q * 256 + (cg & 1) * 128 : cg * 128) + wn * 64;
    char* obase = (char*)P.out[drank] + ((size_t)row0 * NLOC + col0) * 4;
    const unsigned ovoff = (unsigned)(lane >> 4) * (4 * NLOC * 4) + (unsigned)(lane & 15) * 16;
    const float nanv = __builtin_nanf("");
#pragma unroll
    for (int m = 0; m < MF; ++m)
#pragma unroll
        for (int r = 0; r < 4; ++r) {
            f4 o = {silu(acc[m][0][r]), silu(acc[m][1][r]), silu(acc[m][2][r]), silu(acc[m][3][r])};
            if (dead) o = (f4){nanv, nanv, nanv, nanv};
            char* ob = obase + (size_t)(m * 16 + r) * (NLOC * 4);
            const u4 ou = __builtin_bit_cast(u4, o);
            if (remote) { GST4(ovoff, ou, ob, 0); if (MF == 4 && OTHR >= 0) WAIT_V(OTHR < 0 ? 0 : OTHR); }
            else *(GAS u4*)(ob + ovoff) = ou;
        }
    asm volatile("s_waitcnt vmcnt(0)" ::: "memory");
    __builtin_amdgcn_s_barrier();
    asm volatile("" ::: "memory");
    if (remote && wid == 0 && lane == 0) flag_store(P.ws[drank] + OFF_FO + (q * 64 + oflag) * 4, call);
    STAMP(stamp0 + 2);
}

__global__ void __launch_bounds__(512, 2) ag_gemm_kernel(const Params Parg) {
    const CAS Params& P = *(const CAS Params*)__builtin_amdgcn_kernarg_segment_ptr();
    (void)Parg;
    __shared__ __attribute__((aligned(16))) char lds_raw[5 * 32768];
    LAS char* lds = (LAS char*)lds_raw;
    const int lane = threadIdx.x & 63, wid = __builtin_amdgcn_readfirstlane(threadIdx.x >> 6);
    const int xcd = blockIdx.x & 7, slot = blockIdx.x >> 3;
    char* myws = P.ws[P.rank];
    STAMP(0);
    GAS unsigned* seq = (GAS unsigned*)(myws + OFF_SEQ) + (blockIdx.x * 8 + wid) * 4;
    const unsigned call = __builtin_amdgcn_readfirstlane(__hip_atomic_load(seq, __ATOMIC_RELAXED, __HIP_MEMORY_SCOPE_AGENT)) + 1u;
    if (lane == 0) __hip_atomic_store(seq, call, __ATOMIC_RELAXED, __HIP_MEMORY_SCOPE_AGENT);
    const int q = P.rank & 3;

    int s2 = -1;
    if (slot < 24) {
        const int pi = slot >> 3, ci = slot & 7, ps = xcd >> 1, h = xcd & 1;
        const int panel = pi < 2 ? 3 * ps + pi : 12 + ps;
        const int cg = ci < 4 ? 4 * h + ci : 8 + 2 * (ci - 4) + h;
        gemm_tile<4>(P, lds, myws, panel * 2, cg, (ps * 3 + pi) * 2 + h, call, wid, lane, 1);
        if (ci < 4) s2 = pi * 4 + ci;
    } else {
        const int g = (31 - slot) * 8 + xcd;
        const int pw = wid * 64 + g;
        if (pw < NP) pusher(P, myws, pw, g < 32 && wid * 32 + g < NP2 ? wid * 32 + g : -1, call, lane);
        STAMP(1);
        if (slot < 28) s2 = 12 + (slot - 24);
        if (g == 31 && wid == 7) {
            GAS unsigned* tmo = (GAS unsigned*)(myws + OFF_TMO);
            GAS unsigned* fo = (GAS unsigned*)(myws + OFF_FO);
            const int i = lane < 40 ? lane : 0;
            const unsigned long long t0 = __builtin_amdgcn_s_memrealtime();
            for (;;) {
                bool bad = false;
#pragma unroll
                for (int s = 0; s < 4; ++s) bad |= __hip_atomic_load(fo + s * 64 + i, __ATOMIC_RELAXED, __HIP_MEMORY_SCOPE_SYSTEM) < call;
                if (__builtin_amdgcn_ballot_w64(bad) == 0) break;
                if (__builtin_amdgcn_readfirstlane(__hip_atomic_load(tmo, __ATOMIC_RELAXED, __HIP_MEMORY_SCOPE_AGENT)) == call ||
                    __builtin_amdgcn_s_memrealtime() - t0 > DEADLINE) {
                    __hip_atomic_store(tmo, call, __ATOMIC_RELAXED, __HIP_MEMORY_SCOPE_AGENT);
                    P.out[P.rank][(size_t)(P.rank * 1024) * NLOC + lane] = __builtin_nanf("");
                    break;
                }
                __builtin_amdgcn_s_sleep(4);
            }
            STAMP(2);
        }
    }
    if (s2 >= 0) {
        const int qs = (q + 1 + (s2 >> 2)) & 3, ci = s2 & 3, j = xcd >> 1, h = xcd & 1;
        const int cg = ci < 2 ? 2 * j + ci : 8 + 2 * j + (ci - 2);
        gemm_tile<2>(P, lds, myws, (3 * qs + 2) * 2 + h, cg, 24 + (qs * 2 + h) * 2 + (ci & 1), call, wid, lane, 4);
    }
}

}

extern "C" void kernel_launch(int rank, int nranks, ncclComm_t comm, hipStream_t stream,
                              void* const* d_in, const int* in_sizes, int n_in,
                              void* d_out, int out_size, const int* in_dtypes, int out_dtype,
                              void* const* d_out_all, void* const* d_ws_all, size_t ws_bytes) {
    (void)comm; (void)in_dtypes; (void)out_dtype; (void)d_out;
    if (nranks != 8 || (unsigned)rank >= 8u || n_in < 2 || ws_bytes < WS_NEED || out_size != KDIM * NLOC ||
        in_sizes[0] != 1024 * KDIM || in_sizes[1] != KDIM * NLOC) {
        fprintf(stderr, "ag_gemm: unexpected problem shape\n");
        return;
    }
    Params P;
    P.x = (const float*)d_in[0]; P.w = (const float*)d_in[1];
    for (int p = 0; p < 8; ++p) { P.out[p] = (float*)d_out_all[p]; P.ws[p] = (char*)d_ws_all[p]; }
    P.rank = rank; P.pad = 0;
    hipLaunchKernelGGL(ag_gemm_kernel, dim3(256), dim3(512), 0, stream, P);
    HIP_CHECK(hipGetLastError());
}
```

```cpp
#include <hip/hip_runtime.h>
#include <rccl/rccl.h>
#include <cstdio>

#define HIP_CHECK(expr) do { hipError_t _e = (expr); if (_e != hipSuccess) { \
    fprintf(stderr, "HIP error %s at %s:%d\n", hipGetErrorString(_e), __FILE__, __LINE__); \
    return; } } while (0)

namespace {

typedef _Float16 h8 __attribute__((ext_vector_type(8)));
typedef float f4 __attribute__((ext_vector_type(4)));
typedef unsigned u4 __attribute__((ext_vector_type(4)));
#define LAS __attribute__((address_space(3)))
#define GAS __attribute__((address_space(1)))
#define CAS __attribute__((address_space(4)))

constexpr int KDIM = 8192, NLOC = 1024;
constexpr int NKT = KDIM / 64;
constexpr int NCH = NKT / 2;
constexpr size_t PANEL_B = 4u << 20;
constexpr size_t CG_B = 2u << 20;
constexpr size_t OFF_SEQ = 0;
constexpr size_t OFF_TMO = 64u << 10;
constexpr size_t OFF_CTR = OFF_TMO + 256;
constexpr size_t OFF_RATE = OFF_CTR + 256;
constexpr size_t OFF_FO = 68u << 10;
constexpr size_t OFF_FA = 128u << 10;
constexpr size_t OFF_FB = OFF_FA + 16 * 64 * 32;
constexpr size_t OFF_DBG = 640u << 10;
constexpr size_t OFF_DATA = 1u << 20;
constexpr size_t A_B = 16 * PANEL_B, B_B = 16 * CG_B, SLOT_B = A_B + B_B;
constexpr size_t WS_NEED = OFF_DATA + 2 * SLOT_B;
static_assert(OFF_FB + 16 * 64 * 16 <= OFF_DBG, "flag area");
constexpr unsigned long long DEADLINE = 30000000ull;

#define NP 64
#define NP2 16
#define SKEWD 9
#define SKEW1 8
#define OCOST 510
#define AUX 17
constexpr int N1 = 64 * 56, N2 = 512;
static_assert(NP2 <= (NP / 64) * 32 + (NP % 64 < 32 ? NP % 64 : 32), "stage-2 pushers are stage-1 pushers");

struct Params {
    const float* x;
    const float* w;
    float* out[8];
    char* ws[8];
    int rank;
    int pad;
};

#define STAMP(i) do { } while (0)

#define GLD4(dst, voff, sbase) \
    asm volatile("global_load_dwordx4 %0, %1, %2" : "=v"(dst) : "v"(voff), "s"(sbase) : "memory")
#define GST4(voff, data, sbase, imm) \
    asm volatile("global_store_dwordx4 %0, %1, %2 offset:" #imm " sc0 sc1\n\ts_nop 1" :: "v"(voff), "v"(data), "s"(sbase) : "memory")
#define WAIT17(n, v, t) asm volatile("s_waitcnt vmcnt(" #n ")" \
    : "+v"(v[0]), "+v"(v[1]), "+v"(v[2]), "+v"(v[3]), "+v"(v[4]), "+v"(v[5]), "+v"(v[6]), "+v"(v[7]), \
      "+v"(v[8]), "+v"(v[9]), "+v"(v[10]), "+v"(v[11]), "+v"(v[12]), "+v"(v[13]), "+v"(v[14]), "+v"(v[15]), "+v"(t) :: "memory")

__device__ __forceinline__ u4 pack8(float a0, float a1, float a2, float a3, float a4, float a5, float a6, float a7) {
    h8 h = {(_Float16)a0, (_Float16)a1, (_Float16)a2, (_Float16)a3, (_Float16)a4, (_Float16)a5, (_Float16)a6, (_Float16)a7};
    return __builtin_bit_cast(u4, h);
}
__device__ __forceinline__ void flag_store(char* p, unsigned call) {
    __hip_atomic_store((GAS unsigned*)p, call, __ATOMIC_RELAXED, __HIP_MEMORY_SCOPE_SYSTEM);
}
__device__ __forceinline__ void store8(const u4 (&h)[8], char* d0, unsigned half, unsigned vl16) {
    char* d1 = d0 + half;
    GST4(vl16, h[0], d0, 0); GST4(vl16, h[1], d0, 1024); GST4(vl16, h[2], d0, 2048); GST4(vl16, h[3], d0, 3072);
    GST4(vl16, h[4], d1, 0); GST4(vl16, h[5], d1, 1024); GST4(vl16, h[6], d1, 2048); GST4(vl16, h[7], d1, 3072);
}

__device__ __forceinline__ bool unit_decode(int u, int& a, int& b) {
    if (u < N1) {
        constexpr int U0 = 40 * SKEW1, U1 = U0 + 7 * (512 - 8 * SKEW1);
        int set, pos;
        if (u < U0) { set = u / 5; pos = u - set * 5; }
        else if (u < U1) {
            const int j = (u - U0) / 7, r = u - U0 - j * 7;
            pos = (0x6435210 >> (4 * r)) & 7; set = pos < 5 ? j + 8 * SKEW1 : j;
        } else { set = 512 - 8 * SKEW1 + ((u - U1) >> 1); pos = 5 + ((u - U1) & 1); }
        const int ch = set >> 3, g = set & 7;
        if (pos == 0 || pos >= 5) { a = (pos == 0 ? 0 : pos == 5 ? 256 : 768) + 32 * g; b = ch; return true; }
        a = 2 * ch + (g >> 2); b = (pos - 1) * 4 + (g & 3);
        return false;
    }
    const int i = (u - N1) >> 2, j = i - SKEWD;
    const int half = i < SKEWD ? 0 : i >= 128 - SKEWD ? 1 : (~j & 1);
    b = i < SKEWD ? i : i >= 128 - SKEWD ? i - 64 : (j >> 1) + ((j & 1) ? SKEWD : 0);
    a = 512 + 128 * half + 32 * (u & 3);
    return true;
}

__device__ __forceinline__ void pusher(const CAS Params& P, char* myws, int pw, int idx2, unsigned call, int lane) {
    const int rank = P.rank, q = rank & 3;
    const unsigned slot = (unsigned)OFF_DATA + (call & 1) * (unsigned)SLOT_B;
    const unsigned long long wsl = (unsigned long long)P.ws[lane & 7];
    unsigned wlo = (unsigned)wsl, whi = (unsigned)(wsl >> 32);
    asm volatile("" : "+v"(wlo), "+v"(whi));
    auto wsp = [&](int r) {
        return (char*)((unsigned long long)(unsigned)__builtin_amdgcn_readlane(wlo, r) | ((unsigned long long)(unsigned)__builtin_amdgcn_readlane(whi, r) << 32));
    };
    const unsigned voffx = (unsigned)(lane & 15) * (KDIM * 4) + (unsigned)(lane >> 4) * 16;
    const unsigned voffw = (unsigned)(lane >> 4) * (4 * NLOC * 4) + (unsigned)(lane & 15) * 16;
    const unsigned vl16 = (unsigned)lane * 16;
    const char* px = (const char*)P.x; const char* pwm = (const char*)P.w;
    f4 v[16]; u4 h[8];
    auto load = [&](int u) {
        int a, b;
        const bool isx = unit_decode(u, a, b);
        const char* src = isx ? px + (size_t)((unsigned)a * (KDIM * 4) + (unsigned)b * 512) : pwm + (size_t)((unsigned)a * (64 * NLOC * 4) + (unsigned)b * 256);
        const unsigned s0 = isx ? 64 : NLOC * 4, s1 = isx ? 16 * KDIM * 4 : 16 * NLOC * 4, s2 = isx ? 256 : 32 * NLOC * 4;
        const unsigned voff = isx ? voffx : voffw;
#pragma unroll
        for (int i = 0; i < 16; ++i) { const char* s = src + ((i & 3) * s0 + ((i >> 2) & 1) * s1 + (i >> 3) * s2); GLD4(v[i], voff, s); }
    };
    const unsigned base1 = (call - 1u) * (unsigned)(N1 - NP) - 2u * NP, base2 = (call - 1u) * (unsigned)N2 - (unsigned)NP2;
    int cur = pw, nxt = NP + pw, mode = 0;
    unsigned tk = 0, pmask = 0, pfs = 0, pfo = 0;
    load(cur);
    WAIT17(0, v, tk);
    for (;;) {
        int a, b;
        const bool isx = unit_decode(cur, a, b);
        if (isx) {
#pragma unroll
            for (int i = 0; i < 8; ++i)
                h[i] = pack8(v[2 * i][0], v[2 * i][1], v[2 * i][2], v[2 * i][3], v[2 * i + 1][0], v[2 * i + 1][1], v[2 * i + 1][2], v[2 * i + 1][3]);
        } else {
#pragma unroll
            for (int i = 0; i < 8; ++i) {
                const int f = i >> 1, k = (i & 1) * 8;
                h[i] = pack8(v[k][f], v[k + 1][f], v[k + 2][f], v[k + 3][f], v[k + 4][f], v[k + 5][f], v[k + 6][f], v[k + 7][f]);
            }
        }
        asm volatile("" : "+v"(h[0]), "+v"(h[1]), "+v"(h[2]), "+v"(h[3]), "+v"(h[4]), "+v"(h[5]), "+v"(h[6]), "+v"(h[7]));
        load(nxt >= 0 ? nxt : cur);
        {
            const unsigned aoff = (unsigned)OFF_CTR + (unsigned)mode * 128u, inc = nxt >= 0 ? 1u : 0u;
            asm volatile("s_mov_b64 exec, 1\n\tglobal_atomic_add %0, %1, %2, %3 sc0\n\ts_mov_b64 exec, -1"
                         : "=v"(tk) : "v"(aoff), "v"(inc), "s"(myws) : "memory");
        }
        const int gd = (rank & 4) ^ (isx && a < 768 ? 0 : 4);
        unsigned offa, offb, fs, fo, mask; int ra, rb;
        if (isx) {
            const int panel = a < 768 ? 3 * q + (a >> 8) : 12 + q;
            offa = offb = slot + (unsigned)panel * (unsigned)PANEL_B + (unsigned)b * 65536u + (unsigned)((a & 255) >> 4) * 2048u;
            fs = fo = (unsigned)OFF_FA + (unsigned)(panel * 2048 + b * 32 + ((a & 255) >> 5) * 4);
            ra = gd + ((q + 1) & 3); rb = gd + ((q + 2) & 3); mask = (15u << gd) & ~(cur < N1 ? 0u : 1u << rank);
        } else {
            const int cgl = b >> 1, cgr = 8 + 2 * q + (cgl & 1);
            const unsigned o = slot + (unsigned)A_B + (unsigned)a * 16384u + (unsigned)(b & 1) * 8192u;
            offa = o + (unsigned)cgr * (unsigned)CG_B; offb = o + (unsigned)cgl * (unsigned)CG_B;
            const unsigned f = (unsigned)OFF_FB + (unsigned)(a * 8 + (b & 1) * 4);
            fo = f + cgr * 1024; fs = f + cgl * 1024;
            ra = gd + (b >> 2); rb = rank; mask = (1u << ra) | (1u << rank);
        }
        const unsigned half = isx ? 32768u : 4096u;
        store8(h, wsp(ra) + offa, half, vl16);
        store8(h, wsp(rb) + offb, half, vl16);
        WAIT17(16, v, tk);
        if ((pmask >> (lane & 31)) & (lane < 8))
            flag_store((char*)(((unsigned long long)whi << 32) | wlo) + (lane == rank ? pfs : pfo), call);
        if (isx) {
            store8(h, wsp(gd + ((q + 3) & 3)) + offa, half, vl16);
            if (cur < N1) store8(h, wsp(gd + q) + offa, half, vl16);
        }
        pmask = mask; pfs = fs; pfo = fo;
        if (nxt < 0) break;
        cur = nxt;
        const unsigned t = __builtin_amdgcn_readfirstlane(tk);
        if (mode == 0) {
            const unsigned id = t - base1;
            if (id < (unsigned)N1) nxt = (int)id;
            else { nxt = idx2 >= 0 ? N1 + idx2 : -1; mode = 1; }
        } else {
            const unsigned id = t - base2;
            nxt = id < (unsigned)N2 ? N1 + (int)id : -1;
        }
    }
    asm volatile("s_waitcnt vmcnt(0)" ::: "memory");
    if ((pmask >> (lane & 31)) & (lane < 8))
        flag_store((char*)(((unsigned long long)whi << 32) | wlo) + (lane == rank ? pfs : pfo), call);
}

__device__ __forceinline__ void convert_own_b(const CAS Params& P, char* myws, int i, unsigned call, int lane) {
    const int a = 512 + 32 * (i & 7), b = i >> 3, panel = 3 * (P.rank & 3) + 2;
    const unsigned voff = (unsigned)(lane & 15) * (KDIM * 4) + (unsigned)(lane >> 4) * 16;
    const char* src = (const char*)P.x + (size_t)((unsigned)a * (KDIM * 4) + (unsigned)b * 512);
    f4 v[16]; u4 h[8]; unsigned tk = 0;
#pragma unroll
    for (int k = 0; k < 16; ++k) { const char* s = src + ((k & 3) * 64 + ((k >> 2) & 1) * (16 * KDIM * 4) + (k >> 3) * 256); GLD4(v[k], voff, s); }
    WAIT17(0, v, tk);
#pragma unroll
    for (int k = 0; k < 8; ++k)
        h[k] = pack8(v[2 * k][0], v[2 * k][1], v[2 * k][2], v[2 * k][3], v[2 * k + 1][0], v[2 * k + 1][1], v[2 * k + 1][2], v[2 * k + 1][3]);
    store8(h, myws + OFF_DATA + (size_t)(call & 1) * SLOT_B + (size_t)panel * PANEL_B + b * 65536 + (i & 7) * 4096, 32768u, (unsigned)lane * 16);
    asm volatile("s_waitcnt vmcnt(0)" ::: "memory");
    if (lane == 0) flag_store(myws + OFF_FA + (panel * 2048 + b * 32 + (i & 7) * 4), call);
}

__device__ __forceinline__ float silu(float y) {
    const float c = __builtin_amdgcn_fmed3f(y, -60.0f, 60.0f);
    const float e = __builtin_amdgcn_exp2f(c * -1.4426950408889634f);
    return y * __builtin_amdgcn_rcpf(1.0f + e);
}

#define WAIT_V(n) asm volatile("s_waitcnt vmcnt(%0)" :: "n"(n) : "memory")
#define WAIT_L0() asm volatile("s_waitcnt lgkmcnt(0)" ::: "memory")
#define BAR() do { __builtin_amdgcn_sched_barrier(0); __builtin_amdgcn_s_barrier(); asm volatile("" ::: "memory"); __builtin_amdgcn_sched_barrier(0); } while (0)
template <int V> struct IC { static constexpr int value = V; };

template <int MF>
__device__ __forceinline__ void gemm_tile(const CAS Params& P, LAS char* lds, char* myws, int hp0, int cg, int oflag, bool paced,
                                          unsigned call, int wid, int lane, int stamp0) {
    constexpr int ABYTES = MF * 8192, STAGE = ABYTES + 16384, NST = MF == 4 ? 3 : 5;
    const int rank = P.rank, q = rank & 3;
    const size_t slot = OFF_DATA + (size_t)(call & 1) * SLOT_B;
    const char* ga = myws + slot + (size_t)(hp0 >> 1) * PANEL_B + (hp0 & 1) * 16384 + wid * (MF * 1024) + lane * 16;
    const char* gb = myws + slot + A_B + (size_t)cg * CG_B + wid * 2048 + lane * 16;
    const int wm = wid >> 1, wn = wid & 1;
    const unsigned frow = (lane & 2) ? (unsigned)OFF_FB + (unsigned)cg * 1024u
                                     : (unsigned)OFF_FA + (unsigned)(hp0 >> 1) * 2048u + (unsigned)(MF == 4 ? (lane & 1) : (hp0 & 1)) * 16u;
    const unsigned fstep = (lane & 2) ? 16u : 32u;
    GAS unsigned* tmo = (GAS unsigned*)(myws + OFF_TMO);
    int ready = 0;

    auto ensure = [&](int c) {
        if (c < ready) return;
        const unsigned long long t0 = __builtin_amdgcn_s_memrealtime();
        int lim = 15;
        for (;;) {
            int cc = c + ((lane >> 2) < lim ? (lane >> 2) : lim); cc = cc < NCH ? cc : NCH - 1;
            u4 f;
            asm volatile("global_load_dwordx4 %0, %1, %2 sc0 sc1\n\ts_waitcnt vmcnt(0)"
                         : "=&v"(f) : "v"(frow + (unsigned)cc * fstep), "s"(myws) : "memory");
            const bool bad = f[0] < call || f[1] < call || f[2] < call || f[3] < call;
            const unsigned long long m = __builtin_amdgcn_ballot_w64(bad);
            const int n = m ? (__builtin_ctzll(m) >> 2) : lim + 1;
            if (n > 0) { ready = c + n; return; }
            if (__builtin_amdgcn_readfirstlane(__hip_atomic_load(tmo, __ATOMIC_RELAXED, __HIP_MEMORY_SCOPE_AGENT)) == call ||
                __builtin_amdgcn_s_memrealtime() - t0 > DEADLINE) {
                __hip_atomic_store(tmo, call, __ATOMIC_RELAXED, __HIP_MEMORY_SCOPE_AGENT);
                ready = NCH; return;
            }
            lim = 0;
            __builtin_amdgcn_s_sleep(6);
        }
    };
#define GLDS(g, l, imm) __builtin_amdgcn_global_load_lds((const GAS void*)(g), (LAS void*)(l), 16, imm, AUX)
    auto stage = [&](int t, int kf, int bufoff) {
        const char* a = ga + (size_t)t * 32768 + kf * 1024; const char* b = gb + (size_t)t * 16384 + kf * 1024;
        LAS char* la = lds + bufoff + wid * (MF * 1024) + kf * 1024; LAS char* lb = lds + bufoff + ABYTES + wid * 2048 + kf * 1024;
        if (MF == 4) { GLDS(a, la, 0); GLDS(a, la, 2048); GLDS(b, lb, 0); }
        else { GLDS(a, la, 0); GLDS(a, la, 1024); GLDS(b, lb, 0); GLDS(b, lb, 1024); }
    };

    f4 acc[MF][4];
#pragma unroll
    for (int m = 0; m < MF; ++m)
#pragma unroll
        for (int n = 0; n < 4; ++n) acc[m][n] = (f4){0.f, 0.f, 0.f, 0.f};

    const int aoff = wm * (MF * 2048) + lane * 16, boff = ABYTES + wn * 8192 + lane * 16;
    auto phase = [&](auto waitn, int rbuf, int kf, int lt, int lkf, int lbuf, int pc) {
        const LAS char* pa = lds + rbuf + aoff + kf * 1024; const LAS char* pb = lds + rbuf + boff + kf * 1024;
        h8 a[4], b[MF == 4 ? 4 : 8];
        if (MF == 4) {
#pragma unroll
            for (int n = 0; n < 4; ++n) b[n] = *(const LAS h8*)(pb + n * 2048);
#pragma unroll
            for (int m = 0; m < 4; ++m) a[m] = *(const LAS h8*)(pa + m * 2048);
        } else {
#pragma unroll
            for (int n = 0; n < 8; ++n) b[n] = *(const LAS h8*)(pb + n * 1024);
#pragma unroll
            for (int m = 0; m < 4; ++m) a[m] = *(const LAS h8*)(pa + m * 1024);
        }
        __builtin_amdgcn_sched_barrier(0);
        if (lt >= 0) stage(lt, lkf, lbuf);
        if (decltype(waitn)::value >= 0) WAIT_V(decltype(waitn)::value < 0 ? 0 : decltype(waitn)::value);
        WAIT_L0();
        BAR();
        __builtin_amdgcn_s_setprio(1);
        if (MF == 4) {
#pragma unroll
            for (int m = 0; m < 4; ++m)
#pragma unroll
                for (int n = 0; n < 4; ++n) acc[m][n] = __builtin_amdgcn_mfma_f32_16x16x32_f16(a[m], b[n], acc[m][n], 0, 0, 0);
        } else {
#pragma unroll
            for (int k = 0; k < 2; ++k)
#pragma unroll
                for (int m = 0; m < 2; ++m)
#pragma unroll
                    for (int n = 0; n < 4; ++n) acc[m][n] = __builtin_amdgcn_mfma_f32_16x16x32_f16(a[m * 2 + k], b[n * 2 + k], acc[m][n], 0, 0, 0);
        }
        __builtin_amdgcn_s_setprio(0);
        __builtin_amdgcn_sched_barrier(0);
        if (wid == 0 && pc >= 0) {
            ensure(pc);
        }
        BAR();
    };

    if (wid == 0) { ensure(0); ensure(1); if (MF == 2) ensure(2); }
    BAR();
    STAMP(stamp0);
    if (MF == 4) {
        stage(0, 0, 0); stage(0, 1, 0); stage(1, 0, STAGE); stage(1, 1, STAGE); stage(2, 0, 2 * STAGE);
        WAIT_V(12);
        BAR();
        if (wid >= 4) BAR();
        int rb = 0, pb = 2 * STAGE;
        for (int t = 0; t < NKT - 3; ++t) {
            phase(IC<12>{}, rb, 0, t + 2, 1, pb, (t & 1) ? (t + 3) >> 1 : -1);
            phase(IC<12>{}, rb, 1, t + 3, 0, rb, -1);
            pb = rb; rb = rb == 2 * STAGE ? 0 : rb + STAGE;
        }
        static_assert((NKT - 3) % 3 == 2, "tail buffers");
        phase(IC<12>{}, 2 * STAGE, 0, NKT - 1, 1, STAGE, -1);
        phase(IC<9>{}, 2 * STAGE, 1, -1, 0, 0, -1);
        phase(IC<6>{}, 0, 0, -1, 0, 0, -1);
        phase(IC<3>{}, 0, 1, -1, 0, 0, -1);
        phase(IC<0>{}, STAGE, 0, -1, 0, 0, -1);
        phase(IC<-1>{}, STAGE, 1, -1, 0, 0, -1);
    } else {
        stage(0, 0, 0); stage(1, 0, STAGE); stage(2, 0, 2 * STAGE); stage(3, 0, 3 * STAGE);
        WAIT_V(12);
        BAR();
        if (wid >= 4) BAR();
        int rb = 0, pb = 4 * STAGE;
        for (int t = 0; t < NKT - 4; ++t) {
            phase(IC<12>{}, rb, 0, t + 4, 0, pb, (t & 1) && t + 5 < NKT ? (t + 5) >> 1 : -1);
            pb = rb; rb = rb == 4 * STAGE ? 0 : rb + STAGE;
        }
        static_assert((NKT - 4) % 5 == 4, "tail buffers");
        phase(IC<8>{}, 4 * STAGE, 0, -1, 0, 0, -1);
        phase(IC<4>{}, 0, 0, -1, 0, 0, -1);
        phase(IC<0>{}, STAGE, 0, -1, 0, 0, -1);
        phase(IC<-1>{}, 2 * STAGE, 0, -1, 0, 0, -1);
    }
    if (wid < 4) BAR();
    STAMP(stamp0 + 1);

    const bool dead = __builtin_amdgcn_readfirstlane(__hip_atomic_load(tmo, __ATOMIC_RELAXED, __HIP_MEMORY_SCOPE_AGENT)) == call;
    const int panel = hp0 >> 1;
    const int srank = panel < 12 ? (rank & 4) + panel / 3 : ((rank & 4) ^ 4) + (panel - 12);
    const int row0 = srank * 1024 + (panel < 12 ? (panel % 3) * 256 : 768) + (hp0 & 1) * 128 + wm * (MF * 16);
    const bool remote = cg >= 8;
    const int drank = remote ? ((rank & 4) ^ 4) + ((cg - 8) >> 1) : rank;
    const int col0 = (remote ? q * 256 + (cg & 1) * 128 : cg * 128) + wn * 64;
    char* obase = (char*)P.out[drank] + ((size_t)row0 * NLOC + col0) * 4;
    const unsigned ovoff = (unsigned)(lane >> 4) * (4 * NLOC * 4) + (unsigned)(lane & 15) * 16;
    const float nanv = __builtin_nanf("");
    if (remote && paced) {
        GAS unsigned long long* nf = (GAS unsigned long long*)(myws + OFF_RATE + ((cg - 8) >> 1) * 128);
        const unsigned long long now = __builtin_amdgcn_s_memrealtime();
        unsigned long long at = 0;
        if (lane == 0) {
            at = __hip_atomic_fetch_max(nf, now * 16, __ATOMIC_RELAXED, __HIP_MEMORY_SCOPE_AGENT);
            asm volatile("" : "+v"(at));
            at = __hip_atomic_fetch_add(nf, (unsigned long long)(OCOST * MF / 4), __ATOMIC_RELAXED, __HIP_MEMORY_SCOPE_AGENT);
        }
        unsigned wait = __builtin_amdgcn_readfirstlane((unsigned)((at >> 4) - now));
        if (wait > 20000u) wait = 0;
        while (__builtin_amdgcn_s_memrealtime() - now < wait) __builtin_amdgcn_s_sleep(2);
    }
#pragma unroll
    for (int m = 0; m < MF; ++m)
#pragma unroll
        for (int r = 0; r < 4; ++r) {
            f4 o = {silu(acc[m][0][r]), silu(acc[m][1][r]), silu(acc[m][2][r]), silu(acc[m][3][r])};
            if (dead) o = (f4){nanv, nanv, nanv, nanv};
            char* ob = obase + (size_t)(m * 16 + r) * (NLOC * 4);
            const u4 ou = __builtin_bit_cast(u4, o);
            if (remote) GST4(ovoff, ou, ob, 0);
            else *(GAS u4*)(ob + ovoff) = ou;
        }
    asm volatile("s_waitcnt vmcnt(0)" ::: "memory");
    __builtin_amdgcn_s_barrier();
    asm volatile("" ::: "memory");
    if (remote && wid == 0 && lane == 0) flag_store(P.ws[drank] + OFF_FO + (q * 64 + oflag) * 4, call);
    STAMP(stamp0 + 2);
}

__global__ void __launch_bounds__(512, 2) ag_gemm_kernel(const Params Parg) {
    const CAS Params& P = *(const CAS Params*)__builtin_amdgcn_kernarg_segment_ptr();
    (void)Parg;
    __shared__ __attribute__((aligned(16))) char lds_raw[5 * 32768];
    LAS char* lds = (LAS char*)lds_raw;
    const int lane = threadIdx.x & 63, wid = __builtin_amdgcn_readfirstlane(threadIdx.x >> 6);
    const int xcd = blockIdx.x & 7, slot = blockIdx.x >> 3;
    char* myws = P.ws[P.rank];
    STAMP(0);
    GAS unsigned* seq = (GAS unsigned*)(myws + OFF_SEQ) + (blockIdx.x * 8 + wid) * 4;
    const unsigned call = __builtin_amdgcn_readfirstlane(__hip_atomic_load(seq, __ATOMIC_RELAXED, __HIP_MEMORY_SCOPE_AGENT)) + 1u;
    if (lane == 0) __hip_atomic_store(seq, call, __ATOMIC_RELAXED, __HIP_MEMORY_SCOPE_AGENT);
    const int q = P.rank & 3;

    int s2 = -1;
    if (slot < 24) {
        const int pi = slot >> 3, ci = slot & 7, ps = xcd >> 1, h = xcd & 1;
        const int panel = pi < 2 ? 3 * ps + pi : 12 + ps;
        const int cg = ci < 4 ? 4 * h + ci : 8 + 2 * (ci - 4) + h;
        const int ob = (wid - 1) * 192 + (int)blockIdx.x;
        if (wid > 0 && ob < 512) { __builtin_amdgcn_s_sleep(64); convert_own_b(P, myws, ob, call, lane); }
        gemm_tile<4>(P, lds, myws, panel * 2, cg, (ps * 3 + pi) * 2 + h, true, call, wid, lane, 1);
        if (ci < 4) s2 = pi * 4 + ci;
    } else {
        const int g = (31 - slot) * 8 + xcd;
        const int pw = wid * 64 + g;
        if (pw < NP) pusher(P, myws, pw, g < 32 && wid * 32 + g < NP2 ? wid * 32 + g : -1, call, lane);
        STAMP(1);
        if (slot < 28) s2 = 12 + (slot - 24);
        if (g == 31 && wid == 7) {
            GAS unsigned* tmo = (GAS unsigned*)(myws + OFF_TMO);
            GAS unsigned* fo = (GAS unsigned*)(myws + OFF_FO);
            const int i = lane < 40 ? lane : 0;
            const unsigned long long t0 = __builtin_amdgcn_s_memrealtime();
            for (;;) {
                bool bad = false;
#pragma unroll
                for (int s = 0; s < 4; ++s) bad |= __hip_atomic_load(fo + s * 64 + i, __ATOMIC_RELAXED, __HIP_MEMORY_SCOPE_SYSTEM) < call;
                if (__builtin_amdgcn_ballot_w64(bad) == 0) break;
                if (__builtin_amdgcn_readfirstlane(__hip_atomic_load(tmo, __ATOMIC_RELAXED, __HIP_MEMORY_SCOPE_AGENT)) == call ||
                    __builtin_amdgcn_s_memrealtime() - t0 > DEADLINE) {
                    __hip_atomic_store(tmo, call, __ATOMIC_RELAXED, __HIP_MEMORY_SCOPE_AGENT);
                    P.out[P.rank][(size_t)(P.rank * 1024) * NLOC + lane] = __builtin_nanf("");
                    break;
                }
                __builtin_amdgcn_s_sleep(4);
            }
            STAMP(2);
        }
    }
    if (s2 >= 0) {
        const int qs = (q + (s2 >> 2)) & 3, ci = s2 & 3, j = xcd >> 1, h = xcd & 1;
        const int cg = ci < 2 ? 2 * j + ci : 8 + 2 * j + (ci - 2);
        gemm_tile<2>(P, lds, myws, (3 * qs + 2) * 2 + h, cg, 24 + (qs * 2 + h) * 2 + (ci & 1), s2 < 4 || h == 0, call, wid, lane, 4);
    }
}

}

extern "C" void kernel_launch(int rank, int nranks, ncclComm_t comm, hipStream_t stream,
                              void* const* d_in, const int* in_sizes, int n_in,
                              void* d_out, int out_size, const int* in_dtypes, int out_dtype,
                              void* const* d_out_all, void* const* d_ws_all, size_t ws_bytes) {
    (void)comm; (void)in_dtypes; (void)out_dtype; (void)d_out;
    if (nranks != 8 || (unsigned)rank >= 8u || n_in < 2 || ws_bytes < WS_NEED || out_size != KDIM * NLOC ||
        in_sizes[0] != 1024 * KDIM || in_sizes[1] != KDIM * NLOC) {
        fprintf(stderr, "ag_gemm: unexpected problem shape\n");
        return;
    }
    Params P;
    P.x = (const float*)d_in[0]; P.w = (const float*)d_in[1];
    for (int p = 0; p < 8; ++p) { P.out[p] = (float*)d_out_all[p]; P.ws[p] = (char*)d_ws_all[p]; }
    P.rank = rank; P.pad = 0;
    hipLaunchKernelGGL(ag_gemm_kernel, dim3(256), dim3(512), 0, stream, P);
    HIP_CHECK(hipGetLastError());
}
```

```cpp
#include <hip/hip_runtime.h>
#include <rccl/rccl.h>
#include <cstdio>

#define HIP_CHECK(expr) do { hipError_t _e = (expr); if (_e != hipSuccess) { \
    fprintf(stderr, "HIP error %s at %s:%d\n", hipGetErrorString(_e), __FILE__, __LINE__); \
    return; } } while (0)

namespace {

typedef _Float16 h8 __attribute__((ext_vector_type(8)));
typedef float f4 __attribute__((ext_vector_type(4)));
typedef unsigned u4 __attribute__((ext_vector_type(4)));
#define LAS __attribute__((address_space(3)))
#define GAS __attribute__((address_space(1)))
#define CAS __attribute__((address_space(4)))

constexpr int KDIM = 8192, NLOC = 1024;
constexpr int NKT = KDIM / 64;
constexpr int NCH = NKT / 2;
constexpr int NSEQ = 1536;
constexpr size_t PANEL_B = 4u << 20;
constexpr size_t CG_B = 2u << 20;
constexpr size_t OFF_SEQ = 0;
constexpr size_t OFF_TMO = 64u << 10;
constexpr size_t OFF_CTR = OFF_TMO + 256;
constexpr size_t OFF_FA = 128u << 10;
constexpr size_t OFF_FB = OFF_FA + 16 * 64 * 32;
constexpr size_t OFF_FP = 256u << 10;
constexpr size_t FP_B = 5 * NSEQ * 4;
constexpr size_t OFF_DBG = 640u << 10;
constexpr size_t OFF_DATA = 1u << 20;
constexpr size_t A_B = 16 * PANEL_B, B_B = 16 * CG_B;
constexpr size_t OFF_PK = OFF_DATA + A_B + B_B;
constexpr int PK_C = 1536, PK_W = 2048, PK_O = 4096, PK_N = 6144;
constexpr size_t PK_B = (size_t)PK_N * 8192;
constexpr size_t WS_NEED = OFF_PK + 2 * PK_B;
static_assert(OFF_FB + 16 * 64 * 16 <= OFF_FP && OFF_FP + 2 * FP_B <= OFF_DBG, "flag area");
constexpr unsigned long long DEADLINE = 30000000ull;
constexpr unsigned M_RAW = 5, M_DEAD = 6;

#define NPC 35
#define WPC 7
#define SKEWD 12
#define SKEW1 9
#define PACE 10
#define LOOK 64
#define RAW0 0
#define RAW1 0
#define NWIN 10
#define NWIN1 NWIN
#define SWIN 900
#define RELAT 4
#define AUX 17
constexpr int N1 = 64 * 56, N2 = 512;
constexpr int NGEMM = 208;
constexpr int S1 = 8 * SKEW1;

struct Params {
    const float* x;
    const float* w;
    float* out;
    char* ws[8];
    int rank;
    int pad;
};

#define STAMP(i) do { } while (0)

#define GLD4(dst, voff, sbase) \
    asm volatile("global_load_dwordx4 %0, %1, %2" : "=v"(dst) : "v"(voff), "s"(sbase) : "memory")
#define GST4(voff, data, sbase, imm) \
    asm volatile("global_store_dwordx4 %0, %1, %2 offset:" #imm " sc0 sc1\n\ts_nop 1" :: "v"(voff), "v"(data), "s"(sbase) : "memory")
#define GST4P(voff, data, sbase) \
    asm volatile("global_store_dwordx4 %0, %1, %2\n\ts_nop 1" :: "v"(voff), "v"(data), "s"(sbase) : "memory")
#define WAIT16(v) asm volatile("s_waitcnt vmcnt(0)" \
    : "+v"(v[0]), "+v"(v[1]), "+v"(v[2]), "+v"(v[3]), "+v"(v[4]), "+v"(v[5]), "+v"(v[6]), "+v"(v[7]), \
      "+v"(v[8]), "+v"(v[9]), "+v"(v[10]), "+v"(v[11]), "+v"(v[12]), "+v"(v[13]), "+v"(v[14]), "+v"(v[15]) :: "memory")

__device__ __forceinline__ u4 pack8(float a0, float a1, float a2, float a3, float a4, float a5, float a6, float a7) {
    h8 h = {(_Float16)a0, (_Float16)a1, (_Float16)a2, (_Float16)a3, (_Float16)a4, (_Float16)a5, (_Float16)a6, (_Float16)a7};
    return __builtin_bit_cast(u4, h);
}
__device__ __forceinline__ void flag_store(char* p, unsigned v) {
    __hip_atomic_store((GAS unsigned*)p, v, __ATOMIC_RELAXED, __HIP_MEMORY_SCOPE_SYSTEM);
}
__device__ __forceinline__ void store8(const u4 (&h)[8], char* d0, unsigned half, unsigned vl16) {
    char* d1 = d0 + half;
    GST4(vl16, h[0], d0, 0); GST4(vl16, h[1], d0, 1024); GST4(vl16, h[2], d0, 2048); GST4(vl16, h[3], d0, 3072);
    GST4(vl16, h[4], d1, 0); GST4(vl16, h[5], d1, 1024); GST4(vl16, h[6], d1, 2048); GST4(vl16, h[7], d1, 3072);
}
__device__ __forceinline__ unsigned long long mode_mask(unsigned mode) { return mode >= M_RAW ? ~0ull : (1ull << (mode * 4u)) - 1ull; }

__device__ __forceinline__ void b_decode(int u2, int& a, int& b) {
    const int i = u2 >> 2, j = i - SKEWD;
    const int half = i < SKEWD ? 0 : i >= 128 - SKEWD ? 1 : (~j & 1);
    b = i < SKEWD ? i : i >= 128 - SKEWD ? i - 64 : (j >> 1) + ((j & 1) ? SKEWD : 0);
    a = 512 + 128 * half + 32 * (u2 & 3);
}
__device__ __forceinline__ bool set_decode(int set, int pos, int& a, int& b) {
    const int ch = set >> 3, g = set & 7;
    if (pos == 0 || pos >= 5) { a = (pos == 0 ? 0 : pos == 5 ? 256 : 768) + 32 * g; b = ch; return true; }
    a = 2 * ch + (g >> 2); b = (pos - 1) * 4 + (g & 3);
    return false;
}
__device__ __forceinline__ void unit_place(bool isx, int a, int b, int qs, bool own, unsigned& off, unsigned& fl) {
    if (isx) {
        const int panel = a < 768 ? 3 * qs + (a >> 8) : 12 + qs;
        off = (unsigned)OFF_DATA + (unsigned)panel * (unsigned)PANEL_B + (unsigned)b * 65536u + (unsigned)((a & 255) >> 4) * 2048u;
        fl = (unsigned)OFF_FA + (unsigned)(panel * 2048 + b * 32 + ((a & 255) >> 5) * 4);
    } else {
        const int cg = own ? b >> 1 : 8 + 2 * qs + ((b >> 1) & 1);
        off = (unsigned)(OFF_DATA + A_B) + (unsigned)cg * (unsigned)CG_B + (unsigned)a * 16384u + (unsigned)(b & 1) * 8192u;
        fl = (unsigned)OFF_FB + (unsigned)(cg * 1024 + a * 8 + (b & 1) * 4);
    }
}

typedef unsigned short us2 __attribute__((ext_vector_type(2)));
__device__ __forceinline__ unsigned bfi(unsigned m, unsigned x, unsigned y) { return (x & m) | (y & ~m); }
__device__ __forceinline__ unsigned pk_add(unsigned x, unsigned y) { return __builtin_bit_cast(unsigned, (us2)(__builtin_bit_cast(us2, x) + __builtin_bit_cast(us2, y))); }
constexpr unsigned LIST_MAX = 126;
__device__ __forceinline__ unsigned pack_unit(u4 (&h)[8], LAS char* scr, int lane) {
    unsigned es = 0;
#pragma unroll
    for (int j = 0; j < 4; ++j) es += (h[0][j] >> 10) & 0x001F001Fu;
    es = (es & 0xFFFFu) + (es >> 16);
    es += __builtin_amdgcn_update_dpp(0u, es, 0x111, 0xF, 0xF, true);
    es += __builtin_amdgcn_update_dpp(0u, es, 0x112, 0xF, 0xF, true);
    es += __builtin_amdgcn_update_dpp(0u, es, 0x114, 0xF, 0xF, true);
    es += __builtin_amdgcn_update_dpp(0u, es, 0x118, 0xF, 0xF, true);
    const int tot = __builtin_amdgcn_readlane(es, 15) + __builtin_amdgcn_readlane(es, 31) + __builtin_amdgcn_readlane(es, 47) + __builtin_amdgcn_readlane(es, 63);
    int e0 = (tot - 2652 + 511) >> 9;
    e0 = e0 < 0 ? 0 : e0 > 24 ? 24 : e0;
    const unsigned neg = ((0x10000u - ((unsigned)e0 << 10)) & 0xFFFFu) * 0x10001u;
    u4 s, g[7];
#pragma unroll
    for (int j = 0; j < 4; ++j) {
        const unsigned x7 = pk_add(h[7][j], neg), y = bfi(0x20002000u, x7 >> 2, x7);
        unsigned t = (x7 << 1) & 0xC000C000u;
#pragma unroll
        for (int i = 0; i < 7; ++i) {
            const unsigned x = pk_add(h[i][j], neg);
            t |= (x >> (13 - 2 * i)) & (0x00030003u << (2 * i));
            g[i][j] = bfi(0x60006000u, y << (13 - 2 * i), x);
        }
        s[j] = t;
    }
    unsigned n = 0;
#pragma unroll
    for (int j = 0; j < 4; ++j) {
        unsigned t = s[j];
        for (;;) {
            const unsigned long long m = __builtin_amdgcn_ballot_w64(t != 0);
            if (m == 0 || n > LIST_MAX) break;
            if (t != 0) {
                const unsigned b = (unsigned)__builtin_ctz(t) & 30u;
                const unsigned k = n + 2u + __builtin_amdgcn_mbcnt_hi((unsigned)(m >> 32), __builtin_amdgcn_mbcnt_lo((unsigned)m, 0u));
                ((LAS unsigned short*)scr)[k] = (unsigned short)(((unsigned)(lane * 4 + j) << 6) | (b << 1) | ((t >> b) & 3u));
                t &= ~(3u << b);
            }
            n += (unsigned)__builtin_popcountll(m);
        }
    }
    if (n > LIST_MAX) return M_RAW;
    asm volatile("" ::: "memory");
    u4 l = *(const LAS u4*)(scr + (lane & 15) * 16);
    asm volatile("" ::: "memory");
    if (lane == 0) l[0] = ((unsigned)e0 << 16) | n;
#pragma unroll
    for (int i = 0; i < 7; ++i) h[i] = g[i];
    h[7] = l;
    return (n + 33u) >> 5;
}
__device__ __forceinline__ void unpack_unit(u4 (&h)[8], LAS char* scr, int lane) {
    const unsigned hd = __builtin_amdgcn_readfirstlane(h[7][0]);
    const unsigned n = hd & 0xFFFFu, pos = (((hd >> 16) & 31u) << 10) * 0x10001u;
    *(LAS u4*)(scr + 1024 + lane * 16) = h[7];
    *(LAS u4*)(scr + lane * 16) = (u4){0u, 0u, 0u, 0u};
    asm volatile("" ::: "memory");
    const unsigned lst = *(const LAS unsigned*)(scr + 1024 + lane * 4);
    if ((unsigned)(2 * lane - 2) < n)
        __hip_atomic_fetch_or((LAS unsigned*)scr + ((lst >> 6) & 255u), (lst & 3u) << ((lst >> 1) & 30u), __ATOMIC_RELAXED, __HIP_MEMORY_SCOPE_WORKGROUP);
    if ((unsigned)(2 * lane - 1) < n)
        __hip_atomic_fetch_or((LAS unsigned*)scr + ((lst >> 22) & 255u), ((lst >> 16) & 3u) << ((lst >> 17) & 30u), __ATOMIC_RELAXED, __HIP_MEMORY_SCOPE_WORKGROUP);
    asm volatile("" ::: "memory");
    const u4 s = *(const LAS u4*)(scr + lane * 16);
    asm volatile("" ::: "memory");
#pragma unroll
    for (int j = 0; j < 4; ++j) {
        unsigned y = 0;
#pragma unroll
        for (int i = 0; i < 7; ++i) {
            y |= (h[i][j] >> (13 - 2 * i)) & (0x00030003u << (2 * i));
            h[i][j] = pk_add(bfi(0x60006000u, s[j] << (13 - 2 * i), h[i][j]), pos);
        }
        h[7][j] = pk_add(bfi(0x60006000u, s[j] >> 1, bfi(0x80008000u, y << 2, y)), pos);
    }
}

__device__ __forceinline__ void producer(const CAS Params& P, char* myws, LAS char* scr, int u, unsigned call, int lane) {
    const int q = P.rank & 3;
    const unsigned voffx = (unsigned)(lane & 15) * (KDIM * 4) + (unsigned)(lane >> 4) * 16;
    const unsigned voffw = (unsigned)(lane >> 4) * (4 * NLOC * 4) + (unsigned)(lane & 15) * 16;
    const unsigned vl16 = (unsigned)lane * 16;
    const unsigned long long t0 = __builtin_amdgcn_s_memrealtime();
    for (; u < N1 + N2; u += NGEMM * 8) {
        int a, b, idx, pk, rd;
        bool isx = true;
        if (u < N1) {
            constexpr int U0 = 5 * S1, U1 = U0 + 7 * (512 - S1);
            int set, pos;
            if (u < U0) { set = u / 5; pos = u - set * 5; idx = set; }
            else if (u < U1) { const int j = (u - U0) / 7; pos = u - U0 - j * 7; set = pos < 5 ? j + S1 : j; idx = S1 + 2 * j + (pos >= 5); }
            else { set = 512 - S1 + ((u - U1) >> 1); pos = 5 + ((u - U1) & 1); idx = 512 + set; }
            isx = set_decode(set, pos, a, b);
            pk = pos == 0 || pos == 5 ? idx : pos == 6 ? PK_C + set : PK_W + (pos - 1) * 512 + set;
            rd = pos == 0 || pos == 5 ? 4 : pos == 6 ? 5 : pos - 1;
        } else { idx = pk = 1024 + u - N1; rd = 4; b_decode(u - N1, a, b); }
        while (__builtin_amdgcn_s_memrealtime() - t0 < (unsigned)(u * PACE) >> 4) __builtin_amdgcn_s_sleep(2);
        const char* src = isx ? (const char*)P.x + (size_t)((unsigned)a * (KDIM * 4) + (unsigned)b * 512)
                              : (const char*)P.w + (size_t)((unsigned)a * (64 * NLOC * 4) + (unsigned)b * 256);
        const unsigned s0 = isx ? 64 : NLOC * 4, s1 = isx ? 16 * KDIM * 4 : 16 * NLOC * 4, s2 = isx ? 256 : 32 * NLOC * 4;
        const unsigned voff = isx ? voffx : voffw;
        f4 v[16]; u4 h[8];
#pragma unroll
        for (int i = 0; i < 16; ++i) { const char* s = src + ((i & 3) * s0 + ((i >> 2) & 1) * s1 + (i >> 3) * s2); GLD4(v[i], voff, s); }
        WAIT16(v);
        if (isx) {
#pragma unroll
            for (int i = 0; i < 8; ++i)
                h[i] = pack8(v[2 * i][0], v[2 * i][1], v[2 * i][2], v[2 * i][3], v[2 * i + 1][0], v[2 * i + 1][1], v[2 * i + 1][2], v[2 * i + 1][3]);
        } else {
#pragma unroll
            for (int i = 0; i < 8; ++i) {
                const int f = i >> 1, k = (i & 1) * 8;
                h[i] = pack8(v[k][f], v[k + 1][f], v[k + 2][f], v[k + 3][f], v[k + 4][f], v[k + 5][f], v[k + 6][f], v[k + 7][f]);
            }
        }
        unsigned off, fl;
        unit_place(isx, a, b, q, true, off, fl);
        if (rd != 5) store8(h, myws + off, isx ? 32768u : 4096u, vl16);
        unsigned mode = M_RAW;
        if (idx >= RAW0 && idx < NSEQ - RAW1) mode = pack_unit(h, scr, lane);
        {
            char* d0 = myws + (OFF_PK + (call & 1) * PK_B + (size_t)pk * 7168); char* d1 = d0 + 4096;
            char* d2 = myws + (OFF_PK + (call & 1) * PK_B + (size_t)(PK_O * 7168 + pk * 1024 + (mode < M_RAW ? (pk & 3) * 256 : 0)));
            GST4(vl16, h[0], d0, 0); GST4(vl16, h[1], d0, 1024); GST4(vl16, h[2], d0, 2048); GST4(vl16, h[3], d0, 3072);
            GST4(vl16, h[4], d1, 0); GST4(vl16, h[5], d1, 1024); GST4(vl16, h[6], d1, 2048);
            asm volatile("s_mov_b64 exec, %3\n\tglobal_store_dwordx4 %0, %1, %2 sc0 sc1\n\ts_mov_b64 exec, -1\n\ts_waitcnt vmcnt(0)"
                         :: "v"(vl16), "v"(h[7]), "s"(d2), "s"(mode_mask(mode)) : "memory");
        }
        const unsigned fo = lane < 5 ? (unsigned)OFF_FP + (call & 1) * (unsigned)FP_B + (unsigned)(lane * NSEQ + idx) * 4u : fl;
        if (rd == 5 ? lane < 4 : lane == rd || lane == 5) flag_store(myws + fo, lane < 5 ? call * 8u + mode : call);
    }
}

__device__ __forceinline__ void flag_mirror(char* myws, const char* fp, LAS unsigned* info, LAS unsigned* prog, unsigned call, int lane) {
    GAS unsigned* tmo = (GAS unsigned*)(myws + OFF_TMO);
    int pos = 0, nl = 16;
    unsigned long long t0 = __builtin_amdgcn_s_memrealtime();
    while (pos < NSEQ) {
        const int pr = (int)__builtin_amdgcn_readfirstlane(__hip_atomic_load(prog, __ATOMIC_RELAXED, __HIP_MEMORY_SCOPE_WORKGROUP));
        if (pos <= pr + LOOK) {
            const int i = pos + 4 * lane;
            bool bad = false;
            if (lane < nl && i < NSEQ) {
                u4 f;
                asm volatile("global_load_dwordx4 %0, %1, %2 sc0 sc1\n\ts_waitcnt vmcnt(0)" : "=&v"(f) : "v"(i * 4), "s"(fp) : "memory");
                *(LAS u4*)(info + i) = f;
                bad = f[0] < call * 8u || f[1] < call * 8u || f[2] < call * 8u || f[3] < call * 8u;
            }
            const unsigned long long m = __builtin_amdgcn_ballot_w64(bad);
            const int n = m ? (__builtin_ctzll(m) & ~3) : nl;
            nl = m ? 16 : 64;
            if (n) { pos += 4 * n; t0 = __builtin_amdgcn_s_memrealtime(); continue; }
        }
        if (__builtin_amdgcn_readfirstlane(__hip_atomic_load(tmo, __ATOMIC_RELAXED, __HIP_MEMORY_SCOPE_AGENT)) == call ||
            __builtin_amdgcn_s_memrealtime() - t0 > DEADLINE) {
            __hip_atomic_store(tmo, call, __ATOMIC_RELAXED, __HIP_MEMORY_SCOPE_AGENT);
            asm volatile("s_waitcnt vmcnt(0)" ::: "memory");
            for (int i = pos + lane; i < NSEQ; i += 64) info[i] = call * 8u + M_DEAD;
            return;
        }
        if (pos > pr + 24) __builtin_amdgcn_s_sleep(24);
    }
}

__device__ __forceinline__ void puller(const CAS Params& P, char* myws, LAS unsigned* info, LAS unsigned* prog, LAS char* scr,
                                       int link, int peer, int cur, int nw, unsigned win1, unsigned win, unsigned call, int wid, int lane) {
    const int q = P.rank & 3, qs = peer & 3;
    const bool cross = link < 4;
    const char* pk = P.ws[peer] + (OFF_PK + (call & 1) * PK_B + 4096);
    const unsigned vl16 = (unsigned)lane * 16, one = 1, aoff = (unsigned)OFF_CTR + (unsigned)link * 128u;
    const unsigned ovoff = (unsigned)(lane >> 4) * (4 * NLOC * 4) + (unsigned)(lane & 15) * 16;
    const unsigned base = (call - 1u) * (unsigned)NSEQ - (unsigned)nw;
    const unsigned rel = (unsigned)(size_t)(prog + 2);
    for (;;) {
        const bool lag = cur >= 1024 - S1 || (cur >= S1 && ((cur - S1) & 1));
        const int set = cur < S1 ? cur : cur >= 1024 - S1 ? cur - 512 : ((cur - S1) >> 1) + (lag ? 0 : S1);
        const int slot = !cross ? cur : cur >= 1024 ? PK_O - 1024 + q * 512 + cur : lag ? PK_C + set : PK_W + q * 512 + set;
        const char* src = pk + (size_t)(slot * (slot < PK_O ? 7168 : 8192));
        unsigned mode;
        for (;;) {
            mode = __builtin_amdgcn_readfirstlane(__hip_atomic_load(info + cur, __ATOMIC_ACQUIRE, __HIP_MEMORY_SCOPE_WORKGROUP)) - call * 8u;
            if (mode <= 7u) break;
            __builtin_amdgcn_s_sleep(1);
        }
        const unsigned long long mask = mode_mask(mode);
        const char* src7 = slot < PK_O ? pk + (size_t)(PK_O * 7168 - 4096 + slot * 1024 + (mode < M_RAW ? (slot & 3) * 256 : 0)) : src + 3072;
        unsigned my = 0;
        if (lane == 0) my = __hip_atomic_fetch_add(prog + 1, 1u, __ATOMIC_RELAXED, __HIP_MEMORY_SCOPE_WORKGROUP);
        my = __builtin_amdgcn_readfirstlane(my) - (cur < SWIN ? win1 : win);
        while ((int)(my - __builtin_amdgcn_readfirstlane(__hip_atomic_load(prog + 2, __ATOMIC_RELAXED, __HIP_MEMORY_SCOPE_WORKGROUP))) >= 0) { }
        u4 h[8]; unsigned tk;
        asm volatile("global_load_dwordx4 %0, %9, %10 offset:-4096 sc0 sc1\n\tglobal_load_dwordx4 %1, %9, %10 offset:-3072 sc0 sc1\n\t"
                     "global_load_dwordx4 %2, %9, %10 offset:-2048 sc0 sc1\n\tglobal_load_dwordx4 %3, %9, %10 offset:-1024 sc0 sc1\n\t"
                     "global_load_dwordx4 %4, %9, %10 sc0 sc1\n\tglobal_load_dwordx4 %5, %9, %10 offset:1024 sc0 sc1\n\t"
                     "global_load_dwordx4 %6, %9, %10 offset:2048 sc0 sc1\n\t"
                     "s_mov_b64 exec, %11\n\tglobal_load_dwordx4 %7, %9, %17 sc0 sc1\n\t"
                     "s_mov_b64 exec, 1\n\tglobal_atomic_add %8, %12, %13, %14 sc0\n\t"
                     "s_waitcnt vmcnt(%16)\n\t"
                     "ds_add_u32 %15, %13\n\t"
                     "s_mov_b64 exec, -1\n\ts_waitcnt vmcnt(0)"
                     : "=&v"(h[0]), "=&v"(h[1]), "=&v"(h[2]), "=&v"(h[3]), "=&v"(h[4]), "=&v"(h[5]), "=&v"(h[6]), "=&v"(h[7]), "=&v"(tk)
                     : "v"(vl16), "s"(src), "s"(mask), "v"(aoff), "v"(one), "s"(myws), "v"(rel), "n"(RELAT), "s"(src7) : "memory");
        const unsigned nxt = __builtin_amdgcn_readfirstlane(tk) - base;
        if (lane == 0) __hip_atomic_fetch_max(prog, nxt, __ATOMIC_RELAXED, __HIP_MEMORY_SCOPE_WORKGROUP);
        if (cross && cur >= 1024) {
            const int o = cur - 1024, gs = peer & 4;
            int row, cgh, w;
            if (o < 416) {
                const int t = o >> 4;
                w = (o >> 1) & 7; cgh = t & 1;
                row = (t < 8 ? (gs + (t >> 1)) * 1024 : t < 10 ? peer * 1024 + 512 : t < 18 ? (gs + ((t - 10) >> 1)) * 1024 + 256
                                                      : ((gs ^ 4) + ((t - 18) >> 1)) * 1024 + 768) + (w >> 1) * 64 + (o & 1) * 32;
            } else {
                const int hf = o >= 464, r = o - 416 - hf * 48;
                w = r & 7; cgh = (r >> 3) & 1;
                row = (gs + ((qs + 1 + (r >> 4)) & 3)) * 1024 + 512 + hf * 128 + (w >> 1) * 32;
            }
            char* ob = (char*)P.out + (size_t)(row * NLOC + qs * 256 + cgh * 128 + (w & 1) * 64) * 4;
            if (mode == M_DEAD) h[0] = (u4){0x7FC00000u, 0x7FC00000u, 0x7FC00000u, 0x7FC00000u};
#pragma unroll
            for (int k = 0; k < 8; ++k) GST4P(ovoff, h[k], ob + ((k >> 2) * 16 + (k & 3)) * (NLOC * 4));
            asm volatile("s_waitcnt vmcnt(0)" ::: "memory");
        } else {
            if (mode < M_RAW) unpack_unit(h, scr, lane);
            int a, b; bool isx = true;
            if (cur >= 1024) b_decode(cur - 1024, a, b);
            else isx = set_decode(set, lag ? (cross ? 6 : 5) : (cross ? 1 : 0), a, b);
            unsigned off, fl;
            unit_place(isx, a, b, qs, false, off, fl);
            store8(h, myws + off, isx ? 32768u : 4096u, vl16);
            asm volatile("s_waitcnt vmcnt(0)" ::: "memory");
            if (lane == 0) flag_store(myws + fl, call);
        }
        if (nxt >= (unsigned)NSEQ) break;
        cur = (int)nxt;
    }
}

__device__ __forceinline__ float silu(float y) {
    const float c = __builtin_amdgcn_fmed3f(y, -60.0f, 60.0f);
    const float e = __builtin_amdgcn_exp2f(c * -1.4426950408889634f);
    return y * __builtin_amdgcn_rcpf(1.0f + e);
}

#define WAIT_V(n) asm volatile("s_waitcnt vmcnt(%0)" :: "n"(n) : "memory")
#define WAIT_L0() asm volatile("s_waitcnt lgkmcnt(0)" ::: "memory")
#define BAR() do { __builtin_amdgcn_sched_barrier(0); __builtin_amdgcn_s_barrier(); asm volatile("" ::: "memory"); __builtin_amdgcn_sched_barrier(0); } while (0)
template <int V> struct IC { static constexpr int value = V; };

template <int MF>
__device__ __forceinline__ void gemm_tile(const CAS Params& P, LAS char* lds, char* myws, int hp0, int cg, int o0,
                                          unsigned call, int wid, int lane, int stamp0) {
    constexpr int ABYTES = MF * 8192, STAGE = ABYTES + 16384, NST = MF == 4 ? 3 : 5;
    const int rank = P.rank;
    const size_t slot = OFF_DATA;
    const char* ga = myws + slot + (size_t)(hp0 >> 1) * PANEL_B + (hp0 & 1) * 16384 + wid * (MF * 1024) + lane * 16;
    const char* gb = myws + slot + A_B + (size_t)cg * CG_B + wid * 2048 + lane * 16;
    const int wm = wid >> 1, wn = wid & 1;
    const unsigned frow = (lane & 2) ? (unsigned)OFF_FB + (unsigned)cg * 1024u
                                     : (unsigned)OFF_FA + (unsigned)(hp0 >> 1) * 2048u + (unsigned)(MF == 4 ? (lane & 1) : (hp0 & 1)) * 16u;
    const unsigned fstep = (lane & 2) ? 16u : 32u;
    GAS unsigned* tmo = (GAS unsigned*)(myws + OFF_TMO);
    int ready = 0;

    auto ensure = [&](int c) {
        if (c < ready) return;
        const unsigned long long t0 = __builtin_amdgcn_s_memrealtime();
        int lim = 15;
        for (;;) {
            int cc = c + ((lane >> 2) < lim ? (lane >> 2) : lim); cc = cc < NCH ? cc : NCH - 1;
            u4 f;
            asm volatile("global_load_dwordx4 %0, %1, %2 sc0 sc1\n\ts_waitcnt vmcnt(0)"
                         : "=&v"(f) : "v"(frow + (unsigned)cc * fstep), "s"(myws) : "memory");
            const bool bad = f[0] < call || f[1] < call || f[2] < call || f[3] < call;
            const unsigned long long m = __builtin_amdgcn_ballot_w64(bad);
            const int n = m ? (__builtin_ctzll(m) >> 2) : lim + 1;
            if (n > 0) { ready = c + n; return; }
            if (__builtin_amdgcn_readfirstlane(__hip_atomic_load(tmo, __ATOMIC_RELAXED, __HIP_MEMORY_SCOPE_AGENT)) == call ||
                __builtin_amdgcn_s_memrealtime() - t0 > DEADLINE) {
                __hip_atomic_store(tmo, call, __ATOMIC_RELAXED, __HIP_MEMORY_SCOPE_AGENT);
                ready = NCH; return;
            }
            lim = 0;
            __builtin_amdgcn_s_sleep(6);
        }
    };
#define GLDS(g, l, imm) __builtin_amdgcn_global_load_lds((const GAS void*)(g), (LAS void*)(l), 16, imm, AUX)
    auto stage = [&](int t, int kf, int bufoff) {
        const char* a = ga + (size_t)t * 32768 + kf * 1024; const char* b = gb + (size_t)t * 16384 + kf * 1024;
        LAS char* la = lds + bufoff + wid * (MF * 1024) + kf * 1024; LAS char* lb = lds + bufoff + ABYTES + wid * 2048 + kf * 1024;
        if (MF == 4) { GLDS(a, la, 0); GLDS(a, la, 2048); GLDS(b, lb, 0); }
        else { GLDS(a, la, 0); GLDS(a, la, 1024); GLDS(b, lb, 0); GLDS(b, lb, 1024); }
    };

    f4 acc[MF][4];
#pragma unroll
    for (int m = 0; m < MF; ++m)
#pragma unroll
        for (int n = 0; n < 4; ++n) acc[m][n] = (f4){0.f, 0.f, 0.f, 0.f};

    const int aoff = wm * (MF * 2048) + lane * 16, boff = ABYTES + wn * 8192 + lane * 16;
    auto phase = [&](auto waitn, int rbuf, int kf, int lt, int lkf, int lbuf, int pc) {
        const LAS char* pa = lds + rbuf + aoff + kf * 1024; const LAS char* pb = lds + rbuf + boff + kf * 1024;
        h8 a[4], b[MF == 4 ? 4 : 8];
        if (MF == 4) {
#pragma unroll
            for (int n = 0; n < 4; ++n) b[n] = *(const LAS h8*)(pb + n * 2048);
#pragma unroll
            for (int m = 0; m < 4; ++m) a[m] = *(const LAS h8*)(pa + m * 2048);
        } else {
#pragma unroll
            for (int n = 0; n < 8; ++n) b[n] = *(const LAS h8*)(pb + n * 1024);
#pragma unroll
            for (int m = 0; m < 4; ++m) a[m] = *(const LAS h8*)(pa + m * 1024);
        }
        __builtin_amdgcn_sched_barrier(0);
        if (lt >= 0) stage(lt, lkf, lbuf);
        if (decltype(waitn)::value >= 0) WAIT_V(decltype(waitn)::value < 0 ? 0 : decltype(waitn)::value);
        WAIT_L0();
        BAR();
        __builtin_amdgcn_s_setprio(1);
        if (MF == 4) {
#pragma unroll
            for (int m = 0; m < 4; ++m)
#pragma unroll
                for (int n = 0; n < 4; ++n) acc[m][n] = __builtin_amdgcn_mfma_f32_16x16x32_f16(a[m], b[n], acc[m][n], 0, 0, 0);
        } else {
#pragma unroll
            for (int k = 0; k < 2; ++k)
#pragma unroll
                for (int m = 0; m < 2; ++m)
#pragma unroll
                    for (int n = 0; n < 4; ++n) acc[m][n] = __builtin_amdgcn_mfma_f32_16x16x32_f16(a[m * 2 + k], b[n * 2 + k], acc[m][n], 0, 0, 0);
        }
        __builtin_amdgcn_s_setprio(0);
        __builtin_amdgcn_sched_barrier(0);
        if (wid == 0 && pc >= 0) {
            ensure(pc);
        }
        BAR();
    };

    if (wid == 0) { ensure(0); ensure(1); if (MF == 2) ensure(2); }
    BAR();
    STAMP(stamp0);
    if (MF == 4) {
        stage(0, 0, 0); stage(0, 1, 0); stage(1, 0, STAGE); stage(1, 1, STAGE); stage(2, 0, 2 * STAGE);
        WAIT_V(12);
        BAR();
        if (wid >= 4) BAR();
        int rb = 0, pb = 2 * STAGE;
        for (int t = 0; t < NKT - 3; ++t) {
            phase(IC<12>{}, rb, 0, t + 2, 1, pb, (t & 1) ? (t + 3) >> 1 : -1);
            phase(IC<12>{}, rb, 1, t + 3, 0, rb, -1);
            pb = rb; rb = rb == 2 * STAGE ? 0 : rb + STAGE;
        }
        static_assert((NKT - 3) % 3 == 2, "tail buffers");
        phase(IC<12>{}, 2 * STAGE, 0, NKT - 1, 1, STAGE, -1);
        phase(IC<9>{}, 2 * STAGE, 1, -1, 0, 0, -1);
        phase(IC<6>{}, 0, 0, -1, 0, 0, -1);
        phase(IC<3>{}, 0, 1, -1, 0, 0, -1);
        phase(IC<0>{}, STAGE, 0, -1, 0, 0, -1);
        phase(IC<-1>{}, STAGE, 1, -1, 0, 0, -1);
    } else {
        stage(0, 0, 0); stage(1, 0, STAGE); stage(2, 0, 2 * STAGE); stage(3, 0, 3 * STAGE);
        WAIT_V(12);
        BAR();
        if (wid >= 4) BAR();
        int rb = 0, pb = 4 * STAGE;
        for (int t = 0; t < NKT - 4; ++t) {
            phase(IC<12>{}, rb, 0, t + 4, 0, pb, (t & 1) && t + 5 < NKT ? (t + 5) >> 1 : -1);
            pb = rb; rb = rb == 4 * STAGE ? 0 : rb + STAGE;
        }
        static_assert((NKT - 4) % 5 == 4, "tail buffers");
        phase(IC<8>{}, 4 * STAGE, 0, -1, 0, 0, -1);
        phase(IC<4>{}, 0, 0, -1, 0, 0, -1);
        phase(IC<0>{}, STAGE, 0, -1, 0, 0, -1);
        phase(IC<-1>{}, 2 * STAGE, 0, -1, 0, 0, -1);
    }
    if (wid < 4) BAR();
    STAMP(stamp0 + 1);

    const bool dead = __builtin_amdgcn_readfirstlane(__hip_atomic_load(tmo, __ATOMIC_RELAXED, __HIP_MEMORY_SCOPE_AGENT)) == call;
    const int panel = hp0 >> 1;
    const int srank = panel < 12 ? (rank & 4) + panel / 3 : ((rank & 4) ^ 4) + (panel - 12);
    const int row0 = srank * 1024 + (panel < 12 ? (panel % 3) * 256 : 768) + (hp0 & 1) * 128 + wm * (MF * 16);
    const bool remote = cg >= 8;
    const int ou0 = ((cg - 8) >> 1) * 512 + o0 + wid * (MF / 2);
    char* obase = remote ? myws + (OFF_PK + (call & 1) * PK_B + (size_t)(PK_O + ou0) * 8192)
                         : (char*)P.out + ((size_t)row0 * NLOC + cg * 128 + wn * 64) * 4;
    const unsigned ovoff = remote ? (unsigned)lane * 16 : (unsigned)(lane >> 4) * (4 * NLOC * 4) + (unsigned)(lane & 15) * 16;
    const unsigned sm = remote ? 4096 : 16 * NLOC * 4, sr = remote ? 1024 : NLOC * 4;
    const float nanv = __builtin_nanf("");
#pragma unroll
    for (int m = 0; m < MF; ++m)
#pragma unroll
        for (int r = 0; r < 4; ++r) {
            f4 o = {silu(acc[m][0][r]), silu(acc[m][1][r]), silu(acc[m][2][r]), silu(acc[m][3][r])};
            if (dead) o = (f4){nanv, nanv, nanv, nanv};
            char* ob = obase + (m * sm + r * sr);
            const u4 ou = __builtin_bit_cast(u4, o);
            if (remote) GST4(ovoff, ou, ob, 0);
            else GST4P(ovoff, ou, ob);
        }
    if (remote) {
        asm volatile("s_waitcnt vmcnt(0)" ::: "memory");
        if (lane < MF / 2)
            flag_store(myws + (OFF_FP + (call & 1) * FP_B) + (((cg - 8) >> 1) * NSEQ + 1024 + o0 + wid * (MF / 2) + lane) * 4, call * 8u + M_RAW);
    }
    asm volatile("s_waitcnt vmcnt(0)" ::: "memory");
    STAMP(stamp0 + 2);
}

__global__ void __launch_bounds__(512, 2) ag_gemm_kernel(const Params Parg) {
    const CAS Params& P = *(const CAS Params*)__builtin_amdgcn_kernarg_segment_ptr();
    (void)Parg;
    __shared__ __attribute__((aligned(16))) char lds_raw[5 * 32768];
    LAS char* lds = (LAS char*)lds_raw;
    const int lane = threadIdx.x & 63, wid = __builtin_amdgcn_readfirstlane(threadIdx.x >> 6);
    const int xcd = blockIdx.x & 7, slot = blockIdx.x >> 3;
    char* myws = P.ws[P.rank];
    STAMP(0);
    GAS unsigned* seq = (GAS unsigned*)(myws + OFF_SEQ) + (blockIdx.x * 8 + wid) * 4;
    const unsigned call = __builtin_amdgcn_readfirstlane(__hip_atomic_load(seq, __ATOMIC_RELAXED, __HIP_MEMORY_SCOPE_AGENT)) + 1u;
    if (lane == 0) __hip_atomic_store(seq, call, __ATOMIC_RELAXED, __HIP_MEMORY_SCOPE_AGENT);
    const int q = P.rank & 3;

    if (slot < NGEMM / 8) {
        producer(P, myws, lds + wid * 2048, wid * NGEMM + (int)blockIdx.x, call, lane);
        STAMP(7);
        const int pi = slot >> 3, ps = xcd >> 1, h = xcd & 1, ci = pi < 3 ? slot & 7 : (slot & 1) * 4 + ps;
        const int panel = pi < 2 ? 3 * ps + pi : pi == 2 ? 12 + ps : 3 * q + 2;
        const int cg = ci < 4 ? 4 * h + ci : 8 + 2 * (ci - 4) + h;
        gemm_tile<4>(P, lds, myws, panel * 2, cg, pi == 3 ? 128 + h * 16 : (pi == 0 ? 0 : 32 + pi * 128) + xcd * 16, call, wid, lane, 1);
        const int k = slot < 8 ? slot : slot >= 24 ? slot - 16 : slot + 2;
        if (k < 12) {
            const int c3 = k / 3, si = k - c3 * 3, hf = c3 & 1;
            gemm_tile<2>(P, lds, myws, (3 * ((q + 1 + si) & 3) + 2) * 2 + hf, c3 < 2 ? 8 + xcd : xcd, 416 + hf * 48 + (si * 2 + h) * 8, call, wid, lane, 4);
        }
    } else {
        const int g = (int)blockIdx.x - NGEMM;
        LAS unsigned* info = (LAS unsigned*)lds;
        LAS unsigned* prog = info + NSEQ;
        for (int i = threadIdx.x; i < NSEQ + 3; i += 512) info[i] = 0;
        __syncthreads();
        if (g < NPC) {
            const int j = g / 7, link = g - j * 7, ncu = (NPC + 6 - link) / 7;
            const int peer = link < 4 ? ((P.rank & 4) ^ 4) + ((q + link) & 3) : (P.rank & 4) + ((q + link - 3) & 3);
            if (wid < WPC) {
                if (lane == 0) __hip_atomic_fetch_max(prog, (unsigned)(wid * ncu + j), __ATOMIC_RELAXED, __HIP_MEMORY_SCOPE_WORKGROUP);
                puller(P, myws, info, prog, lds + 8192 + wid * 2048, link, peer, wid * ncu + j, ncu * WPC,
                       (unsigned)((NWIN1 + ncu - 1 - j) / ncu), (unsigned)((NWIN + ncu - 1 - j) / ncu), call, wid, lane);
            } else if (wid == 7) {
                flag_mirror(myws, P.ws[peer] + (OFF_FP + (call & 1) * FP_B + (link < 4 ? q : 4) * (NSEQ * 4)), info, prog, call, lane);
            }
            STAMP(3);
        }
    }
}

}

extern "C" void kernel_launch(int rank, int nranks, ncclComm_t comm, hipStream_t stream,
                              void* const* d_in, const int* in_sizes, int n_in,
                              void* d_out, int out_size, const int* in_dtypes, int out_dtype,
                              void* const* d_out_all, void* const* d_ws_all, size_t ws_bytes) {
    (void)comm; (void)in_dtypes; (void)out_dtype; (void)d_out_all;
    if (nranks != 8 || (unsigned)rank >= 8u || n_in < 2 || ws_bytes < WS_NEED || out_size != KDIM * NLOC ||
        in_sizes[0] != 1024 * KDIM || in_sizes[1] != KDIM * NLOC) {
        fprintf(stderr, "ag_gemm: unexpected problem shape\n");
        return;
    }
    Params P;
    P.x = (const float*)d_in[0]; P.w = (const float*)d_in[1];
    P.out = (float*)d_out;
    for (int p = 0; p < 8; ++p) P.ws[p] = (char*)d_ws_all[p];
    P.rank = rank; P.pad = 0;
    hipLaunchKernelGGL(ag_gemm_kernel, dim3(256), dim3(512), 0, stream, P);
    HIP_CHECK(hipGetLastError());
}
```

```cpp
#include <hip/hip_runtime.h>
#include <rccl/rccl.h>
#include <cstdio>

#define HIP_CHECK(expr) do { hipError_t _e = (expr); if (_e != hipSuccess) { \
    fprintf(stderr, "HIP error %s at %s:%d\n", hipGetErrorString(_e), __FILE__, __LINE__); \
    return; } } while (0)

namespace {

typedef _Float16 h8 __attribute__((ext_vector_type(8)));
typedef float f4 __attribute__((ext_vector_type(4)));
typedef unsigned u4 __attribute__((ext_vector_type(4)));
typedef unsigned u2 __attribute__((ext_vector_type(2)));
#define LAS __attribute__((address_space(3)))
#define GAS __attribute__((address_space(1)))
#define CAS __attribute__((address_space(4)))

constexpr int KDIM = 8192, NLOC = 1024;
constexpr int NKT = KDIM / 64;
constexpr int NCH = NKT / 2;
constexpr int NI = 23 * 64;
constexpr int NCI = 17 * 64, NO = 480;
constexpr int NSEQ = NCI + NO;
constexpr int O_A0 = 32, O_A1 = 160, O_C = 288, O_B = 384;
constexpr size_t PANEL_B = 4u << 20;
constexpr size_t CG_B = 2u << 20;
constexpr size_t OFF_SEQ = 0;
constexpr size_t OFF_TMO = 64u << 10;
constexpr size_t OFF_CTR = OFF_TMO + 256;
constexpr size_t OFF_ROLE = OFF_TMO + 1280;
constexpr size_t OFF_MAP = OFF_TMO + 2048;
constexpr size_t OFF_FA = 128u << 10;
constexpr size_t OFF_FB = OFF_FA + 17 * 64 * 32;
constexpr size_t OFF_FP = 256u << 10;
constexpr size_t FP_B = 5 * NSEQ * 4;
constexpr size_t OFF_DBG = 640u << 10;
constexpr size_t OFF_DATA = 1u << 20;
constexpr size_t A_B = 17 * PANEL_B, B_B = 16 * CG_B;
constexpr size_t OFF_PK = OFF_DATA + A_B + B_B + (1u << 20);
constexpr int PK_W = 2048, PK_O = 4096, PK_N = PK_O + 4 * NO;
constexpr int PK_PG = (PK_N + 126) / 127;
constexpr size_t PK_B = (size_t)PK_PG << 21;
constexpr size_t WS_NEED = OFF_PK + 2 * PK_B;
static_assert(OFF_FB + 16 * 64 * 16 <= OFF_FP && OFF_FP + 2 * FP_B <= OFF_DBG, "flag area");
constexpr unsigned long long DEADLINE = 30000000ull;
constexpr unsigned M_RAW = 9, M_DEAD = 10;

#define NPC 35
#define WPC 7
#define SKC 13
#define SKA 5
#define SQ1 4
#define SQ2 10
#define SQ3 17
#define PACE 14
#define LOOK 64
#define RAW0 48
#define RAW1 0
#define DEFER 1
#define PD 2
#define NWIN 10
#define RELAT 4
#define UG 2900
#define QSLEEP 2
#define AUX 17
#define RMAP 0x75416320u
#define EARLY 0
#define ORAW 0
#define SMODE 1
#define XMODE 1
constexpr int NGEMM = 208;
constexpr int NPROD = 256 - NGEMM - NPC;

struct Params {
    const float* x;
    const float* w;
    float* out;
    char* ws[8];
    int rank;
    int pad;
};

#define VBLK blockIdx.x
#define STAMP(i) do { } while (0)

#define TREC(c) do { } while (0)
#define TDUMP(k) do { (void)tv; } while (0)
#define GST4(voff, data, sbase, imm) \
    asm volatile("global_store_dwordx4 %0, %1, %2 offset:" #imm " sc0 sc1\n\ts_nop 1" :: "v"(voff), "v"(data), "s"(sbase) : "memory")
#define GST4P(voff, data, sbase) \
    asm volatile("global_store_dwordx4 %0, %1, %2\n\ts_nop 1" :: "v"(voff), "v"(data), "s"(sbase) : "memory")
#define WAIT_V(n) asm volatile("s_waitcnt vmcnt(%0)" :: "n"(n) : "memory")
#define WAIT_L0() asm volatile("s_waitcnt lgkmcnt(0)" ::: "memory")

__device__ __forceinline__ u4 pack8(float a0, float a1, float a2, float a3, float a4, float a5, float a6, float a7) {
    h8 h = {(_Float16)a0, (_Float16)a1, (_Float16)a2, (_Float16)a3, (_Float16)a4, (_Float16)a5, (_Float16)a6, (_Float16)a7};
    return __builtin_bit_cast(u4, h);
}
__device__ __forceinline__ void flag_store(char* p, unsigned v) {
    __hip_atomic_store((GAS unsigned*)p, v, __ATOMIC_RELAXED, __HIP_MEMORY_SCOPE_SYSTEM);
}
__device__ __forceinline__ void store8(const u4 (&h)[8], char* d0, unsigned half, unsigned vl16) {
    char* d1 = d0 + half;
    GST4(vl16, h[0], d0, 0); GST4(vl16, h[1], d0, 1024); GST4(vl16, h[2], d0, 2048); GST4(vl16, h[3], d0, 3072);
    GST4(vl16, h[4], d1, 0); GST4(vl16, h[5], d1, 1024); GST4(vl16, h[6], d1, 2048); GST4(vl16, h[7], d1, 3072);
}
__device__ __forceinline__ unsigned long long lane_mask(unsigned n) { return n >= 64u ? ~0ull : (1ull << n) - 1ull; }
__device__ __forceinline__ void store_list(char* d, unsigned l, unsigned mode, unsigned vl4) {
    asm volatile("s_mov_b64 exec, %3\n\tglobal_store_dword %0, %1, %2 sc0 sc1\n\ts_mov_b64 exec, -1"
                 :: "v"(vl4), "v"(l), "s"(d), "s"(lane_mask(mode * 16u)) : "memory");
}

__device__ __forceinline__ int clamp64(int v) { return v < 0 ? 0 : v > 64 ? 64 : v; }
__device__ __forceinline__ int pos_w(int kt, int g) { const int ch = kt >> 1; return 9 * ch + 8 * clamp64(ch - SKC) + (kt & 1) * 4 + g; }
__device__ __forceinline__ int pos_x(int rb, int ch) {
    if (rb < 8) return 8 * ch + 8 * clamp64(ch - SKA) + rb;
    if (rb < 16) return 8 * ch + 8 * clamp64(ch + SKA + 1) + rb - 8;
    if (rb == 16) return 9 * ch + 8 * clamp64(ch - SKC) + 8;
    if (rb >= 24) return 8 * ch + 9 * clamp64(ch + SKC + 1) + rb - 24;
    const int j = (rb - 16) >> 1, r = ch + (j == 0 ? 0 : j == 1 ? SQ1 : j == 2 ? SQ2 : SQ3);
    return 1024 + clamp64(r + (j > 0)) + 2 * (clamp64(r - SQ1 + (j > 1)) + clamp64(r - SQ2 + (j > 2)) + clamp64(r - SQ3)) + (j ? rb & 1 : 0);
}
static_assert(NPC % 7 == 0 && 2 * PK_PG <= NGEMM && UG >= 8 * NGEMM && UG + 8 * NPROD <= PK_O && 0 <= SQ1 && SQ1 <= SQ2 && SQ2 <= SQ3 && SKC < 64 && SKA < 64, "skews");
__device__ __forceinline__ void place_x(int panel, int rb, int ch, unsigned& off, unsigned& fl) {
    off = (unsigned)OFF_DATA + (unsigned)panel * (unsigned)PANEL_B + (unsigned)(ch * 65536 + (rb & 7) * 4096);
    fl = (unsigned)OFF_FA + (unsigned)(panel * 2048 + ch * 32 + (rb & 7) * 4);
}
__device__ __forceinline__ void place_w(int cb, int kt, unsigned& off, unsigned& fl) {
    off = (unsigned)(OFF_DATA + A_B) + (unsigned)(cb >> 1) * (unsigned)CG_B + (unsigned)(kt * 16384 + (cb & 1) * 8192);
    fl = (unsigned)OFF_FB + (unsigned)((cb >> 1) * 1024 + kt * 8 + (cb & 1) * 4);
}
__device__ __forceinline__ int panel_row(int pub, int panel, int blk) {
    const int gs = pub & 4, s = panel / 3, k = panel - 3 * s;
    const int r = panel == 16 ? pub : panel >= 12 ? (gs ^ 4) + panel - 12 : (k == 2 && blk == 0 ? gs ^ 4 : gs) + s;
    return (int)((RMAP >> (4 * r)) & 7u) * 1024 + (panel >= 12 ? 768 : k * 256) + blk * 32;
}
__device__ __forceinline__ size_t pk_off(int pk, unsigned call) { return OFF_PK + (call & 1) * PK_B + (size_t)(pk + pk / 127 + 1) * 16384; }
__device__ __forceinline__ unsigned pk_half(char* myws, int pk, unsigned call) {
    return __hip_atomic_load((GAS unsigned*)(myws + OFF_MAP) + ((call & 1) * PK_PG + pk / 127), __ATOMIC_RELAXED, __HIP_MEMORY_SCOPE_SYSTEM);
}

#define PERM(hi, lo, sel) __builtin_amdgcn_perm(hi, lo, sel)
#define ABIT(hi, lo, s) __builtin_amdgcn_alignbit(hi, lo, s)
#define HW(k) h[(k) >> 2][(k) & 3]
#define FENCE() asm volatile("" ::: "memory")
__device__ __forceinline__ unsigned bfi(unsigned m, unsigned x, unsigned y) { return (x & m) | (y & ~m); }
__device__ __forceinline__ unsigned wave_scan(unsigned v) {
    v += __builtin_amdgcn_update_dpp(0u, v, 0x111, 0xF, 0xF, true);
    v += __builtin_amdgcn_update_dpp(0u, v, 0x112, 0xF, 0xF, true);
    v += __builtin_amdgcn_update_dpp(0u, v, 0x114, 0xF, 0xF, true);
    v += __builtin_amdgcn_update_dpp(0u, v, 0x118, 0xF, 0xF, true);
    v += __builtin_amdgcn_update_dpp(0u, v, 0x142, 0xA, 0xF, false);
    v += __builtin_amdgcn_update_dpp(0u, v, 0x143, 0xC, 0xF, false);
    return v;
}
__device__ __forceinline__ int wave_sum(unsigned v) { return __builtin_amdgcn_readlane(wave_scan(v), 63); }
__device__ __forceinline__ unsigned escapes(unsigned c, unsigned acc) { return (unsigned)__builtin_popcount(c & (c >> 1) & 0x55555555u) + acc; }
__device__ __forceinline__ void make_lut(LAS char* lut, bool pack, int lane) {
#pragma unroll
    for (int j = 0; j < 4; ++j) {
        unsigned sel = pack ? 0x0C0C0C0Cu : 0u, k = 0;
#pragma unroll
        for (int i = 0; i < 4; ++i) {
            const unsigned c = i < 3 ? ((unsigned)lane >> (2 * i)) & 3u : (unsigned)j;
            if (pack) sel ^= c == 3u ? (0x0Cu ^ (unsigned)i) << (8 * k) : 0u; else sel |= (c == 3u ? k : 4u + c) << (8 * i);
            k += c == 3u;
        }
        *(LAS u2*)(lut + (j * 64 + lane) * 8) = (u2){sel, pack ? 8u * k : k};
    }
}
__device__ __forceinline__ unsigned code4(unsigned thi, unsigned tlo, unsigned s) { return __builtin_amdgcn_udot4(PERM(thi, tlo, s), 0x80200802u, 0u - 0xF3u * 170u, false); }
template <int N> __device__ __forceinline__ void append(const unsigned* s, const u2* e, unsigned pos) {
#pragma unroll
    for (int g = 0; g < N; ++g) {
        const unsigned long long c = (unsigned long long)PERM(0u, s[g], e[g][0]) << (pos & 31u);
        asm volatile("v_cmpx_ne_u32_e32 vcc, 0, %1\n\tds_or_b32 %0, %1\n\tv_cmpx_ne_u32_e32 vcc, 0, %2\n\tds_or_b32 %0, %2 offset:4\n\ts_mov_b64 exec, -1"
                     :: "v"((pos >> 3) & ~3u), "v"((unsigned)c), "v"((unsigned)(c >> 32)) : "vcc", "memory");
        pos += e[g][1];
    }
}
constexpr int SCR_B = 3328;
constexpr int SCR_3 = 2064, SCR_T = 2592, PBUF = 16384 + SCR_B;
__device__ __forceinline__ unsigned pack_unit(u4 (&h)[8], LAS char* scr, const LAS char* lut, int lane, unsigned& r0, unsigned& r2) {
    const u4 z = {0u, 0u, 0u, 0u};
    *(LAS u4*)(scr + lane * 16) = z; *(LAS u4*)(scr + 1024 + lane * 16) = z; *(LAS u4*)(scr + 2048 + lane * 16) = z;
    FENCE();
    unsigned G[16], s[16], a[16], cw[4];
#pragma unroll
    for (int g = 0; g < 16; ++g) { G[g] = PERM(HW(2 * g + 1), HW(2 * g), 0x07050301u); s[g] = (G[g] >> 1) & 0x3E3E3E3Eu; }
    const int m = wave_sum(__builtin_amdgcn_udot4(s[0] + s[5] + s[10] + s[15], 0x01010101u, 0u, false)) + 205;
    const unsigned b3 = ((unsigned)(m >> 11) + 3u) & 31u, type = (m & 2047) > 1120;
    u2 e[16];
#pragma unroll
    for (int g = 0; g < 16; ++g) {
        s[g] = ((2u * b3 + 65u) * 0x01010101u - s[g]) & 0x3F3F3F3Fu;
        a[g] = code4(0xF300F700u, 0xFB00FF00u, s[g]);
        e[g] = *(const LAS u2*)(lut + a[g]);
    }
    unsigned n = 0;
#pragma unroll
    for (int j = 0; j < 4; ++j) { cw[j] = (a[4 * j] >> 3) | (a[4 * j + 1] << 5) | (a[4 * j + 2] << 13) | (a[4 * j + 3] << 21); n = escapes(cw[j], n); }
    unsigned sc = wave_scan(n);
    r2 = (__builtin_amdgcn_readlane(sc, 63) + 31u) >> 5;
    if (r2 > 62u) return M_RAW;
    append<16>(s, e, (sc - n + (unsigned)(size_t)scr) * 8u);
    FENCE();
    const u4 q0 = *(const LAS u4*)(scr + lane * 32), q1 = *(const LAS u4*)(scr + lane * 32 + 16);
#pragma unroll
    for (int g = 0; g < 8; ++g) {
        const unsigned x = g < 4 ? q0[g & 3] : q1[g & 3];
        s[g] = (x | ~PERM(0xFF000000u, 0xFF00F9F6u, x)) - 0x06060606u;
        a[g] = code4(type ? 0xFF00F700u : 0xFB00F700u, type ? 0xF300FB00u : 0xF300FF00u, s[g]);
        e[g] = *(const LAS u2*)(lut + a[g]);
    }
    const unsigned c0 = (a[0] >> 3) | (a[1] << 5) | (a[2] << 13) | (a[3] << 21), c1 = (a[4] >> 3) | (a[5] << 5) | (a[6] << 13) | (a[7] << 21);
    n = escapes(c1, escapes(c0, 0u));
    sc = wave_scan(n);
    const unsigned r3 = (__builtin_amdgcn_readlane(sc, 63) + 7u) >> 3;
    if (r3 > 64u) return M_RAW;
    append<8>(s, e, (sc - n + (unsigned)(size_t)(scr + SCR_3)) * 8u);
    FENCE();
    u2 y = *(const LAS u2*)(scr + SCR_3 + lane * 8);
#pragma unroll
    for (int j = 0; j < 2; ++j) y[j] = (y[j] | ~PERM(0xFF000000u, 0xFF00F9F8u, y[j])) - 0x07070707u;
    unsigned w = ((y[0] >> 1) & 0x07070707u) | ((y[1] << 2) & 0x38383838u);
    w |= ((w >> 24) * 0x41040u) & 0xC0C0C0u;
    LAS char* t = scr + SCR_T;
    t[4 + lane * 3] = (char)w; t[5 + lane * 3] = (char)(w >> 8); t[6 + lane * 3] = (char)(w >> 16);
    FENCE();
    const unsigned l0 = (3u * r3 + 5u) >> 1;
    unsigned bad = ((y[0] >> 4) & 0x03030303u) | (y[1] & 0x30303030u), nl = 0;
    for (;;) {
        const unsigned long long mk = __builtin_amdgcn_ballot_w64(bad != 0);
        if (mk == 0 || nl > 64u) break;
        if (bad != 0) {
            const unsigned p = (unsigned)__builtin_ctz(bad), k = l0 + nl + __builtin_amdgcn_mbcnt_hi((unsigned)(mk >> 32), __builtin_amdgcn_mbcnt_lo((unsigned)mk, 0u));
            ((LAS unsigned short*)t)[k] = (unsigned short)((((unsigned)lane * 8u + (p & 4u) + (p >> 3)) << 7) | ((4u * b3 - 24u - 2u * (((p & 4u) ? y[1] : y[0]) >> (p & 24u))) & 0x7Cu));
            bad &= ~(15u << (p & 28u));
        }
        nl += (unsigned)__builtin_popcountll(mk);
    }
    r0 = (l0 + nl + 3u) >> 2;
    if (nl > 64u || r0 + r2 > 64u) return M_RAW;
    FENCE();
    u2 f = *(const LAS u2*)(t + lane * 8);
    FENCE();
    if (lane == 0) f[0] = b3 | (type << 5) | (r0 << 8) | (nl << 16) | (l0 << 24);
    unsigned L[16];
#pragma unroll
    for (int g = 0; g < 16; ++g) L[g] = PERM(HW(2 * g + 1), HW(2 * g), 0x06040200u);
#pragma unroll
    for (int g = 0; g < 16; ++g) HW(g) = L[g];
#pragma unroll
    for (int j = 0; j < 6; ++j) {
        const unsigned x = bfi(0x83838383u, G[2 * j], ABIT(G[2 * j + 1], G[2 * j + 1], 29));
        HW(16 + j) = j < 4 ? bfi(0x60606060u, G[12 + j] << 5, x) : bfi(0x20202020u, G[2 * j + 4] >> 2, bfi(0x40404040u, G[2 * j + 5] >> 1, x));
    }
    h[5][2] = cw[0]; h[5][3] = cw[1]; h[6] = (u4){cw[2], cw[3], f[0], f[1]}; h[7][0] = c0; h[7][1] = c1;
    return (r0 + r2 + 7u) >> 3;
}
template <int N> __device__ __forceinline__ void deposit(unsigned* o, const u2* e, unsigned tab, unsigned pos) {
    u2 x[N]; unsigned p[N];
#pragma unroll
    for (int g = 0; g < N; ++g) {
        const LAS unsigned* d = (const LAS unsigned*)(size_t)(pos & ~3u);
        x[g] = (u2){d[0], d[1]}; p[g] = pos;
        pos += e[g][1];
    }
#pragma unroll
    for (int g = 0; g < N; ++g) o[g] = PERM(tab, __builtin_amdgcn_alignbyte(x[g][1], x[g][0], p[g]), e[g][0]);
}
__device__ __forceinline__ void unpack_unit(u4 (&h)[8], LAS char* scr, const LAS char* lut, int lane) {
    LAS char* t = scr + SCR_T;
    *(LAS u2*)(t + lane * 8) = (u2){h[6][2], h[6][3]};
    FENCE();
    const unsigned hd = __builtin_amdgcn_readfirstlane(h[6][2]), r0 = (hd >> 8) & 63u, type = hd & 32u;
    const unsigned kb = (((hd & 31u) << 2) | 0x80u) * 0x01010101u;
    const LAS unsigned* d3 = (const LAS unsigned*)(t + ((4 + lane * 3) & ~3));
    const unsigned w0 = d3[0], w1 = d3[1];
    unsigned le = 0;
    const bool ll = (unsigned)lane < ((hd >> 16) & 127u);
    if (ll) le = ((const LAS unsigned short*)t)[(hd >> 24) + lane];
    const unsigned c0 = (unsigned)lane < r0 ? 0u : h[6][2], c1 = (unsigned)lane < r0 ? 0u : h[6][3];
    const unsigned cw[4] = {h[5][2], h[5][3], h[6][0], h[6][1]};
    u2 e2[8], e1[16];
#pragma unroll
    for (int g = 0; g < 8; ++g) e2[g] = *(const LAS u2*)(lut + (((g < 4 ? c0 : c1) >> (8 * (g & 3))) & 255u) * 8u);
#pragma unroll
    for (int g = 0; g < 16; ++g) e1[g] = *(const LAS u2*)(lut + ((cw[g >> 2] >> (8 * (g & 3))) & 255u) * 8u);
    const unsigned n2 = escapes(c1, escapes(c0, 0u)), n1 = escapes(cw[3], escapes(cw[2], escapes(cw[1], escapes(cw[0], 0u))));
    const unsigned n = n1 | (n2 << 16), sc = wave_scan(n) - n;
    unsigned w = __builtin_amdgcn_alignbyte(w1, w0, (unsigned)lane * 3u);
    w = bfi(0x003F3F3Fu, w, __builtin_amdgcn_udot4(w & 0xC0C0C0u, 0x100401u, 0u, false) << 18);
    const unsigned t3l = (kb - (type ? 0x24201C18u : 0x24201C00u)) & 0x7C7C7C7Cu, t3h = (kb - 0x34302C28u) & 0x7C7C7C7Cu;
    *(LAS u2*)(scr + SCR_3 + lane * 8) = (u2){PERM(t3h, t3l, w & 0x07070707u), PERM(t3h, t3l, (w >> 3) & 0x07070707u)};
    FENCE();
    if (ll) scr[SCR_3 + (le >> 7)] = (char)(le & 0x7Cu);
    FENCE();
    unsigned E[16];
    deposit<8>(E, e2, (kb - (type ? 0x001410u : 0x181410u)) & 0x7C7C7C7Cu, (sc >> 16) + (unsigned)(size_t)(scr + SCR_3));
    FENCE();
    *(LAS u4*)(scr + lane * 32) = (u4){E[0], E[1], E[2], E[3]}; *(LAS u4*)(scr + lane * 32 + 16) = (u4){E[4], E[5], E[6], E[7]};
    FENCE();
    deposit<16>(E, e1, (kb - 0x04080Cu) & 0x7C7C7C7Cu, (sc & 0xFFFFu) + r0 * 32u + (unsigned)(size_t)scr);
    FENCE();
    unsigned L[16];
#pragma unroll
    for (int g = 0; g < 16; ++g) {
        L[g] = HW(g);
        const unsigned x = HW(16 + (g < 12 ? g >> 1 : g - 12));
        E[g] = bfi(0x7C7C7C7Cu, E[g], g >= 12 ? bfi(0x80808080u, HW(14 + (g >> 1)) << (2 - (g & 1)), x >> 5) : (g & 1) ? ABIT(x, x, 3) : x);
    }
#pragma unroll
    for (int g = 0; g < 16; ++g) { HW(2 * g) = PERM(E[g], L[g], 0x05010400u); HW(2 * g + 1) = PERM(E[g], L[g], 0x07030602u); }
}
constexpr unsigned LIST_MAX = 126;
__device__ __forceinline__ unsigned pack_out(u4 (&h)[8], LAS char* scr, int lane) {
    unsigned es = 0;
#pragma unroll
    for (int j = 0; j < 4; ++j) es += ((h[0][j] >> 23) & 255u) + ((h[4][j] >> 23) & 255u);
    int e0 = (wave_sum(es) - 2355 + 511) >> 9;
    e0 = e0 < 0 ? 0 : e0 > 248 ? 248 : e0;
    const unsigned base = (unsigned)e0 << 24;
    unsigned n = 0;
#pragma unroll
    for (int i = 0; i < 8; ++i) {
#pragma unroll
        for (int j = 0; j < 4; ++j) h[i][j] = ABIT(h[i][j], h[i][j], 31) - base;
        unsigned t = __builtin_amdgcn_perm(__builtin_amdgcn_perm(h[i][3], h[i][2], 0x0C0C0703u), __builtin_amdgcn_perm(h[i][1], h[i][0], 0x0C0C0703u), 0x05040100u) & 0xF8F8F8F8u;
        for (;;) {
            const unsigned long long m = __builtin_amdgcn_ballot_w64(t != 0);
            if (m == 0 || n > LIST_MAX) break;
            if (t != 0) {
                const unsigned b = (unsigned)__builtin_ctz(t) & 24u;
                const unsigned k = n + 2u + __builtin_amdgcn_mbcnt_hi((unsigned)(m >> 32), __builtin_amdgcn_mbcnt_lo((unsigned)m, 0u));
                ((LAS unsigned short*)scr)[k] = (unsigned short)(((unsigned)((i >> 2) * 1024 + lane * 16 + (i & 3) * 4) << 5) + (b << 2) + ((t >> (b + 3u)) & 31u));
                t &= ~(255u << b);
            }
            n += (unsigned)__builtin_popcountll(m);
        }
    }
    if (n > LIST_MAX) {
#pragma unroll
        for (int k = 0; k < 32; ++k) { const unsigned x = HW(k) + base; HW(k) = ABIT(x, x, 1); }
        return M_RAW;
    }
    asm volatile("" ::: "memory");
    unsigned l = *(const LAS unsigned*)(scr + lane * 4);
    asm volatile("" ::: "memory");
    if (lane == 0) l = ((unsigned)e0 << 16) | n;
    unsigned lo = 0;
#pragma unroll
    for (int p = 0; p < 5; ++p) {
        lo = ABIT(HW(27 + p), lo, 2);
#pragma unroll
        for (int j = 0; j < 5; ++j) HW(5 * p + j) = ABIT(HW(5 * p + j), HW(27 + p) << (25 - 5 * j), 27);
    }
    HW(25) = ABIT(HW(25), lo << 5, 27); HW(26) = ABIT(HW(26), lo, 27);
    h[6][3] = l;
    return (n + 33u) >> 5;
}
__device__ __forceinline__ void unpack_out(u4 (&h)[8], unsigned lst, LAS char* scr, int lane) {
    const unsigned hd = __builtin_amdgcn_readfirstlane(lst);
    const unsigned n = hd & 0xFFFFu, base = (hd >> 16) << 24;
    *(LAS u4*)(scr + lane * 16) = (u4){0u, 0u, 0u, 0u};
    *(LAS u4*)(scr + 1024 + lane * 16) = (u4){0u, 0u, 0u, 0u};
    asm volatile("" ::: "memory");
    if ((unsigned)(2 * lane - 2) < n) scr[(lst >> 5) & 2047u] = (char)(lst & 31u);
    if ((unsigned)(2 * lane - 1) < n) scr[lst >> 21] = (char)((lst >> 16) & 31u);
    asm volatile("" ::: "memory");
    const u4 s0 = *(const LAS u4*)(scr + lane * 16), s1 = *(const LAS u4*)(scr + 1024 + lane * 16);
    asm volatile("" ::: "memory");
    const unsigned lo = ABIT(HW(26), HW(25) << 27, 5);
#pragma unroll
    for (int p = 0; p < 5; ++p) {
        unsigned y = lo << (8 - 2 * p);
#pragma unroll
        for (int j = 0; j < 5; ++j) y = ABIT(HW(5 * p + j), y, 5);
        HW(27 + p) = y;
    }
#pragma unroll
    for (int k = 0; k < 32; ++k) {
        const unsigned x = ABIT((k < 16 ? s0 : s1)[(k >> 2) & 3] >> (8 * (k & 3)), HW(k), 5) + base;
        HW(k) = ABIT(x, x, 1);
    }
}

__device__ __forceinline__ void unit_decode(int u, int& kind, int& ch, int& g) {
    constexpr int U0 = 56 * SKC, U1 = U0 + 64 * (64 - SKC);
    if (u < U0) { const int s = u / 7; kind = u - 7 * s; ch = s >> 3; g = s & 7; }
    else if (u < U1) { const int v = u - U0; kind = v & 7; g = (v >> 3) & 7; ch = (v >> 6) + (kind == 7 ? 0 : SKC); }
    else { const int v = u - U1; kind = 7; g = v & 7; ch = 64 - SKC + (v >> 3); }
}
__device__ __forceinline__ void fetch_input(const CAS Params& P, LAS char* buf, int u, int lane) {
    int kind, ch, g;
    unit_decode(u, kind, ch, g);
    const bool isx = kind == 0 || kind > 4;
    const char* src = isx ? (const char*)P.x + (size_t)((unsigned)((kind ? kind - 4 : 0) * 8 + g) * (32 * KDIM * 4) + (unsigned)ch * 512)
                          : (const char*)P.w + (size_t)((unsigned)(2 * ch + (g >> 2)) * (64 * NLOC * 4) + (unsigned)((kind - 1) * 4 + (g & 3)) * 256);
    const unsigned s0 = isx ? 64 : NLOC * 4, s1 = isx ? 16 * KDIM * 4 : 16 * NLOC * 4, s2 = isx ? 256 : 32 * NLOC * 4;
    const unsigned voff = isx ? (unsigned)(lane & 15) * (KDIM * 4) + (unsigned)(lane >> 4) * 16 : (unsigned)(lane >> 4) * (4 * NLOC * 4) + (unsigned)(lane & 15) * 16;
#pragma unroll
    for (int i = 0; i < 16; ++i)
        __builtin_amdgcn_global_load_lds((const GAS void*)(src + ((i & 3) * s0 + ((i >> 2) & 1) * s1 + (i >> 3) * s2) + voff), (LAS void*)(buf + i * 1024), 16, 0, 0);
}
__device__ __forceinline__ void pace(unsigned long long t0, int u) {
    if (u < UG) while (__builtin_amdgcn_s_memrealtime() - t0 < (unsigned)(u * PACE) >> 4) __builtin_amdgcn_s_sleep(2);
}
__device__ __forceinline__ void producer(const CAS Params& P, char* myws, LAS char* buf, LAS char* lut, int u, int step, int uend, unsigned long long t0, unsigned call, int lane) {
    const int q = P.rank & 3;
    const unsigned vl16 = (unsigned)lane * 16;
    LAS char* scr = buf + 16384;
    for (; u < uend; u += step) {
        int ch, g, kind;
        unit_decode(u, kind, ch, g);
        const bool isx = kind == 0 || kind > 4;
        const int rb = (kind ? kind - 4 : 0) * 8 + g, kt = 2 * ch + (g >> 2), cb = (kind - 1) * 4 + (g & 3);
        const int idx = isx ? pos_x(rb, ch) : pos_w(kt, g & 3), pk = isx ? ch * 32 + rb : PK_W + (kind - 1) * 512 + ch * 8 + g;
        const int rd = !isx ? kind - 1 : rb == 16 || rb >= 24 ? 5 : 4;
        f4 v[16]; u4 h[8];
        unsigned hv;
        asm volatile("global_load_dword %0, %1, %2 sc0 sc1" : "=&v"(hv) : "v"((unsigned)OFF_MAP + ((call & 1) * PK_PG + pk / 127) * 4u), "s"(myws) : "memory");
#pragma unroll
        for (int i = 0; i < 16; ++i) v[i] = *(const LAS f4*)(buf + i * 1024 + lane * 16);
        __builtin_amdgcn_sched_barrier(0);
        WAIT_L0();
        const bool ahead = u >= UG && u + step < uend;
        if (ahead) fetch_input(P, buf, u + step, lane);
        __builtin_amdgcn_sched_barrier(0);
        if (isx) {
#pragma unroll
            for (int i = 0; i < 8; ++i)
                h[i] = pack8(v[2 * i][0], v[2 * i][1], v[2 * i][2], v[2 * i][3], v[2 * i + 1][0], v[2 * i + 1][1], v[2 * i + 1][2], v[2 * i + 1][3]);
        } else {
#pragma unroll
            for (int i = 0; i < 8; ++i) {
                const int f = i >> 1, k = (i & 1) * 8;
                h[i] = pack8(v[k][f], v[k + 1][f], v[k + 2][f], v[k + 3][f], v[k + 4][f], v[k + 5][f], v[k + 6][f], v[k + 7][f]);
            }
        }
        unsigned off, fl;
        if (isx) place_x(rb < 24 ? 3 * q + (rb >> 3) : 16, rb, ch, off, fl); else place_w(cb, kt, off, fl);
        const bool own = !isx || rb != 16;
        if (ahead) asm volatile("s_waitcnt vmcnt(16)" : "+v"(hv) :: "memory"); else asm volatile("s_waitcnt vmcnt(0)" : "+v"(hv) :: "memory");
        if (own) store8(h, myws + off, isx ? 32768u : 4096u, vl16);
        unsigned mode = M_RAW, r0 = 0, r2 = 0;
        if (idx >= RAW0 && (rd != 4 || idx < NI - RAW1)) mode = pack_unit(h, scr, lut, lane, r0, r2);
        const unsigned half = SMODE == 0 ? idx & 1 : (__builtin_amdgcn_readfirstlane(hv) ^ (rd != 4) ^ (SMODE == 2)) & 1u;
        char* d0 = myws + (pk_off(pk, call) + half * 8192); char* d1 = d0 + 4096;
        GST4(vl16, h[0], d0, 0); GST4(vl16, h[1], d0, 1024); GST4(vl16, h[2], d0, 2048); GST4(vl16, h[3], d0, 3072);
        GST4(vl16, h[4], d1, 0); GST4(vl16, h[5], d1, 1024);
        if (mode < M_RAW) {
            const u2 p0 = {h[6][0], h[6][1]}, p1 = {h[6][2], h[6][3]}, p2 = {h[7][0], h[7][1]};
            asm volatile("global_store_dwordx2 %0, %1, %4 offset:2048 sc0 sc1\n\t"
                         "s_mov_b64 exec, %6\n\tglobal_store_dwordx2 %0, %2, %4 offset:2560 sc0 sc1\n\t"
                         "s_mov_b64 exec, %7\n\tglobal_store_dwordx2 %0, %3, %5 offset:2560 sc0 sc1\n\ts_mov_b64 exec, -1"
                         :: "v"(vl16 >> 1), "v"(p0), "v"(p1), "v"(p2), "s"(d1), "s"(d1 + r0 * 8u), "s"(lane_mask(r0)), "s"(lane_mask(r2)) : "memory");
        } else { GST4(vl16, h[6], d1, 2048); GST4(vl16, h[7], d1, 3072); }
        WAIT_V(0);
        const unsigned fo = lane < 5 ? (unsigned)OFF_FP + (call & 1) * (unsigned)FP_B + (unsigned)(lane * NSEQ + idx) * 4u : fl;
        if ((rd == 5 ? lane < 4 : lane == rd) || (own && lane == 5)) flag_store(myws + fo, lane < 5 ? call * 32u + half * 16u + mode : call);
        if (u + step < UG) { pace(t0, u + step); fetch_input(P, buf, u + step, lane); WAIT_V(0); }
    }
}

__device__ __forceinline__ void flag_mirror(char* myws, const char* fp, int nseq, LAS unsigned* info, LAS unsigned* prog, unsigned call, int lane) {
    GAS unsigned* tmo = (GAS unsigned*)(myws + OFF_TMO);
    int pos = 0, nl = 16;
    unsigned long long t0 = __builtin_amdgcn_s_memrealtime();
    while (pos < nseq) {
        const int pr = (int)__builtin_amdgcn_readfirstlane(__hip_atomic_load(prog, __ATOMIC_RELAXED, __HIP_MEMORY_SCOPE_WORKGROUP));
        if (pos <= pr + LOOK) {
            const int i = pos + 4 * lane;
            bool bad = false;
            if (lane < nl && i < nseq) {
                u4 f;
                asm volatile("global_load_dwordx4 %0, %1, %2 sc0 sc1\n\ts_waitcnt vmcnt(0)" : "=&v"(f) : "v"(i * 4), "s"(fp) : "memory");
                *(LAS u4*)(info + i) = f;
                bad = f[0] < call * 32u || f[1] < call * 32u || f[2] < call * 32u || f[3] < call * 32u;
            }
            const unsigned long long m = __builtin_amdgcn_ballot_w64(bad);
            const int n = m ? (__builtin_ctzll(m) & ~3) : nl;
            nl = m ? 16 : 64;
            if (n) { pos += 4 * n; t0 = __builtin_amdgcn_s_memrealtime(); continue; }
        }
        if (__builtin_amdgcn_readfirstlane(__hip_atomic_load(tmo, __ATOMIC_RELAXED, __HIP_MEMORY_SCOPE_AGENT)) == call ||
            __builtin_amdgcn_s_memrealtime() - t0 > DEADLINE) {
            __hip_atomic_store(tmo, call, __ATOMIC_RELAXED, __HIP_MEMORY_SCOPE_AGENT);
            asm volatile("s_waitcnt vmcnt(0)" ::: "memory");
            for (int i = pos + lane; i < nseq; i += 64) info[i] = call * 32u + M_DEAD;
            return;
        }
        if (pos > pr + 24) __builtin_amdgcn_s_sleep(24);
    }
}

#define STR2(x) #x
#define STR(x) STR2(x)
#define PULL6 "global_load_dwordx4 %[h0], %[v16], %[src] offset:-4096 sc0 sc1\n\tglobal_load_dwordx4 %[h1], %[v16], %[src] offset:-3072 sc0 sc1\n\t" \
              "global_load_dwordx4 %[h2], %[v16], %[src] offset:-2048 sc0 sc1\n\tglobal_load_dwordx4 %[h3], %[v16], %[src] offset:-1024 sc0 sc1\n\t" \
              "global_load_dwordx4 %[h4], %[v16], %[src] sc0 sc1\n\tglobal_load_dwordx4 %[h5], %[v16], %[src] offset:1024 sc0 sc1\n\t"
#define PULLT "s_mov_b64 exec, 1\n\tglobal_atomic_add %[tk], %[ao], %[one], %[ws] sc0\n\t" \
              "s_waitcnt vmcnt(%[rn] + 9 - " STR(RELAT) ")\n\tglobal_store_dword %[pf], %[cl], %[ws] sc0 sc1\n\ts_waitcnt vmcnt(%[rn] + 1)\n\t" \
              "ds_add_u32 %[rel], %[one]\n\ts_mov_b64 exec, -1\n\ts_waitcnt vmcnt(1)"
#define PULLO [h0] "=&v"(h[0]), [h1] "=&v"(h[1]), [h2] "=&v"(h[2]), [h3] "=&v"(h[3]), [h4] "=&v"(h[4]), [h5] "=&v"(h[5]), [tk] "=&v"(tk)
#define PULLI [v16] "v"(vl16), [src] "s"(src), [ao] "v"(aoff), [one] "v"(one), [ws] "s"(myws), [rel] "v"(rel), [pf] "v"(pfl), [cl] "v"(call)
#undef VBLK
#define VBLK (NGEMM + vg)
__device__ __forceinline__ void puller(int vg, const CAS Params& P, char* myws, LAS unsigned* info, LAS unsigned* prog, const LAS unsigned short* tab,
                                       LAS char* scr, const LAS char* lut, int link, int peer, int cur, int nw, unsigned win, unsigned call, int wid, int lane) {
    const int q = P.rank & 3, qs = peer & 3;
    const bool cross = link < 4;
    const int nseq = cross ? NSEQ : NI;
    const char* pk = P.ws[peer] + 4096;
    const unsigned vl16 = (unsigned)lane * 16, one = 1, aoff = (unsigned)OFF_CTR + (unsigned)link * 128u;
    const unsigned ovoff = (unsigned)(lane >> 4) * (4 * NLOC * 4) + (unsigned)(lane & 15) * 16;
    const unsigned base = (call - 1u) * (unsigned)nseq - (unsigned)nw;
    const unsigned rel = (unsigned)(size_t)(prog + 2);
    const unsigned nofl = (unsigned)OFF_SEQ + (blockIdx.x * 8u + (unsigned)wid) * 16u + 4u;
    unsigned pfl = nofl;
    for (;;) {
        const bool isout = cross && cur >= NCI;
        const int id = isout ? 0 : __builtin_amdgcn_readfirstlane((int)tab[cur]);
        const int slot = isout ? PK_O + q * NO + cur - NCI : id < PK_W ? id : id + q * 512;
        unsigned mode;
        if (pfl != nofl) {
            const unsigned i0 = __hip_atomic_load(info + cur, __ATOMIC_RELAXED, __HIP_MEMORY_SCOPE_WORKGROUP);
            const unsigned i1 = __hip_atomic_load(prog + 1, __ATOMIC_RELAXED, __HIP_MEMORY_SCOPE_WORKGROUP), i2 = __hip_atomic_load(prog + 2, __ATOMIC_RELAXED, __HIP_MEMORY_SCOPE_WORKGROUP);
            if (__builtin_amdgcn_readfirstlane(!DEFER || i0 - call * 32u > 31u || i1 - i2 >= win)) {
                WAIT_V(0);
                if (lane == 0) flag_store(myws + pfl, call);
                pfl = nofl;
            }
        }
        for (;;) {
            mode = __builtin_amdgcn_readfirstlane(__hip_atomic_load(info + cur, __ATOMIC_ACQUIRE, __HIP_MEMORY_SCOPE_WORKGROUP)) - call * 32u;
            if (mode <= 31u) break;
            __builtin_amdgcn_s_sleep(1);
        }
        const char* src = pk + (pk_off(slot, call) + (mode & 16u) * 512u);
        mode &= 15u;
        const unsigned long long mask = lane_mask(isout ? mode * 16u : mode * 8u);
        unsigned my = 0;
        if (lane == 0) my = __hip_atomic_fetch_add(prog + 1, 1u, __ATOMIC_RELAXED, __HIP_MEMORY_SCOPE_WORKGROUP);
        my = __builtin_amdgcn_readfirstlane(my) - win;
        while ((int)(my - __builtin_amdgcn_readfirstlane(__hip_atomic_load(prog + 2, __ATOMIC_RELAXED, __HIP_MEMORY_SCOPE_WORKGROUP))) >= 0) { }
        u4 h[8]; unsigned tk, lst = 0;
        if (mode >= M_RAW) {
            asm volatile(PULL6 "global_load_dwordx4 %[h6], %[v16], %[src] offset:2048 sc0 sc1\n\tglobal_load_dwordx4 %[h7], %[v16], %[src] offset:3072 sc0 sc1\n\t" PULLT
                         : PULLO, [h6] "=&v"(h[6]), [h7] "=&v"(h[7]) : PULLI, [rn] "n"(RELAT) : "memory");
        } else if (!isout) {
            u2 p0, p1;
            asm volatile(PULL6 "global_load_dwordx2 %[p0], %[v8], %[src] offset:2048 sc0 sc1\n\t"
                         "s_mov_b64 exec, %[mask]\n\tglobal_load_dwordx2 %[p1], %[v8], %[src] offset:2560 sc0 sc1\n\t" PULLT
                         : PULLO, [p0] "=&v"(p0), [p1] "=&v"(p1)
                         : PULLI, [v8] "v"(vl16 >> 1), [mask] "s"(mask), [rn] "n"(RELAT) : "memory");
            h[6] = (u4){p0[0], p0[1], p1[0], p1[1]};
        } else {
            u2 p2; unsigned p1;
            asm volatile(PULL6 "global_load_dwordx2 %[p2], %[v8], %[src] offset:2048 sc0 sc1\n\tglobal_load_dword %[p1], %[v4], %[src] offset:2560 sc0 sc1\n\t"
                         "s_mov_b64 exec, %[mask]\n\tglobal_load_dword %[lst], %[v4], %[src] offset:2816 sc0 sc1\n\t" PULLT
                         : PULLO, [p2] "=&v"(p2), [p1] "=&v"(p1), [lst] "=&v"(lst)
                         : PULLI, [v8] "v"(vl16 >> 1), [v4] "v"(vl16 >> 2), [mask] "s"(mask), [rn] "n"(RELAT + 1) : "memory");
            h[6][0] = p2[0]; h[6][1] = p2[1]; h[6][2] = p1;
        }
        const unsigned nxt = __builtin_amdgcn_readfirstlane(tk) - base;
        if (lane == 0) __hip_atomic_fetch_max(prog, nxt, __ATOMIC_RELAXED, __HIP_MEMORY_SCOPE_WORKGROUP);
        if (isout) {
            const int o = cur - NCI;
            int panel, blk, col;
            if (o < O_B) {
                const int r = o < O_A0 ? o : o < O_C ? (o - O_A0) & 127 : o - O_C, hn = o < O_A0 ? 16 : o < O_C ? 64 : 48;
                const int hf = r >= hn, t = (r - hf * hn) >> 3;
                panel = o < O_A0 ? 3 * qs + 2 : o < O_C ? 3 * (t >> 1) + (o >= O_A1) : 12 + ((q + 1 + (t >> 1)) & 3);
                blk = (r & 6) + hf; col = (t & 1) * 128 + (r & 1) * 64;
            } else {
                const int j = (o - O_B) / 24, r = o - O_B - j * 24;
                panel = 3 * ((qs + 1 + (r >> 3)) & 3) + 2; blk = 2 * j + ((r >> 1) & 1); col = ((r >> 2) & 1) * 128 + (r & 1) * 64;
            }
            char* ob = (char*)P.out + (size_t)(panel_row(peer, panel, blk) * NLOC + qs * 256 + col) * 4;
            if (mode < M_RAW) unpack_out(h, lst, scr, lane);
            if (mode == M_DEAD) h[0] = (u4){0x7FC00000u, 0x7FC00000u, 0x7FC00000u, 0x7FC00000u};
#pragma unroll
            for (int k = 0; k < 8; ++k) GST4P(ovoff, h[k], ob + ((k >> 2) * 16 + (k & 3)) * (NLOC * 4));
            if (!DEFER) WAIT_V(0);
            pfl = nofl;
        } else {
            if (mode < M_RAW) unpack_unit(h, scr, lut, lane);
            unsigned off, fl;
            const int rb = id & 31;
            if (id < PK_W) place_x(rb < 24 ? 3 * qs + (rb >> 3) : 12 + qs, rb, id >> 5, off, fl);
            else place_w(16 + 4 * qs + (id & 3), (id - PK_W) >> 2, off, fl);
            store8(h, myws + off, id < PK_W ? 32768u : 4096u, vl16);
            pfl = fl;
        }
        if (nxt >= (unsigned)nseq) break;
        cur = (int)nxt;
    }
    WAIT_V(0);
    if (lane == 0) flag_store(myws + pfl, call);
}

#undef VBLK
#define VBLK (blockIdx.x < NGEMM ? blockIdx.x : blockIdx.x + NPC)
__device__ __forceinline__ float silu(float y) {
    const float c = __builtin_amdgcn_fmed3f(y, -60.0f, 60.0f);
    const float e = __builtin_amdgcn_exp2f(c * -1.4426950408889634f);
    return y * __builtin_amdgcn_rcpf(1.0f + e);
}
__device__ __forceinline__ u4 silu4(const f4 (&a)[4], int r, bool dead) {
    f4 o = {silu(a[0][r]), silu(a[1][r]), silu(a[2][r]), silu(a[3][r])};
    const float nanv = __builtin_nanf("");
    if (dead) o = (f4){nanv, nanv, nanv, nanv};
    return __builtin_bit_cast(u4, o);
}

#define BAR() do { __builtin_amdgcn_sched_barrier(0); __builtin_amdgcn_s_barrier(); asm volatile("" ::: "memory"); __builtin_amdgcn_sched_barrier(0); } while (0)
#define GLDS(g, l, imm) __builtin_amdgcn_global_load_lds((const GAS void*)(g), (LAS void*)(l), 16, imm, AUX)
template <int V> struct IC { static constexpr int value = V; };

__device__ __forceinline__ void gemm_panel(const CAS Params& P, LAS char* lds, char* myws, int panel, int cg, int o0, int ohalf,
                                           unsigned call, int wid, int lane) {
    constexpr int ABYTES = 32768, STAGE = ABYTES + 16384;
    const char* ga = myws + OFF_DATA + (size_t)panel * PANEL_B + wid * 4096 + lane * 16;
    const char* gb = myws + OFF_DATA + A_B + (size_t)cg * CG_B + wid * 2048 + lane * 16;
    const int wm = wid >> 1, wn = wid & 1;
    const unsigned frow = (lane & 2) ? (unsigned)OFF_FB + (unsigned)cg * 1024u : (unsigned)OFF_FA + (unsigned)panel * 2048u + (unsigned)(lane & 1) * 16u;
    const unsigned fstep = (lane & 2) ? 16u : 32u;
    GAS unsigned* tmo = (GAS unsigned*)(myws + OFF_TMO);
    char* oflag = myws + (OFF_FP + (call & 1) * FP_B) + (((cg - 8) >> 1) * NSEQ + NCI + o0 + wid) * 4;
    int ready = 0;
    unsigned tv = 0;

    auto early = [&](int c) {
        if (c < ready || c >= NCH) return;
        const int cc = c + (lane >> 2);
        __builtin_amdgcn_global_load_lds((const GAS void*)(myws + (frow + (unsigned)(cc < NCH ? cc : NCH - 1) * fstep)), (LAS void*)(lds + 3 * STAGE), 16, 0, AUX);
    };
    auto ensure = [&](int c, bool pend) {
        if (c < ready) return;
        const unsigned long long t0 = __builtin_amdgcn_s_memrealtime();
        int lim = 15;
        for (;;) {
            int cc = c + ((lane >> 2) < lim ? (lane >> 2) : lim); cc = cc < NCH ? cc : NCH - 1;
            u4 f;
            if (pend) asm volatile("s_waitcnt vmcnt(%2)\n\tds_read_b128 %0, %1\n\ts_waitcnt lgkmcnt(0)"
                                   : "=&v"(f) : "v"((unsigned)(size_t)(lds + 3 * STAGE) + (unsigned)lane * 16u), "n"(3 * PD) : "memory");
            else asm volatile("global_load_dwordx4 %0, %1, %2 sc0 sc1\n\ts_waitcnt vmcnt(0)"
                              : "=&v"(f) : "v"(frow + (unsigned)cc * fstep), "s"(myws) : "memory");
            const bool bad = f[0] < call || f[1] < call || f[2] < call || f[3] < call;
            const unsigned long long m = __builtin_amdgcn_ballot_w64(bad);
            const int n = m ? (__builtin_ctzll(m) >> 2) : lim + 1;
            if (n > 0) { ready = c + n; return; }
            if (pend) { pend = false; continue; }
            if (__builtin_amdgcn_readfirstlane(__hip_atomic_load(tmo, __ATOMIC_RELAXED, __HIP_MEMORY_SCOPE_AGENT)) == call ||
                __builtin_amdgcn_s_memrealtime() - t0 > DEADLINE) {
                __hip_atomic_store(tmo, call, __ATOMIC_RELAXED, __HIP_MEMORY_SCOPE_AGENT);
                ready = NCH; return;
            }
            lim = 0;
            __builtin_amdgcn_s_sleep(6);
        }
    };
    auto stage = [&](int t, int kf, int bufoff) {
        const char* a = ga + (size_t)t * 32768 + kf * 1024; const char* b = gb + (size_t)t * 16384 + kf * 1024;
        LAS char* la = lds + bufoff + wid * 4096 + kf * 1024; LAS char* lb = lds + bufoff + ABYTES + wid * 2048 + kf * 1024;
        GLDS(a, la, 0); GLDS(a, la, 2048); GLDS(b, lb, 0);
    };

    f4 acc[4][4];
#pragma unroll
    for (int m = 0; m < 4; ++m)
#pragma unroll
        for (int n = 0; n < 4; ++n) acc[m][n] = (f4){0.f, 0.f, 0.f, 0.f};

    const int aoff = wm * 8192 + lane * 16, boff = ABYTES + wn * 8192 + lane * 16;
    auto phase = [&](auto waitn, int rbuf, int kf, int lt, int lkf, int lbuf, int pc, int pe = NCH) {
        const LAS char* pa = lds + rbuf + aoff + kf * 1024; const LAS char* pb = lds + rbuf + boff + kf * 1024;
        h8 a[4], b[4];
#pragma unroll
        for (int n = 0; n < 4; ++n) b[n] = *(const LAS h8*)(pb + n * 2048);
#pragma unroll
        for (int m = 0; m < 4; ++m) a[m] = *(const LAS h8*)(pa + m * 2048);
        __builtin_amdgcn_sched_barrier(0);
        if (lt >= 0) stage(lt, lkf, lbuf);
        if (decltype(waitn)::value >= 0) WAIT_V(decltype(waitn)::value < 0 ? 0 : decltype(waitn)::value);
        WAIT_L0();
        BAR();
        __builtin_amdgcn_s_setprio(1);
#pragma unroll
        for (int m = 0; m < 4; ++m)
#pragma unroll
            for (int n = 0; n < 4; ++n) acc[m][n] = __builtin_amdgcn_mfma_f32_16x16x32_f16(a[m], b[n], acc[m][n], 0, 0, 0);
        __builtin_amdgcn_s_setprio(0);
        __builtin_amdgcn_sched_barrier(0);
        if (wid == 0) { if (pc >= 0) { ensure(pc, PD > 0); TREC(pc); } early(pe); }
        BAR();
    };

    if (wid == 0) { ensure(0, false); ensure(1, false); }
    BAR();
    STAMP(1);
    stage(0, 0, 0); stage(0, 1, 0); stage(1, 0, STAGE); stage(1, 1, STAGE); stage(2, 0, 2 * STAGE);
    WAIT_V(12);
    BAR();
    if (wid >= 4) BAR();
    int rb = 0, pb = 2 * STAGE;
    for (int t = 0; t < NKT - 3; ++t) {
        phase(IC<12>{}, rb, 0, t + 2, 1, pb, (t & 1) ? (t + 3) >> 1 : -1, PD == 2 && !(t & 1) ? (t + 4) >> 1 : NCH);
        phase(IC<12>{}, rb, 1, t + 3, 0, rb, -1, PD == 1 && !(t & 1) ? (t + 4) >> 1 : NCH);
        pb = rb; rb = rb == 2 * STAGE ? 0 : rb + STAGE;
    }
    static_assert((NKT - 3) % 3 == 2, "tail buffers");
    phase(IC<12>{}, 2 * STAGE, 0, NKT - 1, 1, STAGE, -1);
    phase(IC<9>{}, 2 * STAGE, 1, -1, 0, 0, -1);
    phase(IC<6>{}, 0, 0, -1, 0, 0, -1);
    phase(IC<3>{}, 0, 1, -1, 0, 0, -1);
    phase(IC<0>{}, STAGE, 0, -1, 0, 0, -1);
    phase(IC<-1>{}, STAGE, 1, -1, 0, 0, -1);
    if (wid < 4) BAR();
    STAMP(2);
    TDUMP(0);

    const bool dead = __builtin_amdgcn_readfirstlane(__hip_atomic_load(tmo, __ATOMIC_RELAXED, __HIP_MEMORY_SCOPE_AGENT)) == call;
    const unsigned vl16 = (unsigned)lane * 16, ovoff = (unsigned)(lane >> 4) * (4 * NLOC * 4) + (unsigned)(lane & 15) * 16;
    const int pk0 = PK_O + ((cg < 8 ? 0 : cg - 8) >> 1) * NO + o0 + wid;
    const unsigned hv[2] = {pk_half(myws, pk0, call), pk_half(myws, pk0 + ohalf, call)};
    unsigned mode = 0;
#pragma unroll
    for (int u = 0; u < 2; ++u) {
        u4 h[8];
#pragma unroll
        for (int k = 0; k < 8; ++k) h[k] = silu4(acc[2 * u + (k >> 2)], k & 3, dead);
        if (cg < 8) {
            char* ob = (char*)P.out + (size_t)(panel_row(P.rank, panel, 2 * wm + u) * NLOC + cg * 128 + wn * 64) * 4;
#pragma unroll
            for (int k = 0; k < 8; ++k) GST4P(ovoff, h[k], ob + ((k >> 2) * 16 + (k & 3)) * (NLOC * 4));
        } else {
            const unsigned prev = mode;
            if (u && o0 < EARLY) { WAIT_V(0); if (lane == 0) flag_store(oflag, call * 32u + prev); }
            mode = M_RAW;
            if (o0 + u * ohalf >= ORAW) mode = pack_out(h, lds + wid * 2048, lane);
            if (u && o0 >= EARLY) { WAIT_V(0); if (lane == 0) flag_store(oflag, call * 32u + prev); }
            const unsigned half = SMODE == 0 ? wid & 1 : (~__builtin_amdgcn_readfirstlane(hv[u]) ^ (SMODE == 2)) & 1u;
            char* d0 = myws + (pk_off(pk0 + u * ohalf, call) + half * 8192); char* d1 = d0 + 4096;
            GST4(vl16, h[0], d0, 0); GST4(vl16, h[1], d0, 1024); GST4(vl16, h[2], d0, 2048); GST4(vl16, h[3], d0, 3072);
            GST4(vl16, h[4], d1, 0); GST4(vl16, h[5], d1, 1024);
            if (mode < M_RAW) {
                const u2 p2 = {h[6][0], h[6][1]};
                asm volatile("global_store_dwordx2 %0, %1, %2 offset:2048 sc0 sc1\n\tglobal_store_dword %3, %4, %2 offset:2560 sc0 sc1"
                             :: "v"(vl16 >> 1), "v"(p2), "s"(d1), "v"(vl16 >> 2), "v"(h[6][2]) : "memory");
                store_list(d1 + 2816, h[6][3], mode, vl16 >> 2);
            } else {
                GST4(vl16, h[6], d1, 2048);
                GST4(vl16, h[7], d1, 3072);
            }
            mode += half * 16u;
        }
    }
    WAIT_V(0);
    if (cg >= 8 && lane == 0) flag_store(oflag + ohalf * 4, call * 32u + mode);
    WAIT_V(0);
    STAMP(3);
}

__device__ __forceinline__ void gemm_quarter(const CAS Params& P, LAS char* lds, char* myws, int panel, int qt, int cg, int o0,
                                             unsigned call, int wid, int lane) {
    constexpr int STAGE = 49152;
    const int kp = wid >> 2, lw = wid & 3, wm = lw >> 1, wn = lw & 1;
    const char* gsrc = myws + OFF_DATA + lane * 16 + (lw < 2 ? (size_t)panel * PANEL_B + kp * 32768 + qt * 8192
                                                             : A_B + (size_t)cg * CG_B + kp * 16384 + (lw - 2) * 8192);
    const unsigned gstep = lw < 2 ? 65536u : 32768u;
    LAS char* ring = lds + kp * 24576;
    LAS unsigned* pl = (LAS unsigned*)(lds + 160768);
    const unsigned frow = (unsigned)OFF_FA + (unsigned)(panel * 2048 + qt * 8 + (lane & 1) * 4);
    GAS unsigned* tmo = (GAS unsigned*)(myws + OFF_TMO);
    char* oflag = myws + (OFF_FP + (call & 1) * FP_B) + (((cg - 8) >> 1) * NSEQ + NCI + o0) * 4;
    unsigned tv = 0;
    int ready = 0; bool pend = false;
    auto paddr = [&]() { const int cc = ready + (lane >> 1); return lane < 32 ? frow + (unsigned)(cc < NCH ? cc : NCH - 1) * 32u : (unsigned)OFF_TMO; };
    auto late = [&](unsigned tm, unsigned long long t0) {
        if (tm != call && __builtin_amdgcn_s_memrealtime() - t0 <= DEADLINE) return false;
        __hip_atomic_store(tmo, call, __ATOMIC_RELAXED, __HIP_MEMORY_SCOPE_AGENT);
        WAIT_V(0);
        ready = NCH;
        return true;
    };
    auto ensure = [&](int c) {
        if (c >= ready) {
            const unsigned long long t0 = __builtin_amdgcn_s_memrealtime();
            for (;;) {
                unsigned f;
                if (pend) { WAIT_V(0); f = pl[lane]; pend = false; }
                else asm volatile("global_load_dword %0, %1, %2 sc0 sc1\n\ts_waitcnt vmcnt(0)" : "=&v"(f) : "v"(paddr()), "s"(myws) : "memory");
                ready += __builtin_ctzll(__builtin_amdgcn_ballot_w64(f < call) | (1ull << 32)) >> 1;
                if (c < ready || late(__builtin_amdgcn_readlane(f, 32), t0)) break;
                __builtin_amdgcn_s_sleep(QSLEEP);
            }
        }
        if (!pend && ready < NCH && ready <= c + 2) {
            __builtin_amdgcn_global_load_lds((const GAS void*)(myws + paddr()), (LAS void*)pl, 4, 0, AUX);
            pend = true;
        }
    };
    auto stage = [&](int c, int buf) {
        const char* g = gsrc + (size_t)((unsigned)c * gstep);
        LAS char* l = ring + buf + (lw - 1) * 8192;
        GLDS(g, l, 0); GLDS(g, l, 1024); GLDS(g, l, 2048); GLDS(g, l, 3072);
        GLDS(g + 4096, l + 4096, 0); GLDS(g + 4096, l + 4096, 1024); GLDS(g + 4096, l + 4096, 2048); GLDS(g + 4096, l + 4096, 3072);
    };

    const int pk0 = PK_O + ((cg < 8 ? 0 : cg - 8) >> 1) * NO + o0;
    if (wid == 0) pl[64] = pk_half(myws, pk0, call);
    f4 acc[2][4];
#pragma unroll
    for (int m = 0; m < 2; ++m)
#pragma unroll
        for (int n = 0; n < 4; ++n) acc[m][n] = (f4){0.f, 0.f, 0.f, 0.f};
    const int aoff = wm * 4096 + lane * 16, boff = 8192 + wn * 8192 + lane * 16;
    auto phase = [&](auto waitn, auto role, int buf, int lc) {
        h8 a[4], b[8];
#pragma unroll
        for (int n = 0; n < 8; ++n) b[n] = *(const LAS h8*)(ring + buf + boff + n * 1024);
#pragma unroll
        for (int m = 0; m < 4; ++m) a[m] = *(const LAS h8*)(ring + buf + aoff + m * 1024);
        __builtin_amdgcn_sched_barrier(0);
        if (decltype(role)::value == 2 && wid == 0) {
            ensure(lc); TREC(lc);
        }
        WAIT_L0();
        BAR();
        if (decltype(role)::value == 1) stage(lc, buf);
#pragma unroll
        for (int k = 0; k < 2; ++k)
#pragma unroll
            for (int m = 0; m < 2; ++m)
#pragma unroll
                for (int n = 0; n < 4; ++n) acc[m][n] = __builtin_amdgcn_mfma_f32_16x16x32_f16(a[m * 2 + k], b[n * 2 + k], acc[m][n], 0, 0, 0);
        if (decltype(role)::value == 1) {
#pragma unroll
            for (int i = 0; i < 8; ++i) { __builtin_amdgcn_sched_group_barrier(0x010, 1, 0); __builtin_amdgcn_sched_group_barrier(0x008, 2, 0); }
        }
        __builtin_amdgcn_sched_barrier(0);
        if (decltype(waitn)::value >= 0) { WAIT_V(decltype(waitn)::value < 0 ? 0 : decltype(waitn)::value); BAR(); }
    };

    if (wid == 0 && ready < NCH) {
        const unsigned long long t0 = __builtin_amdgcn_s_memrealtime();
        for (;;) {
            u4 f;
            asm volatile("global_load_dwordx4 %0, %1, %2 sc0 sc1\n\ts_waitcnt vmcnt(0)"
                         : "=&v"(f) : "v"((unsigned)OFF_FB + (unsigned)(cg * 1024 + lane * 16)), "s"(myws) : "memory");
            if (__builtin_amdgcn_ballot_w64(f[0] < call || f[1] < call || f[2] < call || f[3] < call) == 0 ||
                late(__builtin_amdgcn_readfirstlane(__hip_atomic_load(tmo, __ATOMIC_RELAXED, __HIP_MEMORY_SCOPE_AGENT)), t0)) break;
            __builtin_amdgcn_s_sleep(6);
        }
        ensure(2);
    }
    WAIT_L0();
    BAR();
    STAMP(4);
    const unsigned half = SMODE == 0 ? cg & 1 : (~__builtin_amdgcn_readfirstlane(pl[64]) ^ (SMODE == 2)) & 1u;
    if (lw != 0) { stage(0, 0); stage(1, STAGE); stage(2, 2 * STAGE); }
    WAIT_V(16);
    BAR();
    if (kp) BAR();
    if (lw != 0) for (int c = 3, buf = 0; c < NCH; ++c, buf = buf == 2 * STAGE ? 0 : buf + STAGE) phase(IC<16>{}, IC<1>{}, buf, c);
    else for (int c = 3, buf = 0; c < NCH; ++c, buf = buf == 2 * STAGE ? 0 : buf + STAGE) phase(IC<16>{}, IC<2>{}, buf, c);
    static_assert((NCH - 3) % 3 == 1, "tail buffers");
    const unsigned tm = __hip_atomic_load(tmo, __ATOMIC_RELAXED, __HIP_MEMORY_SCOPE_AGENT);
    phase(IC<8>{}, IC<0>{}, STAGE, -1);
    phase(IC<0>{}, IC<0>{}, 2 * STAGE, -1);
    phase(IC<-1>{}, IC<0>{}, 0, -1);
    f4 r[4];
#pragma unroll
    for (int n = 0; n < 4; ++n) {
        const f4 a0 = acc[0][n], a1 = acc[1][n];
        r[n] = kp ? a1 : a0;
        *(LAS f4*)(ring + lw * 4096 + n * 1024 + lane * 16) = kp ? a0 : a1;
    }
    const bool dead = __builtin_amdgcn_readfirstlane(tm) == call;
    WAIT_L0();
    BAR();
    if (!kp) BAR();
    STAMP(5);
#pragma unroll
    for (int n = 0; n < 4; ++n) r[n] += *(const LAS f4*)(lds + (kp ^ 1) * 24576 + lw * 4096 + n * 1024 + lane * 16);
    if (cg < 8) {
        const unsigned ovoff = (unsigned)(lane >> 4) * (4 * NLOC * 4) + (unsigned)(lane & 15) * 16;
        char* ob = (char*)P.out + (size_t)((panel_row(P.rank, panel, 2 * qt + wm) + kp * 16) * NLOC + cg * 128 + wn * 64) * 4;
#pragma unroll
        for (int k = 0; k < 4; ++k) { const u4 o = silu4(r, k, dead); GST4P(ovoff, o, ob + k * (NLOC * 4)); }
    } else {
        char* d0 = myws + (pk_off(pk0 + lw, call) + half * 8192 + kp * 4096);
#pragma unroll
        for (int k = 0; k < 4; ++k) { const u4 o = silu4(r, k, dead); GST4((unsigned)lane * 16, o, d0 + k * 1024, 0); }
        WAIT_V(0);
        BAR();
        if (wid == 0 && lane < 4) flag_store(oflag + lane * 4, call * 32u + half * 16u + M_RAW);
    }
    STAMP(6);
    TDUMP(1);
}

__global__ void __launch_bounds__(512, 2) ag_gemm_kernel(const Params Parg) {
    const CAS Params& P = *(const CAS Params*)__builtin_amdgcn_kernarg_segment_ptr();
    (void)Parg;
    __shared__ __attribute__((aligned(16))) char lds_raw[5 * 32768];
    LAS char* lds = (LAS char*)lds_raw;
    const int lane = threadIdx.x & 63, wid = __builtin_amdgcn_readfirstlane(threadIdx.x >> 6);
    const int xcd = blockIdx.x & 7, slot = blockIdx.x >> 3;
    char* myws = P.ws[P.rank];
    const int q = P.rank & 3;
    const unsigned long long t0 = __builtin_amdgcn_s_memrealtime();
    const unsigned side = (__builtin_amdgcn_s_getreg(20 | (3 << 11)) >> 2) & 1u;
    int g = (int)blockIdx.x - NGEMM;
    if (slot == NGEMM / 8) g = NPC + xcd;
    else if (g >= 0) {
        if (threadIdx.x == 0) {
            GAS unsigned* rt = (GAS unsigned*)(myws + OFF_ROLE);
            const unsigned t = __hip_atomic_fetch_add(rt, side ? 65536u : 1u, __ATOMIC_RELAXED, __HIP_MEMORY_SCOPE_AGENT);
            if ((t & 65535u) + (t >> 16) >= 247u - NGEMM) __hip_atomic_store(rt, 0u, __ATOMIC_RELAXED, __HIP_MEMORY_SCOPE_AGENT);
            *(LAS unsigned*)(lds + 8 * PBUF) = t;
        }
        __syncthreads();
        const unsigned t = __builtin_amdgcn_readfirstlane(*(const LAS unsigned*)(lds + 8 * PBUF));
        constexpr unsigned N0 = NPC / 7 * 3, NL = 247 - NGEMM;
        const unsigned r = (side ? NL - (t >> 16) : t & 65535u) % (NL + 1), b = NL - r;
        g = r < N0 ? r / 3 * 7 + 4 + r % 3 : r < N0 + NPROD - 8 ? NPC + 8 + r - N0 : b / 4 * 7 + b % 4;
        if (XMODE == 0) { g = (int)blockIdx.x - NGEMM; g = g < NPC + 8 ? g - 8 : g; }
    }
#undef VBLK
#define VBLK (NGEMM + g)
    if (wid == 7 && blockIdx.x < 2 * PK_PG) {
        char* a = myws + (OFF_PK + ((size_t)blockIdx.x << 21));
        unsigned t[2] = {~0u, ~0u};
#pragma unroll
        for (int i = 0; i < 6; ++i) {
            const unsigned long long c0 = __builtin_amdgcn_s_memtime();
            asm volatile("s_waitcnt lgkmcnt(0)\n\tglobal_store_dword %0, %0, %1 sc0 sc1\n\ts_waitcnt vmcnt(0)" :: "v"((i & 1) * 8192), "s"(a), "s"(c0) : "memory");
            const unsigned d = (unsigned)(__builtin_amdgcn_s_memtime() - c0);
            t[i & 1] = d < t[i & 1] ? d : t[i & 1];
        }
        if (lane == 0) flag_store(myws + OFF_MAP + blockIdx.x * 4, (t[1] < t[0]) ^ side);
        STAMP(10);
    }
    const int u0 = g < 0 ? wid * NGEMM + (int)blockIdx.x : UG + wid * NPROD + g - NPC;
    if (g < 0 || g >= NPC) { pace(t0, u0); fetch_input(P, lds + wid * PBUF, u0, lane); make_lut(lds + (8 * PBUF + 16), true, lane); }
    GAS unsigned* seq = (GAS unsigned*)(myws + OFF_SEQ) + (blockIdx.x * 8 + wid) * 4;
    const unsigned call = __builtin_amdgcn_readfirstlane(__hip_atomic_load(seq, __ATOMIC_RELAXED, __HIP_MEMORY_SCOPE_AGENT)) + 1u;
    WAIT_V(0);
    if (lane == 0) __hip_atomic_store(seq, call, __ATOMIC_RELAXED, __HIP_MEMORY_SCOPE_AGENT);

    if (g < 0 || g >= NPC) {
        producer(P, myws, lds + wid * PBUF, lds + (8 * PBUF + 16), u0, g < 0 ? 8 * NGEMM : 8 * NPROD, g < 0 ? UG : PK_O, t0, call, lane);
        STAMP(7);
    }
    if (slot <= NGEMM / 8) {
        const int pi = slot >> 3, ps = xcd >> 1, h = xcd & 1, sown = 20 + ps;
        const int ci = pi == 3 ? (slot & 1) * 4 + ps : slot == sown ? ps : slot & 7;
        const int panel = pi < 2 ? 3 * ps + pi : pi == 3 ? 3 * q + 2 : slot == sown ? 16 : 12 + ps;
        if (g < 0) gemm_panel(P, lds, myws, panel, ci < 4 ? 4 * h + ci : 2 * ci + h,
                   pi == 3 ? h * 8 : pi == 2 ? O_C + (((ps - ci - 1) & 3) * 2 + h) * 8 : O_A0 + pi * 128 + xcd * 8, pi == 3 ? 16 : pi == 2 ? 48 : 64, call, wid, lane);
        const int k = slot == 26 ? 23 : slot >= 24 ? slot - 24 : slot == sown ? 2 : slot < 4 ? slot + 3 : slot < 8 ? slot + 7 : slot < 12 ? slot - 1
                    : slot < 20 ? slot + 3 : 24;
        if (k < 24) {
            const int kk = k < 12 ? k : k - 12, qt = kk / 3, si = kk - qt * 3;
            gemm_quarter(P, lds, myws, 3 * ((q + 1 + si) & 3) + 2, qt, k < 12 ? 8 + xcd : xcd, O_B + qt * 24 + (si * 2 + h) * 4, call, wid, lane);
        }
    } else if (g < NPC) {
        LAS unsigned* info = (LAS unsigned*)lds;
        LAS unsigned* prog = info + NSEQ;
        LAS unsigned short* tab = (LAS unsigned short*)(lds + 8192);
        const int j = g / 7, link = g - j * 7, ncu = (NPC + 6 - link) / 7;
        for (int i = threadIdx.x; i < NSEQ + 3; i += 512) info[i] = 0;
        for (int i = threadIdx.x; i < PK_W; i += 512)
            if (((i & 31) == 16 || (i & 31) >= 24) == (link < 4)) tab[pos_x(i & 31, i >> 5)] = (unsigned short)i;
        if (link < 4) tab[pos_w(threadIdx.x >> 2, threadIdx.x & 3)] = (unsigned short)(PK_W + threadIdx.x);
        if (wid == 0) make_lut(lds + 14336, false, lane);
        __syncthreads();
        const int peer = link < 4 ? ((P.rank & 4) ^ 4) + ((q + link) & 3) : (P.rank & 4) + ((q + link - 3) & 3);
        if (wid < WPC) {
            if (lane == 0) __hip_atomic_fetch_max(prog, (unsigned)(wid * ncu + j), __ATOMIC_RELAXED, __HIP_MEMORY_SCOPE_WORKGROUP);
            puller(g, P, myws, info, prog, tab, lds + 16384 + wid * 8192, lds + 14336, link, peer, wid * ncu + j, ncu * WPC,
                   (unsigned)((NWIN + ncu - 1 - j) / ncu), call, wid, lane);
        } else if (wid == 7) {
            flag_mirror(myws, P.ws[peer] + (OFF_FP + (call & 1) * FP_B + (link < 4 ? q : 4) * (NSEQ * 4)), link < 4 ? NSEQ : NI, info, prog, call, lane);
        }
        STAMP(3);
    }
}

}

extern "C" void kernel_launch(int rank, int nranks, ncclComm_t comm, hipStream_t stream,
                              void* const* d_in, const int* in_sizes, int n_in,
                              void* d_out, int out_size, const int* in_dtypes, int out_dtype,
                              void* const* d_out_all, void* const* d_ws_all, size_t ws_bytes) {
    (void)comm; (void)in_dtypes; (void)out_dtype; (void)d_out_all;
    if (nranks != 8 || (unsigned)rank >= 8u || n_in < 2 || ws_bytes < WS_NEED || out_size != KDIM * NLOC ||
        in_sizes[0] != 1024 * KDIM || in_sizes[1] != KDIM * NLOC) {
        fprintf(stderr, "ag_gemm: unexpected problem shape\n");
        return;
    }
    Params P;
    P.x = (const float*)d_in[0]; P.w = (const float*)d_in[1];
    P.out = (float*)d_out;
    P.rank = 0; P.pad = 0;
    for (int p = 0; p < 8; ++p) {
        const int r = (int)((RMAP >> (4 * p)) & 7u);
        P.ws[p] = (char*)d_ws_all[r];
        if (r == rank) P.rank = p;
    }
    hipLaunchKernelGGL(ag_gemm_kernel, dim3(256), dim3(512), 0, stream, P);
    HIP_CHECK(hipGetLastError());
}
```

```cpp
#include <hip/hip_runtime.h>
#include <rccl/rccl.h>
#include <cstdio>

#define HIP_CHECK(expr) do { hipError_t _e = (expr); if (_e != hipSuccess) { \
    fprintf(stderr, "HIP error %s at %s:%d\n", hipGetErrorString(_e), __FILE__, __LINE__); \
    return; } } while (0)

namespace {

typedef _Float16 h8 __attribute__((ext_vector_type(8)));
typedef float f4 __attribute__((ext_vector_type(4)));
typedef unsigned u4 __attribute__((ext_vector_type(4)));
typedef unsigned u2 __attribute__((ext_vector_type(2)));
#define LAS __attribute__((address_space(3)))
#define GAS __attribute__((address_space(1)))
#define CAS __attribute__((address_space(4)))

constexpr int KDIM = 8192, NLOC = 1024;
constexpr int NKT = KDIM / 64;
constexpr int NCH = NKT / 2;
constexpr int NI = 23 * 64;
constexpr int NCI = 17 * 64, NO = 480;
constexpr int NSEQ = NCI + NO;
constexpr int O_A0 = 32, O_A1 = 160, O_C = 288, O_B = 384;
constexpr size_t PANEL_B = 4u << 20;
constexpr size_t CG_B = 2u << 20;
constexpr size_t OFF_SEQ = 0;
constexpr size_t OFF_TMO = 64u << 10;
constexpr size_t OFF_CTR = OFF_TMO + 256;
constexpr size_t OFF_ROLE = OFF_TMO + 1280;
constexpr size_t OFF_MAP = OFF_TMO + 2048;
#define MAPS 256u
constexpr size_t OFF_FA = 128u << 10;
constexpr size_t OFF_FB = OFF_FA + 17 * 64 * 32;
constexpr size_t OFF_FP = 256u << 10;
constexpr size_t FP_B = 5 * NSEQ * 4;
constexpr size_t OFF_DBG = 640u << 10;
constexpr size_t OFF_DATA = 1u << 20;
constexpr size_t A_B = 21 * PANEL_B, B_B = 16 * CG_B;
constexpr size_t OFF_PK = OFF_DATA + A_B + B_B + (1u << 20);
constexpr int PK_W = 2048, PK_O = 4096, PK_N = PK_O + 4 * NO;
constexpr int PK_PG = (PK_N + 126) / 127;
constexpr size_t PK_B = (size_t)PK_PG << 21;
constexpr size_t WS_NEED = OFF_PK + 2 * PK_B;
static_assert(OFF_MAP + 2 * PK_PG * MAPS <= OFF_FA && OFF_FB + 16 * 64 * 16 <= OFF_FP && OFF_FP + 2 * FP_B <= OFF_DBG, "flag area");
constexpr unsigned long long DEADLINE = 30000000ull;
constexpr unsigned M_RAW = 9, M_DEAD = 10;

#define NOTMO
#define NOTMO1
#define NOTMO2
#define NPC 35
#define WPC 7
#define SKC 14
#define SKA 5
#define SQ1 4
#define SQ2 10
#define SQ3 17
#define PACE 14
#define U1ST 448
#define PACE1 PACE
#define PACE2 PACE1
#define PROBE1 1
#define LOOK 64
#define PCH 2
#define PT0 180
#define PDT 13
#define CHMAP 0x76543210u
#define PROBET 0
#define PROBEX 0
#define TOUCH 0
#define RAW0 40
#define RAW1 0
#define DEFER 1
#define PD 2
#define NWIN 10
#define NWINI NWIN
#define PRATE 134
#define PRATEI PRATE
#define LGAIN 64
#define LCAP 30
#define GLINK 1
#define PSLACK 0
#define RELAT 8
#define UG 3200
#define B1 64
#define AHEAD UG
#define QSLEEP 2
#define AUX 17
#define RMAP 0x75416320u
#define EARLY 0
#define ORAW 0
#define SMODE 1
#define XMODE 1
constexpr int NGEMM = 208;
constexpr int NPROD = 256 - NGEMM - NPC;

struct Params {
    const float* x;
    const float* w;
    float* out;
    char* ws[8];
    int rank;
    int pad;
};

#define VBLK blockIdx.x
#define STAMP(i) do { } while (0)

#define TREC(c) do { } while (0)
#define TDUMP(k) do { (void)tv; } while (0)
#define HST(i, v) do { } while (0)
#define NOW() __builtin_amdgcn_s_memrealtime()
#define GST4(voff, data, sbase, imm) \
    asm volatile("global_store_dwordx4 %0, %1, %2 offset:" #imm " sc0 sc1\n\ts_nop 1" :: "v"(voff), "v"(data), "s"(sbase) : "memory")
#define OSUF " sc0 sc1"
#define GST4P(voff, data, sbase) \
    asm volatile("global_store_dwordx4 %0, %1, %2" OSUF "\n\ts_nop 1" :: "v"(voff), "v"(data), "s"(sbase) : "memory")
#define WAIT_V(n) asm volatile("s_waitcnt vmcnt(%0)" :: "n"(n) : "memory")
#define WAIT_L0() asm volatile("s_waitcnt lgkmcnt(0)" ::: "memory")

__device__ __forceinline__ u4 pack8(float a0, float a1, float a2, float a3, float a4, float a5, float a6, float a7) {
    h8 h = {(_Float16)a0, (_Float16)a1, (_Float16)a2, (_Float16)a3, (_Float16)a4, (_Float16)a5, (_Float16)a6, (_Float16)a7};
    return __builtin_bit_cast(u4, h);
}
__device__ __forceinline__ void flag_store(char* p, unsigned v) {
    __hip_atomic_store((GAS unsigned*)p, v, __ATOMIC_RELAXED, __HIP_MEMORY_SCOPE_SYSTEM);
}
__device__ __forceinline__ void store8(const u4 (&h)[8], char* d0, unsigned half, unsigned vl16) {
    char* d1 = d0 + half;
    GST4(vl16, h[0], d0, 0); GST4(vl16, h[1], d0, 1024); GST4(vl16, h[2], d0, 2048); GST4(vl16, h[3], d0, 3072);
    GST4(vl16, h[4], d1, 0); GST4(vl16, h[5], d1, 1024); GST4(vl16, h[6], d1, 2048); GST4(vl16, h[7], d1, 3072);
}
__device__ __forceinline__ unsigned long long lane_mask(unsigned n) { return n >= 64u ? ~0ull : (1ull << n) - 1ull; }
__device__ __forceinline__ void store_list(char* d, unsigned l, unsigned mode, unsigned vl4) {
    asm volatile("s_mov_b64 exec, %3\n\tglobal_store_dword %0, %1, %2 sc0 sc1\n\ts_mov_b64 exec, -1"
                 :: "v"(vl4), "v"(l), "s"(d), "s"(lane_mask(mode * 16u)) : "memory");
}

__device__ __forceinline__ int clamp64(int v) { return v < 0 ? 0 : v > 64 ? 64 : v; }
__device__ __forceinline__ int pos_w(int kt, int g) { const int ch = kt >> 1; return (ch < PCH ? 8 : 9) * ch + 8 * clamp64(ch - SKC) + (kt & 1) * 4 + g; }
__device__ __forceinline__ int pos_x(int rb, int ch) {
    if (rb < 8) return 8 * ch + 8 * clamp64(ch - SKA) + rb;
    if (rb < 16) return 8 * ch + 8 * clamp64(ch + SKA + 1) + rb - 8;
    if (rb == 16) return ch < PCH ? 8 * PCH + ch : 9 * ch + 8 * clamp64(ch - SKC) + 8;
    if (rb >= 24) return 8 * ch + 9 * clamp64(ch + SKC + 1) + rb - 24;
    const int j = (rb - 16) >> 1, r = ch + (j == 0 ? 0 : j == 1 ? SQ1 : j == 2 ? SQ2 : SQ3);
    return 1024 + clamp64(r + (j > 0)) + 2 * (clamp64(r - SQ1 + (j > 1)) + clamp64(r - SQ2 + (j > 2)) + clamp64(r - SQ3)) + (j ? rb & 1 : 0);
}
static_assert(NPC % 7 == 0 && 2 * PK_PG <= NGEMM && UG >= 8 * NGEMM && UG + 8 * NPROD <= PK_O && 0 <= SQ1 && SQ1 <= SQ2 && SQ2 <= SQ3 && SKC < 64 && SKA < 64 && PCH <= SKA && PCH <= SKC && SKC >= 8, "skews");
constexpr int NPUSH = 8 * PCH;
__device__ __forceinline__ unsigned panel_off(int panel, unsigned call) {
    return (unsigned)OFF_DATA + (unsigned)((call & 1) && panel < 12 && panel % 3 == 0 ? 17 + panel / 3 : panel) * (unsigned)PANEL_B;
}
__device__ __forceinline__ void place_x(int panel, int rb, int ch, unsigned call, unsigned& off, unsigned& fl) {
    off = panel_off(panel, call) + (unsigned)(ch * 65536 + (rb & 7) * 4096);
    fl = (unsigned)OFF_FA + (unsigned)(panel * 2048 + ch * 32 + (rb & 7) * 4);
}
__device__ __forceinline__ void place_w(int cb, int kt, unsigned& off, unsigned& fl) {
    off = (unsigned)(OFF_DATA + A_B) + (unsigned)(cb >> 1) * (unsigned)CG_B + (unsigned)(kt * 16384 + (cb & 1) * 8192);
    fl = (unsigned)OFF_FB + (unsigned)((cb >> 1) * 1024 + kt * 8 + (cb & 1) * 4);
}
__device__ __forceinline__ int panel_row(int pub, int panel, int blk) {
    const int gs = pub & 4, s = panel / 3, k = panel - 3 * s;
    const int r = panel == 16 ? pub : panel >= 12 ? (gs ^ 4) + panel - 12 : (k == 2 && blk == 0 ? gs ^ 4 : gs) + s;
    return (int)((RMAP >> (4 * r)) & 7u) * 1024 + (panel >= 12 ? 768 : k * 256) + blk * 32;
}
__device__ __forceinline__ size_t pk_off(int pk, unsigned call) { return OFF_PK + (call & 1) * PK_B + (size_t)(pk + pk / 127 + 1) * 16384; }
__device__ __forceinline__ unsigned pk_half(char* myws, int pk, unsigned call) {
    return __hip_atomic_load((GAS unsigned*)(myws + OFF_MAP) + ((call & 1) * PK_PG + pk / 127) * (MAPS / 4), __ATOMIC_RELAXED, __HIP_MEMORY_SCOPE_SYSTEM);
}

#define PERM(hi, lo, sel) __builtin_amdgcn_perm(hi, lo, sel)
#define ABIT(hi, lo, s) __builtin_amdgcn_alignbit(hi, lo, s)
#define HW(k) h[(k) >> 2][(k) & 3]
#define FENCE() asm volatile("" ::: "memory")
__device__ __forceinline__ unsigned bfi(unsigned m, unsigned x, unsigned y) { return (x & m) | (y & ~m); }
__device__ __forceinline__ unsigned wave_scan(unsigned v) {
    v += __builtin_amdgcn_update_dpp(0u, v, 0x111, 0xF, 0xF, true);
    v += __builtin_amdgcn_update_dpp(0u, v, 0x112, 0xF, 0xF, true);
    v += __builtin_amdgcn_update_dpp(0u, v, 0x114, 0xF, 0xF, true);
    v += __builtin_amdgcn_update_dpp(0u, v, 0x118, 0xF, 0xF, true);
    v += __builtin_amdgcn_update_dpp(0u, v, 0x142, 0xA, 0xF, false);
    v += __builtin_amdgcn_update_dpp(0u, v, 0x143, 0xC, 0xF, false);
    return v;
}
__device__ __forceinline__ int wave_sum(unsigned v) { return __builtin_amdgcn_readlane(wave_scan(v), 63); }
__device__ __forceinline__ unsigned escapes(unsigned c, unsigned acc) { return (unsigned)__builtin_popcount(c & (c >> 1) & 0x55555555u) + acc; }
__device__ __forceinline__ void make_lut(LAS char* lut, bool pack, int lane) {
#pragma unroll
    for (int j = 0; j < 4; ++j) {
        unsigned sel = pack ? 0x0C0C0C0Cu : 0u, k = 0;
#pragma unroll
        for (int i = 0; i < 4; ++i) {
            const unsigned c = i < 3 ? ((unsigned)lane >> (2 * i)) & 3u : (unsigned)j;
            if (pack) sel ^= c == 3u ? (0x0Cu ^ (unsigned)i) << (8 * k) : 0u; else sel |= (c == 3u ? k : 4u + c) << (8 * i);
            k += c == 3u;
        }
        *(LAS u2*)(lut + (j * 64 + lane) * 8) = (u2){sel, pack ? 8u * k : k};
    }
}
__device__ __forceinline__ unsigned code4(unsigned thi, unsigned tlo, unsigned s) { return __builtin_amdgcn_udot4(PERM(thi, tlo, s), 0x80200802u, 0u - 0xF3u * 170u, false); }
template <int N> __device__ __forceinline__ void append(const unsigned* s, const u2* e, unsigned pos) {
#pragma unroll
    for (int g = 0; g < N; ++g) {
        const unsigned long long c = (unsigned long long)PERM(0u, s[g], e[g][0]) << (pos & 31u);
        asm volatile("v_cmpx_ne_u32_e32 vcc, 0, %1\n\tds_or_b32 %0, %1\n\tv_cmpx_ne_u32_e32 vcc, 0, %2\n\tds_or_b32 %0, %2 offset:4\n\ts_mov_b64 exec, -1"
                     :: "v"((pos >> 3) & ~3u), "v"((unsigned)c), "v"((unsigned)(c >> 32)) : "vcc", "memory");
        pos += e[g][1];
    }
}
constexpr int SCR_B = 3328;
constexpr int SCR_3 = 2064, SCR_T = 2592, PBUF = 16384 + SCR_B;
__device__ __forceinline__ unsigned pack_unit(u4 (&h)[8], LAS char* scr, const LAS char* lut, int lane, unsigned& r0, unsigned& r2) {
    const u4 z = {0u, 0u, 0u, 0u};
    *(LAS u4*)(scr + lane * 16) = z; *(LAS u4*)(scr + 1024 + lane * 16) = z; *(LAS u4*)(scr + 2048 + lane * 16) = z;
    FENCE();
    unsigned G[16], s[16], a[16], cw[4];
#pragma unroll
    for (int g = 0; g < 16; ++g) { G[g] = PERM(HW(2 * g + 1), HW(2 * g), 0x07050301u); s[g] = (G[g] >> 1) & 0x3E3E3E3Eu; }
    const int m = wave_sum(__builtin_amdgcn_udot4(s[0] + s[5] + s[10] + s[15], 0x01010101u, 0u, false)) + 205;
    const unsigned b3 = ((unsigned)(m >> 11) + 3u) & 31u, type = (m & 2047) > 1120;
    u2 e[16];
#pragma unroll
    for (int g = 0; g < 16; ++g) {
        s[g] = ((2u * b3 + 65u) * 0x01010101u - s[g]) & 0x3F3F3F3Fu;
        a[g] = code4(0xF300F700u, 0xFB00FF00u, s[g]);
        e[g] = *(const LAS u2*)(lut + a[g]);
    }
    unsigned n = 0;
#pragma unroll
    for (int j = 0; j < 4; ++j) { cw[j] = (a[4 * j] >> 3) | (a[4 * j + 1] << 5) | (a[4 * j + 2] << 13) | (a[4 * j + 3] << 21); n = escapes(cw[j], n); }
    unsigned sc = wave_scan(n);
    r2 = (__builtin_amdgcn_readlane(sc, 63) + 31u) >> 5;
    if (r2 > 62u) return M_RAW;
    append<16>(s, e, (sc - n + (unsigned)(size_t)scr) * 8u);
    FENCE();
    const u4 q0 = *(const LAS u4*)(scr + lane * 32), q1 = *(const LAS u4*)(scr + lane * 32 + 16);
#pragma unroll
    for (int g = 0; g < 8; ++g) {
        const unsigned x = g < 4 ? q0[g & 3] : q1[g & 3];
        s[g] = (x | ~PERM(0xFF000000u, 0xFF00F9F6u, x)) - 0x06060606u;
        a[g] = code4(type ? 0xFF00F700u : 0xFB00F700u, type ? 0xF300FB00u : 0xF300FF00u, s[g]);
        e[g] = *(const LAS u2*)(lut + a[g]);
    }
    const unsigned c0 = (a[0] >> 3) | (a[1] << 5) | (a[2] << 13) | (a[3] << 21), c1 = (a[4] >> 3) | (a[5] << 5) | (a[6] << 13) | (a[7] << 21);
    n = escapes(c1, escapes(c0, 0u));
    sc = wave_scan(n);
    const unsigned r3 = (__builtin_amdgcn_readlane(sc, 63) + 7u) >> 3;
    if (r3 > 64u) return M_RAW;
    append<8>(s, e, (sc - n + (unsigned)(size_t)(scr + SCR_3)) * 8u);
    FENCE();
    u2 y = *(const LAS u2*)(scr + SCR_3 + lane * 8);
#pragma unroll
    for (int j = 0; j < 2; ++j) y[j] = (y[j] | ~PERM(0xFF000000u, 0xFF00F9F8u, y[j])) - 0x07070707u;
    unsigned w = ((y[0] >> 1) & 0x07070707u) | ((y[1] << 2) & 0x38383838u);
    w |= ((w >> 24) * 0x41040u) & 0xC0C0C0u;
    LAS char* t = scr + SCR_T;
    t[4 + lane * 3] = (char)w; t[5 + lane * 3] = (char)(w >> 8); t[6 + lane * 3] = (char)(w >> 16);
    FENCE();
    const unsigned l0 = (3u * r3 + 5u) >> 1;
    unsigned bad = ((y[0] >> 4) & 0x03030303u) | (y[1] & 0x30303030u), nl = 0;
    for (;;) {
        const unsigned long long mk = __builtin_amdgcn_ballot_w64(bad != 0);
        if (mk == 0 || nl > 64u) break;
        if (bad != 0) {
            const unsigned p = (unsigned)__builtin_ctz(bad), k = l0 + nl + __builtin_amdgcn_mbcnt_hi((unsigned)(mk >> 32), __builtin_amdgcn_mbcnt_lo((unsigned)mk, 0u));
            ((LAS unsigned short*)t)[k] = (unsigned short)((((unsigned)lane * 8u + (p & 4u) + (p >> 3)) << 7) | ((4u * b3 - 24u - 2u * (((p & 4u) ? y[1] : y[0]) >> (p & 24u))) & 0x7Cu));
            bad &= ~(15u << (p & 28u));
        }
        nl += (unsigned)__builtin_popcountll(mk);
    }
    r0 = (l0 + nl + 3u) >> 2;
    if (nl > 64u || r0 + r2 > 64u) return M_RAW;
    FENCE();
    u2 f = *(const LAS u2*)(t + lane * 8);
    FENCE();
    if (lane == 0) f[0] = b3 | (type << 5) | (r0 << 8) | (nl << 16) | (l0 << 24);
    unsigned L[16];
#pragma unroll
    for (int g = 0; g < 16; ++g) L[g] = PERM(HW(2 * g + 1), HW(2 * g), 0x06040200u);
#pragma unroll
    for (int g = 0; g < 16; ++g) HW(g) = L[g];
#pragma unroll
    for (int j = 0; j < 6; ++j) {
        const unsigned x = bfi(0x83838383u, G[2 * j], ABIT(G[2 * j + 1], G[2 * j + 1], 29));
        HW(16 + j) = j < 4 ? bfi(0x60606060u, G[12 + j] << 5, x) : bfi(0x20202020u, G[2 * j + 4] >> 2, bfi(0x40404040u, G[2 * j + 5] >> 1, x));
    }
    h[5][2] = cw[0]; h[5][3] = cw[1]; h[6] = (u4){cw[2], cw[3], f[0], f[1]}; h[7][0] = c0; h[7][1] = c1;
    return (r0 + r2 + 7u) >> 3;
}
template <int N> __device__ __forceinline__ void deposit(unsigned* o, const u2* e, unsigned tab, unsigned pos) {
    u2 x[N]; unsigned p[N];
#pragma unroll
    for (int g = 0; g < N; ++g) {
        const LAS unsigned* d = (const LAS unsigned*)(size_t)(pos & ~3u);
        x[g] = (u2){d[0], d[1]}; p[g] = pos;
        pos += e[g][1];
    }
#pragma unroll
    for (int g = 0; g < N; ++g) o[g] = PERM(tab, __builtin_amdgcn_alignbyte(x[g][1], x[g][0], p[g]), e[g][0]);
}
__device__ __forceinline__ void unpack_unit(u4 (&h)[8], LAS char* scr, const LAS char* lut, int lane) {
    LAS char* t = scr + SCR_T;
    *(LAS u2*)(t + lane * 8) = (u2){h[6][2], h[6][3]};
    FENCE();
    const unsigned hd = __builtin_amdgcn_readfirstlane(h[6][2]), r0 = (hd >> 8) & 63u, type = hd & 32u;
    const unsigned kb = (((hd & 31u) << 2) | 0x80u) * 0x01010101u;
    const LAS unsigned* d3 = (const LAS unsigned*)(t + ((4 + lane * 3) & ~3));
    const unsigned w0 = d3[0], w1 = d3[1];
    unsigned le = 0;
    const bool ll = (unsigned)lane < ((hd >> 16) & 127u);
    if (ll) le = ((const LAS unsigned short*)t)[(hd >> 24) + lane];
    const unsigned c0 = (unsigned)lane < r0 ? 0u : h[6][2], c1 = (unsigned)lane < r0 ? 0u : h[6][3];
    const unsigned cw[4] = {h[5][2], h[5][3], h[6][0], h[6][1]};
    u2 e2[8], e1[16];
#pragma unroll
    for (int g = 0; g < 8; ++g) e2[g] = *(const LAS u2*)(lut + (((g < 4 ? c0 : c1) >> (8 * (g & 3))) & 255u) * 8u);
#pragma unroll
    for (int g = 0; g < 16; ++g) e1[g] = *(const LAS u2*)(lut + ((cw[g >> 2] >> (8 * (g & 3))) & 255u) * 8u);
    const unsigned n2 = escapes(c1, escapes(c0, 0u)), n1 = escapes(cw[3], escapes(cw[2], escapes(cw[1], escapes(cw[0], 0u))));
    const unsigned n = n1 | (n2 << 16), sc = wave_scan(n) - n;
    unsigned w = __builtin_amdgcn_alignbyte(w1, w0, (unsigned)lane * 3u);
    w = bfi(0x003F3F3Fu, w, __builtin_amdgcn_udot4(w & 0xC0C0C0u, 0x100401u, 0u, false) << 18);
    const unsigned t3l = (kb - (type ? 0x24201C18u : 0x24201C00u)) & 0x7C7C7C7Cu, t3h = (kb - 0x34302C28u) & 0x7C7C7C7Cu;
    *(LAS u2*)(scr + SCR_3 + lane * 8) = (u2){PERM(t3h, t3l, w & 0x07070707u), PERM(t3h, t3l, (w >> 3) & 0x07070707u)};
    FENCE();
    if (ll) scr[SCR_3 + (le >> 7)] = (char)(le & 0x7Cu);
    FENCE();
    unsigned E[16];
    deposit<8>(E, e2, (kb - (type ? 0x001410u : 0x181410u)) & 0x7C7C7C7Cu, (sc >> 16) + (unsigned)(size_t)(scr + SCR_3));
    FENCE();
    *(LAS u4*)(scr + lane * 32) = (u4){E[0], E[1], E[2], E[3]}; *(LAS u4*)(scr + lane * 32 + 16) = (u4){E[4], E[5], E[6], E[7]};
    FENCE();
    deposit<16>(E, e1, (kb - 0x04080Cu) & 0x7C7C7C7Cu, (sc & 0xFFFFu) + r0 * 32u + (unsigned)(size_t)scr);
    FENCE();
    unsigned L[16];
#pragma unroll
    for (int g = 0; g < 16; ++g) {
        L[g] = HW(g);
        const unsigned x = HW(16 + (g < 12 ? g >> 1 : g - 12));
        E[g] = bfi(0x7C7C7C7Cu, E[g], g >= 12 ? bfi(0x80808080u, HW(14 + (g >> 1)) << (2 - (g & 1)), x >> 5) : (g & 1) ? ABIT(x, x, 3) : x);
    }
#pragma unroll
    for (int g = 0; g < 16; ++g) { HW(2 * g) = PERM(E[g], L[g], 0x05010400u); HW(2 * g + 1) = PERM(E[g], L[g], 0x07030602u); }
}
constexpr unsigned LIST_MAX = 126;
__device__ __forceinline__ unsigned pack_out(u4 (&h)[8], LAS char* scr, int lane) {
    unsigned es = 0;
#pragma unroll
    for (int j = 0; j < 4; ++j) es += ((h[0][j] >> 23) & 255u) + ((h[4][j] >> 23) & 255u);
    int e0 = (wave_sum(es) - 2355 + 511) >> 9;
    e0 = e0 < 0 ? 0 : e0 > 248 ? 248 : e0;
    const unsigned base = (unsigned)e0 << 24;
    unsigned n = 0;
#pragma unroll
    for (int i = 0; i < 8; ++i) {
#pragma unroll
        for (int j = 0; j < 4; ++j) h[i][j] = ABIT(h[i][j], h[i][j], 31) - base;
        unsigned t = __builtin_amdgcn_perm(__builtin_amdgcn_perm(h[i][3], h[i][2], 0x0C0C0703u), __builtin_amdgcn_perm(h[i][1], h[i][0], 0x0C0C0703u), 0x05040100u) & 0xF8F8F8F8u;
        for (;;) {
            const unsigned long long m = __builtin_amdgcn_ballot_w64(t != 0);
            if (m == 0 || n > LIST_MAX) break;
            if (t != 0) {
                const unsigned b = (unsigned)__builtin_ctz(t) & 24u;
                const unsigned k = n + 2u + __builtin_amdgcn_mbcnt_hi((unsigned)(m >> 32), __builtin_amdgcn_mbcnt_lo((unsigned)m, 0u));
                ((LAS unsigned short*)scr)[k] = (unsigned short)(((unsigned)((i >> 2) * 1024 + lane * 16 + (i & 3) * 4) << 5) + (b << 2) + ((t >> (b + 3u)) & 31u));
                t &= ~(255u << b);
            }
            n += (unsigned)__builtin_popcountll(m);
        }
    }
    if (n > LIST_MAX) {
#pragma unroll
        for (int k = 0; k < 32; ++k) { const unsigned x = HW(k) + base; HW(k) = ABIT(x, x, 1); }
        return M_RAW;
    }
    asm volatile("" ::: "memory");
    unsigned l = *(const LAS unsigned*)(scr + lane * 4);
    asm volatile("" ::: "memory");
    if (lane == 0) l = ((unsigned)e0 << 16) | n;
    unsigned lo = 0;
#pragma unroll
    for (int p = 0; p < 5; ++p) {
        lo = ABIT(HW(27 + p), lo, 2);
#pragma unroll
        for (int j = 0; j < 5; ++j) HW(5 * p + j) = ABIT(HW(5 * p + j), HW(27 + p) << (25 - 5 * j), 27);
    }
    HW(25) = ABIT(HW(25), lo << 5, 27); HW(26) = ABIT(HW(26), lo, 27);
    h[6][3] = l;
    return (n + 33u) >> 5;
}
__device__ __forceinline__ void unpack_out(u4 (&h)[8], unsigned lst, LAS char* scr, int lane) {
    const unsigned hd = __builtin_amdgcn_readfirstlane(lst);
    const unsigned n = hd & 0xFFFFu, base = (hd >> 16) << 24;
    *(LAS u4*)(scr + lane * 16) = (u4){0u, 0u, 0u, 0u};
    *(LAS u4*)(scr + 1024 + lane * 16) = (u4){0u, 0u, 0u, 0u};
    asm volatile("" ::: "memory");
    if ((unsigned)(2 * lane - 2) < n) scr[(lst >> 5) & 2047u] = (char)(lst & 31u);
    if ((unsigned)(2 * lane - 1) < n) scr[lst >> 21] = (char)((lst >> 16) & 31u);
    asm volatile("" ::: "memory");
    const u4 s0 = *(const LAS u4*)(scr + lane * 16), s1 = *(const LAS u4*)(scr + 1024 + lane * 16);
    asm volatile("" ::: "memory");
    const unsigned lo = ABIT(HW(26), HW(25) << 27, 5);
#pragma unroll
    for (int p = 0; p < 5; ++p) {
        unsigned y = lo << (8 - 2 * p);
#pragma unroll
        for (int j = 0; j < 5; ++j) y = ABIT(HW(5 * p + j), y, 5);
        HW(27 + p) = y;
    }
#pragma unroll
    for (int k = 0; k < 32; ++k) {
        const unsigned x = ABIT((k < 16 ? s0 : s1)[(k >> 2) & 3] >> (8 * (k & 3)), HW(k), 5) + base;
        HW(k) = ABIT(x, x, 1);
    }
}

__device__ __forceinline__ void unit_decode(int u, int& kind, int& ch, int& g) {
    constexpr int U0 = 56 * SKC, U1 = U0 + 64 * (64 - SKC);
    if (u < U0) { const int s = u / 7; kind = u - 7 * s; ch = s >> 3; g = s & 7; if (ch < 8 && kind < 5) ch = (CHMAP >> (4 * ch)) & 7; }
    else if (u < U1) { const int v = u - U0; kind = v & 7; g = (v >> 3) & 7; ch = (v >> 6) + (kind == 7 ? 0 : SKC); }
    else { const int v = u - U1; kind = 7; g = v & 7; ch = 64 - SKC + (v >> 3); }
}
__device__ __forceinline__ void fetch_input(const CAS Params& P, LAS char* buf, int u, int lane) {
    int kind, ch, g;
    unit_decode(u, kind, ch, g);
    const bool isx = kind == 0 || kind > 4;
    const char* src = isx ? (const char*)P.x + (size_t)((unsigned)((kind ? kind - 4 : 0) * 8 + g) * (32 * KDIM * 4) + (unsigned)ch * 512)
                          : (const char*)P.w + (size_t)((unsigned)(2 * ch + (g >> 2)) * (64 * NLOC * 4) + (unsigned)((kind - 1) * 4 + (g & 3)) * 256);
    const unsigned s0 = isx ? 64 : NLOC * 4, s1 = isx ? 16 * KDIM * 4 : 16 * NLOC * 4, s2 = isx ? 256 : 32 * NLOC * 4;
    const unsigned voff = isx ? (unsigned)(lane & 15) * (KDIM * 4) + (unsigned)(lane >> 4) * 16 : (unsigned)(lane >> 4) * (4 * NLOC * 4) + (unsigned)(lane & 15) * 16;
#pragma unroll
    for (int i = 0; i < 16; ++i)
        __builtin_amdgcn_global_load_lds((const GAS void*)(src + ((i & 3) * s0 + ((i >> 2) & 1) * s1 + (i >> 3) * s2) + voff), (LAS void*)(buf + i * 1024), 16, 0, 0);
}
#define TOUCHU 448
__device__ __forceinline__ void touch(const CAS Params& P, char* myws, LAS char* scr, int u, int lane) {
    int ch, g, kind;
    unit_decode(u, kind, ch, g);
    const bool isx = kind == 0 || kind > 4;
    const int rb = (kind ? kind - 4 : 0) * 8 + g, kt = 2 * ch + (g >> 2), cb = (kind - 1) * 4 + (g & 3);
    const int pk = isx ? ch * 32 + rb : PK_W + (kind - 1) * 512 + ch * 8 + g;
    unsigned off, fl;
    if (isx) place_x(rb < 24 ? 3 * (P.rank & 3) + (rb >> 3) : 16, rb, ch, lane & 1, off, fl); else place_w(cb, kt, off, fl);
    const unsigned a = lane & 2 ? off : (unsigned)pk_off(pk, lane & 1);
    if (lane < 4) __builtin_amdgcn_global_load_lds((const GAS void*)(myws + a), (LAS void*)scr, 4, 0, 17);
}
__device__ __forceinline__ void pace(unsigned long long t0, int u) {
    if (u < UG) while (__builtin_amdgcn_s_memrealtime() - t0 < (unsigned)(u < U1ST ? u * PACE1 : (u - U1ST) * PACE + U1ST * PACE2) >> 4) __builtin_amdgcn_s_sleep(2);
}
__device__ __forceinline__ int next_unit(int u, int wid) {
    constexpr int R1 = 8 * NGEMM, R2 = R1 + 8 * (192 - B1);
    const int b = blockIdx.x;
    if (u >= UG) return u + 8 * NPROD;
    u = u >= R2 ? u + 512 : b >= 192 || b < (u < R1 ? B1 : 128) ? UG : u < R1 ? R1 + wid * (192 - B1) + b - B1 : R2 + wid * 64 + b - 128;
    return u < UG ? u : PK_O;
}
__device__ __forceinline__ void producer(const CAS Params& P, char* myws, LAS char* buf, LAS char* lut, int u, unsigned long long t0, unsigned call, int wid, int lane) {
    const int q = P.rank & 3;
    const unsigned vl16 = (unsigned)lane * 16;
    LAS char* scr = buf + 16384;
    for (int un; u < PK_O; u = un) {
        un = next_unit(u, wid);
        int ch, g, kind;
        unit_decode(u, kind, ch, g);
        const bool isx = kind == 0 || kind > 4;
        const int rb = (kind ? kind - 4 : 0) * 8 + g, kt = 2 * ch + (g >> 2), cb = (kind - 1) * 4 + (g & 3);
        const int idx = isx ? pos_x(rb, ch) : pos_w(kt, g & 3), pk = isx ? ch * 32 + rb : PK_W + (kind - 1) * 512 + ch * 8 + g;
        const int rd = !isx ? kind - 1 : rb == 16 || rb >= 24 ? 5 : 4;
        f4 v[16]; u4 h[8];
        unsigned hv;
        asm volatile("global_load_dword %0, %1, %2 sc0 sc1" : "=&v"(hv) : "v"((unsigned)OFF_MAP + ((call & 1) * PK_PG + pk / 127) * MAPS), "s"(myws) : "memory");
#pragma unroll
        for (int i = 0; i < 16; ++i) v[i] = *(const LAS f4*)(buf + i * 1024 + lane * 16);
        __builtin_amdgcn_sched_barrier(0);
        WAIT_L0();
        const bool ahead = u >= AHEAD && un < PK_O;
        if (ahead) { pace(t0, un); fetch_input(P, buf, un, lane); }
        __builtin_amdgcn_sched_barrier(0);
        if (isx) {
#pragma unroll
            for (int i = 0; i < 8; ++i)
                h[i] = pack8(v[2 * i][0], v[2 * i][1], v[2 * i][2], v[2 * i][3], v[2 * i + 1][0], v[2 * i + 1][1], v[2 * i + 1][2], v[2 * i + 1][3]);
        } else {
#pragma unroll
            for (int i = 0; i < 8; ++i) {
                const int f = i >> 1, k = (i & 1) * 8;
                h[i] = pack8(v[k][f], v[k + 1][f], v[k + 2][f], v[k + 3][f], v[k + 4][f], v[k + 5][f], v[k + 6][f], v[k + 7][f]);
            }
        }
        unsigned off, fl;
        if (isx) place_x(rb < 24 ? 3 * q + (rb >> 3) : 16, rb, ch, call, off, fl); else place_w(cb, kt, off, fl);
        const bool own = !isx || rb != 16;
        const unsigned long long h11 = NOW();
        if (idx < NPUSH) {
            unsigned off2 = off, fl2 = fl;
            if (!isx) place_w(16 + 4 * q + (g & 3), kt, off2, fl2);
            while (__builtin_amdgcn_s_memrealtime() - t0 < (unsigned)(PT0 + idx * PDT)) __builtin_amdgcn_s_sleep(1);
            char* fp = myws + fl;
            const unsigned long long h14 = NOW();
            for (int i = 1; i < (isx ? 4 : 2); ++i) {
                char* pw = P.ws[isx ? (P.rank & 4) + ((q + i) & 3) : ((P.rank & 4) ^ 4) + rd];
                store8(h, pw + off2, isx ? 32768u : 4096u, vl16);
                fp = lane == i ? pw + fl2 : fp;
            }
            store8(h, myws + off, isx ? 32768u : 4096u, vl16);
            asm volatile("s_waitcnt vmcnt(0)" : "+v"(hv) :: "memory");
            if (lane < (isx ? 4 : 2)) flag_store(fp, call);
            HST(12, NOW()); HST(11, h11); HST(14, h14); HST(13, (unsigned long long)(rd * 4096 + idx + 1));
            if (un < PK_O) { pace(t0, un); fetch_input(P, buf, un, lane); WAIT_V(0); }
            continue;
        }
        if (own) store8(h, myws + off, isx ? 32768u : 4096u, vl16);
        unsigned mode = M_RAW, r0 = 0, r2 = 0;
        if (idx >= RAW0 && (rd != 4 || idx < NI - RAW1)) mode = pack_unit(h, scr, lut, lane, r0, r2);
        const unsigned long long h14 = NOW();
        if (!ahead) { if (own) asm volatile("s_waitcnt vmcnt(8)" : "+v"(hv) :: "memory"); else asm volatile("s_waitcnt vmcnt(0)" : "+v"(hv) :: "memory"); }
        else if (own) asm volatile("s_waitcnt vmcnt(24)" : "+v"(hv) :: "memory"); else asm volatile("s_waitcnt vmcnt(16)" : "+v"(hv) :: "memory");
        const unsigned half = SMODE == 0 ? idx & 1 : (__builtin_amdgcn_readfirstlane(hv) ^ (rd != 4) ^ (SMODE == 2)) & 1u;
        char* d0 = myws + (pk_off(pk, call) + half * 8192); char* d1 = d0 + 4096;
        GST4(vl16, h[0], d0, 0); GST4(vl16, h[1], d0, 1024); GST4(vl16, h[2], d0, 2048); GST4(vl16, h[3], d0, 3072);
        GST4(vl16, h[4], d1, 0); GST4(vl16, h[5], d1, 1024);
        if (mode < M_RAW) {
            const u2 p0 = {h[6][0], h[6][1]}, p1 = {h[6][2], h[6][3]}, p2 = {h[7][0], h[7][1]};
            asm volatile("global_store_dwordx2 %0, %1, %4 offset:2048 sc0 sc1\n\t"
                         "s_mov_b64 exec, %6\n\tglobal_store_dwordx2 %0, %2, %4 offset:2560 sc0 sc1\n\t"
                         "s_mov_b64 exec, %7\n\tglobal_store_dwordx2 %0, %3, %5 offset:2560 sc0 sc1\n\ts_mov_b64 exec, -1"
                         :: "v"(vl16 >> 1), "v"(p0), "v"(p1), "v"(p2), "s"(d1), "s"(d1 + r0 * 8u), "s"(lane_mask(r0)), "s"(lane_mask(r2)) : "memory");
        } else { GST4(vl16, h[6], d1, 2048); GST4(vl16, h[7], d1, 3072); }
        WAIT_V(0);
        const unsigned fo = lane < 5 ? (unsigned)OFF_FP + (call & 1) * (unsigned)FP_B + (unsigned)(lane * NSEQ + idx) * 4u : fl;
        if ((rd == 5 ? lane < 4 : lane == rd) || (own && lane == 5)) flag_store(myws + fo, lane < 5 ? call * 32u + half * 16u + mode : call);
        HST(12, NOW()); HST(11, h11); HST(14, h14); HST(13, (unsigned long long)(rd * 4096 + idx + 1));
        if (!ahead && un < PK_O) { pace(t0, un); fetch_input(P, buf, un, lane); WAIT_V(0); }
    }
}

__device__ __forceinline__ void flag_mirror(char* myws, const char* fp, int nseq, LAS unsigned* info, LAS unsigned* prog, unsigned call, int lane) {
    GAS unsigned* tmo = (GAS unsigned*)(myws + OFF_TMO);
    int pos = NPUSH, nl = 16;
    unsigned long long t0 = __builtin_amdgcn_s_memrealtime();
    while (pos < nseq) {
        const int pr = (int)__builtin_amdgcn_readfirstlane(__hip_atomic_load(prog, __ATOMIC_RELAXED, __HIP_MEMORY_SCOPE_WORKGROUP));
        if (pos <= pr + LOOK) {
            const int i = pos + 4 * lane;
            bool bad = false;
            if (lane < nl && i < nseq) {
                u4 f;
                asm volatile("global_load_dwordx4 %0, %1, %2 sc0 sc1\n\ts_waitcnt vmcnt(0)" : "=&v"(f) : "v"(i * 4), "s"(fp) : "memory");
                *(LAS u4*)(info + i) = f;
                bad = f[0] < call * 32u || f[1] < call * 32u || f[2] < call * 32u || f[3] < call * 32u;
            }
            const unsigned long long m = __builtin_amdgcn_ballot_w64(bad);
            const int n = m ? (__builtin_ctzll(m) & ~3) : nl;
            nl = m ? 16 : 64;
            if (n) { pos += 4 * n; t0 = __builtin_amdgcn_s_memrealtime(); continue; }
        }
        if (
            __builtin_amdgcn_s_memrealtime() - t0 > DEADLINE) {
            __hip_atomic_store(tmo, call, __ATOMIC_RELAXED, __HIP_MEMORY_SCOPE_AGENT);
            asm volatile("s_waitcnt vmcnt(0)" ::: "memory");
            for (int i = pos + lane; i < nseq; i += 64) info[i] = call * 32u + M_DEAD;
            return;
        }
        if (pos > pr + 24) __builtin_amdgcn_s_sleep(24);
    }
}

#define STR2(x) #x
#define STR(x) STR2(x)
#define SP ""
#define RD ""
#define PULL6 "global_load_dwordx4 %[h0], %[v16], %[src] offset:-4096 sc0 sc1\n\t" SP "global_load_dwordx4 %[h1], %[v16], %[src] offset:-3072 sc0 sc1\n\t" SP \
              "global_load_dwordx4 %[h2], %[v16], %[src] offset:-2048 sc0 sc1\n\t" SP "global_load_dwordx4 %[h3], %[v16], %[src] offset:-1024 sc0 sc1\n\t" SP \
              "global_load_dwordx4 %[h4], %[v16], %[src] sc0 sc1\n\t" SP "global_load_dwordx4 %[h5], %[v16], %[src] offset:1024 sc0 sc1\n\t" SP
#define PULLT "s_mov_b64 exec, 1\n\tglobal_atomic_add %[tk], %[ao], %[one], %[ws] sc0\n\t" \
              "s_waitcnt vmcnt(%[rn] + 9 - " STR(RELAT) ")\n\tglobal_store_dword %[pf], %[cl], %[ws] sc0 sc1\n\ts_waitcnt vmcnt(%[rn] + 1)\n\t" RD \
              "ds_add_u32 %[rel], %[one]\n\ts_mov_b64 exec, -1\n\ts_waitcnt vmcnt(1)"
#define PULLO [h0] "=&v"(h[0]), [h1] "=&v"(h[1]), [h2] "=&v"(h[2]), [h3] "=&v"(h[3]), [h4] "=&v"(h[4]), [h5] "=&v"(h[5]), [tk] "=&v"(tk)
#define PULLI [v16] "v"(vl16), [src] "s"(src), [ao] "v"(aoff), [one] "v"(one), [ws] "s"(myws), [rel] "v"(rel), [pf] "v"(pfl), [cl] "v"(call)
#undef VBLK
#define VBLK (NGEMM + vg)
__device__ __forceinline__ void puller(int vg, const CAS Params& P, char* myws, LAS unsigned* info, LAS unsigned* prog, const LAS unsigned short* tab,
                                       LAS char* scr, const LAS char* lut, int link, int peer, int cur, int nw, unsigned win, unsigned call, int wid, int lane) {
    const int q = P.rank & 3, qs = peer & 3;
    const bool cross = link < 4;
    const int nseq = cross ? NSEQ : NI;
    const char* pk = P.ws[peer] + 4096;
    const unsigned vl16 = (unsigned)lane * 16, one = 1, aoff = (unsigned)OFF_CTR + (unsigned)link * 128u;
    const unsigned ovoff = (unsigned)(lane >> 4) * (4 * NLOC * 4) + (unsigned)(lane & 15) * 16;
    const unsigned base = (call - 1u) * (unsigned)(nseq - NPUSH) - (unsigned)(nw + NPUSH);
    const unsigned rel = (unsigned)(size_t)(prog + 2);
    const unsigned nofl = (unsigned)OFF_SEQ + (blockIdx.x * 8u + (unsigned)wid) * 16u + 4u;
    unsigned pfl = nofl;
    const unsigned tz = __builtin_amdgcn_readfirstlane(prog[4]);
    if (TOUCH & 2) {
        const int id = __builtin_amdgcn_readfirstlane((int)tab[cur]), rb = id & 31;
        unsigned off, fl, d;
        if (id < PK_W) place_x(rb < 24 ? 3 * qs + (rb >> 3) : 12 + qs, rb, id >> 5, call, off, fl);
        else place_w(16 + 4 * qs + (id & 3), (id - PK_W) >> 2, off, fl);
        const char* a = lane ? myws + off : pk + pk_off(id < PK_W ? id : id + q * 512, call);
        if (lane < 2) asm volatile("global_load_dword %0, %1, off sc0 sc1\n\ts_waitcnt vmcnt(0)" : "=&v"(d) : "v"(a) : "memory");
    }
    for (;;) {
        const bool isout = cross && cur >= NCI;
        const int id = isout ? 0 : __builtin_amdgcn_readfirstlane((int)tab[cur]);
        const int slot = isout ? PK_O + q * NO + cur - NCI : id < PK_W ? id : id + q * 512;
        unsigned mode;
        if (pfl != nofl) {
            const unsigned i0 = __hip_atomic_load(info + cur, __ATOMIC_RELAXED, __HIP_MEMORY_SCOPE_WORKGROUP);
            const unsigned i1 = __hip_atomic_load(prog + 1, __ATOMIC_RELAXED, __HIP_MEMORY_SCOPE_WORKGROUP), i2 = __hip_atomic_load(prog + 2, __ATOMIC_RELAXED, __HIP_MEMORY_SCOPE_WORKGROUP);
            if (__builtin_amdgcn_readfirstlane(!DEFER || i0 - call * 32u > 31u || i1 - i2 >= win)) {
                WAIT_V(0);
                if (lane == 0) flag_store(myws + pfl, call);
                pfl = nofl;
            }
        }
        for (;;) {
            mode = __builtin_amdgcn_readfirstlane(__hip_atomic_load(info + cur, __ATOMIC_ACQUIRE, __HIP_MEMORY_SCOPE_WORKGROUP)) - call * 32u;
            if (mode <= 31u) break;
            __builtin_amdgcn_s_sleep(1);
        }
        const char* src = pk + (pk_off(slot, call) + (mode & 16u) * 512u);
        mode &= 15u;
        const unsigned long long mask = lane_mask(isout ? mode * 16u : mode * 8u);
        unsigned my = 0;
        if (lane == 0) my = __hip_atomic_fetch_add(prog + 1, 1u, __ATOMIC_RELAXED, __HIP_MEMORY_SCOPE_WORKGROUP);
        my = __builtin_amdgcn_readfirstlane(my) - win;
        while ((int)(my - __builtin_amdgcn_readfirstlane(__hip_atomic_load(prog + 2, __ATOMIC_RELAXED, __HIP_MEMORY_SCOPE_WORKGROUP))) >= 0) { }
        {
            unsigned ts = 0;
            if (lane == 0) {
                __hip_atomic_fetch_max(prog + 3, ((unsigned)__builtin_amdgcn_s_memrealtime() - tz) << 8, __ATOMIC_RELAXED, __HIP_MEMORY_SCOPE_WORKGROUP);
                ts = __hip_atomic_fetch_add(prog + 3, (mode >= M_RAW ? 128u : isout ? 108u + mode : 104u + mode) * (cross ? PRATE : PRATEI), __ATOMIC_RELAXED, __HIP_MEMORY_SCOPE_WORKGROUP);
            }
            ts = __builtin_amdgcn_readfirstlane(ts);
            ts = (ts >> 8) + tz - PSLACK;
            while ((int)((unsigned)__builtin_amdgcn_s_memrealtime() - ts) < 0) { }
        }
        u4 h[8]; unsigned tk, lst = 0;
        const unsigned tis = (unsigned)__builtin_amdgcn_s_memrealtime();
        if (mode >= M_RAW) {
            asm volatile(PULL6 "global_load_dwordx4 %[h6], %[v16], %[src] offset:2048 sc0 sc1\n\tglobal_load_dwordx4 %[h7], %[v16], %[src] offset:3072 sc0 sc1\n\t" PULLT
                         : PULLO, [h6] "=&v"(h[6]), [h7] "=&v"(h[7]) : PULLI, [rn] "n"(RELAT) : "memory");
        } else if (!isout) {
            u2 p0, p1;
            asm volatile(PULL6 "global_load_dwordx2 %[p0], %[v8], %[src] offset:2048 sc0 sc1\n\t"
                         "s_mov_b64 exec, %[mask]\n\tglobal_load_dwordx2 %[p1], %[v8], %[src] offset:2560 sc0 sc1\n\t" PULLT
                         : PULLO, [p0] "=&v"(p0), [p1] "=&v"(p1)
                         : PULLI, [v8] "v"(vl16 >> 1), [mask] "s"(mask), [rn] "n"(RELAT) : "memory");
            h[6] = (u4){p0[0], p0[1], p1[0], p1[1]};
        } else {
            u2 p2; unsigned p1;
            asm volatile(PULL6 "global_load_dwordx2 %[p2], %[v8], %[src] offset:2048 sc0 sc1\n\tglobal_load_dword %[p1], %[v4], %[src] offset:2560 sc0 sc1\n\t"
                         "s_mov_b64 exec, %[mask]\n\tglobal_load_dword %[lst], %[v4], %[src] offset:2816 sc0 sc1\n\t" PULLT
                         : PULLO, [p2] "=&v"(p2), [p1] "=&v"(p1), [lst] "=&v"(lst)
                         : PULLI, [v8] "v"(vl16 >> 1), [v4] "v"(vl16 >> 2), [mask] "s"(mask), [rn] "n"(RELAT + 1) : "memory");
            h[6][0] = p2[0]; h[6][1] = p2[1]; h[6][2] = p1;
        }
        const unsigned lat = (unsigned)__builtin_amdgcn_s_memrealtime() - tis;
        const unsigned nxt = __builtin_amdgcn_readfirstlane(tk) - base;
        if (lane == 0) __hip_atomic_fetch_max(prog, nxt, __ATOMIC_RELAXED, __HIP_MEMORY_SCOPE_WORKGROUP);
        if (isout) {
            const int o = cur - NCI;
            int panel, blk, col;
            if (o < O_B) {
                const int r = o < O_A0 ? o : o < O_C ? (o - O_A0) & 127 : o - O_C, hn = o < O_A0 ? 16 : o < O_C ? 64 : 48;
                const int hf = r >= hn, t = (r - hf * hn) >> 3;
                panel = o < O_A0 ? 3 * qs + 2 : o < O_C ? 3 * (t >> 1) + (o >= O_A1) : 12 + ((q + 1 + (t >> 1)) & 3);
                blk = (r & 6) + hf; col = (t & 1) * 128 + (r & 1) * 64;
            } else {
                const int j = (o - O_B) / 24, r = o - O_B - j * 24;
                panel = 3 * ((qs + 1 + (r >> 3)) & 3) + 2; blk = 2 * j + ((r >> 1) & 1); col = ((r >> 2) & 1) * 128 + (r & 1) * 64;
            }
            char* ob = (char*)P.out + (size_t)(panel_row(peer, panel, blk) * NLOC + qs * 256 + col) * 4;
            if (mode < M_RAW) unpack_out(h, lst, scr, lane);
            if (mode == M_DEAD) h[0] = (u4){0x7FC00000u, 0x7FC00000u, 0x7FC00000u, 0x7FC00000u};
#pragma unroll
            for (int k = 0; k < 8; ++k) GST4P(ovoff, h[k], ob + ((k >> 2) * 16 + (k & 3)) * (NLOC * 4));
            if (!DEFER) WAIT_V(0);
            pfl = nofl;
        } else {
            if (mode < M_RAW) unpack_unit(h, scr, lut, lane);
            unsigned off, fl;
            const int rb = id & 31;
            if (id < PK_W) place_x(rb < 24 ? 3 * qs + (rb >> 3) : 12 + qs, rb, id >> 5, call, off, fl);
            else place_w(16 + 4 * qs + (id & 3), (id - PK_W) >> 2, off, fl);
            store8(h, myws + off, id < PK_W ? 32768u : 4096u, vl16);
            pfl = fl;
        }
        if (nxt >= (unsigned)nseq) break;
        cur = (int)nxt;
    }
    WAIT_V(0);
    if (lane == 0) flag_store(myws + pfl, call);
}

#undef VBLK
#define VBLK (blockIdx.x < NGEMM ? blockIdx.x : blockIdx.x + NPC)
__device__ __forceinline__ float silu(float y) {
    const float c = __builtin_amdgcn_fmed3f(y, -60.0f, 60.0f);
    const float e = __builtin_amdgcn_exp2f(c * -1.4426950408889634f);
    return y * __builtin_amdgcn_rcpf(1.0f + e);
}
__device__ __forceinline__ u4 silu4(const f4 (&a)[4], int r, bool dead) {
    f4 o = {silu(a[0][r]), silu(a[1][r]), silu(a[2][r]), silu(a[3][r])};
    const float nanv = __builtin_nanf("");
    if (dead) o = (f4){nanv, nanv, nanv, nanv};
    return __builtin_bit_cast(u4, o);
}

#define BAR() do { __builtin_amdgcn_sched_barrier(0); __builtin_amdgcn_s_barrier(); asm volatile("" ::: "memory"); __builtin_amdgcn_sched_barrier(0); } while (0)
#define GLDS(g, l, imm) __builtin_amdgcn_global_load_lds((const GAS void*)(g), (LAS void*)(l), 16, imm, AUX)
template <int V> struct IC { static constexpr int value = V; };

__device__ __forceinline__ void gemm_panel(const CAS Params& P, LAS char* lds, char* myws, int panel, int cg, int o0, int ohalf,
                                           unsigned call, int wid, int lane) {
    constexpr int ABYTES = 32768, STAGE = ABYTES + 16384;
    const char* ga = myws + panel_off(panel, call) + wid * 4096 + lane * 16;
    const char* gb = myws + OFF_DATA + A_B + (size_t)cg * CG_B + wid * 2048 + lane * 16;
    const int wm = wid >> 1, wn = wid & 1;
    const unsigned frow = (lane & 2) ? (unsigned)OFF_FB + (unsigned)cg * 1024u : (unsigned)OFF_FA + (unsigned)panel * 2048u + (unsigned)(lane & 1) * 16u;
    const unsigned fstep = (lane & 2) ? 16u : 32u;
    GAS unsigned* tmo = (GAS unsigned*)(myws + OFF_TMO);
    char* oflag = myws + (OFF_FP + (call & 1) * FP_B) + (((cg - 8) >> 1) * NSEQ + NCI + o0 + wid) * 4;
    int ready = 0;
    unsigned tv = 0;

    auto early = [&](int c) {
        if (c < ready || c >= NCH) return;
        const int cc = c + (lane >> 2);
        __builtin_amdgcn_global_load_lds((const GAS void*)(myws + (frow + (unsigned)(cc < NCH ? cc : NCH - 1) * fstep)), (LAS void*)(lds + 3 * STAGE), 16, 0, AUX);
    };
    auto ensure = [&](int c, bool pend) {
        if (c < ready) return;
        const unsigned long long t0 = __builtin_amdgcn_s_memrealtime();
        int lim = 15;
        for (;;) {
            int cc = c + ((lane >> 2) < lim ? (lane >> 2) : lim); cc = cc < NCH ? cc : NCH - 1;
            u4 f;
            if (pend) asm volatile("s_waitcnt vmcnt(%2)\n\tds_read_b128 %0, %1\n\ts_waitcnt lgkmcnt(0)"
                                   : "=&v"(f) : "v"((unsigned)(size_t)(lds + 3 * STAGE) + (unsigned)lane * 16u), "n"(3 * PD) : "memory");
            else asm volatile("global_load_dwordx4 %0, %1, %2 sc0 sc1\n\ts_waitcnt vmcnt(0)"
                              : "=&v"(f) : "v"(frow + (unsigned)cc * fstep), "s"(myws) : "memory");
            const bool bad = f[0] < call || f[1] < call || f[2] < call || f[3] < call;
            const unsigned long long m = __builtin_amdgcn_ballot_w64(bad);
            const int n = m ? (__builtin_ctzll(m) >> 2) : lim + 1;
            if (n > 0) { ready = c + n; return; }
            if (pend) { pend = false; continue; }
            if (
                __builtin_amdgcn_s_memrealtime() - t0 > DEADLINE) {
                __hip_atomic_store(tmo, call, __ATOMIC_RELAXED, __HIP_MEMORY_SCOPE_AGENT);
                ready = NCH; return;
            }
            lim = 0;
            __builtin_amdgcn_s_sleep(6);
        }
    };
    auto stage = [&](int t, int kf, int bufoff) {
        const char* a = ga + (size_t)t * 32768 + kf * 1024; const char* b = gb + (size_t)t * 16384 + kf * 1024;
        LAS char* la = lds + bufoff + wid * 4096 + kf * 1024; LAS char* lb = lds + bufoff + ABYTES + wid * 2048 + kf * 1024;
        GLDS(a, la, 0); GLDS(a, la, 2048); GLDS(b, lb, 0);
    };

    f4 acc[4][4];
#pragma unroll
    for (int m = 0; m < 4; ++m)
#pragma unroll
        for (int n = 0; n < 4; ++n) acc[m][n] = (f4){0.f, 0.f, 0.f, 0.f};

    const int aoff = wm * 8192 + lane * 16, boff = ABYTES + wn * 8192 + lane * 16;
    auto phase = [&](auto waitn, int rbuf, int kf, int lt, int lkf, int lbuf, int pc, int pe = NCH) {
        const LAS char* pa = lds + rbuf + aoff + kf * 1024; const LAS char* pb = lds + rbuf + boff + kf * 1024;
        h8 a[4], b[4];
#pragma unroll
        for (int n = 0; n < 4; ++n) b[n] = *(const LAS h8*)(pb + n * 2048);
#pragma unroll
        for (int m = 0; m < 4; ++m) a[m] = *(const LAS h8*)(pa + m * 2048);
        __builtin_amdgcn_sched_barrier(0);
        if (lt >= 0) stage(lt, lkf, lbuf);
        if (decltype(waitn)::value >= 0) WAIT_V(decltype(waitn)::value < 0 ? 0 : decltype(waitn)::value);
        WAIT_L0();
        BAR();
        __builtin_amdgcn_s_setprio(1);
#pragma unroll
        for (int m = 0; m < 4; ++m)
#pragma unroll
            for (int n = 0; n < 4; ++n) acc[m][n] = __builtin_amdgcn_mfma_f32_16x16x32_f16(a[m], b[n], acc[m][n], 0, 0, 0);
        __builtin_amdgcn_s_setprio(0);
        __builtin_amdgcn_sched_barrier(0);
        if (wid == 0) { if (pc >= 0) { ensure(pc, PD > 0); TREC(pc); } early(pe); }
        BAR();
    };

    if (wid == 0) { ensure(0, false); ensure(1, false); }
    BAR();
    STAMP(1);
    stage(0, 0, 0); stage(0, 1, 0); stage(1, 0, STAGE); stage(1, 1, STAGE); stage(2, 0, 2 * STAGE);
    WAIT_V(12);
    BAR();
    if (wid >= 4) BAR();
    int rb = 0, pb = 2 * STAGE;
    for (int t = 0; t < NKT - 3; ++t) {
        phase(IC<12>{}, rb, 0, t + 2, 1, pb, (t & 1) ? (t + 3) >> 1 : -1, PD == 2 && !(t & 1) ? (t + 4) >> 1 : NCH);
        phase(IC<12>{}, rb, 1, t + 3, 0, rb, -1, PD == 1 && !(t & 1) ? (t + 4) >> 1 : NCH);
        pb = rb; rb = rb == 2 * STAGE ? 0 : rb + STAGE;
    }
    static_assert((NKT - 3) % 3 == 2, "tail buffers");
    phase(IC<12>{}, 2 * STAGE, 0, NKT - 1, 1, STAGE, -1);
    phase(IC<9>{}, 2 * STAGE, 1, -1, 0, 0, -1);
    phase(IC<6>{}, 0, 0, -1, 0, 0, -1);
    phase(IC<3>{}, 0, 1, -1, 0, 0, -1);
    phase(IC<0>{}, STAGE, 0, -1, 0, 0, -1);
    phase(IC<-1>{}, STAGE, 1, -1, 0, 0, -1);
    if (wid < 4) BAR();
    STAMP(2);
    TDUMP(0);

    const bool dead = __builtin_amdgcn_readfirstlane(__hip_atomic_load(tmo, __ATOMIC_RELAXED, __HIP_MEMORY_SCOPE_AGENT)) == call;
    const unsigned vl16 = (unsigned)lane * 16, ovoff = (unsigned)(lane >> 4) * (4 * NLOC * 4) + (unsigned)(lane & 15) * 16;
    const int pk0 = PK_O + ((cg < 8 ? 0 : cg - 8) >> 1) * NO + o0 + wid;
    const unsigned hv[2] = {pk_half(myws, pk0, call), pk_half(myws, pk0 + ohalf, call)};
    unsigned mode = 0;
#pragma unroll
    for (int u = 0; u < 2; ++u) {
        u4 h[8];
#pragma unroll
        for (int k = 0; k < 8; ++k) h[k] = silu4(acc[2 * u + (k >> 2)], k & 3, dead);
        if (cg < 8) {
            char* ob = (char*)P.out + (size_t)(panel_row(P.rank, panel, 2 * wm + u) * NLOC + cg * 128 + wn * 64) * 4;
#pragma unroll
            for (int k = 0; k < 8; ++k) GST4P(ovoff, h[k], ob + ((k >> 2) * 16 + (k & 3)) * (NLOC * 4));
        } else {
            const unsigned prev = mode;
            if (u && o0 < EARLY) { WAIT_V(0); if (lane == 0) flag_store(oflag, call * 32u + prev); }
            mode = M_RAW;
            if (o0 + u * ohalf >= ORAW) mode = pack_out(h, lds + wid * 2048, lane);
            if (u && o0 >= EARLY) { WAIT_V(0); if (lane == 0) flag_store(oflag, call * 32u + prev); }
            const unsigned half = SMODE == 0 ? wid & 1 : (~__builtin_amdgcn_readfirstlane(hv[u]) ^ (SMODE == 2)) & 1u;
            char* d0 = myws + (pk_off(pk0 + u * ohalf, call) + half * 8192); char* d1 = d0 + 4096;
            GST4(vl16, h[0], d0, 0); GST4(vl16, h[1], d0, 1024); GST4(vl16, h[2], d0, 2048); GST4(vl16, h[3], d0, 3072);
            GST4(vl16, h[4], d1, 0); GST4(vl16, h[5], d1, 1024);
            if (mode < M_RAW) {
                const u2 p2 = {h[6][0], h[6][1]};
                asm volatile("global_store_dwordx2 %0, %1, %2 offset:2048 sc0 sc1\n\tglobal_store_dword %3, %4, %2 offset:2560 sc0 sc1"
                             :: "v"(vl16 >> 1), "v"(p2), "s"(d1), "v"(vl16 >> 2), "v"(h[6][2]) : "memory");
                store_list(d1 + 2816, h[6][3], mode, vl16 >> 2);
            } else {
                GST4(vl16, h[6], d1, 2048);
                GST4(vl16, h[7], d1, 3072);
            }
            mode += half * 16u;
        }
    }
    WAIT_V(0);
    if (cg >= 8 && lane == 0) flag_store(oflag + ohalf * 4, call * 32u + mode);
    WAIT_V(0);
    STAMP(3);
}

__device__ __forceinline__ void gemm_quarter(const CAS Params& P, LAS char* lds, char* myws, int panel, int qt, int cg, int o0,
                                             unsigned call, int wid, int lane) {
    constexpr int STAGE = 49152;
    const int kp = wid >> 2, lw = wid & 3, wm = lw >> 1, wn = lw & 1;
    const char* gsrc = myws + OFF_DATA + lane * 16 + (lw < 2 ? (size_t)panel * PANEL_B + kp * 32768 + qt * 8192
                                                             : A_B + (size_t)cg * CG_B + kp * 16384 + (lw - 2) * 8192);
    const unsigned gstep = lw < 2 ? 65536u : 32768u;
    LAS char* ring = lds + kp * 24576;
    LAS unsigned* pl = (LAS unsigned*)(lds + 160768);
    const unsigned frow = (unsigned)OFF_FA + (unsigned)(panel * 2048 + qt * 8 + (lane & 1) * 4);
    GAS unsigned* tmo = (GAS unsigned*)(myws + OFF_TMO);
    char* oflag = myws + (OFF_FP + (call & 1) * FP_B) + (((cg - 8) >> 1) * NSEQ + NCI + o0) * 4;
    unsigned tv = 0;
    int ready = 0; bool pend = false;
#define QPOLL 8
    auto paddr = [&]() { const int cc = ready + ((lane >> 1) < QPOLL ? (lane >> 1) : QPOLL - 1); return frow + (unsigned)(cc < NCH ? cc : NCH - 1) * 32u; };
    auto late = [&](unsigned tm, unsigned long long t0) {
        if (tm != call && __builtin_amdgcn_s_memrealtime() - t0 <= DEADLINE) return false;
        __hip_atomic_store(tmo, call, __ATOMIC_RELAXED, __HIP_MEMORY_SCOPE_AGENT);
        WAIT_V(0);
        ready = NCH;
        return true;
    };
    auto ensure = [&](int c) {
        if (c >= ready) {
            const unsigned long long t0 = __builtin_amdgcn_s_memrealtime();
            for (;;) {
                unsigned f;
                if (pend) { WAIT_V(0); f = pl[lane]; pend = false; }
                else asm volatile("global_load_dword %0, %1, %2 sc0 sc1\n\ts_waitcnt vmcnt(0)" : "=&v"(f) : "v"(paddr()), "s"(myws) : "memory");
                ready += __builtin_ctzll(__builtin_amdgcn_ballot_w64(f < call) | (1ull << (2 * QPOLL))) >> 1;
                if (c < ready || late(0u, t0)) break;
                __builtin_amdgcn_s_sleep(QSLEEP);
            }
        }
        if (!pend && ready < NCH && ready <= c + 2) {
            __builtin_amdgcn_global_load_lds((const GAS void*)(myws + paddr()), (LAS void*)pl, 4, 0, AUX);
            pend = true;
        }
    };
    auto stage = [&](int c, int buf) {
        const char* g = gsrc + (size_t)((unsigned)c * gstep);
        LAS char* l = ring + buf + (lw - 1) * 8192;
        GLDS(g, l, 0); GLDS(g, l, 1024); GLDS(g, l, 2048); GLDS(g, l, 3072);
        GLDS(g + 4096, l + 4096, 0); GLDS(g + 4096, l + 4096, 1024); GLDS(g + 4096, l + 4096, 2048); GLDS(g + 4096, l + 4096, 3072);
    };

    const int pk0 = PK_O + ((cg < 8 ? 0 : cg - 8) >> 1) * NO + o0;
    if (wid == 0) pl[64] = pk_half(myws, pk0, call);
    f4 acc[2][4];
#pragma unroll
    for (int m = 0; m < 2; ++m)
#pragma unroll
        for (int n = 0; n < 4; ++n) acc[m][n] = (f4){0.f, 0.f, 0.f, 0.f};
    const int aoff = wm * 4096 + lane * 16, boff = 8192 + wn * 8192 + lane * 16;
    auto phase = [&](auto waitn, auto role, int buf, int lc) {
        h8 a[4], b[8];
#pragma unroll
        for (int n = 0; n < 8; ++n) b[n] = *(const LAS h8*)(ring + buf + boff + n * 1024);
#pragma unroll
        for (int m = 0; m < 4; ++m) a[m] = *(const LAS h8*)(ring + buf + aoff + m * 1024);
        __builtin_amdgcn_sched_barrier(0);
        if (decltype(role)::value == 2 && wid == 0) {
            ensure(lc); TREC(lc);
        }
        WAIT_L0();
        BAR();
        if (decltype(role)::value == 1) stage(lc, buf);
#pragma unroll
        for (int k = 0; k < 2; ++k)
#pragma unroll
            for (int m = 0; m < 2; ++m)
#pragma unroll
                for (int n = 0; n < 4; ++n) acc[m][n] = __builtin_amdgcn_mfma_f32_16x16x32_f16(a[m * 2 + k], b[n * 2 + k], acc[m][n], 0, 0, 0);
        if (decltype(role)::value == 1) {
#pragma unroll
            for (int i = 0; i < 8; ++i) { __builtin_amdgcn_sched_group_barrier(0x010, 1, 0); __builtin_amdgcn_sched_group_barrier(0x008, 2, 0); }
        }
        __builtin_amdgcn_sched_barrier(0);
        if (decltype(waitn)::value >= 0) { WAIT_V(decltype(waitn)::value < 0 ? 0 : decltype(waitn)::value); BAR(); }
    };

    if (wid == 0 && ready < NCH) {
        const unsigned long long t0 = __builtin_amdgcn_s_memrealtime();
        for (;;) {
            u4 f;
            asm volatile("global_load_dwordx4 %0, %1, %2 sc0 sc1\n\ts_waitcnt vmcnt(0)"
                         : "=&v"(f) : "v"((unsigned)OFF_FB + (unsigned)(cg * 1024 + lane * 16)), "s"(myws) : "memory");
            if (__builtin_amdgcn_ballot_w64(f[0] < call || f[1] < call || f[2] < call || f[3] < call) == 0 ||
                late(__builtin_amdgcn_readfirstlane(__hip_atomic_load(tmo, __ATOMIC_RELAXED, __HIP_MEMORY_SCOPE_AGENT)), t0)) break;
            __builtin_amdgcn_s_sleep(6);
        }
        ensure(2);
    }
    WAIT_L0();
    BAR();
    STAMP(4);
    const unsigned half = SMODE == 0 ? cg & 1 : (~__builtin_amdgcn_readfirstlane(pl[64]) ^ (SMODE == 2)) & 1u;
    if (lw != 0) { stage(0, 0); stage(1, STAGE); stage(2, 2 * STAGE); }
    WAIT_V(16);
    BAR();
    if (kp) BAR();
    if (lw != 0) for (int c = 3, buf = 0; c < NCH; ++c, buf = buf == 2 * STAGE ? 0 : buf + STAGE) phase(IC<16>{}, IC<1>{}, buf, c);
    else for (int c = 3, buf = 0; c < NCH; ++c, buf = buf == 2 * STAGE ? 0 : buf + STAGE) phase(IC<16>{}, IC<2>{}, buf, c);
    static_assert((NCH - 3) % 3 == 1, "tail buffers");
    const unsigned tm = __hip_atomic_load(tmo, __ATOMIC_RELAXED, __HIP_MEMORY_SCOPE_AGENT);
    phase(IC<8>{}, IC<0>{}, STAGE, -1);
    phase(IC<0>{}, IC<0>{}, 2 * STAGE, -1);
    phase(IC<-1>{}, IC<0>{}, 0, -1);
    f4 r[4];
#pragma unroll
    for (int n = 0; n < 4; ++n) {
        const f4 a0 = acc[0][n], a1 = acc[1][n];
        r[n] = kp ? a1 : a0;
        *(LAS f4*)(ring + lw * 4096 + n * 1024 + lane * 16) = kp ? a0 : a1;
    }
    const bool dead = __builtin_amdgcn_readfirstlane(tm) == call;
    WAIT_L0();
    BAR();
    if (!kp) BAR();
    STAMP(5);
#pragma unroll
    for (int n = 0; n < 4; ++n) r[n] += *(const LAS f4*)(lds + (kp ^ 1) * 24576 + lw * 4096 + n * 1024 + lane * 16);
    if (cg < 8) {
        const unsigned ovoff = (unsigned)(lane >> 4) * (4 * NLOC * 4) + (unsigned)(lane & 15) * 16;
        char* ob = (char*)P.out + (size_t)((panel_row(P.rank, panel, 2 * qt + wm) + kp * 16) * NLOC + cg * 128 + wn * 64) * 4;
#pragma unroll
        for (int k = 0; k < 4; ++k) { const u4 o = silu4(r, k, dead); GST4P(ovoff, o, ob + k * (NLOC * 4)); }
    } else {
        char* d0 = myws + (pk_off(pk0 + lw, call) + half * 8192 + kp * 4096);
#pragma unroll
        for (int k = 0; k < 4; ++k) { const u4 o = silu4(r, k, dead); GST4((unsigned)lane * 16, o, d0 + k * 1024, 0); }
        WAIT_V(0);
        BAR();
        if (wid == 0 && lane < 4) flag_store(oflag + lane * 4, call * 32u + half * 16u + M_RAW);
    }
    STAMP(6);
    TDUMP(1);
}

__global__ void __launch_bounds__(512, 2) ag_gemm_kernel(const Params Parg) {
    const CAS Params& P = *(const CAS Params*)__builtin_amdgcn_kernarg_segment_ptr();
    (void)Parg;
    __shared__ __attribute__((aligned(16))) char lds_raw[5 * 32768];
    LAS char* lds = (LAS char*)lds_raw;
    const int lane = threadIdx.x & 63, wid = __builtin_amdgcn_readfirstlane(threadIdx.x >> 6);
    const int xcd = blockIdx.x & 7, slot = blockIdx.x >> 3;
    char* myws = P.ws[P.rank];
    const int q = P.rank & 3;
    const unsigned long long t0 = __builtin_amdgcn_s_memrealtime();
    const unsigned side = (__builtin_amdgcn_s_getreg(20 | (3 << 11)) >> 2) & 1u;
    int g = (int)blockIdx.x - NGEMM;
    if (slot == NGEMM / 8) g = NPC + xcd;
    else if (g >= 0) {
        if (threadIdx.x == 0) {
            GAS unsigned* rt = (GAS unsigned*)(myws + OFF_ROLE);
            const unsigned t = __hip_atomic_fetch_add(rt, side ? 65536u : 1u, __ATOMIC_RELAXED, __HIP_MEMORY_SCOPE_AGENT);
            if ((t & 65535u) + (t >> 16) >= 247u - NGEMM) __hip_atomic_store(rt, 0u, __ATOMIC_RELAXED, __HIP_MEMORY_SCOPE_AGENT);
            *(LAS unsigned*)(lds + 8 * PBUF) = t;
        }
        __syncthreads();
        const unsigned t = __builtin_amdgcn_readfirstlane(*(const LAS unsigned*)(lds + 8 * PBUF));
        constexpr unsigned N0 = NPC / 7 * 3, NL = 247 - NGEMM;
        const unsigned r = (side ? NL - (t >> 16) : t & 65535u) % (NL + 1), b = NL - r;
        g = r < N0 ? r / 3 * 7 + 4 + r % 3 : r < N0 + NPROD - 8 ? NPC + 8 + r - N0 : b / 4 * 7 + b % 4;
        if (XMODE == 0) { g = (int)blockIdx.x - NGEMM; g = g < NPC + 8 ? g - 8 : g; }
    }
#undef VBLK
#define VBLK (NGEMM + g)
    if (PROBET >= 0 && wid == 7 && blockIdx.x < 2 * PK_PG) {
        char* a = myws + (OFF_PK + ((size_t)blockIdx.x << 21) + PROBEX * (blockIdx.x & 31) * 256);
        unsigned t[2] = {~0u, ~0u};
        while (__builtin_amdgcn_s_memrealtime() - t0 < PROBET) __builtin_amdgcn_s_sleep(4);
#pragma unroll
        for (int i = 0; i < 6; ++i) {
            const unsigned long long c0 = __builtin_amdgcn_s_memtime();
            if (PROBE1) asm volatile("s_waitcnt lgkmcnt(0)\n\ts_mov_b64 exec, 1\n\tglobal_store_dword %0, %0, %1 sc0 sc1\n\ts_mov_b64 exec, -1\n\ts_waitcnt vmcnt(0)" :: "v"((i & 1) * 8192), "s"(a), "s"(c0) : "memory");
            else asm volatile("s_waitcnt lgkmcnt(0)\n\tglobal_store_dword %0, %0, %1 sc0 sc1\n\ts_waitcnt vmcnt(0)" :: "v"((i & 1) * 8192), "s"(a), "s"(c0) : "memory");
            const unsigned d = (unsigned)(__builtin_amdgcn_s_memtime() - c0);
            t[i & 1] = d < t[i & 1] ? d : t[i & 1];
        }
        if (lane == 0) flag_store(myws + OFF_MAP + blockIdx.x * MAPS, (t[1] < t[0]) ^ side);
        STAMP(10);
    }
    const int u0 = g < 0 ? wid * NGEMM + (int)blockIdx.x : UG + wid * NPROD + g - NPC;
    if (g < 0 || g >= NPC) {
        pace(t0, u0); fetch_input(P, lds + wid * PBUF, u0, lane);
        if ((TOUCH & 1) && u0 < TOUCHU) touch(P, myws, lds + wid * PBUF + 16384, u0, lane);
        make_lut(lds + (8 * PBUF + 16), true, lane);
    }
    GAS unsigned* seq = (GAS unsigned*)(myws + OFF_SEQ) + (blockIdx.x * 8 + wid) * 4;
    const unsigned call = __builtin_amdgcn_readfirstlane(__hip_atomic_load(seq, __ATOMIC_RELAXED, __HIP_MEMORY_SCOPE_AGENT)) + 1u;
    WAIT_V(0);
    if (lane == 0) __hip_atomic_store(seq, call, __ATOMIC_RELAXED, __HIP_MEMORY_SCOPE_AGENT);

    if (g < 0 || g >= NPC) {
        producer(P, myws, lds + wid * PBUF, lds + (8 * PBUF + 16), u0, t0, call, wid, lane);
        STAMP(7);
    }
    if (slot <= NGEMM / 8) {
        const int pi = slot >> 3, ps = xcd >> 1, h = xcd & 1, sown = 20 + ps;
        const int ci = pi == 3 ? (slot & 1) * 4 + ps : slot == sown ? ps : slot & 7;
        const int panel = pi < 2 ? 3 * ps + pi : pi == 3 ? 3 * q + 2 : slot == sown ? 16 : 12 + ps;
        if (g < 0) gemm_panel(P, lds, myws, panel, ci < 4 ? 4 * h + ci : 2 * ci + h,
                   pi == 3 ? h * 8 : pi == 2 ? O_C + (((ps - ci - 1) & 3) * 2 + h) * 8 : O_A0 + pi * 128 + xcd * 8, pi == 3 ? 16 : pi == 2 ? 48 : 64, call, wid, lane);
        const int k = slot == 26 ? 23 : slot >= 24 ? slot - 24 : slot == sown ? 2 : slot < 4 ? slot + 3 : slot < 8 ? slot + 7 : slot < 12 ? slot - 1
                    : slot < 20 ? slot + 3 : 24;
        if (k < 24) {
            const int kk = k < 12 ? k : k - 12, qt = kk / 3, si = kk - qt * 3;
            gemm_quarter(P, lds, myws, 3 * ((q + 1 + si) & 3) + 2, qt, k < 12 ? 8 + xcd : xcd, O_B + qt * 24 + (si * 2 + h) * 4, call, wid, lane);
        }
    } else if (g < NPC) {
        LAS unsigned* info = (LAS unsigned*)lds;
        LAS unsigned* prog = info + NSEQ;
        LAS unsigned short* tab = (LAS unsigned short*)(lds + 8192);
        const int j = g / 7, link = g - j * 7, ncu = (NPC + 6 - link) / 7;
        for (int i = threadIdx.x; i < NSEQ + 4; i += 512) info[i] = 0;
        if (threadIdx.x == 0) prog[4] = (unsigned)t0;
        for (int i = threadIdx.x; i < PK_W; i += 512)
            if (((i & 31) == 16 || (i & 31) >= 24) == (link < 4)) tab[pos_x(i & 31, i >> 5)] = (unsigned short)i;
        if (link < 4) tab[pos_w(threadIdx.x >> 2, threadIdx.x & 3)] = (unsigned short)(PK_W + threadIdx.x);
        if (wid == 0) make_lut(lds + 14336, false, lane);
        __syncthreads();
        const int peer = link < 4 ? ((P.rank & 4) ^ 4) + ((q + link) & 3) : (P.rank & 4) + ((q + link - 3) & 3);
        if (wid < WPC) {
            if (lane == 0) __hip_atomic_fetch_max(prog, (unsigned)(NPUSH + wid * ncu + j), __ATOMIC_RELAXED, __HIP_MEMORY_SCOPE_WORKGROUP);
            puller(g, P, myws, info, prog, tab, lds + 16384 + wid * 8192, lds + 14336, link, peer, NPUSH + wid * ncu + j, ncu * WPC,
                   (unsigned)(((link < 4 ? NWIN : NWINI) + ncu - 1 - j) / ncu), call, wid, lane);
        } else if (wid == 7) {
            flag_mirror(myws, P.ws[peer] + (OFF_FP + (call & 1) * FP_B + (link < 4 ? q : 4) * (NSEQ * 4)), link < 4 ? NSEQ : NI, info, prog, call, lane);
        }
        STAMP(3);
    }
}

}

extern "C" void kernel_launch(int rank, int nranks, ncclComm_t comm, hipStream_t stream,
                              void* const* d_in, const int* in_sizes, int n_in,
                              void* d_out, int out_size, const int* in_dtypes, int out_dtype,
                              void* const* d_out_all, void* const* d_ws_all, size_t ws_bytes) {
    (void)comm; (void)in_dtypes; (void)out_dtype; (void)d_out_all;
    if (nranks != 8 || (unsigned)rank >= 8u || n_in < 2 || ws_bytes < WS_NEED || out_size != KDIM * NLOC ||
        in_sizes[0] != 1024 * KDIM || in_sizes[1] != KDIM * NLOC) {
        fprintf(stderr, "ag_gemm: unexpected problem shape\n");
        return;
    }
    Params P;
    P.x = (const float*)d_in[0]; P.w = (const float*)d_in[1];
    P.out = (float*)d_out;
    P.rank = 0; P.pad = 0;
    for (int p = 0; p < 8; ++p) {
        const int r = (int)((RMAP >> (4 * p)) & 7u);
        P.ws[p] = (char*)d_ws_all[r];
        if (r == rank) P.rank = p;
    }
    hipLaunchKernelGGL(ag_gemm_kernel, dim3(256), dim3(512), 0, stream, P);
    HIP_CHECK(hipGetLastError());
}
```

```cpp
#include <hip/hip_runtime.h>
#include <rccl/rccl.h>
#include <cstdio>

#define HIP_CHECK(expr) do { hipError_t _e = (expr); if (_e != hipSuccess) { \
    fprintf(stderr, "HIP error %s at %s:%d\n", hipGetErrorString(_e), __FILE__, __LINE__); \
    return; } } while (0)

namespace {

typedef _Float16 h8 __attribute__((ext_vector_type(8)));
typedef float f4 __attribute__((ext_vector_type(4)));
typedef unsigned u4 __attribute__((ext_vector_type(4)));
typedef unsigned u2 __attribute__((ext_vector_type(2)));
#define LAS __attribute__((address_space(3)))
#define GAS __attribute__((address_space(1)))
#define CAS __attribute__((address_space(4)))

constexpr int KDIM = 8192, NLOC = 1024;
constexpr int NKT = KDIM / 64;
constexpr int NCH = NKT / 2;
constexpr int NI = 23 * 64;
constexpr int NCI = 17 * 64, NO = 480;
constexpr int NSEQ = NCI + NO;
constexpr int O_A0 = 32, O_A1 = 160, O_C = 288, O_B = 384;
constexpr size_t PANEL_B = 4u << 20;
constexpr size_t CG_B = 2u << 20;
constexpr size_t OFF_SEQ = 0;
constexpr size_t OFF_TMO = 64u << 10;
constexpr size_t OFF_CTR = OFF_TMO + 256;
constexpr size_t OFF_ROLE = OFF_TMO + 1280;
constexpr size_t OFF_MAP = OFF_TMO + 2048;
#define MAPS 256u
constexpr size_t OFF_FA = 128u << 10;
constexpr size_t OFF_FB = OFF_FA + 17 * 64 * 32;
constexpr size_t OFF_FP = 256u << 10;
constexpr size_t FP_B = 5 * NSEQ * 4;
constexpr size_t OFF_DBG = 640u << 10;
constexpr size_t OFF_DATA = 1u << 20;
constexpr size_t A_B = 21 * PANEL_B, B_B = 16 * CG_B;
constexpr size_t OFF_PK = OFF_DATA + A_B + B_B + (1u << 20);
constexpr int PK_W = 2048, PK_O = 4096, PK_N = PK_O + 4 * NO;
constexpr int PK_PG = (PK_N + 126) / 127;
constexpr size_t PK_B = (size_t)PK_PG << 21;
constexpr size_t WS_NEED = OFF_PK + 2 * PK_B;
static_assert(OFF_MAP + 2 * PK_PG * MAPS <= OFF_FA && OFF_FB + 16 * 64 * 16 <= OFF_FP && OFF_FP + 2 * FP_B <= OFF_DBG, "flag area");
constexpr unsigned long long DEADLINE = 30000000ull;
constexpr unsigned M_RAW = 9, M_DEAD = 10;

#define NOTMO
#define NOTMO1
#define NOTMO2
#define NPC 35
#define WPC 7
#define SKC 14
#define SKA 5
#define SQ1 4
#define SQ2 10
#define SQ3 17
#define PACE 16
#define U1ST 448
#define PACE1 PACE
#define PACE2 PACE1
#define PROBE1 1
#define LOOK 64
#define PCH 2
#define PT0 180
#define PDT 13
#define CHMAP 0x76543210u
#define PROBET 0
#define PROBEX 0
#define TOUCH 0
#define RAW0 40
#define RAW1 0
#define DEFER 1
#define PD 2
#define NWIN 10
#define NWINI NWIN
#define PRATE 135
#define PRATEI PRATE
#define LGAIN 64
#define LCAP 30
#define HWIN 280
#define HWIN0 0
#define GLINK 1
#define PSLACK 10
#define RELAT 8
#define UG 3000
#define NB2 200
#define AHEAD UG
#define QSLEEP 2
#define AUX 17
#define RMAP 0x75416320u
#define EARLY 0
#define ORAW 0
#define SMODE 1
#define XMODE 1
constexpr int NGEMM = 208;
constexpr int NPROD = 256 - NGEMM - NPC;

struct Params {
    const float* x;
    const float* w;
    float* out;
    char* ws[8];
    int rank;
    int pad;
};

#define VBLK blockIdx.x
#define STAMP(i) do { } while (0)

#define TREC(c) do { } while (0)
#define TDUMP(k) do { (void)tv; } while (0)
#define HST(i, v) do { } while (0)
#define NOW() __builtin_amdgcn_s_memrealtime()
#define GST4(voff, data, sbase, imm) \
    asm volatile("global_store_dwordx4 %0, %1, %2 offset:" #imm " sc0 sc1\n\ts_nop 1" :: "v"(voff), "v"(data), "s"(sbase) : "memory")
#define OSUF " sc0 sc1"
#define GST4P(voff, data, sbase) \
    asm volatile("global_store_dwordx4 %0, %1, %2" OSUF "\n\ts_nop 1" :: "v"(voff), "v"(data), "s"(sbase) : "memory")
#define WAIT_V(n) asm volatile("s_waitcnt vmcnt(%0)" :: "n"(n) : "memory")
#define WAIT_L0() asm volatile("s_waitcnt lgkmcnt(0)" ::: "memory")

__device__ __forceinline__ u4 pack8(float a0, float a1, float a2, float a3, float a4, float a5, float a6, float a7) {
    h8 h = {(_Float16)a0, (_Float16)a1, (_Float16)a2, (_Float16)a3, (_Float16)a4, (_Float16)a5, (_Float16)a6, (_Float16)a7};
    return __builtin_bit_cast(u4, h);
}
__device__ __forceinline__ void flag_store(char* p, unsigned v) {
    __hip_atomic_store((GAS unsigned*)p, v, __ATOMIC_RELAXED, __HIP_MEMORY_SCOPE_SYSTEM);
}
__device__ __forceinline__ void store8(const u4 (&h)[8], char* d0, unsigned half, unsigned vl16) {
    char* d1 = d0 + half;
    GST4(vl16, h[0], d0, 0); GST4(vl16, h[1], d0, 1024); GST4(vl16, h[2], d0, 2048); GST4(vl16, h[3], d0, 3072);
    GST4(vl16, h[4], d1, 0); GST4(vl16, h[5], d1, 1024); GST4(vl16, h[6], d1, 2048); GST4(vl16, h[7], d1, 3072);
}
__device__ __forceinline__ unsigned long long lane_mask(unsigned n) { return n >= 64u ? ~0ull : (1ull << n) - 1ull; }
__device__ __forceinline__ void store_list(char* d, unsigned l, unsigned mode, unsigned vl4) {
    asm volatile("s_mov_b64 exec, %3\n\tglobal_store_dword %0, %1, %2 sc0 sc1\n\ts_mov_b64 exec, -1"
                 :: "v"(vl4), "v"(l), "s"(d), "s"(lane_mask(mode * 16u)) : "memory");
}

__device__ __forceinline__ int clamp64(int v) { return v < 0 ? 0 : v > 64 ? 64 : v; }
__device__ __forceinline__ int pos_w(int kt, int g) { const int ch = kt >> 1; return (ch < PCH ? 8 : 9) * ch + 8 * clamp64(ch - SKC) + (kt & 1) * 4 + g; }
__device__ __forceinline__ int pos_x(int rb, int ch) {
    if (rb < 8) return 8 * ch + 8 * clamp64(ch - SKA) + rb;
    if (rb < 16) return 8 * ch + 8 * clamp64(ch + SKA + 1) + rb - 8;
    if (rb == 16) return ch < PCH ? 8 * PCH + ch : 9 * ch + 8 * clamp64(ch - SKC) + 8;
    if (rb >= 24) return 8 * ch + 9 * clamp64(ch + SKC + 1) + rb - 24;
    const int j = (rb - 16) >> 1, r = ch + (j == 0 ? 0 : j == 1 ? SQ1 : j == 2 ? SQ2 : SQ3);
    return 1024 + clamp64(r + (j > 0)) + 2 * (clamp64(r - SQ1 + (j > 1)) + clamp64(r - SQ2 + (j > 2)) + clamp64(r - SQ3)) + (j ? rb & 1 : 0);
}
static_assert(NPC % 7 == 0 && 2 * PK_PG <= NGEMM && UG >= 8 * NGEMM && UG <= 8 * (NGEMM + NB2) && UG + 8 * NPROD <= PK_O && 0 <= SQ1 && SQ1 <= SQ2 && SQ2 <= SQ3 && SKC < 64 && SKA < 64 && PCH <= SKA && PCH <= SKC && SKC >= 8, "skews");
constexpr int NPUSH = 8 * PCH;
__device__ __forceinline__ unsigned panel_off(int panel, unsigned call) {
    return (unsigned)OFF_DATA + (unsigned)((call & 1) && panel < 12 && panel % 3 == 0 ? 17 + panel / 3 : panel) * (unsigned)PANEL_B;
}
__device__ __forceinline__ void place_x(int panel, int rb, int ch, unsigned call, unsigned& off, unsigned& fl) {
    off = panel_off(panel, call) + (unsigned)(ch * 65536 + (rb & 7) * 4096);
    fl = (unsigned)OFF_FA + (unsigned)(panel * 2048 + ch * 32 + (rb & 7) * 4);
}
__device__ __forceinline__ void place_w(int cb, int kt, unsigned& off, unsigned& fl) {
    off = (unsigned)(OFF_DATA + A_B) + (unsigned)(cb >> 1) * (unsigned)CG_B + (unsigned)(kt * 16384 + (cb & 1) * 8192);
    fl = (unsigned)OFF_FB + (unsigned)((cb >> 1) * 1024 + kt * 8 + (cb & 1) * 4);
}
__device__ __forceinline__ int panel_row(int pub, int panel, int blk) {
    const int gs = pub & 4, s = panel / 3, k = panel - 3 * s;
    const int r = panel == 16 ? pub : panel >= 12 ? (gs ^ 4) + panel - 12 : (k == 2 && blk == 0 ? gs ^ 4 : gs) + s;
    return (int)((RMAP >> (4 * r)) & 7u) * 1024 + (panel >= 12 ? 768 : k * 256) + blk * 32;
}
__device__ __forceinline__ size_t pk_off(int pk, unsigned call) { return OFF_PK + (call & 1) * PK_B + (size_t)(pk + pk / 127 + 1) * 16384; }
__device__ __forceinline__ unsigned pk_half(char* myws, int pk, unsigned call) {
    return __hip_atomic_load((GAS unsigned*)(myws + OFF_MAP) + ((call & 1) * PK_PG + pk / 127) * (MAPS / 4), __ATOMIC_RELAXED, __HIP_MEMORY_SCOPE_SYSTEM);
}

#define PERM(hi, lo, sel) __builtin_amdgcn_perm(hi, lo, sel)
#define ABIT(hi, lo, s) __builtin_amdgcn_alignbit(hi, lo, s)
#define HW(k) h[(k) >> 2][(k) & 3]
#define FENCE() asm volatile("" ::: "memory")
__device__ __forceinline__ unsigned bfi(unsigned m, unsigned x, unsigned y) { return (x & m) | (y & ~m); }
__device__ __forceinline__ unsigned wave_scan(unsigned v) {
    v += __builtin_amdgcn_update_dpp(0u, v, 0x111, 0xF, 0xF, true);
    v += __builtin_amdgcn_update_dpp(0u, v, 0x112, 0xF, 0xF, true);
    v += __builtin_amdgcn_update_dpp(0u, v, 0x114, 0xF, 0xF, true);
    v += __builtin_amdgcn_update_dpp(0u, v, 0x118, 0xF, 0xF, true);
    v += __builtin_amdgcn_update_dpp(0u, v, 0x142, 0xA, 0xF, false);
    v += __builtin_amdgcn_update_dpp(0u, v, 0x143, 0xC, 0xF, false);
    return v;
}
__device__ __forceinline__ int wave_sum(unsigned v) { return __builtin_amdgcn_readlane(wave_scan(v), 63); }
__device__ __forceinline__ unsigned escapes(unsigned c, unsigned acc) { return (unsigned)__builtin_popcount(c & (c >> 1) & 0x55555555u) + acc; }
__device__ __forceinline__ void make_lut(LAS char* lut, bool pack, int lane) {
#pragma unroll
    for (int j = 0; j < 4; ++j) {
        unsigned sel = pack ? 0x0C0C0C0Cu : 0u, k = 0;
#pragma unroll
        for (int i = 0; i < 4; ++i) {
            const unsigned c = i < 3 ? ((unsigned)lane >> (2 * i)) & 3u : (unsigned)j;
            if (pack) sel ^= c == 3u ? (0x0Cu ^ (unsigned)i) << (8 * k) : 0u; else sel |= (c == 3u ? k : 4u + c) << (8 * i);
            k += c == 3u;
        }
        *(LAS u2*)(lut + (j * 64 + lane) * 8) = (u2){sel, pack ? 8u * k : k};
    }
}
__device__ __forceinline__ unsigned code4(unsigned thi, unsigned tlo, unsigned s) { return __builtin_amdgcn_udot4(PERM(thi, tlo, s), 0x80200802u, 0u - 0xF3u * 170u, false); }
template <int N> __device__ __forceinline__ void append(const unsigned* s, const u2* e, unsigned pos) {
#pragma unroll
    for (int g = 0; g < N; ++g) {
        const unsigned long long c = (unsigned long long)PERM(0u, s[g], e[g][0]) << (pos & 31u);
        asm volatile("v_cmpx_ne_u32_e32 vcc, 0, %1\n\tds_or_b32 %0, %1\n\tv_cmpx_ne_u32_e32 vcc, 0, %2\n\tds_or_b32 %0, %2 offset:4\n\ts_mov_b64 exec, -1"
                     :: "v"((pos >> 3) & ~3u), "v"((unsigned)c), "v"((unsigned)(c >> 32)) : "vcc", "memory");
        pos += e[g][1];
    }
}
constexpr int SCR_B = 3328;
constexpr int SCR_3 = 2064, SCR_T = 2592, PBUF = 16384 + SCR_B;
__device__ __forceinline__ unsigned pack_unit(u4 (&h)[8], LAS char* scr, const LAS char* lut, int lane, unsigned& r0, unsigned& r2) {
    const u4 z = {0u, 0u, 0u, 0u};
    *(LAS u4*)(scr + lane * 16) = z; *(LAS u4*)(scr + 1024 + lane * 16) = z; *(LAS u4*)(scr + 2048 + lane * 16) = z;
    FENCE();
    unsigned G[16], s[16], a[16], cw[4];
#pragma unroll
    for (int g = 0; g < 16; ++g) { G[g] = PERM(HW(2 * g + 1), HW(2 * g), 0x07050301u); s[g] = (G[g] >> 1) & 0x3E3E3E3Eu; }
    const int m = wave_sum(__builtin_amdgcn_udot4(s[0] + s[5] + s[10] + s[15], 0x01010101u, 0u, false)) + 205;
    const unsigned b3 = ((unsigned)(m >> 11) + 3u) & 31u, type = (m & 2047) > 1120;
    u2 e[16];
#pragma unroll
    for (int g = 0; g < 16; ++g) {
        s[g] = ((2u * b3 + 65u) * 0x01010101u - s[g]) & 0x3F3F3F3Fu;
        a[g] = code4(0xF300F700u, 0xFB00FF00u, s[g]);
        e[g] = *(const LAS u2*)(lut + a[g]);
    }
    unsigned n = 0;
#pragma unroll
    for (int j = 0; j < 4; ++j) { cw[j] = (a[4 * j] >> 3) | (a[4 * j + 1] << 5) | (a[4 * j + 2] << 13) | (a[4 * j + 3] << 21); n = escapes(cw[j], n); }
    unsigned sc = wave_scan(n);
    r2 = (__builtin_amdgcn_readlane(sc, 63) + 31u) >> 5;
    if (r2 > 62u) return M_RAW;
    append<16>(s, e, (sc - n + (unsigned)(size_t)scr) * 8u);
    FENCE();
    const u4 q0 = *(const LAS u4*)(scr + lane * 32), q1 = *(const LAS u4*)(scr + lane * 32 + 16);
#pragma unroll
    for (int g = 0; g < 8; ++g) {
        const unsigned x = g < 4 ? q0[g & 3] : q1[g & 3];
        s[g] = (x | ~PERM(0xFF000000u, 0xFF00F9F6u, x)) - 0x06060606u;
        a[g] = code4(type ? 0xFF00F700u : 0xFB00F700u, type ? 0xF300FB00u : 0xF300FF00u, s[g]);
        e[g] = *(const LAS u2*)(lut + a[g]);
    }
    const unsigned c0 = (a[0] >> 3) | (a[1] << 5) | (a[2] << 13) | (a[3] << 21), c1 = (a[4] >> 3) | (a[5] << 5) | (a[6] << 13) | (a[7] << 21);
    n = escapes(c1, escapes(c0, 0u));
    sc = wave_scan(n);
    const unsigned r3 = (__builtin_amdgcn_readlane(sc, 63) + 7u) >> 3;
    if (r3 > 64u) return M_RAW;
    append<8>(s, e, (sc - n + (unsigned)(size_t)(scr + SCR_3)) * 8u);
    FENCE();
    u2 y = *(const LAS u2*)(scr + SCR_3 + lane * 8);
#pragma unroll
    for (int j = 0; j < 2; ++j) y[j] = (y[j] | ~PERM(0xFF000000u, 0xFF00F9F8u, y[j])) - 0x07070707u;
    unsigned w = ((y[0] >> 1) & 0x07070707u) | ((y[1] << 2) & 0x38383838u);
    w |= ((w >> 24) * 0x41040u) & 0xC0C0C0u;
    LAS char* t = scr + SCR_T;
    t[4 + lane * 3] = (char)w; t[5 + lane * 3] = (char)(w >> 8); t[6 + lane * 3] = (char)(w >> 16);
    FENCE();
    const unsigned l0 = (3u * r3 + 5u) >> 1;
    unsigned bad = ((y[0] >> 4) & 0x03030303u) | (y[1] & 0x30303030u), nl = 0;
    for (;;) {
        const unsigned long long mk = __builtin_amdgcn_ballot_w64(bad != 0);
        if (mk == 0 || nl > 64u) break;
        if (bad != 0) {
            const unsigned p = (unsigned)__builtin_ctz(bad), k = l0 + nl + __builtin_amdgcn_mbcnt_hi((unsigned)(mk >> 32), __builtin_amdgcn_mbcnt_lo((unsigned)mk, 0u));
            ((LAS unsigned short*)t)[k] = (unsigned short)((((unsigned)lane * 8u + (p & 4u) + (p >> 3)) << 7) | ((4u * b3 - 24u - 2u * (((p & 4u) ? y[1] : y[0]) >> (p & 24u))) & 0x7Cu));
            bad &= ~(15u << (p & 28u));
        }
        nl += (unsigned)__builtin_popcountll(mk);
    }
    r0 = (l0 + nl + 3u) >> 2;
    if (nl > 64u || r0 + r2 > 64u) return M_RAW;
    FENCE();
    u2 f = *(const LAS u2*)(t + lane * 8);
    FENCE();
    if (lane == 0) f[0] = b3 | (type << 5) | (r0 << 8) | (nl << 16) | (l0 << 24);
    unsigned L[16];
#pragma unroll
    for (int g = 0; g < 16; ++g) L[g] = PERM(HW(2 * g + 1), HW(2 * g), 0x06040200u);
#pragma unroll
    for (int g = 0; g < 16; ++g) HW(g) = L[g];
#pragma unroll
    for (int j = 0; j < 6; ++j) {
        const unsigned x = bfi(0x83838383u, G[2 * j], ABIT(G[2 * j + 1], G[2 * j + 1], 29));
        HW(16 + j) = j < 4 ? bfi(0x60606060u, G[12 + j] << 5, x) : bfi(0x20202020u, G[2 * j + 4] >> 2, bfi(0x40404040u, G[2 * j + 5] >> 1, x));
    }
    h[5][2] = cw[0]; h[5][3] = cw[1]; h[6] = (u4){cw[2], cw[3], f[0], f[1]}; h[7][0] = c0; h[7][1] = c1;
    return (r0 + r2 + 7u) >> 3;
}
template <int N> __device__ __forceinline__ void deposit(unsigned* o, const u2* e, unsigned tab, unsigned pos) {
    u2 x[N]; unsigned p[N];
#pragma unroll
    for (int g = 0; g < N; ++g) {
        const LAS unsigned* d = (const LAS unsigned*)(size_t)(pos & ~3u);
        x[g] = (u2){d[0], d[1]}; p[g] = pos;
        pos += e[g][1];
    }
#pragma unroll
    for (int g = 0; g < N; ++g) o[g] = PERM(tab, __builtin_amdgcn_alignbyte(x[g][1], x[g][0], p[g]), e[g][0]);
}
__device__ __forceinline__ void unpack_unit(u4 (&h)[8], LAS char* scr, const LAS char* lut, int lane) {
    LAS char* t = scr + SCR_T;
    *(LAS u2*)(t + lane * 8) = (u2){h[6][2], h[6][3]};
    FENCE();
    const unsigned hd = __builtin_amdgcn_readfirstlane(h[6][2]), r0 = (hd >> 8) & 63u, type = hd & 32u;
    const unsigned kb = (((hd & 31u) << 2) | 0x80u) * 0x01010101u;
    const LAS unsigned* d3 = (const LAS unsigned*)(t + ((4 + lane * 3) & ~3));
    const unsigned w0 = d3[0], w1 = d3[1];
    unsigned le = 0;
    const bool ll = (unsigned)lane < ((hd >> 16) & 127u);
    if (ll) le = ((const LAS unsigned short*)t)[(hd >> 24) + lane];
    const unsigned c0 = (unsigned)lane < r0 ? 0u : h[6][2], c1 = (unsigned)lane < r0 ? 0u : h[6][3];
    const unsigned cw[4] = {h[5][2], h[5][3], h[6][0], h[6][1]};
    u2 e2[8], e1[16];
#pragma unroll
    for (int g = 0; g < 8; ++g) e2[g] = *(const LAS u2*)(lut + (((g < 4 ? c0 : c1) >> (8 * (g & 3))) & 255u) * 8u);
#pragma unroll
    for (int g = 0; g < 16; ++g) e1[g] = *(const LAS u2*)(lut + ((cw[g >> 2] >> (8 * (g & 3))) & 255u) * 8u);
    const unsigned n2 = escapes(c1, escapes(c0, 0u)), n1 = escapes(cw[3], escapes(cw[2], escapes(cw[1], escapes(cw[0], 0u))));
    const unsigned n = n1 | (n2 << 16), sc = wave_scan(n) - n;
    unsigned w = __builtin_amdgcn_alignbyte(w1, w0, (unsigned)lane * 3u);
    w = bfi(0x003F3F3Fu, w, __builtin_amdgcn_udot4(w & 0xC0C0C0u, 0x100401u, 0u, false) << 18);
    const unsigned t3l = (kb - (type ? 0x24201C18u : 0x24201C00u)) & 0x7C7C7C7Cu, t3h = (kb - 0x34302C28u) & 0x7C7C7C7Cu;
    *(LAS u2*)(scr + SCR_3 + lane * 8) = (u2){PERM(t3h, t3l, w & 0x07070707u), PERM(t3h, t3l, (w >> 3) & 0x07070707u)};
    FENCE();
    if (ll) scr[SCR_3 + (le >> 7)] = (char)(le & 0x7Cu);
    FENCE();
    unsigned E[16];
    deposit<8>(E, e2, (kb - (type ? 0x001410u : 0x181410u)) & 0x7C7C7C7Cu, (sc >> 16) + (unsigned)(size_t)(scr + SCR_3));
    FENCE();
    *(LAS u4*)(scr + lane * 32) = (u4){E[0], E[1], E[2], E[3]}; *(LAS u4*)(scr + lane * 32 + 16) = (u4){E[4], E[5], E[6], E[7]};
    FENCE();
    deposit<16>(E, e1, (kb - 0x04080Cu) & 0x7C7C7C7Cu, (sc & 0xFFFFu) + r0 * 32u + (unsigned)(size_t)scr);
    FENCE();
    unsigned L[16];
#pragma unroll
    for (int g = 0; g < 16; ++g) {
        L[g] = HW(g);
        const unsigned x = HW(16 + (g < 12 ? g >> 1 : g - 12));
        E[g] = bfi(0x7C7C7C7Cu, E[g], g >= 12 ? bfi(0x80808080u, HW(14 + (g >> 1)) << (2 - (g & 1)), x >> 5) : (g & 1) ? ABIT(x, x, 3) : x);
    }
#pragma unroll
    for (int g = 0; g < 16; ++g) { HW(2 * g) = PERM(E[g], L[g], 0x05010400u); HW(2 * g + 1) = PERM(E[g], L[g], 0x07030602u); }
}
constexpr unsigned LIST_MAX = 126;
__device__ __forceinline__ unsigned pack_out(u4 (&h)[8], LAS char* scr, int lane) {
    unsigned es = 0;
#pragma unroll
    for (int j = 0; j < 4; ++j) es += ((h[0][j] >> 23) & 255u) + ((h[4][j] >> 23) & 255u);
    int e0 = (wave_sum(es) - 2355 + 511) >> 9;
    e0 = e0 < 0 ? 0 : e0 > 248 ? 248 : e0;
    const unsigned base = (unsigned)e0 << 24;
    unsigned n = 0;
#pragma unroll
    for (int i = 0; i < 8; ++i) {
#pragma unroll
        for (int j = 0; j < 4; ++j) h[i][j] = ABIT(h[i][j], h[i][j], 31) - base;
        unsigned t = __builtin_amdgcn_perm(__builtin_amdgcn_perm(h[i][3], h[i][2], 0x0C0C0703u), __builtin_amdgcn_perm(h[i][1], h[i][0], 0x0C0C0703u), 0x05040100u) & 0xF8F8F8F8u;
        for (;;) {
            const unsigned long long m = __builtin_amdgcn_ballot_w64(t != 0);
            if (m == 0 || n > LIST_MAX) break;
            if (t != 0) {
                const unsigned b = (unsigned)__builtin_ctz(t) & 24u;
                const unsigned k = n + 2u + __builtin_amdgcn_mbcnt_hi((unsigned)(m >> 32), __builtin_amdgcn_mbcnt_lo((unsigned)m, 0u));
                ((LAS unsigned short*)scr)[k] = (unsigned short)(((unsigned)((i >> 2) * 1024 + lane * 16 + (i & 3) * 4) << 5) + (b << 2) + ((t >> (b + 3u)) & 31u));
                t &= ~(255u << b);
            }
            n += (unsigned)__builtin_popcountll(m);
        }
    }
    if (n > LIST_MAX) {
#pragma unroll
        for (int k = 0; k < 32; ++k) { const unsigned x = HW(k) + base; HW(k) = ABIT(x, x, 1); }
        return M_RAW;
    }
    asm volatile("" ::: "memory");
    unsigned l = *(const LAS unsigned*)(scr + lane * 4);
    asm volatile("" ::: "memory");
    if (lane == 0) l = ((unsigned)e0 << 16) | n;
    unsigned lo = 0;
#pragma unroll
    for (int p = 0; p < 5; ++p) {
        lo = ABIT(HW(27 + p), lo, 2);
#pragma unroll
        for (int j = 0; j < 5; ++j) HW(5 * p + j) = ABIT(HW(5 * p + j), HW(27 + p) << (25 - 5 * j), 27);
    }
    HW(25) = ABIT(HW(25), lo << 5, 27); HW(26) = ABIT(HW(26), lo, 27);
    h[6][3] = l;
    return (n + 33u) >> 5;
}
__device__ __forceinline__ void unpack_out(u4 (&h)[8], unsigned lst, LAS char* scr, int lane) {
    const unsigned hd = __builtin_amdgcn_readfirstlane(lst);
    const unsigned n = hd & 0xFFFFu, base = (hd >> 16) << 24;
    *(LAS u4*)(scr + lane * 16) = (u4){0u, 0u, 0u, 0u};
    *(LAS u4*)(scr + 1024 + lane * 16) = (u4){0u, 0u, 0u, 0u};
    asm volatile("" ::: "memory");
    if ((unsigned)(2 * lane - 2) < n) scr[(lst >> 5) & 2047u] = (char)(lst & 31u);
    if ((unsigned)(2 * lane - 1) < n) scr[lst >> 21] = (char)((lst >> 16) & 31u);
    asm volatile("" ::: "memory");
    const u4 s0 = *(const LAS u4*)(scr + lane * 16), s1 = *(const LAS u4*)(scr + 1024 + lane * 16);
    asm volatile("" ::: "memory");
    const unsigned lo = ABIT(HW(26), HW(25) << 27, 5);
#pragma unroll
    for (int p = 0; p < 5; ++p) {
        unsigned y = lo << (8 - 2 * p);
#pragma unroll
        for (int j = 0; j < 5; ++j) y = ABIT(HW(5 * p + j), y, 5);
        HW(27 + p) = y;
    }
#pragma unroll
    for (int k = 0; k < 32; ++k) {
        const unsigned x = ABIT((k < 16 ? s0 : s1)[(k >> 2) & 3] >> (8 * (k & 3)), HW(k), 5) + base;
        HW(k) = ABIT(x, x, 1);
    }
}

__device__ __forceinline__ void unit_decode(int u, int& kind, int& ch, int& g) {
    constexpr int U0 = 56 * SKC, U1 = U0 + 64 * (64 - SKC);
    if (u < U0) { const int s = u / 7; kind = u - 7 * s; ch = s >> 3; g = s & 7; if (ch < 8 && kind < 5) ch = (CHMAP >> (4 * ch)) & 7; }
    else if (u < U1) { const int v = u - U0; kind = v & 7; g = (v >> 3) & 7; ch = (v >> 6) + (kind == 7 ? 0 : SKC); }
    else { const int v = u - U1; kind = 7; g = v & 7; ch = 64 - SKC + (v >> 3); }
}
__device__ __forceinline__ void fetch_input(const CAS Params& P, LAS char* buf, int u, int lane) {
    int kind, ch, g;
    unit_decode(u, kind, ch, g);
    const bool isx = kind == 0 || kind > 4;
    const char* src = isx ? (const char*)P.x + (size_t)((unsigned)((kind ? kind - 4 : 0) * 8 + g) * (32 * KDIM * 4) + (unsigned)ch * 512)
                          : (const char*)P.w + (size_t)((unsigned)(2 * ch + (g >> 2)) * (64 * NLOC * 4) + (unsigned)((kind - 1) * 4 + (g & 3)) * 256);
    const unsigned s0 = isx ? 64 : NLOC * 4, s1 = isx ? 16 * KDIM * 4 : 16 * NLOC * 4, s2 = isx ? 256 : 32 * NLOC * 4;
    const unsigned voff = isx ? (unsigned)(lane & 15) * (KDIM * 4) + (unsigned)(lane >> 4) * 16 : (unsigned)(lane >> 4) * (4 * NLOC * 4) + (unsigned)(lane & 15) * 16;
#pragma unroll
    for (int i = 0; i < 16; ++i)
        __builtin_amdgcn_global_load_lds((const GAS void*)(src + ((i & 3) * s0 + ((i >> 2) & 1) * s1 + (i >> 3) * s2) + voff), (LAS void*)(buf + i * 1024), 16, 0, 0);
}
#define TOUCHU 448
__device__ __forceinline__ void touch(const CAS Params& P, char* myws, LAS char* scr, int u, int lane) {
    int ch, g, kind;
    unit_decode(u, kind, ch, g);
    const bool isx = kind == 0 || kind > 4;
    const int rb = (kind ? kind - 4 : 0) * 8 + g, kt = 2 * ch + (g >> 2), cb = (kind - 1) * 4 + (g & 3);
    const int pk = isx ? ch * 32 + rb : PK_W + (kind - 1) * 512 + ch * 8 + g;
    unsigned off, fl;
    if (isx) place_x(rb < 24 ? 3 * (P.rank & 3) + (rb >> 3) : 16, rb, ch, lane & 1, off, fl); else place_w(cb, kt, off, fl);
    const unsigned a = lane & 2 ? off : (unsigned)pk_off(pk, lane & 1);
    if (lane < 4) __builtin_amdgcn_global_load_lds((const GAS void*)(myws + a), (LAS void*)scr, 4, 0, 17);
}
__device__ __forceinline__ void pace(unsigned long long t0, int u) {
    if (u < UG) while (__builtin_amdgcn_s_memrealtime() - t0 < (unsigned)(u < U1ST ? u * PACE1 : (u - U1ST) * PACE + U1ST * PACE2) >> 4) __builtin_amdgcn_s_sleep(2);
}
__device__ __forceinline__ int next_unit(int u, int wid) {
    if (u >= UG) return u + 8 * NPROD;
    u = u < 8 * NGEMM && blockIdx.x < NB2 ? 8 * NGEMM + wid * NB2 + (int)blockIdx.x : UG;
    return u < UG ? u : PK_O;
}
__device__ __forceinline__ void producer(const CAS Params& P, char* myws, LAS char* buf, LAS char* lut, int u, unsigned long long t0, unsigned call, int wid, int lane) {
    const int q = P.rank & 3;
    const unsigned vl16 = (unsigned)lane * 16;
    LAS char* scr = buf + 16384;
    for (int un; u < PK_O; u = un) {
        un = next_unit(u, wid);
        int ch, g, kind;
        unit_decode(u, kind, ch, g);
        const bool isx = kind == 0 || kind > 4;
        const int rb = (kind ? kind - 4 : 0) * 8 + g, kt = 2 * ch + (g >> 2), cb = (kind - 1) * 4 + (g & 3);
        const int idx = isx ? pos_x(rb, ch) : pos_w(kt, g & 3), pk = isx ? ch * 32 + rb : PK_W + (kind - 1) * 512 + ch * 8 + g;
        const int rd = !isx ? kind - 1 : rb == 16 || rb >= 24 ? 5 : 4;
        f4 v[16]; u4 h[8];
        unsigned hv;
        asm volatile("global_load_dword %0, %1, %2 sc0 sc1" : "=&v"(hv) : "v"((unsigned)OFF_MAP + ((call & 1) * PK_PG + pk / 127) * MAPS), "s"(myws) : "memory");
#pragma unroll
        for (int i = 0; i < 16; ++i) v[i] = *(const LAS f4*)(buf + i * 1024 + lane * 16);
        __builtin_amdgcn_sched_barrier(0);
        WAIT_L0();
        const bool ahead = u >= AHEAD && un < PK_O;
        if (ahead) { pace(t0, un); fetch_input(P, buf, un, lane); }
        __builtin_amdgcn_sched_barrier(0);
        if (isx) {
#pragma unroll
            for (int i = 0; i < 8; ++i)
                h[i] = pack8(v[2 * i][0], v[2 * i][1], v[2 * i][2], v[2 * i][3], v[2 * i + 1][0], v[2 * i + 1][1], v[2 * i + 1][2], v[2 * i + 1][3]);
        } else {
#pragma unroll
            for (int i = 0; i < 8; ++i) {
                const int f = i >> 1, k = (i & 1) * 8;
                h[i] = pack8(v[k][f], v[k + 1][f], v[k + 2][f], v[k + 3][f], v[k + 4][f], v[k + 5][f], v[k + 6][f], v[k + 7][f]);
            }
        }
        unsigned off, fl;
        if (isx) place_x(rb < 24 ? 3 * q + (rb >> 3) : 16, rb, ch, call, off, fl); else place_w(cb, kt, off, fl);
        const bool own = !isx || rb != 16;
        const unsigned long long h11 = NOW();
        if (idx < NPUSH) {
            unsigned off2 = off, fl2 = fl;
            if (!isx) place_w(16 + 4 * q + (g & 3), kt, off2, fl2);
            while (__builtin_amdgcn_s_memrealtime() - t0 < (unsigned)(PT0 + idx * PDT)) __builtin_amdgcn_s_sleep(1);
            char* fp = myws + fl;
            const unsigned long long h14 = NOW();
            for (int i = 1; i < (isx ? 4 : 2); ++i) {
                char* pw = P.ws[isx ? (P.rank & 4) + ((q + i) & 3) : ((P.rank & 4) ^ 4) + rd];
                store8(h, pw + off2, isx ? 32768u : 4096u, vl16);
                fp = lane == i ? pw + fl2 : fp;
            }
            store8(h, myws + off, isx ? 32768u : 4096u, vl16);
            asm volatile("s_waitcnt vmcnt(0)" : "+v"(hv) :: "memory");
            if (lane < (isx ? 4 : 2)) flag_store(fp, call);
            HST(12, NOW()); HST(11, h11); HST(14, h14); HST(13, (unsigned long long)(rd * 4096 + idx + 1));
            if (un < PK_O) { pace(t0, un); fetch_input(P, buf, un, lane); WAIT_V(0); }
            continue;
        }
        if (own) store8(h, myws + off, isx ? 32768u : 4096u, vl16);
        unsigned mode = M_RAW, r0 = 0, r2 = 0;
        if (idx >= RAW0 && (rd != 4 || idx < NI - RAW1)) mode = pack_unit(h, scr, lut, lane, r0, r2);
        const unsigned long long h14 = NOW();
        if (!ahead) { if (own) asm volatile("s_waitcnt vmcnt(8)" : "+v"(hv) :: "memory"); else asm volatile("s_waitcnt vmcnt(0)" : "+v"(hv) :: "memory"); }
        else if (own) asm volatile("s_waitcnt vmcnt(24)" : "+v"(hv) :: "memory"); else asm volatile("s_waitcnt vmcnt(16)" : "+v"(hv) :: "memory");
        const unsigned half = SMODE == 0 ? idx & 1 : (__builtin_amdgcn_readfirstlane(hv) ^ (rd != 4) ^ (SMODE == 2)) & 1u;
        char* d0 = myws + (pk_off(pk, call) + half * 8192); char* d1 = d0 + 4096;
        GST4(vl16, h[0], d0, 0); GST4(vl16, h[1], d0, 1024); GST4(vl16, h[2], d0, 2048); GST4(vl16, h[3], d0, 3072);
        GST4(vl16, h[4], d1, 0); GST4(vl16, h[5], d1, 1024);
        if (mode < M_RAW) {
            const u2 p0 = {h[6][0], h[6][1]}, p1 = {h[6][2], h[6][3]}, p2 = {h[7][0], h[7][1]};
            asm volatile("global_store_dwordx2 %0, %1, %4 offset:2048 sc0 sc1\n\t"
                         "s_mov_b64 exec, %6\n\tglobal_store_dwordx2 %0, %2, %4 offset:2560 sc0 sc1\n\t"
                         "s_mov_b64 exec, %7\n\tglobal_store_dwordx2 %0, %3, %5 offset:2560 sc0 sc1\n\ts_mov_b64 exec, -1"
                         :: "v"(vl16 >> 1), "v"(p0), "v"(p1), "v"(p2), "s"(d1), "s"(d1 + r0 * 8u), "s"(lane_mask(r0)), "s"(lane_mask(r2)) : "memory");
        } else { GST4(vl16, h[6], d1, 2048); GST4(vl16, h[7], d1, 3072); }
        WAIT_V(0);
        const unsigned fo = lane < 5 ? (unsigned)OFF_FP + (call & 1) * (unsigned)FP_B + (unsigned)(lane * NSEQ + idx) * 4u : fl;
        if ((rd == 5 ? lane < 4 : lane == rd) || (own && lane == 5)) flag_store(myws + fo, lane < 5 ? call * 32u + half * 16u + mode : call);
        HST(12, NOW()); HST(11, h11); HST(14, h14); HST(13, (unsigned long long)(rd * 4096 + idx + 1));
        if (!ahead && un < PK_O) { pace(t0, un); fetch_input(P, buf, un, lane); WAIT_V(0); }
    }
}

__device__ __forceinline__ void flag_mirror(char* myws, const char* fp, int nseq, LAS unsigned* info, LAS unsigned* prog, unsigned call, int lane) {
    GAS unsigned* tmo = (GAS unsigned*)(myws + OFF_TMO);
    int pos = NPUSH, nl = 16;
    unsigned long long t0 = __builtin_amdgcn_s_memrealtime();
    while (pos < nseq) {
        const int pr = (int)__builtin_amdgcn_readfirstlane(__hip_atomic_load(prog, __ATOMIC_RELAXED, __HIP_MEMORY_SCOPE_WORKGROUP));
        if (pos <= pr + LOOK) {
            const int i = pos + 4 * lane;
            bool bad = false;
            if (lane < nl && i < nseq) {
                u4 f;
                asm volatile("global_load_dwordx4 %0, %1, %2 sc0 sc1\n\ts_waitcnt vmcnt(0)" : "=&v"(f) : "v"(i * 4), "s"(fp) : "memory");
                *(LAS u4*)(info + i) = f;
                bad = f[0] < call * 32u || f[1] < call * 32u || f[2] < call * 32u || f[3] < call * 32u;
            }
            const unsigned long long m = __builtin_amdgcn_ballot_w64(bad);
            const int n = m ? (__builtin_ctzll(m) & ~3) : nl;
            nl = m ? 16 : 64;
            if (n) { pos += 4 * n; t0 = __builtin_amdgcn_s_memrealtime(); continue; }
        }
        if (
            __builtin_amdgcn_s_memrealtime() - t0 > DEADLINE) {
            __hip_atomic_store(tmo, call, __ATOMIC_RELAXED, __HIP_MEMORY_SCOPE_AGENT);
            asm volatile("s_waitcnt vmcnt(0)" ::: "memory");
            for (int i = pos + lane; i < nseq; i += 64) info[i] = call * 32u + M_DEAD;
            return;
        }
        if (pos > pr + 24) __builtin_amdgcn_s_sleep(24);
    }
}

#define STR2(x) #x
#define STR(x) STR2(x)
#define SP ""
#define RD ""
#define PULL6 "global_load_dwordx4 %[h0], %[v16], %[src] offset:-4096 sc0 sc1\n\t" SP "global_load_dwordx4 %[h1], %[v16], %[src] offset:-3072 sc0 sc1\n\t" SP \
              "global_load_dwordx4 %[h2], %[v16], %[src] offset:-2048 sc0 sc1\n\t" SP "global_load_dwordx4 %[h3], %[v16], %[src] offset:-1024 sc0 sc1\n\t" SP \
              "global_load_dwordx4 %[h4], %[v16], %[src] sc0 sc1\n\t" SP "global_load_dwordx4 %[h5], %[v16], %[src] offset:1024 sc0 sc1\n\t" SP
#define PULLT "s_mov_b64 exec, 1\n\tglobal_atomic_add %[tk], %[ao], %[one], %[ws] sc0\n\t" \
              "s_waitcnt vmcnt(%[rn] + 9 - " STR(RELAT) ")\n\tglobal_store_dword %[pf], %[cl], %[ws] sc0 sc1\n\ts_waitcnt vmcnt(%[rn] + 1)\n\t" RD \
              "ds_add_u32 %[rel], %[one]\n\ts_mov_b64 exec, -1\n\ts_waitcnt vmcnt(1)"
#define PULLO [h0] "=&v"(h[0]), [h1] "=&v"(h[1]), [h2] "=&v"(h[2]), [h3] "=&v"(h[3]), [h4] "=&v"(h[4]), [h5] "=&v"(h[5]), [tk] "=&v"(tk)
#define PULLI [v16] "v"(vl16), [src] "s"(src), [ao] "v"(aoff), [one] "v"(one), [ws] "s"(myws), [rel] "v"(rel), [pf] "v"(pfl), [cl] "v"(call)
#undef VBLK
#define VBLK (NGEMM + vg)
__device__ __forceinline__ void puller(int vg, const CAS Params& P, char* myws, LAS unsigned* info, LAS unsigned* prog, const LAS unsigned short* tab,
                                       LAS char* scr, const LAS char* lut, int link, int peer, int cur, int nw, unsigned win, unsigned call, int wid, int lane) {
    const int q = P.rank & 3, qs = peer & 3;
    const bool cross = link < 4;
    const int nseq = cross ? NSEQ : NI;
    const char* pk = P.ws[peer] + 4096;
    const unsigned vl16 = (unsigned)lane * 16, one = 1, aoff = (unsigned)OFF_CTR + (unsigned)link * 128u;
    const unsigned ovoff = (unsigned)(lane >> 4) * (4 * NLOC * 4) + (unsigned)(lane & 15) * 16;
    const unsigned base = (call - 1u) * (unsigned)(nseq - NPUSH) - (unsigned)(nw + NPUSH);
    const unsigned rel = (unsigned)(size_t)(prog + 2);
    const unsigned nofl = (unsigned)OFF_SEQ + (blockIdx.x * 8u + (unsigned)wid) * 16u + 4u;
    unsigned pfl = nofl;
    const unsigned tz = __builtin_amdgcn_readfirstlane(prog[4]);
    if (TOUCH & 2) {
        const int id = __builtin_amdgcn_readfirstlane((int)tab[cur]), rb = id & 31;
        unsigned off, fl, d;
        if (id < PK_W) place_x(rb < 24 ? 3 * qs + (rb >> 3) : 12 + qs, rb, id >> 5, call, off, fl);
        else place_w(16 + 4 * qs + (id & 3), (id - PK_W) >> 2, off, fl);
        const char* a = lane ? myws + off : pk + pk_off(id < PK_W ? id : id + q * 512, call);
        if (lane < 2) asm volatile("global_load_dword %0, %1, off sc0 sc1\n\ts_waitcnt vmcnt(0)" : "=&v"(d) : "v"(a) : "memory");
    }
    for (;;) {
        const bool isout = cross && cur >= NCI;
        const int id = isout ? 0 : __builtin_amdgcn_readfirstlane((int)tab[cur]);
        const int slot = isout ? PK_O + q * NO + cur - NCI : id < PK_W ? id : id + q * 512;
        unsigned mode;
        if (pfl != nofl) {
            const unsigned i0 = __hip_atomic_load(info + cur, __ATOMIC_RELAXED, __HIP_MEMORY_SCOPE_WORKGROUP);
            const unsigned i1 = __hip_atomic_load(prog + 1, __ATOMIC_RELAXED, __HIP_MEMORY_SCOPE_WORKGROUP), i2 = __hip_atomic_load(prog + 2, __ATOMIC_RELAXED, __HIP_MEMORY_SCOPE_WORKGROUP);
            if (__builtin_amdgcn_readfirstlane(!DEFER || i0 - call * 32u > 31u || i1 - i2 >= win)) {
                WAIT_V(0);
                if (lane == 0) flag_store(myws + pfl, call);
                pfl = nofl;
            }
        }
        for (;;) {
            mode = __builtin_amdgcn_readfirstlane(__hip_atomic_load(info + cur, __ATOMIC_ACQUIRE, __HIP_MEMORY_SCOPE_WORKGROUP)) - call * 32u;
            if (mode <= 31u) break;
            __builtin_amdgcn_s_sleep(1);
        }
        const char* src = pk + (pk_off(slot, call) + (mode & 16u) * 512u);
        mode &= 15u;
        const unsigned long long mask = lane_mask(isout ? mode * 16u : mode * 8u);
        unsigned my = 0;
        if (lane == 0) my = __hip_atomic_fetch_add(prog + 1, 1u, __ATOMIC_RELAXED, __HIP_MEMORY_SCOPE_WORKGROUP);
        my = __builtin_amdgcn_readfirstlane(my) - win - (cur < HWIN && cur >= HWIN0);
        while ((int)(my - __builtin_amdgcn_readfirstlane(__hip_atomic_load(prog + 2, __ATOMIC_RELAXED, __HIP_MEMORY_SCOPE_WORKGROUP))) >= 0) { }
        {
            unsigned ts = 0;
            if (lane == 0) {
                __hip_atomic_fetch_max(prog + 3, ((unsigned)__builtin_amdgcn_s_memrealtime() - tz) << 8, __ATOMIC_RELAXED, __HIP_MEMORY_SCOPE_WORKGROUP);
                ts = __hip_atomic_fetch_add(prog + 3, (mode >= M_RAW ? 128u : isout ? 108u + mode : 104u + mode) * (cross ? PRATE : PRATEI), __ATOMIC_RELAXED, __HIP_MEMORY_SCOPE_WORKGROUP);
            }
            ts = __builtin_amdgcn_readfirstlane(ts);
            ts = (ts >> 8) + tz - PSLACK;
            while ((int)((unsigned)__builtin_amdgcn_s_memrealtime() - ts) < 0) { }
        }
        u4 h[8]; unsigned tk, lst = 0;
        const unsigned tis = (unsigned)__builtin_amdgcn_s_memrealtime();
        if (mode >= M_RAW) {
            asm volatile(PULL6 "global_load_dwordx4 %[h6], %[v16], %[src] offset:2048 sc0 sc1\n\tglobal_load_dwordx4 %[h7], %[v16], %[src] offset:3072 sc0 sc1\n\t" PULLT
                         : PULLO, [h6] "=&v"(h[6]), [h7] "=&v"(h[7]) : PULLI, [rn] "n"(RELAT) : "memory");
        } else if (!isout) {
            u2 p0, p1;
            asm volatile(PULL6 "global_load_dwordx2 %[p0], %[v8], %[src] offset:2048 sc0 sc1\n\t"
                         "s_mov_b64 exec, %[mask]\n\tglobal_load_dwordx2 %[p1], %[v8], %[src] offset:2560 sc0 sc1\n\t" PULLT
                         : PULLO, [p0] "=&v"(p0), [p1] "=&v"(p1)
                         : PULLI, [v8] "v"(vl16 >> 1), [mask] "s"(mask), [rn] "n"(RELAT) : "memory");
            h[6] = (u4){p0[0], p0[1], p1[0], p1[1]};
        } else {
            u2 p2; unsigned p1;
            asm volatile(PULL6 "global_load_dwordx2 %[p2], %[v8], %[src] offset:2048 sc0 sc1\n\tglobal_load_dword %[p1], %[v4], %[src] offset:2560 sc0 sc1\n\t"
                         "s_mov_b64 exec, %[mask]\n\tglobal_load_dword %[lst], %[v4], %[src] offset:2816 sc0 sc1\n\t" PULLT
                         : PULLO, [p2] "=&v"(p2), [p1] "=&v"(p1), [lst] "=&v"(lst)
                         : PULLI, [v8] "v"(vl16 >> 1), [v4] "v"(vl16 >> 2), [mask] "s"(mask), [rn] "n"(RELAT + 1) : "memory");
            h[6][0] = p2[0]; h[6][1] = p2[1]; h[6][2] = p1;
        }
        const unsigned lat = (unsigned)__builtin_amdgcn_s_memrealtime() - tis;
        const unsigned nxt = __builtin_amdgcn_readfirstlane(tk) - base;
        if (lane == 0) __hip_atomic_fetch_max(prog, nxt, __ATOMIC_RELAXED, __HIP_MEMORY_SCOPE_WORKGROUP);
        if (isout) {
            const int o = cur - NCI;
            int panel, blk, col;
            if (o < O_B) {
                const int r = o < O_A0 ? o : o < O_C ? (o - O_A0) & 127 : o - O_C, hn = o < O_A0 ? 16 : o < O_C ? 64 : 48;
                const int hf = r >= hn, t = (r - hf * hn) >> 3;
                panel = o < O_A0 ? 3 * qs + 2 : o < O_C ? 3 * (t >> 1) + (o >= O_A1) : 12 + ((q + 1 + (t >> 1)) & 3);
                blk = (r & 6) + hf; col = (t & 1) * 128 + (r & 1) * 64;
            } else {
                const int j = (o - O_B) / 24, r = o - O_B - j * 24;
                panel = 3 * ((qs + 1 + (r >> 3)) & 3) + 2; blk = 2 * j + ((r >> 1) & 1); col = ((r >> 2) & 1) * 128 + (r & 1) * 64;
            }
            char* ob = (char*)P.out + (size_t)(panel_row(peer, panel, blk) * NLOC + qs * 256 + col) * 4;
            if (mode < M_RAW) unpack_out(h, lst, scr, lane);
            if (mode == M_DEAD) h[0] = (u4){0x7FC00000u, 0x7FC00000u, 0x7FC00000u, 0x7FC00000u};
#pragma unroll
            for (int k = 0; k < 8; ++k) GST4P(ovoff, h[k], ob + ((k >> 2) * 16 + (k & 3)) * (NLOC * 4));
            if (!DEFER) WAIT_V(0);
            pfl = nofl;
        } else {
            if (mode < M_RAW) unpack_unit(h, scr, lut, lane);
            unsigned off, fl;
            const int rb = id & 31;
            if (id < PK_W) place_x(rb < 24 ? 3 * qs + (rb >> 3) : 12 + qs, rb, id >> 5, call, off, fl);
            else place_w(16 + 4 * qs + (id & 3), (id - PK_W) >> 2, off, fl);
            store8(h, myws + off, id < PK_W ? 32768u : 4096u, vl16);
            pfl = fl;
        }
        if (nxt >= (unsigned)nseq) break;
        cur = (int)nxt;
    }
    WAIT_V(0);
    if (lane == 0) flag_store(myws + pfl, call);
}

#undef VBLK
#define VBLK (blockIdx.x < NGEMM ? blockIdx.x : blockIdx.x + NPC)
__device__ __forceinline__ float silu(float y) {
    const float c = __builtin_amdgcn_fmed3f(y, -60.0f, 60.0f);
    const float e = __builtin_amdgcn_exp2f(c * -1.4426950408889634f);
    return y * __builtin_amdgcn_rcpf(1.0f + e);
}
__device__ __forceinline__ u4 silu4(const f4 (&a)[4], int r, bool dead) {
    f4 o = {silu(a[0][r]), silu(a[1][r]), silu(a[2][r]), silu(a[3][r])};
    const float nanv = __builtin_nanf("");
    if (dead) o = (f4){nanv, nanv, nanv, nanv};
    return __builtin_bit_cast(u4, o);
}

#define BAR() do { __builtin_amdgcn_sched_barrier(0); __builtin_amdgcn_s_barrier(); asm volatile("" ::: "memory"); __builtin_amdgcn_sched_barrier(0); } while (0)
#define GLDS(g, l, imm) __builtin_amdgcn_global_load_lds((const GAS void*)(g), (LAS void*)(l), 16, imm, AUX)
template <int V> struct IC { static constexpr int value = V; };

__device__ __forceinline__ void gemm_panel(const CAS Params& P, LAS char* lds, char* myws, int panel, int cg, int o0, int ohalf,
                                           unsigned call, int wid, int lane) {
    constexpr int ABYTES = 32768, STAGE = ABYTES + 16384;
    const char* ga = myws + panel_off(panel, call) + wid * 4096 + lane * 16;
    const char* gb = myws + OFF_DATA + A_B + (size_t)cg * CG_B + wid * 2048 + lane * 16;
    const int wm = wid >> 1, wn = wid & 1;
    const unsigned frow = (lane & 2) ? (unsigned)OFF_FB + (unsigned)cg * 1024u : (unsigned)OFF_FA + (unsigned)panel * 2048u + (unsigned)(lane & 1) * 16u;
    const unsigned fstep = (lane & 2) ? 16u : 32u;
    GAS unsigned* tmo = (GAS unsigned*)(myws + OFF_TMO);
    char* oflag = myws + (OFF_FP + (call & 1) * FP_B) + (((cg - 8) >> 1) * NSEQ + NCI + o0 + wid) * 4;
    int ready = 0;
    unsigned tv = 0;

    auto early = [&](int c) {
        if (c < ready || c >= NCH) return;
        const int cc = c + (lane >> 2);
        __builtin_amdgcn_global_load_lds((const GAS void*)(myws + (frow + (unsigned)(cc < NCH ? cc : NCH - 1) * fstep)), (LAS void*)(lds + 3 * STAGE), 16, 0, AUX);
    };
    auto ensure = [&](int c, bool pend) {
        if (c < ready) return;
        const unsigned long long t0 = __builtin_amdgcn_s_memrealtime();
        int lim = 15;
        for (;;) {
            int cc = c + ((lane >> 2) < lim ? (lane >> 2) : lim); cc = cc < NCH ? cc : NCH - 1;
            u4 f;
            if (pend) asm volatile("s_waitcnt vmcnt(%2)\n\tds_read_b128 %0, %1\n\ts_waitcnt lgkmcnt(0)"
                                   : "=&v"(f) : "v"((unsigned)(size_t)(lds + 3 * STAGE) + (unsigned)lane * 16u), "n"(3 * PD) : "memory");
            else asm volatile("global_load_dwordx4 %0, %1, %2 sc0 sc1\n\ts_waitcnt vmcnt(0)"
                              : "=&v"(f) : "v"(frow + (unsigned)cc * fstep), "s"(myws) : "memory");
            const bool bad = f[0] < call || f[1] < call || f[2] < call || f[3] < call;
            const unsigned long long m = __builtin_amdgcn_ballot_w64(bad);
            const int n = m ? (__builtin_ctzll(m) >> 2) : lim + 1;
            if (n > 0) { ready = c + n; return; }
            if (pend) { pend = false; continue; }
            if (
                __builtin_amdgcn_s_memrealtime() - t0 > DEADLINE) {
                __hip_atomic_store(tmo, call, __ATOMIC_RELAXED, __HIP_MEMORY_SCOPE_AGENT);
                ready = NCH; return;
            }
            lim = 0;
            __builtin_amdgcn_s_sleep(6);
        }
    };
    auto stage = [&](int t, int kf, int bufoff) {
        const char* a = ga + (size_t)t * 32768 + kf * 1024; const char* b = gb + (size_t)t * 16384 + kf * 1024;
        LAS char* la = lds + bufoff + wid * 4096 + kf * 1024; LAS char* lb = lds + bufoff + ABYTES + wid * 2048 + kf * 1024;
        GLDS(a, la, 0); GLDS(a, la, 2048); GLDS(b, lb, 0);
    };

    f4 acc[4][4];
#pragma unroll
    for (int m = 0; m < 4; ++m)
#pragma unroll
        for (int n = 0; n < 4; ++n) acc[m][n] = (f4){0.f, 0.f, 0.f, 0.f};

    const int aoff = wm * 8192 + lane * 16, boff = ABYTES + wn * 8192 + lane * 16;
    auto phase = [&](auto waitn, int rbuf, int kf, int lt, int lkf, int lbuf, int pc, int pe = NCH) {
        const LAS char* pa = lds + rbuf + aoff + kf * 1024; const LAS char* pb = lds + rbuf + boff + kf * 1024;
        h8 a[4], b[4];
#pragma unroll
        for (int n = 0; n < 4; ++n) b[n] = *(const LAS h8*)(pb + n * 2048);
#pragma unroll
        for (int m = 0; m < 4; ++m) a[m] = *(const LAS h8*)(pa + m * 2048);
        __builtin_amdgcn_sched_barrier(0);
        if (lt >= 0) stage(lt, lkf, lbuf);
        if (decltype(waitn)::value >= 0) WAIT_V(decltype(waitn)::value < 0 ? 0 : decltype(waitn)::value);
        WAIT_L0();
        BAR();
        __builtin_amdgcn_s_setprio(1);
#pragma unroll
        for (int m = 0; m < 4; ++m)
#pragma unroll
            for (int n = 0; n < 4; ++n) acc[m][n] = __builtin_amdgcn_mfma_f32_16x16x32_f16(a[m], b[n], acc[m][n], 0, 0, 0);
        __builtin_amdgcn_s_setprio(0);
        __builtin_amdgcn_sched_barrier(0);
        if (wid == 0) { if (pc >= 0) { ensure(pc, PD > 0); TREC(pc); } early(pe); }
        BAR();
    };

    if (wid == 0) { ensure(0, false); ensure(1, false); }
    BAR();
    STAMP(1);
    stage(0, 0, 0); stage(0, 1, 0); stage(1, 0, STAGE); stage(1, 1, STAGE); stage(2, 0, 2 * STAGE);
    WAIT_V(12);
    BAR();
    if (wid >= 4) BAR();
    int rb = 0, pb = 2 * STAGE;
    for (int t = 0; t < NKT - 3; ++t) {
        phase(IC<12>{}, rb, 0, t + 2, 1, pb, (t & 1) ? (t + 3) >> 1 : -1, PD == 2 && !(t & 1) ? (t + 4) >> 1 : NCH);
        phase(IC<12>{}, rb, 1, t + 3, 0, rb, -1, PD == 1 && !(t & 1) ? (t + 4) >> 1 : NCH);
        pb = rb; rb = rb == 2 * STAGE ? 0 : rb + STAGE;
    }
    static_assert((NKT - 3) % 3 == 2, "tail buffers");
    phase(IC<12>{}, 2 * STAGE, 0, NKT - 1, 1, STAGE, -1);
    phase(IC<9>{}, 2 * STAGE, 1, -1, 0, 0, -1);
    phase(IC<6>{}, 0, 0, -1, 0, 0, -1);
    phase(IC<3>{}, 0, 1, -1, 0, 0, -1);
    phase(IC<0>{}, STAGE, 0, -1, 0, 0, -1);
    phase(IC<-1>{}, STAGE, 1, -1, 0, 0, -1);
    if (wid < 4) BAR();
    STAMP(2);
    TDUMP(0);

    const bool dead = __builtin_amdgcn_readfirstlane(__hip_atomic_load(tmo, __ATOMIC_RELAXED, __HIP_MEMORY_SCOPE_AGENT)) == call;
    const unsigned vl16 = (unsigned)lane * 16, ovoff = (unsigned)(lane >> 4) * (4 * NLOC * 4) + (unsigned)(lane & 15) * 16;
    const int pk0 = PK_O + ((cg < 8 ? 0 : cg - 8) >> 1) * NO + o0 + wid;
    const unsigned hv[2] = {pk_half(myws, pk0, call), pk_half(myws, pk0 + ohalf, call)};
    unsigned mode = 0;
#pragma unroll
    for (int u = 0; u < 2; ++u) {
        u4 h[8];
#pragma unroll
        for (int k = 0; k < 8; ++k) h[k] = silu4(acc[2 * u + (k >> 2)], k & 3, dead);
        if (cg < 8) {
            char* ob = (char*)P.out + (size_t)(panel_row(P.rank, panel, 2 * wm + u) * NLOC + cg * 128 + wn * 64) * 4;
#pragma unroll
            for (int k = 0; k < 8; ++k) GST4P(ovoff, h[k], ob + ((k >> 2) * 16 + (k & 3)) * (NLOC * 4));
        } else {
            const unsigned prev = mode;
            if (u && o0 < EARLY) { WAIT_V(0); if (lane == 0) flag_store(oflag, call * 32u + prev); }
            mode = M_RAW;
            if (o0 + u * ohalf >= ORAW) mode = pack_out(h, lds + wid * 2048, lane);
            if (u && o0 >= EARLY) { WAIT_V(0); if (lane == 0) flag_store(oflag, call * 32u + prev); }
            const unsigned half = SMODE == 0 ? wid & 1 : (~__builtin_amdgcn_readfirstlane(hv[u]) ^ (SMODE == 2)) & 1u;
            char* d0 = myws + (pk_off(pk0 + u * ohalf, call) + half * 8192); char* d1 = d0 + 4096;
            GST4(vl16, h[0], d0, 0); GST4(vl16, h[1], d0, 1024); GST4(vl16, h[2], d0, 2048); GST4(vl16, h[3], d0, 3072);
            GST4(vl16, h[4], d1, 0); GST4(vl16, h[5], d1, 1024);
            if (mode < M_RAW) {
                const u2 p2 = {h[6][0], h[6][1]};
                asm volatile("global_store_dwordx2 %0, %1, %2 offset:2048 sc0 sc1\n\tglobal_store_dword %3, %4, %2 offset:2560 sc0 sc1"
                             :: "v"(vl16 >> 1), "v"(p2), "s"(d1), "v"(vl16 >> 2), "v"(h[6][2]) : "memory");
                store_list(d1 + 2816, h[6][3], mode, vl16 >> 2);
            } else {
                GST4(vl16, h[6], d1, 2048);
                GST4(vl16, h[7], d1, 3072);
            }
            mode += half * 16u;
        }
    }
    WAIT_V(0);
    if (cg >= 8 && lane == 0) flag_store(oflag + ohalf * 4, call * 32u + mode);
    WAIT_V(0);
    STAMP(3);
}

__device__ __forceinline__ void gemm_quarter(const CAS Params& P, LAS char* lds, char* myws, int panel, int qt, int cg, int o0,
                                             unsigned call, int wid, int lane) {
    constexpr int STAGE = 49152;
    const int kp = wid >> 2, lw = wid & 3, wm = lw >> 1, wn = lw & 1;
    const char* gsrc = myws + OFF_DATA + lane * 16 + (lw < 2 ? (size_t)panel * PANEL_B + kp * 32768 + qt * 8192
                                                             : A_B + (size_t)cg * CG_B + kp * 16384 + (lw - 2) * 8192);
    const unsigned gstep = lw < 2 ? 65536u : 32768u;
    LAS char* ring = lds + kp * 24576;
    LAS unsigned* pl = (LAS unsigned*)(lds + 160768);
    const unsigned frow = (unsigned)OFF_FA + (unsigned)(panel * 2048 + qt * 8 + (lane & 1) * 4);
    GAS unsigned* tmo = (GAS unsigned*)(myws + OFF_TMO);
    char* oflag = myws + (OFF_FP + (call & 1) * FP_B) + (((cg - 8) >> 1) * NSEQ + NCI + o0) * 4;
    unsigned tv = 0;
    int ready = 0; bool pend = false;
#define QPOLL 8
    auto paddr = [&]() { const int cc = ready + ((lane >> 1) < QPOLL ? (lane >> 1) : QPOLL - 1); return frow + (unsigned)(cc < NCH ? cc : NCH - 1) * 32u; };
    auto late = [&](unsigned tm, unsigned long long t0) {
        if (tm != call && __builtin_amdgcn_s_memrealtime() - t0 <= DEADLINE) return false;
        __hip_atomic_store(tmo, call, __ATOMIC_RELAXED, __HIP_MEMORY_SCOPE_AGENT);
        WAIT_V(0);
        ready = NCH;
        return true;
    };
    auto ensure = [&](int c) {
        if (c >= ready) {
            const unsigned long long t0 = __builtin_amdgcn_s_memrealtime();
            for (;;) {
                unsigned f;
                if (pend) { WAIT_V(0); f = pl[lane]; pend = false; }
                else asm volatile("global_load_dword %0, %1, %2 sc0 sc1\n\ts_waitcnt vmcnt(0)" : "=&v"(f) : "v"(paddr()), "s"(myws) : "memory");
                ready += __builtin_ctzll(__builtin_amdgcn_ballot_w64(f < call) | (1ull << (2 * QPOLL))) >> 1;
                if (c < ready || late(0u, t0)) break;
                __builtin_amdgcn_s_sleep(QSLEEP);
            }
        }
        if (!pend && ready < NCH && ready <= c + 2) {
            __builtin_amdgcn_global_load_lds((const GAS void*)(myws + paddr()), (LAS void*)pl, 4, 0, AUX);
            pend = true;
        }
    };
    auto stage = [&](int c, int buf) {
        const char* g = gsrc + (size_t)((unsigned)c * gstep);
        LAS char* l = ring + buf + (lw - 1) * 8192;
        GLDS(g, l, 0); GLDS(g, l, 1024); GLDS(g, l, 2048); GLDS(g, l, 3072);
        GLDS(g + 4096, l + 4096, 0); GLDS(g + 4096, l + 4096, 1024); GLDS(g + 4096, l + 4096, 2048); GLDS(g + 4096, l + 4096, 3072);
    };

    const int pk0 = PK_O + ((cg < 8 ? 0 : cg - 8) >> 1) * NO + o0;
    if (wid == 0) pl[64] = pk_half(myws, pk0, call);
    f4 acc[2][4];
#pragma unroll
    for (int m = 0; m < 2; ++m)
#pragma unroll
        for (int n = 0; n < 4; ++n) acc[m][n] = (f4){0.f, 0.f, 0.f, 0.f};
    const int aoff = wm * 4096 + lane * 16, boff = 8192 + wn * 8192 + lane * 16;
    auto phase = [&](auto waitn, auto role, int buf, int lc) {
        h8 a[4], b[8];
#pragma unroll
        for (int n = 0; n < 8; ++n) b[n] = *(const LAS h8*)(ring + buf + boff + n * 1024);
#pragma unroll
        for (int m = 0; m < 4; ++m) a[m] = *(const LAS h8*)(ring + buf + aoff + m * 1024);
        __builtin_amdgcn_sched_barrier(0);
        if (decltype(role)::value == 2 && wid == 0) {
            ensure(lc); TREC(lc);
        }
        WAIT_L0();
        BAR();
        if (decltype(role)::value == 1) stage(lc, buf);
#pragma unroll
        for (int k = 0; k < 2; ++k)
#pragma unroll
            for (int m = 0; m < 2; ++m)
#pragma unroll
                for (int n = 0; n < 4; ++n) acc[m][n] = __builtin_amdgcn_mfma_f32_16x16x32_f16(a[m * 2 + k], b[n * 2 + k], acc[m][n], 0, 0, 0);
        if (decltype(role)::value == 1) {
#pragma unroll
            for (int i = 0; i < 8; ++i) { __builtin_amdgcn_sched_group_barrier(0x010, 1, 0); __builtin_amdgcn_sched_group_barrier(0x008, 2, 0); }
        }
        __builtin_amdgcn_sched_barrier(0);
        if (decltype(waitn)::value >= 0) { WAIT_V(decltype(waitn)::value < 0 ? 0 : decltype(waitn)::value); BAR(); }
    };

    if (wid == 0 && ready < NCH) {
        const unsigned long long t0 = __builtin_amdgcn_s_memrealtime();
        for (;;) {
            u4 f;
            asm volatile("global_load_dwordx4 %0, %1, %2 sc0 sc1\n\ts_waitcnt vmcnt(0)"
                         : "=&v"(f) : "v"((unsigned)OFF_FB + (unsigned)(cg * 1024 + lane * 16)), "s"(myws) : "memory");
            if (__builtin_amdgcn_ballot_w64(f[0] < call || f[1] < call || f[2] < call || f[3] < call) == 0 ||
                late(__builtin_amdgcn_readfirstlane(__hip_atomic_load(tmo, __ATOMIC_RELAXED, __HIP_MEMORY_SCOPE_AGENT)), t0)) break;
            __builtin_amdgcn_s_sleep(6);
        }
        ensure(2);
    }
    WAIT_L0();
    BAR();
    STAMP(4);
    const unsigned half = SMODE == 0 ? cg & 1 : (~__builtin_amdgcn_readfirstlane(pl[64]) ^ (SMODE == 2)) & 1u;
    if (lw != 0) { stage(0, 0); stage(1, STAGE); stage(2, 2 * STAGE); }
    WAIT_V(16);
    BAR();
    if (kp) BAR();
    if (lw != 0) for (int c = 3, buf = 0; c < NCH; ++c, buf = buf == 2 * STAGE ? 0 : buf + STAGE) phase(IC<16>{}, IC<1>{}, buf, c);
    else for (int c = 3, buf = 0; c < NCH; ++c, buf = buf == 2 * STAGE ? 0 : buf + STAGE) phase(IC<16>{}, IC<2>{}, buf, c);
    static_assert((NCH - 3) % 3 == 1, "tail buffers");
    const unsigned tm = __hip_atomic_load(tmo, __ATOMIC_RELAXED, __HIP_MEMORY_SCOPE_AGENT);
    phase(IC<8>{}, IC<0>{}, STAGE, -1);
    phase(IC<0>{}, IC<0>{}, 2 * STAGE, -1);
    phase(IC<-1>{}, IC<0>{}, 0, -1);
    f4 r[4];
#pragma unroll
    for (int n = 0; n < 4; ++n) {
        const f4 a0 = acc[0][n], a1 = acc[1][n];
        r[n] = kp ? a1 : a0;
        *(LAS f4*)(ring + lw * 4096 + n * 1024 + lane * 16) = kp ? a0 : a1;
    }
    const bool dead = __builtin_amdgcn_readfirstlane(tm) == call;
    WAIT_L0();
    BAR();
    if (!kp) BAR();
    STAMP(5);
#pragma unroll
    for (int n = 0; n < 4; ++n) r[n] += *(const LAS f4*)(lds + (kp ^ 1) * 24576 + lw * 4096 + n * 1024 + lane * 16);
    if (cg < 8) {
        const unsigned ovoff = (unsigned)(lane >> 4) * (4 * NLOC * 4) + (unsigned)(lane & 15) * 16;
        char* ob = (char*)P.out + (size_t)((panel_row(P.rank, panel, 2 * qt + wm) + kp * 16) * NLOC + cg * 128 + wn * 64) * 4;
#pragma unroll
        for (int k = 0; k < 4; ++k) { const u4 o = silu4(r, k, dead); GST4P(ovoff, o, ob + k * (NLOC * 4)); }
    } else {
        char* d0 = myws + (pk_off(pk0 + lw, call) + half * 8192 + kp * 4096);
#pragma unroll
        for (int k = 0; k < 4; ++k) { const u4 o = silu4(r, k, dead); GST4((unsigned)lane * 16, o, d0 + k * 1024, 0); }
        WAIT_V(0);
        BAR();
        if (wid == 0 && lane < 4) flag_store(oflag + lane * 4, call * 32u + half * 16u + M_RAW);
    }
    STAMP(6);
    TDUMP(1);
}

__global__ void __launch_bounds__(512, 2) ag_gemm_kernel(const Params Parg) {
    const CAS Params& P = *(const CAS Params*)__builtin_amdgcn_kernarg_segment_ptr();
    (void)Parg;
    __shared__ __attribute__((aligned(16))) char lds_raw[5 * 32768];
    LAS char* lds = (LAS char*)lds_raw;
    const int lane = threadIdx.x & 63, wid = __builtin_amdgcn_readfirstlane(threadIdx.x >> 6);
    const int xcd = blockIdx.x & 7, slot = blockIdx.x >> 3;
    char* myws = P.ws[P.rank];
    const int q = P.rank & 3;
    const unsigned long long t0 = __builtin_amdgcn_s_memrealtime();
    const unsigned side = (__builtin_amdgcn_s_getreg(20 | (3 << 11)) >> 2) & 1u;
    int g = (int)blockIdx.x - NGEMM;
    if (slot == NGEMM / 8) g = NPC + xcd;
    else if (g >= 0) {
        if (threadIdx.x == 0) {
            GAS unsigned* rt = (GAS unsigned*)(myws + OFF_ROLE);
            const unsigned t = __hip_atomic_fetch_add(rt, side ? 65536u : 1u, __ATOMIC_RELAXED, __HIP_MEMORY_SCOPE_AGENT);
            if ((t & 65535u) + (t >> 16) >= 247u - NGEMM) __hip_atomic_store(rt, 0u, __ATOMIC_RELAXED, __HIP_MEMORY_SCOPE_AGENT);
            *(LAS unsigned*)(lds + 8 * PBUF) = t;
        }
        __syncthreads();
        const unsigned t = __builtin_amdgcn_readfirstlane(*(const LAS unsigned*)(lds + 8 * PBUF));
        constexpr unsigned N0 = NPC / 7 * 3, NL = 247 - NGEMM;
        const unsigned r = (side ? NL - (t >> 16) : t & 65535u) % (NL + 1), b = NL - r;
        g = r < N0 ? r / 3 * 7 + 4 + r % 3 : r < N0 + NPROD - 8 ? NPC + 8 + r - N0 : b / 4 * 7 + b % 4;
        if (XMODE == 0) { g = (int)blockIdx.x - NGEMM; g = g < NPC + 8 ? g - 8 : g; }
    }
#undef VBLK
#define VBLK (NGEMM + g)
    if (PROBET >= 0 && wid == 7 && blockIdx.x < 2 * PK_PG) {
        char* a = myws + (OFF_PK + ((size_t)blockIdx.x << 21) + PROBEX * (blockIdx.x & 31) * 256);
        unsigned t[2] = {~0u, ~0u};
        while (__builtin_amdgcn_s_memrealtime() - t0 < PROBET) __builtin_amdgcn_s_sleep(4);
#pragma unroll
        for (int i = 0; i < 6; ++i) {
            const unsigned long long c0 = __builtin_amdgcn_s_memtime();
            if (PROBE1) asm volatile("s_waitcnt lgkmcnt(0)\n\ts_mov_b64 exec, 1\n\tglobal_store_dword %0, %0, %1 sc0 sc1\n\ts_mov_b64 exec, -1\n\ts_waitcnt vmcnt(0)" :: "v"((i & 1) * 8192), "s"(a), "s"(c0) : "memory");
            else asm volatile("s_waitcnt lgkmcnt(0)\n\tglobal_store_dword %0, %0, %1 sc0 sc1\n\ts_waitcnt vmcnt(0)" :: "v"((i & 1) * 8192), "s"(a), "s"(c0) : "memory");
            const unsigned d = (unsigned)(__builtin_amdgcn_s_memtime() - c0);
            t[i & 1] = d < t[i & 1] ? d : t[i & 1];
        }
        if (lane == 0) flag_store(myws + OFF_MAP + blockIdx.x * MAPS, (t[1] < t[0]) ^ side);
        STAMP(10);
    }
    const int u0 = g < 0 ? wid * NGEMM + (int)blockIdx.x : UG + wid * NPROD + g - NPC;
    if (g < 0 || g >= NPC) {
        pace(t0, u0); fetch_input(P, lds + wid * PBUF, u0, lane);
        if ((TOUCH & 1) && u0 < TOUCHU) touch(P, myws, lds + wid * PBUF + 16384, u0, lane);
        make_lut(lds + (8 * PBUF + 16), true, lane);
    }
    GAS unsigned* seq = (GAS unsigned*)(myws + OFF_SEQ) + (blockIdx.x * 8 + wid) * 4;
    const unsigned call = __builtin_amdgcn_readfirstlane(__hip_atomic_load(seq, __ATOMIC_RELAXED, __HIP_MEMORY_SCOPE_AGENT)) + 1u;
    WAIT_V(0);
    if (lane == 0) __hip_atomic_store(seq, call, __ATOMIC_RELAXED, __HIP_MEMORY_SCOPE_AGENT);

    if (g < 0 || g >= NPC) {
        producer(P, myws, lds + wid * PBUF, lds + (8 * PBUF + 16), u0, t0, call, wid, lane);
        STAMP(7);
    }
    if (slot <= NGEMM / 8) {
        const int pi = slot >> 3, ps = xcd >> 1, h = xcd & 1, sown = 20 + ps;
        const int ci = pi == 3 ? (slot & 1) * 4 + ps : slot == sown ? ps : slot & 7;
        const int panel = pi < 2 ? 3 * ps + pi : pi == 3 ? 3 * q + 2 : slot == sown ? 16 : 12 + ps;
        if (g < 0) gemm_panel(P, lds, myws, panel, ci < 4 ? 4 * h + ci : 2 * ci + h,
                   pi == 3 ? h * 8 : pi == 2 ? O_C + (((ps - ci - 1) & 3) * 2 + h) * 8 : O_A0 + pi * 128 + xcd * 8, pi == 3 ? 16 : pi == 2 ? 48 : 64, call, wid, lane);
        const int k = slot == 26 ? 23 : slot >= 24 ? slot - 24 : slot == sown ? 2 : slot < 4 ? slot + 3 : slot < 8 ? slot + 7 : slot < 12 ? slot - 1
                    : slot < 20 ? slot + 3 : 24;
        if (k < 24) {
            const int kk = k < 12 ? k : k - 12, qt = kk / 3, si = kk - qt * 3;
            gemm_quarter(P, lds, myws, 3 * ((q + 1 + si) & 3) + 2, qt, k < 12 ? 8 + xcd : xcd, O_B + qt * 24 + (si * 2 + h) * 4, call, wid, lane);
        }
    } else if (g < NPC) {
        LAS unsigned* info = (LAS unsigned*)lds;
        LAS unsigned* prog = info + NSEQ;
        LAS unsigned short* tab = (LAS unsigned short*)(lds + 8192);
        const int j = g / 7, link = g - j * 7, ncu = (NPC + 6 - link) / 7;
        for (int i = threadIdx.x; i < NSEQ + 4; i += 512) info[i] = 0;
        if (threadIdx.x == 0) prog[4] = (unsigned)t0;
        for (int i = threadIdx.x; i < PK_W; i += 512)
            if (((i & 31) == 16 || (i & 31) >= 24) == (link < 4)) tab[pos_x(i & 31, i >> 5)] = (unsigned short)i;
        if (link < 4) tab[pos_w(threadIdx.x >> 2, threadIdx.x & 3)] = (unsigned short)(PK_W + threadIdx.x);
        if (wid == 0) make_lut(lds + 14336, false, lane);
        __syncthreads();
        const int peer = link < 4 ? ((P.rank & 4) ^ 4) + ((q + link) & 3) : (P.rank & 4) + ((q + link - 3) & 3);
        if (wid < WPC) {
            if (lane == 0) __hip_atomic_fetch_max(prog, (unsigned)(NPUSH + wid * ncu + j), __ATOMIC_RELAXED, __HIP_MEMORY_SCOPE_WORKGROUP);
            puller(g, P, myws, info, prog, tab, lds + 16384 + wid * 8192, lds + 14336, link, peer, NPUSH + wid * ncu + j, ncu * WPC,
                   (unsigned)(((link < 4 ? NWIN : NWINI) + ncu - 1 - j) / ncu), call, wid, lane);
        } else if (wid == 7) {
            flag_mirror(myws, P.ws[peer] + (OFF_FP + (call & 1) * FP_B + (link < 4 ? q : 4) * (NSEQ * 4)), link < 4 ? NSEQ : NI, info, prog, call, lane);
        }
        STAMP(3);
    }
}

}

extern "C" void kernel_launch(int rank, int nranks, ncclComm_t comm, hipStream_t stream,
                              void* const* d_in, const int* in_sizes, int n_in,
                              void* d_out, int out_size, const int* in_dtypes, int out_dtype,
                              void* const* d_out_all, void* const* d_ws_all, size_t ws_bytes) {
    (void)comm; (void)in_dtypes; (void)out_dtype; (void)d_out_all;
    if (nranks != 8 || (unsigned)rank >= 8u || n_in < 2 || ws_bytes < WS_NEED || out_size != KDIM * NLOC ||
        in_sizes[0] != 1024 * KDIM || in_sizes[1] != KDIM * NLOC) {
        fprintf(stderr, "ag_gemm: unexpected problem shape\n");
        return;
    }
    Params P;
    P.x = (const float*)d_in[0]; P.w = (const float*)d_in[1];
    P.out = (float*)d_out;
    P.rank = 0; P.pad = 0;
    for (int p = 0; p < 8; ++p) {
        const int r = (int)((RMAP >> (4 * p)) & 7u);
        P.ws[p] = (char*)d_ws_all[r];
        if (r == rank) P.rank = p;
    }
    hipLaunchKernelGGL(ag_gemm_kernel, dim3(256), dim3(512), 0, stream, P);
    HIP_CHECK(hipGetLastError());
}
```

```cpp
#include <hip/hip_runtime.h>
#include <rccl/rccl.h>
#include <cstdio>

#define HIP_CHECK(expr) do { hipError_t _e = (expr); if (_e != hipSuccess) { \
    fprintf(stderr, "HIP error %s at %s:%d\n", hipGetErrorString(_e), __FILE__, __LINE__); \
    return; } } while (0)

namespace {

typedef _Float16 h8 __attribute__((ext_vector_type(8)));
typedef float f4 __attribute__((ext_vector_type(4)));
typedef unsigned u4 __attribute__((ext_vector_type(4)));
typedef unsigned u2 __attribute__((ext_vector_type(2)));
#define LAS __attribute__((address_space(3)))
#define GAS __attribute__((address_space(1)))
#define CAS __attribute__((address_space(4)))

constexpr int KDIM = 8192, NLOC = 1024;
constexpr int NKT = KDIM / 64;
constexpr int NCH = NKT / 2;
constexpr int NI = 23 * 64;
constexpr int NCI = 17 * 64, NO = 480;
constexpr int NSEQ = NCI + NO;
constexpr int O_A0 = 32, O_A1 = 160, O_C = 288, O_B = 384;
constexpr size_t PANEL_B = 4u << 20;
constexpr size_t CG_B = 2u << 20;
constexpr size_t OFF_SEQ = 0;
constexpr size_t OFF_TMO = 64u << 10;
constexpr size_t OFF_CTR = OFF_TMO + 256;
constexpr size_t OFF_ROLE = OFF_TMO + 1280;
constexpr size_t OFF_MAP = OFF_TMO + 2048;
#define MAPS 256u
constexpr size_t OFF_FA = 128u << 10;
constexpr size_t OFF_FB = OFF_FA + 17 * 64 * 32;
constexpr size_t OFF_FP = 256u << 10;
constexpr size_t FP_B = 5 * NSEQ * 4;
constexpr size_t OFF_DBG = 640u << 10;
constexpr size_t OFF_DATA = 1u << 20;
constexpr size_t A_B = 21 * PANEL_B, B_B = 16 * CG_B;
constexpr size_t OFF_PK = OFF_DATA + A_B + B_B + (1u << 20);
constexpr int PK_W = 2048, PK_O = 4096, PK_N = PK_O + 4 * NO;
constexpr int PK_PG = (PK_N + 126) / 127;
constexpr size_t PK_B = (size_t)PK_PG << 21;
constexpr size_t WS_NEED = OFF_PK + 2 * PK_B;
static_assert(OFF_MAP + 2 * PK_PG * MAPS <= OFF_FA && OFF_FB + 16 * 64 * 16 <= OFF_FP && OFF_FP + 2 * FP_B <= OFF_DBG, "flag area");
constexpr unsigned long long DEADLINE = 30000000ull;
constexpr unsigned M_RAW = 9, M_DEAD = 10;

#define NOTMO
#define NOTMO1
#define NOTMO2
#define NPC 35
#define WPC 7
#define SKC 14
#define SKA 5
#define SQ1 4
#define SQ2 10
#define SQ3 17
#define PACE 12
#define U1ST 448
#define PACE1 PACE
#define PACE2 PACE1
#define PROBE1 1
#define LOOK 64
#define PCH 2
#define PT0 180
#define PDT 13
#define CHMAP 0x76543210u
#define PROBET 0
#define PROBEX 0
#define TOUCH 0
#define RAW0 32
#define RAW1 0
#define DEFER 1
#define PD 2
#define NWIN 10
#define NWINI NWIN
#define PRATE 134
#define PRATEI PRATE
#define LGAIN 64
#define LCAP 30
#define HWIN 280
#define HWIN0 0
#define GLINK 1
#define PSLACK 0
#define RELAT 8
#define UG 3000
#define NB2 200
#define AHEAD UG
#define PDEF 0
#define LOOK2 4
#define MALL 4
#define MSL 8
#define QSLEEP 2
#define AUX 17
#define RMAP 0x75416320u
#define EARLY 0
#define ORAW 0
#define SMODE 1
#define XMODE 1
constexpr int NGEMM = 208;
constexpr int NPROD = 256 - NGEMM - NPC;

struct Params {
    const float* x;
    const float* w;
    float* out;
    char* ws[8];
    int rank;
    int pad;
};

#define VBLK blockIdx.x
#define STAMP(i) do { } while (0)

#define TREC(c) do { } while (0)
#define TDUMP(k) do { (void)tv; } while (0)
#define HST(i, v) do { } while (0)
#define NOW() __builtin_amdgcn_s_memrealtime()
#define GST4(voff, data, sbase, imm) \
    asm volatile("global_store_dwordx4 %0, %1, %2 offset:" #imm " sc0 sc1\n\ts_nop 1" :: "v"(voff), "v"(data), "s"(sbase) : "memory")
#define OSUF " sc0 sc1"
#define GST4P(voff, data, sbase) \
    asm volatile("global_store_dwordx4 %0, %1, %2" OSUF "\n\ts_nop 1" :: "v"(voff), "v"(data), "s"(sbase) : "memory")
#define WAIT_V(n) asm volatile("s_waitcnt vmcnt(%0)" :: "n"(n) : "memory")
#define WAIT_L0() asm volatile("s_waitcnt lgkmcnt(0)" ::: "memory")

__device__ __forceinline__ u4 pack8(float a0, float a1, float a2, float a3, float a4, float a5, float a6, float a7) {
    h8 h = {(_Float16)a0, (_Float16)a1, (_Float16)a2, (_Float16)a3, (_Float16)a4, (_Float16)a5, (_Float16)a6, (_Float16)a7};
    return __builtin_bit_cast(u4, h);
}
__device__ __forceinline__ void flag_store(char* p, unsigned v) {
    __hip_atomic_store((GAS unsigned*)p, v, __ATOMIC_RELAXED, __HIP_MEMORY_SCOPE_SYSTEM);
}
__device__ __forceinline__ void store8(const u4 (&h)[8], char* d0, unsigned half, unsigned vl16) {
    char* d1 = d0 + half;
    GST4(vl16, h[0], d0, 0); GST4(vl16, h[1], d0, 1024); GST4(vl16, h[2], d0, 2048); GST4(vl16, h[3], d0, 3072);
    GST4(vl16, h[4], d1, 0); GST4(vl16, h[5], d1, 1024); GST4(vl16, h[6], d1, 2048); GST4(vl16, h[7], d1, 3072);
}
__device__ __forceinline__ unsigned long long lane_mask(unsigned n) { return n >= 64u ? ~0ull : (1ull << n) - 1ull; }
__device__ __forceinline__ void store_list(char* d, unsigned l, unsigned mode, unsigned vl4) {
    asm volatile("s_mov_b64 exec, %3\n\tglobal_store_dword %0, %1, %2 sc0 sc1\n\ts_mov_b64 exec, -1"
                 :: "v"(vl4), "v"(l), "s"(d), "s"(lane_mask(mode * 16u)) : "memory");
}

__device__ __forceinline__ int clamp64(int v) { return v < 0 ? 0 : v > 64 ? 64 : v; }
__device__ __forceinline__ int pos_w(int kt, int g) { const int ch = kt >> 1; return (ch < PCH ? 8 : 9) * ch + 8 * clamp64(ch - SKC) + (kt & 1) * 4 + g; }
__device__ __forceinline__ int pos_x(int rb, int ch) {
    if (rb < 8) return 8 * ch + 8 * clamp64(ch - SKA) + rb;
    if (rb < 16) return 8 * ch + 8 * clamp64(ch + SKA + 1) + rb - 8;
    if (rb == 16) return ch < PCH ? 8 * PCH + ch : 9 * ch + 8 * clamp64(ch - SKC) + 8;
    if (rb >= 24) return 8 * ch + 9 * clamp64(ch + SKC + 1) + rb - 24;
    const int j = (rb - 16) >> 1, r = ch + (j == 0 ? 0 : j == 1 ? SQ1 : j == 2 ? SQ2 : SQ3);
    return 1024 + clamp64(r + (j > 0)) + 2 * (clamp64(r - SQ1 + (j > 1)) + clamp64(r - SQ2 + (j > 2)) + clamp64(r - SQ3)) + (j ? rb & 1 : 0);
}
static_assert(NPC % 7 == 0 && 2 * PK_PG <= NGEMM && UG >= 8 * NGEMM && UG <= 8 * (NGEMM + NB2) && UG + 8 * NPROD <= PK_O && 0 <= SQ1 && SQ1 <= SQ2 && SQ2 <= SQ3 && SKC < 64 && SKA < 64 && PCH <= SKA && PCH <= SKC && SKC >= 8, "skews");
constexpr int NPUSH = 8 * PCH;
__device__ __forceinline__ unsigned panel_off(int panel, unsigned call) {
    return (unsigned)OFF_DATA + (unsigned)((call & 1) && panel < 12 && panel % 3 == 0 ? 17 + panel / 3 : panel) * (unsigned)PANEL_B;
}
__device__ __forceinline__ void place_x(int panel, int rb, int ch, unsigned call, unsigned& off, unsigned& fl) {
    off = panel_off(panel, call) + (unsigned)(ch * 65536 + (rb & 7) * 4096);
    fl = (unsigned)OFF_FA + (unsigned)(panel * 2048 + ch * 32 + (rb & 7) * 4);
}
__device__ __forceinline__ void place_w(int cb, int kt, unsigned& off, unsigned& fl) {
    off = (unsigned)(OFF_DATA + A_B) + (unsigned)(cb >> 1) * (unsigned)CG_B + (unsigned)(kt * 16384 + (cb & 1) * 8192);
    fl = (unsigned)OFF_FB + (unsigned)((cb >> 1) * 1024 + kt * 8 + (cb & 1) * 4);
}
__device__ __forceinline__ int panel_row(int pub, int panel, int blk) {
    const int gs = pub & 4, s = panel / 3, k = panel - 3 * s;
    const int r = panel == 16 ? pub : panel >= 12 ? (gs ^ 4) + panel - 12 : (k == 2 && blk == 0 ? gs ^ 4 : gs) + s;
    return (int)((RMAP >> (4 * r)) & 7u) * 1024 + (panel >= 12 ? 768 : k * 256) + blk * 32;
}
__device__ __forceinline__ size_t pk_off(int pk, unsigned call) { return OFF_PK + (call & 1) * PK_B + (size_t)(pk + pk / 127 + 1) * 16384; }
__device__ __forceinline__ unsigned pk_half(char* myws, int pk, unsigned call) {
    return __hip_atomic_load((GAS unsigned*)(myws + OFF_MAP) + ((call & 1) * PK_PG + pk / 127) * (MAPS / 4), __ATOMIC_RELAXED, __HIP_MEMORY_SCOPE_SYSTEM);
}

#define PERM(hi, lo, sel) __builtin_amdgcn_perm(hi, lo, sel)
#define ABIT(hi, lo, s) __builtin_amdgcn_alignbit(hi, lo, s)
#define HW(k) h[(k) >> 2][(k) & 3]
#define FENCE() asm volatile("" ::: "memory")
__device__ __forceinline__ unsigned bfi(unsigned m, unsigned x, unsigned y) { return (x & m) | (y & ~m); }
__device__ __forceinline__ unsigned wave_scan(unsigned v) {
    v += __builtin_amdgcn_update_dpp(0u, v, 0x111, 0xF, 0xF, true);
    v += __builtin_amdgcn_update_dpp(0u, v, 0x112, 0xF, 0xF, true);
    v += __builtin_amdgcn_update_dpp(0u, v, 0x114, 0xF, 0xF, true);
    v += __builtin_amdgcn_update_dpp(0u, v, 0x118, 0xF, 0xF, true);
    v += __builtin_amdgcn_update_dpp(0u, v, 0x142, 0xA, 0xF, false);
    v += __builtin_amdgcn_update_dpp(0u, v, 0x143, 0xC, 0xF, false);
    return v;
}
__device__ __forceinline__ int wave_sum(unsigned v) { return __builtin_amdgcn_readlane(wave_scan(v), 63); }
__device__ __forceinline__ unsigned escapes(unsigned c, unsigned acc) { return (unsigned)__builtin_popcount(c & (c >> 1) & 0x55555555u) + acc; }
__device__ __forceinline__ void make_lut(LAS char* lut, bool pack, int lane) {
#pragma unroll
    for (int j = 0; j < 4; ++j) {
        unsigned sel = pack ? 0x0C0C0C0Cu : 0u, k = 0;
#pragma unroll
        for (int i = 0; i < 4; ++i) {
            const unsigned c = i < 3 ? ((unsigned)lane >> (2 * i)) & 3u : (unsigned)j;
            if (pack) sel ^= c == 3u ? (0x0Cu ^ (unsigned)i) << (8 * k) : 0u; else sel |= (c == 3u ? k : 4u + c) << (8 * i);
            k += c == 3u;
        }
        *(LAS u2*)(lut + (j * 64 + lane) * 8) = (u2){sel, pack ? 8u * k : k};
    }
}
__device__ __forceinline__ unsigned code4(unsigned thi, unsigned tlo, unsigned s) { return __builtin_amdgcn_udot4(PERM(thi, tlo, s), 0x80200802u, 0u - 0xF3u * 170u, false); }
template <int N> __device__ __forceinline__ void append(const unsigned* s, const u2* e, unsigned pos) {
#pragma unroll
    for (int g = 0; g < N; ++g) {
        const unsigned long long c = (unsigned long long)PERM(0u, s[g], e[g][0]) << (pos & 31u);
        asm volatile("v_cmpx_ne_u32_e32 vcc, 0, %1\n\tds_or_b32 %0, %1\n\tv_cmpx_ne_u32_e32 vcc, 0, %2\n\tds_or_b32 %0, %2 offset:4\n\ts_mov_b64 exec, -1"
                     :: "v"((pos >> 3) & ~3u), "v"((unsigned)c), "v"((unsigned)(c >> 32)) : "vcc", "memory");
        pos += e[g][1];
    }
}
constexpr int SCR_B = 3328;
constexpr int SCR_3 = 2064, SCR_T = 2592, PBUF = 16384 + SCR_B;
__device__ __forceinline__ unsigned pack_unit(u4 (&h)[8], LAS char* scr, const LAS char* lut, int lane, unsigned& r0, unsigned& r2) {
    const u4 z = {0u, 0u, 0u, 0u};
    *(LAS u4*)(scr + lane * 16) = z; *(LAS u4*)(scr + 1024 + lane * 16) = z; *(LAS u4*)(scr + 2048 + lane * 16) = z;
    FENCE();
    unsigned G[16], s[16], a[16], cw[4];
#pragma unroll
    for (int g = 0; g < 16; ++g) { G[g] = PERM(HW(2 * g + 1), HW(2 * g), 0x07050301u); s[g] = (G[g] >> 1) & 0x3E3E3E3Eu; }
    const int m = wave_sum(__builtin_amdgcn_udot4(s[0] + s[5] + s[10] + s[15], 0x01010101u, 0u, false)) + 205;
    const unsigned b3 = ((unsigned)(m >> 11) + 3u) & 31u, type = (m & 2047) > 1120;
    u2 e[16];
#pragma unroll
    for (int g = 0; g < 16; ++g) {
        s[g] = ((2u * b3 + 65u) * 0x01010101u - s[g]) & 0x3F3F3F3Fu;
        a[g] = code4(0xF300F700u, 0xFB00FF00u, s[g]);
        e[g] = *(const LAS u2*)(lut + a[g]);
    }
    unsigned n = 0;
#pragma unroll
    for (int j = 0; j < 4; ++j) { cw[j] = (a[4 * j] >> 3) | (a[4 * j + 1] << 5) | (a[4 * j + 2] << 13) | (a[4 * j + 3] << 21); n = escapes(cw[j], n); }
    unsigned sc = wave_scan(n);
    r2 = (__builtin_amdgcn_readlane(sc, 63) + 31u) >> 5;
    if (r2 > 62u) return M_RAW;
    append<16>(s, e, (sc - n + (unsigned)(size_t)scr) * 8u);
    FENCE();
    const u4 q0 = *(const LAS u4*)(scr + lane * 32), q1 = *(const LAS u4*)(scr + lane * 32 + 16);
#pragma unroll
    for (int g = 0; g < 8; ++g) {
        const unsigned x = g < 4 ? q0[g & 3] : q1[g & 3];
        s[g] = (x | ~PERM(0xFF000000u, 0xFF00F9F6u, x)) - 0x06060606u;
        a[g] = code4(type ? 0xFF00F700u : 0xFB00F700u, type ? 0xF300FB00u : 0xF300FF00u, s[g]);
        e[g] = *(const LAS u2*)(lut + a[g]);
    }
    const unsigned c0 = (a[0] >> 3) | (a[1] << 5) | (a[2] << 13) | (a[3] << 21), c1 = (a[4] >> 3) | (a[5] << 5) | (a[6] << 13) | (a[7] << 21);
    n = escapes(c1, escapes(c0, 0u));
    sc = wave_scan(n);
    const unsigned r3 = (__builtin_amdgcn_readlane(sc, 63) + 7u) >> 3;
    if (r3 > 64u) return M_RAW;
    append<8>(s, e, (sc - n + (unsigned)(size_t)(scr + SCR_3)) * 8u);
    FENCE();
    u2 y = *(const LAS u2*)(scr + SCR_3 + lane * 8);
#pragma unroll
    for (int j = 0; j < 2; ++j) y[j] = (y[j] | ~PERM(0xFF000000u, 0xFF00F9F8u, y[j])) - 0x07070707u;
    unsigned w = ((y[0] >> 1) & 0x07070707u) | ((y[1] << 2) & 0x38383838u);
    w |= ((w >> 24) * 0x41040u) & 0xC0C0C0u;
    LAS char* t = scr + SCR_T;
    t[4 + lane * 3] = (char)w; t[5 + lane * 3] = (char)(w >> 8); t[6 + lane * 3] = (char)(w >> 16);
    FENCE();
    const unsigned l0 = (3u * r3 + 5u) >> 1;
    unsigned bad = ((y[0] >> 4) & 0x03030303u) | (y[1] & 0x30303030u), nl = 0;
    for (;;) {
        const unsigned long long mk = __builtin_amdgcn_ballot_w64(bad != 0);
        if (mk == 0 || nl > 64u) break;
        if (bad != 0) {
            const unsigned p = (unsigned)__builtin_ctz(bad), k = l0 + nl + __builtin_amdgcn_mbcnt_hi((unsigned)(mk >> 32), __builtin_amdgcn_mbcnt_lo((unsigned)mk, 0u));
            ((LAS unsigned short*)t)[k] = (unsigned short)((((unsigned)lane * 8u + (p & 4u) + (p >> 3)) << 7) | ((4u * b3 - 24u - 2u * (((p & 4u) ? y[1] : y[0]) >> (p & 24u))) & 0x7Cu));
            bad &= ~(15u << (p & 28u));
        }
        nl += (unsigned)__builtin_popcountll(mk);
    }
    r0 = (l0 + nl + 3u) >> 2;
    if (nl > 64u || r0 + r2 > 64u) return M_RAW;
    FENCE();
    u2 f = *(const LAS u2*)(t + lane * 8);
    FENCE();
    if (lane == 0) f[0] = b3 | (type << 5) | (r0 << 8) | (nl << 16) | (l0 << 24);
    unsigned L[16];
#pragma unroll
    for (int g = 0; g < 16; ++g) L[g] = PERM(HW(2 * g + 1), HW(2 * g), 0x06040200u);
#pragma unroll
    for (int g = 0; g < 16; ++g) HW(g) = L[g];
#pragma unroll
    for (int j = 0; j < 6; ++j) {
        const unsigned x = bfi(0x83838383u, G[2 * j], ABIT(G[2 * j + 1], G[2 * j + 1], 29));
        HW(16 + j) = j < 4 ? bfi(0x60606060u, G[12 + j] << 5, x) : bfi(0x20202020u, G[2 * j + 4] >> 2, bfi(0x40404040u, G[2 * j + 5] >> 1, x));
    }
    h[5][2] = cw[0]; h[5][3] = cw[1]; h[6] = (u4){cw[2], cw[3], f[0], f[1]}; h[7][0] = c0; h[7][1] = c1;
    return (r0 + r2 + 7u) >> 3;
}
template <int N> __device__ __forceinline__ void deposit(unsigned* o, const u2* e, unsigned tab, unsigned pos) {
    u2 x[N]; unsigned p[N];
#pragma unroll
    for (int g = 0; g < N; ++g) {
        const LAS unsigned* d = (const LAS unsigned*)(size_t)(pos & ~3u);
        x[g] = (u2){d[0], d[1]}; p[g] = pos;
        pos += e[g][1];
    }
#pragma unroll
    for (int g = 0; g < N; ++g) o[g] = PERM(tab, __builtin_amdgcn_alignbyte(x[g][1], x[g][0], p[g]), e[g][0]);
}
__device__ __forceinline__ void unpack_unit(u4 (&h)[8], LAS char* scr, const LAS char* lut, int lane) {
    LAS char* t = scr + SCR_T;
    *(LAS u2*)(t + lane * 8) = (u2){h[6][2], h[6][3]};
    FENCE();
    const unsigned hd = __builtin_amdgcn_readfirstlane(h[6][2]), r0 = (hd >> 8) & 63u, type = hd & 32u;
    const unsigned kb = (((hd & 31u) << 2) | 0x80u) * 0x01010101u;
    const LAS unsigned* d3 = (const LAS unsigned*)(t + ((4 + lane * 3) & ~3));
    const unsigned w0 = d3[0], w1 = d3[1];
    unsigned le = 0;
    const bool ll = (unsigned)lane < ((hd >> 16) & 127u);
    if (ll) le = ((const LAS unsigned short*)t)[(hd >> 24) + lane];
    const unsigned c0 = (unsigned)lane < r0 ? 0u : h[6][2], c1 = (unsigned)lane < r0 ? 0u : h[6][3];
    const unsigned cw[4] = {h[5][2], h[5][3], h[6][0], h[6][1]};
    u2 e2[8], e1[16];
#pragma unroll
    for (int g = 0; g < 8; ++g) e2[g] = *(const LAS u2*)(lut + (((g < 4 ? c0 : c1) >> (8 * (g & 3))) & 255u) * 8u);
#pragma unroll
    for (int g = 0; g < 16; ++g) e1[g] = *(const LAS u2*)(lut + ((cw[g >> 2] >> (8 * (g & 3))) & 255u) * 8u);
    const unsigned n2 = escapes(c1, escapes(c0, 0u)), n1 = escapes(cw[3], escapes(cw[2], escapes(cw[1], escapes(cw[0], 0u))));
    const unsigned n = n1 | (n2 << 16), sc = wave_scan(n) - n;
    unsigned w = __builtin_amdgcn_alignbyte(w1, w0, (unsigned)lane * 3u);
    w = bfi(0x003F3F3Fu, w, __builtin_amdgcn_udot4(w & 0xC0C0C0u, 0x100401u, 0u, false) << 18);
    const unsigned t3l = (kb - (type ? 0x24201C18u : 0x24201C00u)) & 0x7C7C7C7Cu, t3h = (kb - 0x34302C28u) & 0x7C7C7C7Cu;
    *(LAS u2*)(scr + SCR_3 + lane * 8) = (u2){PERM(t3h, t3l, w & 0x07070707u), PERM(t3h, t3l, (w >> 3) & 0x07070707u)};
    FENCE();
    if (ll) scr[SCR_3 + (le >> 7)] = (char)(le & 0x7Cu);
    FENCE();
    unsigned E[16];
    deposit<8>(E, e2, (kb - (type ? 0x001410u : 0x181410u)) & 0x7C7C7C7Cu, (sc >> 16) + (unsigned)(size_t)(scr + SCR_3));
    FENCE();
    *(LAS u4*)(scr + lane * 32) = (u4){E[0], E[1], E[2], E[3]}; *(LAS u4*)(scr + lane * 32 + 16) = (u4){E[4], E[5], E[6], E[7]};
    FENCE();
    deposit<16>(E, e1, (kb - 0x04080Cu) & 0x7C7C7C7Cu, (sc & 0xFFFFu) + r0 * 32u + (unsigned)(size_t)scr);
    FENCE();
    unsigned L[16];
#pragma unroll
    for (int g = 0; g < 16; ++g) {
        L[g] = HW(g);
        const unsigned x = HW(16 + (g < 12 ? g >> 1 : g - 12));
        E[g] = bfi(0x7C7C7C7Cu, E[g], g >= 12 ? bfi(0x80808080u, HW(14 + (g >> 1)) << (2 - (g & 1)), x >> 5) : (g & 1) ? ABIT(x, x, 3) : x);
    }
#pragma unroll
    for (int g = 0; g < 16; ++g) { HW(2 * g) = PERM(E[g], L[g], 0x05010400u); HW(2 * g + 1) = PERM(E[g], L[g], 0x07030602u); }
}
constexpr unsigned LIST_MAX = 126;
__device__ __forceinline__ unsigned pack_out(u4 (&h)[8], LAS char* scr, int lane) {
    unsigned es = 0;
#pragma unroll
    for (int j = 0; j < 4; ++j) es += ((h[0][j] >> 23) & 255u) + ((h[4][j] >> 23) & 255u);
    int e0 = (wave_sum(es) - 2355 + 511) >> 9;
    e0 = e0 < 0 ? 0 : e0 > 248 ? 248 : e0;
    const unsigned base = (unsigned)e0 << 24;
    unsigned n = 0;
#pragma unroll
    for (int i = 0; i < 8; ++i) {
#pragma unroll
        for (int j = 0; j < 4; ++j) h[i][j] = ABIT(h[i][j], h[i][j], 31) - base;
        unsigned t = __builtin_amdgcn_perm(__builtin_amdgcn_perm(h[i][3], h[i][2], 0x0C0C0703u), __builtin_amdgcn_perm(h[i][1], h[i][0], 0x0C0C0703u), 0x05040100u) & 0xF8F8F8F8u;
        for (;;) {
            const unsigned long long m = __builtin_amdgcn_ballot_w64(t != 0);
            if (m == 0 || n > LIST_MAX) break;
            if (t != 0) {
                const unsigned b = (unsigned)__builtin_ctz(t) & 24u;
                const unsigned k = n + 2u + __builtin_amdgcn_mbcnt_hi((unsigned)(m >> 32), __builtin_amdgcn_mbcnt_lo((unsigned)m, 0u));
                ((LAS unsigned short*)scr)[k] = (unsigned short)(((unsigned)((i >> 2) * 1024 + lane * 16 + (i & 3) * 4) << 5) + (b << 2) + ((t >> (b + 3u)) & 31u));
                t &= ~(255u << b);
            }
            n += (unsigned)__builtin_popcountll(m);
        }
    }
    if (n > LIST_MAX) {
#pragma unroll
        for (int k = 0; k < 32; ++k) { const unsigned x = HW(k) + base; HW(k) = ABIT(x, x, 1); }
        return M_RAW;
    }
    asm volatile("" ::: "memory");
    unsigned l = *(const LAS unsigned*)(scr + lane * 4);
    asm volatile("" ::: "memory");
    if (lane == 0) l = ((unsigned)e0 << 16) | n;
    unsigned lo = 0;
#pragma unroll
    for (int p = 0; p < 5; ++p) {
        lo = ABIT(HW(27 + p), lo, 2);
#pragma unroll
        for (int j = 0; j < 5; ++j) HW(5 * p + j) = ABIT(HW(5 * p + j), HW(27 + p) << (25 - 5 * j), 27);
    }
    HW(25) = ABIT(HW(25), lo << 5, 27); HW(26) = ABIT(HW(26), lo, 27);
    h[6][3] = l;
    return (n + 33u) >> 5;
}
__device__ __forceinline__ void unpack_out(u4 (&h)[8], unsigned lst, LAS char* scr, int lane) {
    const unsigned hd = __builtin_amdgcn_readfirstlane(lst);
    const unsigned n = hd & 0xFFFFu, base = (hd >> 16) << 24;
    *(LAS u4*)(scr + lane * 16) = (u4){0u, 0u, 0u, 0u};
    *(LAS u4*)(scr + 1024 + lane * 16) = (u4){0u, 0u, 0u, 0u};
    asm volatile("" ::: "memory");
    if ((unsigned)(2 * lane - 2) < n) scr[(lst >> 5) & 2047u] = (char)(lst & 31u);
    if ((unsigned)(2 * lane - 1) < n) scr[lst >> 21] = (char)((lst >> 16) & 31u);
    asm volatile("" ::: "memory");
    const u4 s0 = *(const LAS u4*)(scr + lane * 16), s1 = *(const LAS u4*)(scr + 1024 + lane * 16);
    asm volatile("" ::: "memory");
    const unsigned lo = ABIT(HW(26), HW(25) << 27, 5);
#pragma unroll
    for (int p = 0; p < 5; ++p) {
        unsigned y = lo << (8 - 2 * p);
#pragma unroll
        for (int j = 0; j < 5; ++j) y = ABIT(HW(5 * p + j), y, 5);
        HW(27 + p) = y;
    }
#pragma unroll
    for (int k = 0; k < 32; ++k) {
        const unsigned x = ABIT((k < 16 ? s0 : s1)[(k >> 2) & 3] >> (8 * (k & 3)), HW(k), 5) + base;
        HW(k) = ABIT(x, x, 1);
    }
}

__device__ __forceinline__ void unit_decode(int u, int& kind, int& ch, int& g) {
    constexpr int U0 = 56 * SKC, U1 = U0 + 64 * (64 - SKC);
    if (u < U0) { const int s = u / 7; kind = u - 7 * s; ch = s >> 3; g = s & 7; if (ch < 8 && kind < 5) ch = (CHMAP >> (4 * ch)) & 7; }
    else if (u < U1) { const int v = u - U0; kind = v & 7; g = (v >> 3) & 7; ch = (v >> 6) + (kind == 7 ? 0 : SKC); }
    else { const int v = u - U1; kind = 7; g = v & 7; ch = 64 - SKC + (v >> 3); }
}
__device__ __forceinline__ void fetch_input(const CAS Params& P, LAS char* buf, int u, int lane) {
    int kind, ch, g;
    unit_decode(u, kind, ch, g);
    const bool isx = kind == 0 || kind > 4;
    const char* src = isx ? (const char*)P.x + (size_t)((unsigned)((kind ? kind - 4 : 0) * 8 + g) * (32 * KDIM * 4) + (unsigned)ch * 512)
                          : (const char*)P.w + (size_t)((unsigned)(2 * ch + (g >> 2)) * (64 * NLOC * 4) + (unsigned)((kind - 1) * 4 + (g & 3)) * 256);
    const unsigned s0 = isx ? 64 : NLOC * 4, s1 = isx ? 16 * KDIM * 4 : 16 * NLOC * 4, s2 = isx ? 256 : 32 * NLOC * 4;
    const unsigned voff = isx ? (unsigned)(lane & 15) * (KDIM * 4) + (unsigned)(lane >> 4) * 16 : (unsigned)(lane >> 4) * (4 * NLOC * 4) + (unsigned)(lane & 15) * 16;
#pragma unroll
    for (int i = 0; i < 16; ++i)
        __builtin_amdgcn_global_load_lds((const GAS void*)(src + ((i & 3) * s0 + ((i >> 2) & 1) * s1 + (i >> 3) * s2) + voff), (LAS void*)(buf + i * 1024), 16, 0, 0);
}
#define TOUCHU 448
__device__ __forceinline__ void touch(const CAS Params& P, char* myws, LAS char* scr, int u, int lane) {
    int ch, g, kind;
    unit_decode(u, kind, ch, g);
    const bool isx = kind == 0 || kind > 4;
    const int rb = (kind ? kind - 4 : 0) * 8 + g, kt = 2 * ch + (g >> 2), cb = (kind - 1) * 4 + (g & 3);
    const int pk = isx ? ch * 32 + rb : PK_W + (kind - 1) * 512 + ch * 8 + g;
    unsigned off, fl;
    if (isx) place_x(rb < 24 ? 3 * (P.rank & 3) + (rb >> 3) : 16, rb, ch, lane & 1, off, fl); else place_w(cb, kt, off, fl);
    const unsigned a = lane & 2 ? off : (unsigned)pk_off(pk, lane & 1);
    if (lane < 4) __builtin_amdgcn_global_load_lds((const GAS void*)(myws + a), (LAS void*)scr, 4, 0, 17);
}
__device__ __forceinline__ void pace(unsigned long long t0, int u) {
    if (u < UG) while (__builtin_amdgcn_s_memrealtime() - t0 < (unsigned)(u < U1ST ? u * PACE1 : (u - U1ST) * PACE + U1ST * PACE2) >> 4) __builtin_amdgcn_s_sleep(2);
}
__device__ __forceinline__ int next_unit(int u, int wid) {
    if (u >= UG) return u + 8 * NPROD;
    u = u < 8 * NGEMM && blockIdx.x < NB2 ? 8 * NGEMM + wid * NB2 + (int)blockIdx.x : UG;
    return u < UG ? u : PK_O;
}
__device__ __forceinline__ void producer(const CAS Params& P, char* myws, LAS char* buf, LAS char* lut, int u, unsigned long long t0, unsigned call, int wid, int lane) {
    const int q = P.rank & 3;
    const unsigned vl16 = (unsigned)lane * 16;
    LAS char* scr = buf + 16384;
#define PT(i) do { } while (0)
    unsigned pfo = 0, pfv = 0;
    for (int un; u < PK_O; u = un) {
        un = next_unit(u, wid);
        int ch, g, kind;
        unit_decode(u, kind, ch, g);
        const bool isx = kind == 0 || kind > 4;
        const int rb = (kind ? kind - 4 : 0) * 8 + g, kt = 2 * ch + (g >> 2), cb = (kind - 1) * 4 + (g & 3);
        const int idx = isx ? pos_x(rb, ch) : pos_w(kt, g & 3), pk = isx ? ch * 32 + rb : PK_W + (kind - 1) * 512 + ch * 8 + g;
        const int rd = !isx ? kind - 1 : rb == 16 || rb >= 24 ? 5 : 4;
        f4 v[16]; u4 h[8];
        unsigned hv;
        asm volatile("global_load_dword %0, %1, %2 sc0 sc1" : "=&v"(hv) : "v"((unsigned)OFF_MAP + ((call & 1) * PK_PG + pk / 127) * MAPS), "s"(myws) : "memory");
#pragma unroll
        for (int i = 0; i < 16; ++i) v[i] = *(const LAS f4*)(buf + i * 1024 + lane * 16);
        __builtin_amdgcn_sched_barrier(0);
        WAIT_L0();
        const bool ahead = u >= AHEAD && un < PK_O;
        PT(0);
        if (ahead) { pace(t0, un); fetch_input(P, buf, un, lane); }
        PT(1);
        __builtin_amdgcn_sched_barrier(0);
        if (isx) {
#pragma unroll
            for (int i = 0; i < 8; ++i)
                h[i] = pack8(v[2 * i][0], v[2 * i][1], v[2 * i][2], v[2 * i][3], v[2 * i + 1][0], v[2 * i + 1][1], v[2 * i + 1][2], v[2 * i + 1][3]);
        } else {
#pragma unroll
            for (int i = 0; i < 8; ++i) {
                const int f = i >> 1, k = (i & 1) * 8;
                h[i] = pack8(v[k][f], v[k + 1][f], v[k + 2][f], v[k + 3][f], v[k + 4][f], v[k + 5][f], v[k + 6][f], v[k + 7][f]);
            }
        }
        unsigned off, fl;
        if (isx) place_x(rb < 24 ? 3 * q + (rb >> 3) : 16, rb, ch, call, off, fl); else place_w(cb, kt, off, fl);
        const bool own = !isx || rb != 16;
        const unsigned long long h11 = NOW();
        if (idx < NPUSH) {
            unsigned off2 = off, fl2 = fl;
            if (!isx) place_w(16 + 4 * q + (g & 3), kt, off2, fl2);
            while (__builtin_amdgcn_s_memrealtime() - t0 < (unsigned)(PT0 + idx * PDT)) __builtin_amdgcn_s_sleep(1);
            char* fp = myws + fl;
            const unsigned long long h14 = NOW();
            for (int i = 1; i < (isx ? 4 : 2); ++i) {
                char* pw = P.ws[isx ? (P.rank & 4) + ((q + i) & 3) : ((P.rank & 4) ^ 4) + rd];
                store8(h, pw + off2, isx ? 32768u : 4096u, vl16);
                fp = lane == i ? pw + fl2 : fp;
            }
            store8(h, myws + off, isx ? 32768u : 4096u, vl16);
            asm volatile("s_waitcnt vmcnt(0)" : "+v"(hv) :: "memory");
            if (lane < (isx ? 4 : 2)) flag_store(fp, call);
            HST(12, NOW()); HST(11, h11); HST(14, h14); HST(13, (unsigned long long)(rd * 4096 + idx + 1));
            if (un < PK_O) { pace(t0, un); fetch_input(P, buf, un, lane); WAIT_V(0); }
            continue;
        }
        if (own) store8(h, myws + off, isx ? 32768u : 4096u, vl16);
        unsigned mode = M_RAW, r0 = 0, r2 = 0;
        PT(2);
        if (idx >= RAW0 && (rd != 4 || idx < NI - RAW1)) mode = pack_unit(h, scr, lut, lane, r0, r2);
        PT(3);
        const unsigned long long h14 = NOW();
        if (own) asm volatile("s_waitcnt vmcnt(8)" : "+v"(hv) :: "memory"); else asm volatile("s_waitcnt vmcnt(0)" : "+v"(hv) :: "memory");
        PT(4);
        if (pfo) flag_store(myws + pfo, pfv);
        const unsigned half = SMODE == 0 ? idx & 1 : (__builtin_amdgcn_readfirstlane(hv) ^ (rd != 4) ^ (SMODE == 2)) & 1u;
        char* d0 = myws + (pk_off(pk, call) + half * 8192); char* d1 = d0 + 4096;
        GST4(vl16, h[0], d0, 0); GST4(vl16, h[1], d0, 1024); GST4(vl16, h[2], d0, 2048); GST4(vl16, h[3], d0, 3072);
        GST4(vl16, h[4], d1, 0); GST4(vl16, h[5], d1, 1024);
        if (mode < M_RAW) {
            const u2 p0 = {h[6][0], h[6][1]}, p1 = {h[6][2], h[6][3]}, p2 = {h[7][0], h[7][1]};
            asm volatile("global_store_dwordx2 %0, %1, %4 offset:2048 sc0 sc1\n\t"
                         "s_mov_b64 exec, %6\n\tglobal_store_dwordx2 %0, %2, %4 offset:2560 sc0 sc1\n\t"
                         "s_mov_b64 exec, %7\n\tglobal_store_dwordx2 %0, %3, %5 offset:2560 sc0 sc1\n\ts_mov_b64 exec, -1"
                         :: "v"(vl16 >> 1), "v"(p0), "v"(p1), "v"(p2), "s"(d1), "s"(d1 + r0 * 8u), "s"(lane_mask(r0)), "s"(lane_mask(r2)) : "memory");
        } else { GST4(vl16, h[6], d1, 2048); GST4(vl16, h[7], d1, 3072); }
        pfo = lane < 5 ? (unsigned)OFF_FP + (call & 1) * (unsigned)FP_B + (unsigned)(lane * NSEQ + idx) * 4u : fl;
        pfv = lane < 5 ? call * 32u + half * 16u + mode : call;
        if (!((rd == 5 ? lane < 4 : lane == rd) || (own && lane == 5))) pfo = 0;
        if (!PDEF || !ahead) {
            WAIT_V(0);
            if (pfo) flag_store(myws + pfo, pfv);
            pfo = 0;
        }
        HST(12, NOW()); HST(11, h11); HST(14, h14); HST(13, (unsigned long long)(rd * 4096 + idx + 1));
        if (!ahead && un < PK_O) { pace(t0, un); fetch_input(P, buf, un, lane); WAIT_V(0); }
        PT(5);
    }
}

__device__ __forceinline__ void flag_mirror(char* myws, const char* fp, int nseq, LAS unsigned* info, LAS unsigned* prog, unsigned call, int lane) {
    GAS unsigned* tmo = (GAS unsigned*)(myws + OFF_TMO);
    int pos = NPUSH;
    bool all = false;
    unsigned long long t0 = __builtin_amdgcn_s_memrealtime();
    while (pos < nseq) {
        const int pr = (int)__builtin_amdgcn_readfirstlane(__hip_atomic_load(prog, __ATOMIC_RELAXED, __HIP_MEMORY_SCOPE_WORKGROUP));
        if (pos <= pr + (all ? LOOK : LOOK2)) {
            const int lo = pos & ~31, hi = all ? lo + 32 * MALL : ((pr + LOOK2) | 31) + 1, i = lo + 4 * lane;
            bool bad = i >= hi;
            if (!bad && i < nseq) {
                u4 f = *(const LAS u4*)(info + i);
                if (i >= pos && (f[0] < call * 32u || f[1] < call * 32u || f[2] < call * 32u || f[3] < call * 32u)) {
                    asm volatile("global_load_dwordx4 %0, %1, %2 sc0 sc1\n\ts_waitcnt vmcnt(0)" : "=&v"(f) : "v"(i * 4), "s"(fp) : "memory");
                    *(LAS u4*)(info + i) = f;
                    bad = f[0] < call * 32u || f[1] < call * 32u || f[2] < call * 32u || f[3] < call * 32u;
                }
            }
            const unsigned long long m = __builtin_amdgcn_ballot_w64(bad);
            const int n = lo + 4 * (m ? __builtin_ctzll(m) : 64);
            all = n >= hi;
            if (n > pos) { pos = n; t0 = __builtin_amdgcn_s_memrealtime(); if (all) continue; }
        }
        if (
            __builtin_amdgcn_s_memrealtime() - t0 > DEADLINE) {
            __hip_atomic_store(tmo, call, __ATOMIC_RELAXED, __HIP_MEMORY_SCOPE_AGENT);
            asm volatile("s_waitcnt vmcnt(0)" ::: "memory");
            for (int i = pos + lane; i < nseq; i += 64) info[i] = call * 32u + M_DEAD;
            return;
        }
        if (pos > pr) __builtin_amdgcn_s_sleep(MSL);
    }
}

#define STR2(x) #x
#define STR(x) STR2(x)
#define SP ""
#define RD ""
#define PULL6 "global_load_dwordx4 %[h0], %[v16], %[src] offset:-4096 sc0 sc1\n\t" SP "global_load_dwordx4 %[h1], %[v16], %[src] offset:-3072 sc0 sc1\n\t" SP \
              "global_load_dwordx4 %[h2], %[v16], %[src] offset:-2048 sc0 sc1\n\t" SP "global_load_dwordx4 %[h3], %[v16], %[src] offset:-1024 sc0 sc1\n\t" SP \
              "global_load_dwordx4 %[h4], %[v16], %[src] sc0 sc1\n\t" SP "global_load_dwordx4 %[h5], %[v16], %[src] offset:1024 sc0 sc1\n\t" SP
#define PULLT "s_mov_b64 exec, 1\n\tglobal_atomic_add %[tk], %[ao], %[one], %[ws] sc0\n\t" \
              "s_waitcnt vmcnt(%[rn] + 9 - " STR(RELAT) ")\n\tglobal_store_dword %[pf], %[cl], %[ws] sc0 sc1\n\ts_waitcnt vmcnt(%[rn] + 1)\n\t" RD \
              "ds_add_u32 %[rel], %[one]\n\ts_mov_b64 exec, -1\n\ts_waitcnt vmcnt(1)"
#define PULLO [h0] "=&v"(h[0]), [h1] "=&v"(h[1]), [h2] "=&v"(h[2]), [h3] "=&v"(h[3]), [h4] "=&v"(h[4]), [h5] "=&v"(h[5]), [tk] "=&v"(tk)
#define PULLI [v16] "v"(vl16), [src] "s"(src), [ao] "v"(aoff), [one] "v"(one), [ws] "s"(myws), [rel] "v"(rel), [pf] "v"(pfl), [cl] "v"(call)
#undef VBLK
#define VBLK (NGEMM + vg)
__device__ __forceinline__ void puller(int vg, const CAS Params& P, char* myws, LAS unsigned* info, LAS unsigned* prog, const LAS unsigned short* tab,
                                       LAS char* scr, const LAS char* lut, int link, int peer, int cur, int nw, unsigned win, unsigned call, int wid, int lane) {
    const int q = P.rank & 3, qs = peer & 3;
    const bool cross = link < 4;
    const int nseq = cross ? NSEQ : NI;
    const char* pk = P.ws[peer] + 4096;
    const unsigned vl16 = (unsigned)lane * 16, one = 1, aoff = (unsigned)OFF_CTR + (unsigned)link * 128u;
    const unsigned ovoff = (unsigned)(lane >> 4) * (4 * NLOC * 4) + (unsigned)(lane & 15) * 16;
    const unsigned base = (call - 1u) * (unsigned)(nseq - NPUSH) - (unsigned)(nw + NPUSH);
    const unsigned rel = (unsigned)(size_t)(prog + 2);
    const unsigned nofl = (unsigned)OFF_SEQ + (blockIdx.x * 8u + (unsigned)wid) * 16u + 4u;
    unsigned pfl = nofl;
    const unsigned tz = __builtin_amdgcn_readfirstlane(prog[4]);
    if (TOUCH & 2) {
        const int id = __builtin_amdgcn_readfirstlane((int)tab[cur]), rb = id & 31;
        unsigned off, fl, d;
        if (id < PK_W) place_x(rb < 24 ? 3 * qs + (rb >> 3) : 12 + qs, rb, id >> 5, call, off, fl);
        else place_w(16 + 4 * qs + (id & 3), (id - PK_W) >> 2, off, fl);
        const char* a = lane ? myws + off : pk + pk_off(id < PK_W ? id : id + q * 512, call);
        if (lane < 2) asm volatile("global_load_dword %0, %1, off sc0 sc1\n\ts_waitcnt vmcnt(0)" : "=&v"(d) : "v"(a) : "memory");
    }
    for (;;) {
        const bool isout = cross && cur >= NCI;
        const int id = isout ? 0 : __builtin_amdgcn_readfirstlane((int)tab[cur]);
        const int slot = isout ? PK_O + q * NO + cur - NCI : id < PK_W ? id : id + q * 512;
        unsigned mode;
        if (pfl != nofl) {
            const unsigned i0 = __hip_atomic_load(info + cur, __ATOMIC_RELAXED, __HIP_MEMORY_SCOPE_WORKGROUP);
            const unsigned i1 = __hip_atomic_load(prog + 1, __ATOMIC_RELAXED, __HIP_MEMORY_SCOPE_WORKGROUP), i2 = __hip_atomic_load(prog + 2, __ATOMIC_RELAXED, __HIP_MEMORY_SCOPE_WORKGROUP);
            if (__builtin_amdgcn_readfirstlane(!DEFER || i0 - call * 32u > 31u || i1 - i2 >= win)) {
                WAIT_V(0);
                if (lane == 0) flag_store(myws + pfl, call);
                pfl = nofl;
            }
        }
        for (;;) {
            mode = __builtin_amdgcn_readfirstlane(__hip_atomic_load(info + cur, __ATOMIC_ACQUIRE, __HIP_MEMORY_SCOPE_WORKGROUP)) - call * 32u;
            if (mode <= 31u) break;
            __builtin_amdgcn_s_sleep(1);
        }
        const char* src = pk + (pk_off(slot, call) + (mode & 16u) * 512u);
        mode &= 15u;
        const unsigned long long mask = lane_mask(isout ? mode * 16u : mode * 8u);
        unsigned my = 0;
        if (lane == 0) my = __hip_atomic_fetch_add(prog + 1, 1u, __ATOMIC_RELAXED, __HIP_MEMORY_SCOPE_WORKGROUP);
        my = __builtin_amdgcn_readfirstlane(my) - win - (cur < HWIN && cur >= HWIN0);
        while ((int)(my - __builtin_amdgcn_readfirstlane(__hip_atomic_load(prog + 2, __ATOMIC_RELAXED, __HIP_MEMORY_SCOPE_WORKGROUP))) >= 0) { }
        {
            unsigned ts = 0;
            if (lane == 0) {
                __hip_atomic_fetch_max(prog + 3, ((unsigned)__builtin_amdgcn_s_memrealtime() - tz) << 8, __ATOMIC_RELAXED, __HIP_MEMORY_SCOPE_WORKGROUP);
                ts = __hip_atomic_fetch_add(prog + 3, (mode >= M_RAW ? 128u : isout ? 108u + mode : 104u + mode) * (cross ? PRATE : PRATEI), __ATOMIC_RELAXED, __HIP_MEMORY_SCOPE_WORKGROUP);
            }
            ts = __builtin_amdgcn_readfirstlane(ts);
            ts = (ts >> 8) + tz - PSLACK;
            while ((int)((unsigned)__builtin_amdgcn_s_memrealtime() - ts) < 0) { }
        }
        u4 h[8]; unsigned tk, lst = 0;
        const unsigned tis = (unsigned)__builtin_amdgcn_s_memrealtime();
        if (mode >= M_RAW) {
            asm volatile(PULL6 "global_load_dwordx4 %[h6], %[v16], %[src] offset:2048 sc0 sc1\n\tglobal_load_dwordx4 %[h7], %[v16], %[src] offset:3072 sc0 sc1\n\t" PULLT
                         : PULLO, [h6] "=&v"(h[6]), [h7] "=&v"(h[7]) : PULLI, [rn] "n"(RELAT) : "memory");
        } else if (!isout) {
            u2 p0, p1;
            asm volatile(PULL6 "global_load_dwordx2 %[p0], %[v8], %[src] offset:2048 sc0 sc1\n\t"
                         "s_mov_b64 exec, %[mask]\n\tglobal_load_dwordx2 %[p1], %[v8], %[src] offset:2560 sc0 sc1\n\t" PULLT
                         : PULLO, [p0] "=&v"(p0), [p1] "=&v"(p1)
                         : PULLI, [v8] "v"(vl16 >> 1), [mask] "s"(mask), [rn] "n"(RELAT) : "memory");
            h[6] = (u4){p0[0], p0[1], p1[0], p1[1]};
        } else {
            u2 p2; unsigned p1;
            asm volatile(PULL6 "global_load_dwordx2 %[p2], %[v8], %[src] offset:2048 sc0 sc1\n\tglobal_load_dword %[p1], %[v4], %[src] offset:2560 sc0 sc1\n\t"
                         "s_mov_b64 exec, %[mask]\n\tglobal_load_dword %[lst], %[v4], %[src] offset:2816 sc0 sc1\n\t" PULLT
                         : PULLO, [p2] "=&v"(p2), [p1] "=&v"(p1), [lst] "=&v"(lst)
                         : PULLI, [v8] "v"(vl16 >> 1), [v4] "v"(vl16 >> 2), [mask] "s"(mask), [rn] "n"(RELAT + 1) : "memory");
            h[6][0] = p2[0]; h[6][1] = p2[1]; h[6][2] = p1;
        }
        const unsigned lat = (unsigned)__builtin_amdgcn_s_memrealtime() - tis;
        const unsigned nxt = __builtin_amdgcn_readfirstlane(tk) - base;
        if (lane == 0) __hip_atomic_fetch_max(prog, nxt, __ATOMIC_RELAXED, __HIP_MEMORY_SCOPE_WORKGROUP);
        if (isout) {
            const int o = cur - NCI;
            int panel, blk, col;
            if (o < O_B) {
                const int r = o < O_A0 ? o : o < O_C ? (o - O_A0) & 127 : o - O_C, hn = o < O_A0 ? 16 : o < O_C ? 64 : 48;
                const int hf = r >= hn, t = (r - hf * hn) >> 3;
                panel = o < O_A0 ? 3 * qs + 2 : o < O_C ? 3 * (t >> 1) + (o >= O_A1) : 12 + ((q + 1 + (t >> 1)) & 3);
                blk = (r & 6) + hf; col = (t & 1) * 128 + (r & 1) * 64;
            } else {
                const int j = (o - O_B) / 24, r = o - O_B - j * 24;
                panel = 3 * ((qs + 1 + (r >> 3)) & 3) + 2; blk = 2 * j + ((r >> 1) & 1); col = ((r >> 2) & 1) * 128 + (r & 1) * 64;
            }
            char* ob = (char*)P.out + (size_t)(panel_row(peer, panel, blk) * NLOC + qs * 256 + col) * 4;
            if (mode < M_RAW) unpack_out(h, lst, scr, lane);
            if (mode == M_DEAD) h[0] = (u4){0x7FC00000u, 0x7FC00000u, 0x7FC00000u, 0x7FC00000u};
#pragma unroll
            for (int k = 0; k < 8; ++k) GST4P(ovoff, h[k], ob + ((k >> 2) * 16 + (k & 3)) * (NLOC * 4));
            if (!DEFER) WAIT_V(0);
            pfl = nofl;
        } else {
            if (mode < M_RAW) unpack_unit(h, scr, lut, lane);
            unsigned off, fl;
            const int rb = id & 31;
            if (id < PK_W) place_x(rb < 24 ? 3 * qs + (rb >> 3) : 12 + qs, rb, id >> 5, call, off, fl);
            else place_w(16 + 4 * qs + (id & 3), (id - PK_W) >> 2, off, fl);
            store8(h, myws + off, id < PK_W ? 32768u : 4096u, vl16);
            pfl = fl;
        }
        if (nxt >= (unsigned)nseq) break;
        cur = (int)nxt;
    }
    WAIT_V(0);
    if (lane == 0) flag_store(myws + pfl, call);
}

#undef VBLK
#define VBLK (blockIdx.x < NGEMM ? blockIdx.x : blockIdx.x + NPC)
__device__ __forceinline__ float silu(float y) {
    const float c = __builtin_amdgcn_fmed3f(y, -60.0f, 60.0f);
    const float e = __builtin_amdgcn_exp2f(c * -1.4426950408889634f);
    return y * __builtin_amdgcn_rcpf(1.0f + e);
}
__device__ __forceinline__ u4 silu4(const f4 (&a)[4], int r, bool dead) {
    f4 o = {silu(a[0][r]), silu(a[1][r]), silu(a[2][r]), silu(a[3][r])};
    const float nanv = __builtin_nanf("");
    if (dead) o = (f4){nanv, nanv, nanv, nanv};
    return __builtin_bit_cast(u4, o);
}

#define BAR() do { __builtin_amdgcn_sched_barrier(0); __builtin_amdgcn_s_barrier(); asm volatile("" ::: "memory"); __builtin_amdgcn_sched_barrier(0); } while (0)
#define GLDS(g, l, imm) __builtin_amdgcn_global_load_lds((const GAS void*)(g), (LAS void*)(l), 16, imm, AUX)
template <int V> struct IC { static constexpr int value = V; };

__device__ __forceinline__ void gemm_panel(const CAS Params& P, LAS char* lds, char* myws, int panel, int cg, int o0, int ohalf,
                                           unsigned call, int wid, int lane) {
    constexpr int ABYTES = 32768, STAGE = ABYTES + 16384;
    const char* ga = myws + panel_off(panel, call) + wid * 4096 + lane * 16;
    const char* gb = myws + OFF_DATA + A_B + (size_t)cg * CG_B + wid * 2048 + lane * 16;
    const int wm = wid >> 1, wn = wid & 1;
    const unsigned frow = (lane & 2) ? (unsigned)OFF_FB + (unsigned)cg * 1024u : (unsigned)OFF_FA + (unsigned)panel * 2048u + (unsigned)(lane & 1) * 16u;
    const unsigned fstep = (lane & 2) ? 16u : 32u;
    GAS unsigned* tmo = (GAS unsigned*)(myws + OFF_TMO);
    char* oflag = myws + (OFF_FP + (call & 1) * FP_B) + (((cg - 8) >> 1) * NSEQ + NCI + o0 + wid) * 4;
    int ready = 0;
    unsigned tv = 0;

    auto early = [&](int c) {
        if (c < ready || c >= NCH) return;
        const int cc = c + (lane >> 2);
        __builtin_amdgcn_global_load_lds((const GAS void*)(myws + (frow + (unsigned)(cc < NCH ? cc : NCH - 1) * fstep)), (LAS void*)(lds + 3 * STAGE), 16, 0, AUX);
    };
    auto ensure = [&](int c, bool pend) {
        if (c < ready) return;
        const unsigned long long t0 = __builtin_amdgcn_s_memrealtime();
        int lim = 15;
        for (;;) {
            int cc = c + ((lane >> 2) < lim ? (lane >> 2) : lim); cc = cc < NCH ? cc : NCH - 1;
            u4 f;
            if (pend) asm volatile("s_waitcnt vmcnt(%2)\n\tds_read_b128 %0, %1\n\ts_waitcnt lgkmcnt(0)"
                                   : "=&v"(f) : "v"((unsigned)(size_t)(lds + 3 * STAGE) + (unsigned)lane * 16u), "n"(3 * PD) : "memory");
            else asm volatile("global_load_dwordx4 %0, %1, %2 sc0 sc1\n\ts_waitcnt vmcnt(0)"
                              : "=&v"(f) : "v"(frow + (unsigned)cc * fstep), "s"(myws) : "memory");
            const bool bad = f[0] < call || f[1] < call || f[2] < call || f[3] < call;
            const unsigned long long m = __builtin_amdgcn_ballot_w64(bad);
            const int n = m ? (__builtin_ctzll(m) >> 2) : lim + 1;
            if (n > 0) { ready = c + n; return; }
            if (pend) { pend = false; continue; }
            if (
                __builtin_amdgcn_s_memrealtime() - t0 > DEADLINE) {
                __hip_atomic_store(tmo, call, __ATOMIC_RELAXED, __HIP_MEMORY_SCOPE_AGENT);
                ready = NCH; return;
            }
            lim = 0;
            __builtin_amdgcn_s_sleep(6);
        }
    };
    auto stage = [&](int t, int kf, int bufoff) {
        const char* a = ga + (size_t)t * 32768 + kf * 1024; const char* b = gb + (size_t)t * 16384 + kf * 1024;
        LAS char* la = lds + bufoff + wid * 4096 + kf * 1024; LAS char* lb = lds + bufoff + ABYTES + wid * 2048 + kf * 1024;
        GLDS(a, la, 0); GLDS(a, la, 2048); GLDS(b, lb, 0);
    };

    f4 acc[4][4];
#pragma unroll
    for (int m = 0; m < 4; ++m)
#pragma unroll
        for (int n = 0; n < 4; ++n) acc[m][n] = (f4){0.f, 0.f, 0.f, 0.f};

    const int aoff = wm * 8192 + lane * 16, boff = ABYTES + wn * 8192 + lane * 16;
    auto phase = [&](auto waitn, int rbuf, int kf, int lt, int lkf, int lbuf, int pc, int pe = NCH) {
        const LAS char* pa = lds + rbuf + aoff + kf * 1024; const LAS char* pb = lds + rbuf + boff + kf * 1024;
        h8 a[4], b[4];
#pragma unroll
        for (int n = 0; n < 4; ++n) b[n] = *(const LAS h8*)(pb + n * 2048);
#pragma unroll
        for (int m = 0; m < 4; ++m) a[m] = *(const LAS h8*)(pa + m * 2048);
        __builtin_amdgcn_sched_barrier(0);
        if (lt >= 0) stage(lt, lkf, lbuf);
        if (decltype(waitn)::value >= 0) WAIT_V(decltype(waitn)::value < 0 ? 0 : decltype(waitn)::value);
        WAIT_L0();
        BAR();
        __builtin_amdgcn_s_setprio(1);
#pragma unroll
        for (int m = 0; m < 4; ++m)
#pragma unroll
            for (int n = 0; n < 4; ++n) acc[m][n] = __builtin_amdgcn_mfma_f32_16x16x32_f16(a[m], b[n], acc[m][n], 0, 0, 0);
        __builtin_amdgcn_s_setprio(0);
        __builtin_amdgcn_sched_barrier(0);
        if (wid == 0) { if (pc >= 0) { ensure(pc, PD > 0); TREC(pc); } early(pe); }
        BAR();
    };

    if (wid == 0) { ensure(0, false); ensure(1, false); }
    BAR();
    STAMP(1);
    stage(0, 0, 0); stage(0, 1, 0); stage(1, 0, STAGE); stage(1, 1, STAGE); stage(2, 0, 2 * STAGE);
    WAIT_V(12);
    BAR();
    if (wid >= 4) BAR();
    int rb = 0, pb = 2 * STAGE;
    for (int t = 0; t < NKT - 3; ++t) {
        phase(IC<12>{}, rb, 0, t + 2, 1, pb, (t & 1) ? (t + 3) >> 1 : -1, PD == 2 && !(t & 1) ? (t + 4) >> 1 : NCH);
        phase(IC<12>{}, rb, 1, t + 3, 0, rb, -1, PD == 1 && !(t & 1) ? (t + 4) >> 1 : NCH);
        pb = rb; rb = rb == 2 * STAGE ? 0 : rb + STAGE;
    }
    static_assert((NKT - 3) % 3 == 2, "tail buffers");
    phase(IC<12>{}, 2 * STAGE, 0, NKT - 1, 1, STAGE, -1);
    phase(IC<9>{}, 2 * STAGE, 1, -1, 0, 0, -1);
    phase(IC<6>{}, 0, 0, -1, 0, 0, -1);
    phase(IC<3>{}, 0, 1, -1, 0, 0, -1);
    phase(IC<0>{}, STAGE, 0, -1, 0, 0, -1);
    phase(IC<-1>{}, STAGE, 1, -1, 0, 0, -1);
    if (wid < 4) BAR();
    STAMP(2);
    TDUMP(0);

    const bool dead = __builtin_amdgcn_readfirstlane(__hip_atomic_load(tmo, __ATOMIC_RELAXED, __HIP_MEMORY_SCOPE_AGENT)) == call;
    const unsigned vl16 = (unsigned)lane * 16, ovoff = (unsigned)(lane >> 4) * (4 * NLOC * 4) + (unsigned)(lane & 15) * 16;
    const int pk0 = PK_O + ((cg < 8 ? 0 : cg - 8) >> 1) * NO + o0 + wid;
    const unsigned hv[2] = {pk_half(myws, pk0, call), pk_half(myws, pk0 + ohalf, call)};
    unsigned mode = 0;
#pragma unroll
    for (int u = 0; u < 2; ++u) {
        u4 h[8];
#pragma unroll
        for (int k = 0; k < 8; ++k) h[k] = silu4(acc[2 * u + (k >> 2)], k & 3, dead);
        if (cg < 8) {
            char* ob = (char*)P.out + (size_t)(panel_row(P.rank, panel, 2 * wm + u) * NLOC + cg * 128 + wn * 64) * 4;
#pragma unroll
            for (int k = 0; k < 8; ++k) GST4P(ovoff, h[k], ob + ((k >> 2) * 16 + (k & 3)) * (NLOC * 4));
        } else {
            const unsigned prev = mode;
            if (u && o0 < EARLY) { WAIT_V(0); if (lane == 0) flag_store(oflag, call * 32u + prev); }
            mode = M_RAW;
            if (o0 + u * ohalf >= ORAW) mode = pack_out(h, lds + wid * 2048, lane);
            if (u && o0 >= EARLY) { WAIT_V(0); if (lane == 0) flag_store(oflag, call * 32u + prev); }
            const unsigned half = SMODE == 0 ? wid & 1 : (~__builtin_amdgcn_readfirstlane(hv[u]) ^ (SMODE == 2)) & 1u;
            char* d0 = myws + (pk_off(pk0 + u * ohalf, call) + half * 8192); char* d1 = d0 + 4096;
            GST4(vl16, h[0], d0, 0); GST4(vl16, h[1], d0, 1024); GST4(vl16, h[2], d0, 2048); GST4(vl16, h[3], d0, 3072);
            GST4(vl16, h[4], d1, 0); GST4(vl16, h[5], d1, 1024);
            if (mode < M_RAW) {
                const u2 p2 = {h[6][0], h[6][1]};
                asm volatile("global_store_dwordx2 %0, %1, %2 offset:2048 sc0 sc1\n\tglobal_store_dword %3, %4, %2 offset:2560 sc0 sc1"
                             :: "v"(vl16 >> 1), "v"(p2), "s"(d1), "v"(vl16 >> 2), "v"(h[6][2]) : "memory");
                store_list(d1 + 2816, h[6][3], mode, vl16 >> 2);
            } else {
                GST4(vl16, h[6], d1, 2048);
                GST4(vl16, h[7], d1, 3072);
            }
            mode += half * 16u;
        }
    }
    WAIT_V(0);
    if (cg >= 8 && lane == 0) flag_store(oflag + ohalf * 4, call * 32u + mode);
    WAIT_V(0);
    STAMP(3);
}

__device__ __forceinline__ void gemm_quarter(const CAS Params& P, LAS char* lds, char* myws, int panel, int qt, int cg, int o0,
                                             unsigned call, int wid, int lane) {
    constexpr int STAGE = 49152;
    const int kp = wid >> 2, lw = wid & 3, wm = lw >> 1, wn = lw & 1;
    const char* gsrc = myws + OFF_DATA + lane * 16 + (lw < 2 ? (size_t)panel * PANEL_B + kp * 32768 + qt * 8192
                                                             : A_B + (size_t)cg * CG_B + kp * 16384 + (lw - 2) * 8192);
    const unsigned gstep = lw < 2 ? 65536u : 32768u;
    LAS char* ring = lds + kp * 24576;
    LAS unsigned* pl = (LAS unsigned*)(lds + 160768);
    const unsigned frow = (unsigned)OFF_FA + (unsigned)(panel * 2048 + qt * 8 + (lane & 1) * 4);
    GAS unsigned* tmo = (GAS unsigned*)(myws + OFF_TMO);
    char* oflag = myws + (OFF_FP + (call & 1) * FP_B) + (((cg - 8) >> 1) * NSEQ + NCI + o0) * 4;
    unsigned tv = 0;
    int ready = 0; bool pend = false;
#define QPOLL 8
    auto paddr = [&]() { const int cc = ready + ((lane >> 1) < QPOLL ? (lane >> 1) : QPOLL - 1); return frow + (unsigned)(cc < NCH ? cc : NCH - 1) * 32u; };
    auto late = [&](unsigned tm, unsigned long long t0) {
        if (tm != call && __builtin_amdgcn_s_memrealtime() - t0 <= DEADLINE) return false;
        __hip_atomic_store(tmo, call, __ATOMIC_RELAXED, __HIP_MEMORY_SCOPE_AGENT);
        WAIT_V(0);
        ready = NCH;
        return true;
    };
    auto ensure = [&](int c) {
        if (c >= ready) {
            const unsigned long long t0 = __builtin_amdgcn_s_memrealtime();
            for (;;) {
                unsigned f;
                if (pend) { WAIT_V(0); f = pl[lane]; pend = false; }
                else asm volatile("global_load_dword %0, %1, %2 sc0 sc1\n\ts_waitcnt vmcnt(0)" : "=&v"(f) : "v"(paddr()), "s"(myws) : "memory");
                ready += __builtin_ctzll(__builtin_amdgcn_ballot_w64(f < call) | (1ull << (2 * QPOLL))) >> 1;
                if (c < ready || late(0u, t0)) break;
                __builtin_amdgcn_s_sleep(QSLEEP);
            }
        }
        if (!pend && ready < NCH && ready <= c + 2) {
            __builtin_amdgcn_global_load_lds((const GAS void*)(myws + paddr()), (LAS void*)pl, 4, 0, AUX);
            pend = true;
        }
    };
    auto stage = [&](int c, int buf) {
        const char* g = gsrc + (size_t)((unsigned)c * gstep);
        LAS char* l = ring + buf + (lw - 1) * 8192;
        GLDS(g, l, 0); GLDS(g, l, 1024); GLDS(g, l, 2048); GLDS(g, l, 3072);
        GLDS(g + 4096, l + 4096, 0); GLDS(g + 4096, l + 4096, 1024); GLDS(g + 4096, l + 4096, 2048); GLDS(g + 4096, l + 4096, 3072);
    };

    const int pk0 = PK_O + ((cg < 8 ? 0 : cg - 8) >> 1) * NO + o0;
    if (wid == 0) pl[64] = pk_half(myws, pk0, call);
    f4 acc[2][4];
#pragma unroll
    for (int m = 0; m < 2; ++m)
#pragma unroll
        for (int n = 0; n < 4; ++n) acc[m][n] = (f4){0.f, 0.f, 0.f, 0.f};
    const int aoff = wm * 4096 + lane * 16, boff = 8192 + wn * 8192 + lane * 16;
    auto phase = [&](auto waitn, auto role, int buf, int lc) {
        h8 a[4], b[8];
#pragma unroll
        for (int n = 0; n < 8; ++n) b[n] = *(const LAS h8*)(ring + buf + boff + n * 1024);
#pragma unroll
        for (int m = 0; m < 4; ++m) a[m] = *(const LAS h8*)(ring + buf + aoff + m * 1024);
        __builtin_amdgcn_sched_barrier(0);
        if (decltype(role)::value == 2 && wid == 0) {
            ensure(lc); TREC(lc);
        }
        WAIT_L0();
        BAR();
        if (decltype(role)::value == 1) stage(lc, buf);
#pragma unroll
        for (int k = 0; k < 2; ++k)
#pragma unroll
            for (int m = 0; m < 2; ++m)
#pragma unroll
                for (int n = 0; n < 4; ++n) acc[m][n] = __builtin_amdgcn_mfma_f32_16x16x32_f16(a[m * 2 + k], b[n * 2 + k], acc[m][n], 0, 0, 0);
        if (decltype(role)::value == 1) {
#pragma unroll
            for (int i = 0; i < 8; ++i) { __builtin_amdgcn_sched_group_barrier(0x010, 1, 0); __builtin_amdgcn_sched_group_barrier(0x008, 2, 0); }
        }
        __builtin_amdgcn_sched_barrier(0);
        if (decltype(waitn)::value >= 0) { WAIT_V(decltype(waitn)::value < 0 ? 0 : decltype(waitn)::value); BAR(); }
    };

    if (wid == 0 && ready < NCH) {
        const unsigned long long t0 = __builtin_amdgcn_s_memrealtime();
        for (;;) {
            u4 f;
            asm volatile("global_load_dwordx4 %0, %1, %2 sc0 sc1\n\ts_waitcnt vmcnt(0)"
                         : "=&v"(f) : "v"((unsigned)OFF_FB + (unsigned)(cg * 1024 + lane * 16)), "s"(myws) : "memory");
            if (__builtin_amdgcn_ballot_w64(f[0] < call || f[1] < call || f[2] < call || f[3] < call) == 0 ||
                late(__builtin_amdgcn_readfirstlane(__hip_atomic_load(tmo, __ATOMIC_RELAXED, __HIP_MEMORY_SCOPE_AGENT)), t0)) break;
            __builtin_amdgcn_s_sleep(6);
        }
        ensure(2);
    }
    WAIT_L0();
    BAR();
    STAMP(4);
    const unsigned half = SMODE == 0 ? cg & 1 : (~__builtin_amdgcn_readfirstlane(pl[64]) ^ (SMODE == 2)) & 1u;
    if (lw != 0) { stage(0, 0); stage(1, STAGE); stage(2, 2 * STAGE); }
    WAIT_V(16);
    BAR();
    if (kp) BAR();
    if (lw != 0) for (int c = 3, buf = 0; c < NCH; ++c, buf = buf == 2 * STAGE ? 0 : buf + STAGE) phase(IC<16>{}, IC<1>{}, buf, c);
    else for (int c = 3, buf = 0; c < NCH; ++c, buf = buf == 2 * STAGE ? 0 : buf + STAGE) phase(IC<16>{}, IC<2>{}, buf, c);
    static_assert((NCH - 3) % 3 == 1, "tail buffers");
    const unsigned tm = __hip_atomic_load(tmo, __ATOMIC_RELAXED, __HIP_MEMORY_SCOPE_AGENT);
    phase(IC<8>{}, IC<0>{}, STAGE, -1);
    phase(IC<0>{}, IC<0>{}, 2 * STAGE, -1);
    phase(IC<-1>{}, IC<0>{}, 0, -1);
    f4 r[4];
#pragma unroll
    for (int n = 0; n < 4; ++n) {
        const f4 a0 = acc[0][n], a1 = acc[1][n];
        r[n] = kp ? a1 : a0;
        *(LAS f4*)(ring + lw * 4096 + n * 1024 + lane * 16) = kp ? a0 : a1;
    }
    const bool dead = __builtin_amdgcn_readfirstlane(tm) == call;
    WAIT_L0();
    BAR();
    if (!kp) BAR();
    STAMP(5);
#pragma unroll
    for (int n = 0; n < 4; ++n) r[n] += *(const LAS f4*)(lds + (kp ^ 1) * 24576 + lw * 4096 + n * 1024 + lane * 16);
    if (cg < 8) {
        const unsigned ovoff = (unsigned)(lane >> 4) * (4 * NLOC * 4) + (unsigned)(lane & 15) * 16;
        char* ob = (char*)P.out + (size_t)((panel_row(P.rank, panel, 2 * qt + wm) + kp * 16) * NLOC + cg * 128 + wn * 64) * 4;
#pragma unroll
        for (int k = 0; k < 4; ++k) { const u4 o = silu4(r, k, dead); GST4P(ovoff, o, ob + k * (NLOC * 4)); }
    } else {
        char* d0 = myws + (pk_off(pk0 + lw, call) + half * 8192 + kp * 4096);
#pragma unroll
        for (int k = 0; k < 4; ++k) { const u4 o = silu4(r, k, dead); GST4((unsigned)lane * 16, o, d0 + k * 1024, 0); }
        WAIT_V(0);
        BAR();
        if (wid == 0 && lane < 4) flag_store(oflag + lane * 4, call * 32u + half * 16u + M_RAW);
    }
    STAMP(6);
    TDUMP(1);
}

__global__ void __launch_bounds__(512, 2) ag_gemm_kernel(const Params Parg) {
    const CAS Params& P = *(const CAS Params*)__builtin_amdgcn_kernarg_segment_ptr();
    (void)Parg;
    __shared__ __attribute__((aligned(16))) char lds_raw[5 * 32768];
    LAS char* lds = (LAS char*)lds_raw;
    const int lane = threadIdx.x & 63, wid = __builtin_amdgcn_readfirstlane(threadIdx.x >> 6);
    const int xcd = blockIdx.x & 7, slot = blockIdx.x >> 3;
    char* myws = P.ws[P.rank];
    const int q = P.rank & 3;
    const unsigned long long t0 = __builtin_amdgcn_s_memrealtime();
    const unsigned side = (__builtin_amdgcn_s_getreg(20 | (3 << 11)) >> 2) & 1u;
    int g = (int)blockIdx.x - NGEMM;
    if (slot == NGEMM / 8) g = NPC + xcd;
    else if (g >= 0) {
        if (threadIdx.x == 0) {
            GAS unsigned* rt = (GAS unsigned*)(myws + OFF_ROLE);
            const unsigned t = __hip_atomic_fetch_add(rt, side ? 65536u : 1u, __ATOMIC_RELAXED, __HIP_MEMORY_SCOPE_AGENT);
            if ((t & 65535u) + (t >> 16) >= 247u - NGEMM) __hip_atomic_store(rt, 0u, __ATOMIC_RELAXED, __HIP_MEMORY_SCOPE_AGENT);
            *(LAS unsigned*)(lds + 8 * PBUF) = t;
        }
        __syncthreads();
        const unsigned t = __builtin_amdgcn_readfirstlane(*(const LAS unsigned*)(lds + 8 * PBUF));
        constexpr unsigned N0 = NPC / 7 * 3, NL = 247 - NGEMM;
        const unsigned r = (side ? NL - (t >> 16) : t & 65535u) % (NL + 1), b = NL - r;
        g = r < N0 ? r / 3 * 7 + 4 + r % 3 : r < N0 + NPROD - 8 ? NPC + 8 + r - N0 : b / 4 * 7 + b % 4;
        if (XMODE == 0) { g = (int)blockIdx.x - NGEMM; g = g < NPC + 8 ? g - 8 : g; }
    }
#undef VBLK
#define VBLK (NGEMM + g)
    if (PROBET >= 0 && wid == 7 && blockIdx.x < 2 * PK_PG) {
        char* a = myws + (OFF_PK + ((size_t)blockIdx.x << 21) + PROBEX * (blockIdx.x & 31) * 256);
        unsigned t[2] = {~0u, ~0u};
        while (__builtin_amdgcn_s_memrealtime() - t0 < PROBET) __builtin_amdgcn_s_sleep(4);
#pragma unroll
        for (int i = 0; i < 6; ++i) {
            const unsigned long long c0 = __builtin_amdgcn_s_memtime();
            if (PROBE1) asm volatile("s_waitcnt lgkmcnt(0)\n\ts_mov_b64 exec, 1\n\tglobal_store_dword %0, %0, %1 sc0 sc1\n\ts_mov_b64 exec, -1\n\ts_waitcnt vmcnt(0)" :: "v"((i & 1) * 8192), "s"(a), "s"(c0) : "memory");
            else asm volatile("s_waitcnt lgkmcnt(0)\n\tglobal_store_dword %0, %0, %1 sc0 sc1\n\ts_waitcnt vmcnt(0)" :: "v"((i & 1) * 8192), "s"(a), "s"(c0) : "memory");
            const unsigned d = (unsigned)(__builtin_amdgcn_s_memtime() - c0);
            t[i & 1] = d < t[i & 1] ? d : t[i & 1];
        }
        if (lane == 0) flag_store(myws + OFF_MAP + blockIdx.x * MAPS, (t[1] < t[0]) ^ side);
        STAMP(10);
    }
    const int u0 = g < 0 ? wid * NGEMM + (int)blockIdx.x : UG + wid * NPROD + g - NPC;
    if (g < 0 || g >= NPC) {
        pace(t0, u0); fetch_input(P, lds + wid * PBUF, u0, lane);
        if ((TOUCH & 1) && u0 < TOUCHU) touch(P, myws, lds + wid * PBUF + 16384, u0, lane);
        make_lut(lds + (8 * PBUF + 16), true, lane);
    }
    GAS unsigned* seq = (GAS unsigned*)(myws + OFF_SEQ) + (blockIdx.x * 8 + wid) * 4;
    const unsigned call = __builtin_amdgcn_readfirstlane(__hip_atomic_load(seq, __ATOMIC_RELAXED, __HIP_MEMORY_SCOPE_AGENT)) + 1u;
    WAIT_V(0);
    if (lane == 0) __hip_atomic_store(seq, call, __ATOMIC_RELAXED, __HIP_MEMORY_SCOPE_AGENT);

    if (g < 0 || g >= NPC) {
        producer(P, myws, lds + wid * PBUF, lds + (8 * PBUF + 16), u0, t0, call, wid, lane);
        STAMP(7);
    }
    if (slot <= NGEMM / 8) {
        const int pi = slot >> 3, ps = xcd >> 1, h = xcd & 1, sown = 20 + ps;
        const int ci = pi == 3 ? (slot & 1) * 4 + ps : slot == sown ? ps : slot & 7;
        const int panel = pi < 2 ? 3 * ps + pi : pi == 3 ? 3 * q + 2 : slot == sown ? 16 : 12 + ps;
        if (g < 0) gemm_panel(P, lds, myws, panel, ci < 4 ? 4 * h + ci : 2 * ci + h,
                   pi == 3 ? h * 8 : pi == 2 ? O_C + (((ps - ci - 1) & 3) * 2 + h) * 8 : O_A0 + pi * 128 + xcd * 8, pi == 3 ? 16 : pi == 2 ? 48 : 64, call, wid, lane);
        const int k = slot == 26 ? 23 : slot >= 24 ? slot - 24 : slot == sown ? 2 : slot < 4 ? slot + 3 : slot < 8 ? slot + 7 : slot < 12 ? slot - 1
                    : slot < 20 ? slot + 3 : 24;
        if (k < 24) {
            const int kk = k < 12 ? k : k - 12, qt = kk / 3, si = kk - qt * 3;
            gemm_quarter(P, lds, myws, 3 * ((q + 1 + si) & 3) + 2, qt, k < 12 ? 8 + xcd : xcd, O_B + qt * 24 + (si * 2 + h) * 4, call, wid, lane);
        }
    } else if (g < NPC) {
        LAS unsigned* info = (LAS unsigned*)lds;
        LAS unsigned* prog = info + NSEQ;
        LAS unsigned short* tab = (LAS unsigned short*)(lds + 8192);
        const int j = g / 7, link = g - j * 7, ncu = (NPC + 6 - link) / 7;
        for (int i = threadIdx.x; i < NSEQ + 4; i += 512) info[i] = 0;
        if (threadIdx.x == 0) prog[4] = (unsigned)t0;
        for (int i = threadIdx.x; i < PK_W; i += 512)
            if (((i & 31) == 16 || (i & 31) >= 24) == (link < 4)) tab[pos_x(i & 31, i >> 5)] = (unsigned short)i;
        if (link < 4) tab[pos_w(threadIdx.x >> 2, threadIdx.x & 3)] = (unsigned short)(PK_W + threadIdx.x);
        if (wid == 0) make_lut(lds + 14336, false, lane);
        __syncthreads();
        const int peer = link < 4 ? ((P.rank & 4) ^ 4) + ((q + link) & 3) : (P.rank & 4) + ((q + link - 3) & 3);
        if (wid < WPC) {
            if (lane == 0) __hip_atomic_fetch_max(prog, (unsigned)(NPUSH + wid * ncu + j), __ATOMIC_RELAXED, __HIP_MEMORY_SCOPE_WORKGROUP);
            puller(g, P, myws, info, prog, tab, lds + 16384 + wid * 8192, lds + 14336, link, peer, NPUSH + wid * ncu + j, ncu * WPC,
                   (unsigned)(((link < 4 ? NWIN : NWINI) + ncu - 1 - j) / ncu), call, wid, lane);
        } else if (wid == 7) {
            flag_mirror(myws, P.ws[peer] + (OFF_FP + (call & 1) * FP_B + (link < 4 ? q : 4) * (NSEQ * 4)), link < 4 ? NSEQ : NI, info, prog, call, lane);
        }
        STAMP(3);
    }
}

}

extern "C" void kernel_launch(int rank, int nranks, ncclComm_t comm, hipStream_t stream,
                              void* const* d_in, const int* in_sizes, int n_in,
                              void* d_out, int out_size, const int* in_dtypes, int out_dtype,
                              void* const* d_out_all, void* const* d_ws_all, size_t ws_bytes) {
    (void)comm; (void)in_dtypes; (void)out_dtype; (void)d_out_all;
    if (nranks != 8 || (unsigned)rank >= 8u || n_in < 2 || ws_bytes < WS_NEED || out_size != KDIM * NLOC ||
        in_sizes[0] != 1024 * KDIM || in_sizes[1] != KDIM * NLOC) {
        fprintf(stderr, "ag_gemm: unexpected problem shape\n");
        return;
    }
    Params P;
    P.x = (const float*)d_in[0]; P.w = (const float*)d_in[1];
    P.out = (float*)d_out;
    P.rank = 0; P.pad = 0;
    for (int p = 0; p < 8; ++p) {
        const int r = (int)((RMAP >> (4 * p)) & 7u);
        P.ws[p] = (char*)d_ws_all[r];
        if (r == rank) P.rank = p;
    }
    hipLaunchKernelGGL(ag_gemm_kernel, dim3(256), dim3(512), 0, stream, P);
    HIP_CHECK(hipGetLastError());
}
```

```cpp
#include <hip/hip_runtime.h>
#include <rccl/rccl.h>
#include <cstdio>

#define HIP_CHECK(expr) do { hipError_t _e = (expr); if (_e != hipSuccess) { \
    fprintf(stderr, "HIP error %s at %s:%d\n", hipGetErrorString(_e), __FILE__, __LINE__); \
    return; } } while (0)

namespace {

typedef _Float16 h8 __attribute__((ext_vector_type(8)));
typedef float f4 __attribute__((ext_vector_type(4)));
typedef unsigned u4 __attribute__((ext_vector_type(4)));
typedef unsigned u2 __attribute__((ext_vector_type(2)));
#define LAS __attribute__((address_space(3)))
#define GAS __attribute__((address_space(1)))
#define CAS __attribute__((address_space(4)))

constexpr int KDIM = 8192, NLOC = 1024;
constexpr int NKT = KDIM / 64;
constexpr int NCH = NKT / 2;
constexpr int NI = 23 * 64;
constexpr int NCI = 17 * 64, NO = 480;
constexpr int NSEQ = NCI + NO;
constexpr int O_A0 = 32, O_A1 = 160, O_C = 288, O_B = 384;
constexpr size_t PANEL_B = 4u << 20;
constexpr size_t CG_B = 2u << 20;
constexpr size_t OFF_SEQ = 0;
constexpr size_t OFF_TMO = 64u << 10;
constexpr size_t OFF_CTR = OFF_TMO + 256;
constexpr size_t OFF_ROLE = OFF_TMO + 1280;
constexpr size_t OFF_MAP = OFF_TMO + 2048;
#define MAPS 256u
constexpr size_t OFF_FP = 256u << 10;
constexpr size_t FP_B = 5 * NSEQ * 4;
constexpr size_t OFF_DBG = 640u << 10;
constexpr size_t OFF_DATA = 1u << 20;
constexpr size_t A_B = 21 * PANEL_B, B_B = 16 * CG_B;
constexpr size_t OFF_FA = OFF_DATA + A_B + B_B;
constexpr size_t OFF_FB = OFF_FA + 17 * 64 * 32;
constexpr size_t OFF_FN = OFF_FB + 16 * 64 * 16;
constexpr unsigned FSTR = 219 * 256;
constexpr size_t OFF_PK = OFF_DATA + A_B + B_B + (1u << 20);
constexpr int PK_W = 2048, PK_O = 4096, PK_N = PK_O + 4 * NO;
constexpr int PK_PG = (PK_N + 126) / 127;
constexpr size_t PK_B = (size_t)PK_PG << 21;
constexpr size_t WS_NEED = OFF_PK + 2 * PK_B;
static_assert(OFF_MAP + 2 * PK_PG * MAPS <= OFF_FP && OFF_FN + 35 * 128 <= OFF_FA + FSTR && 16 * FSTR <= (1u << 20) && OFF_FP + 2 * FP_B <= OFF_DBG, "flag area");
constexpr unsigned long long DEADLINE = 30000000ull;
constexpr unsigned M_RAW = 9, M_DEAD = 10;

#define NOTMO
#define NOTMO1
#define NOTMO2
#define NPC 35
#define WPC 7
#define SKC 14
#define SKA 5
#define SQ1 5
#define SQ2 15
#define SQ3 28
#define PACE 12
#define U1ST 448
#define PACE1 PACE
#define PACE2 PACE1
#define PROBE1 1
#define LOOK 64
#define PCH 2
#define PT0 180
#define PDT 13
#define CHMAP 0x76543210u
#define PROBET 0
#define PROBEX 0
#define TOUCH 0
#define RAW0 32
#define RAW1 0
#define DEFER 1
#define PD 2
#define NWIN 10
#define NWINI NWIN
#define PRATE 134
#define PRATEI PRATE
#define LGAIN 64
#define LCAP 30
#define HWIN 280
#define HWIN0 0
#define GLINK 1
#define PSLACK 0
#define RELAT 8
#define UG 3000
#define NB2 200
#define AHEAD UG
#define PDEF 0
#define LOOK2 4
#define MALL 4
#define MSL 8
#define QSLEEP 0
#define AUX 17
#define RMAP 0x75416320u
#define EARLY 0
#define ORAW 0
#define SMODE 1
#define XMODE 1
constexpr int NGEMM = 208;
constexpr int NPROD = 256 - NGEMM - NPC;

struct Params {
    const float* x;
    const float* w;
    float* out;
    char* ws[8];
    int rank;
    int pad;
};

#define VBLK blockIdx.x
#define STAMP(i) do { } while (0)

#define TREC(c) do { } while (0)
#define TDUMP(k) do { (void)tv; } while (0)
#define HST(i, v) do { } while (0)
#define NOW() __builtin_amdgcn_s_memrealtime()
#define GST4(voff, data, sbase, imm) \
    asm volatile("global_store_dwordx4 %0, %1, %2 offset:" #imm " sc0 sc1\n\ts_nop 1" :: "v"(voff), "v"(data), "s"(sbase) : "memory")
#define OSUF " sc0 sc1"
#define GST4P(voff, data, sbase) \
    asm volatile("global_store_dwordx4 %0, %1, %2" OSUF "\n\ts_nop 1" :: "v"(voff), "v"(data), "s"(sbase) : "memory")
#define WAIT_V(n) asm volatile("s_waitcnt vmcnt(%0)" :: "n"(n) : "memory")
#define WAIT_L0() asm volatile("s_waitcnt lgkmcnt(0)" ::: "memory")

__device__ __forceinline__ u4 pack8(float a0, float a1, float a2, float a3, float a4, float a5, float a6, float a7) {
    h8 h = {(_Float16)a0, (_Float16)a1, (_Float16)a2, (_Float16)a3, (_Float16)a4, (_Float16)a5, (_Float16)a6, (_Float16)a7};
    return __builtin_bit_cast(u4, h);
}
__device__ __forceinline__ void flag_store(char* p, unsigned v) {
    __hip_atomic_store((GAS unsigned*)p, v, __ATOMIC_RELAXED, __HIP_MEMORY_SCOPE_SYSTEM);
}
__device__ __forceinline__ unsigned xcc_id() { return __builtin_amdgcn_s_getreg(20 | (3 << 11)) & 7u; }
__device__ __forceinline__ void flag_local(char* base, unsigned off, unsigned v) {
    asm volatile("global_store_dword %0, %1, %2" :: "v"(off + (8u + xcc_id()) * FSTR), "v"(v), "s"(base) : "memory");
}
__device__ __forceinline__ unsigned flag_copy(int lane) { return (xcc_id() + (unsigned)(lane >> 5) * 8u) * FSTR; }
__device__ __forceinline__ unsigned b_flags(int cg, int lane) { return flag_copy(lane) + (unsigned)OFF_FB + (unsigned)(cg * 1024 + (lane & 31) * 16); }
constexpr int BFLG = 148480;
__device__ __forceinline__ unsigned unset(unsigned f, unsigned call) {
    const unsigned long long m = __builtin_amdgcn_ballot_w64(f < call);
    return (unsigned)m & (unsigned)(m >> 32);
}
__device__ __forceinline__ void store8(const u4 (&h)[8], char* d0, unsigned half, unsigned vl16) {
    char* d1 = d0 + half;
    GST4(vl16, h[0], d0, 0); GST4(vl16, h[1], d0, 1024); GST4(vl16, h[2], d0, 2048); GST4(vl16, h[3], d0, 3072);
    GST4(vl16, h[4], d1, 0); GST4(vl16, h[5], d1, 1024); GST4(vl16, h[6], d1, 2048); GST4(vl16, h[7], d1, 3072);
}
__device__ __forceinline__ unsigned long long lane_mask(unsigned n) { return n >= 64u ? ~0ull : (1ull << n) - 1ull; }

__device__ __forceinline__ int clamp64(int v) { return v < 0 ? 0 : v > 64 ? 64 : v; }
__device__ __forceinline__ int pos_w(int kt, int g) { const int ch = kt >> 1; return (ch < PCH ? 8 : 9) * ch + 8 * clamp64(ch - SKC) + (kt & 1) * 4 + g; }
__device__ __forceinline__ int pos_x(int rb, int ch) {
    if (rb < 8) return 8 * ch + 8 * clamp64(ch - SKA) + rb;
    if (rb < 16) return 8 * ch + 8 * clamp64(ch + SKA + 1) + rb - 8;
    if (rb == 16) return ch < PCH ? 8 * PCH + ch : 9 * ch + 8 * clamp64(ch - SKC) + 8;
    if (rb >= 24) return 8 * ch + 9 * clamp64(ch + SKC + 1) + rb - 24;
    const int j = (rb - 16) >> 1, r = ch + (j == 0 ? 0 : j == 1 ? SQ1 : j == 2 ? SQ2 : SQ3);
    return 1024 + clamp64(r + (j > 0)) + 2 * (clamp64(r - SQ1 + (j > 1)) + clamp64(r - SQ2 + (j > 2)) + clamp64(r - SQ3)) + (j ? rb & 1 : 0);
}
static_assert(NPC % 7 == 0 && 2 * PK_PG <= NGEMM && UG >= 8 * NGEMM && UG <= 8 * (NGEMM + NB2) && UG + 8 * NPROD <= PK_O && 0 <= SQ1 && SQ1 <= SQ2 && SQ2 <= SQ3 && SKC < 64 && SKA < 64 && PCH <= SKA && PCH <= SKC && SKC >= 8, "skews");
constexpr int NPUSH = 8 * PCH;
__device__ __forceinline__ unsigned panel_off(int panel, unsigned call) {
    return (unsigned)OFF_DATA + (unsigned)((call & 1) && panel < 12 && panel % 3 == 0 ? 17 + panel / 3 : panel) * (unsigned)PANEL_B;
}
__device__ __forceinline__ void place_x(int panel, int rb, int ch, unsigned call, unsigned& off, unsigned& fl) {
    off = panel_off(panel, call) + (unsigned)(ch * 65536 + (rb & 7) * 4096);
    fl = (unsigned)OFF_FA + (unsigned)(panel * 2048 + ch * 32 + (rb & 7) * 4);
}
__device__ __forceinline__ void place_w(int cb, int kt, unsigned& off, unsigned& fl) {
    off = (unsigned)(OFF_DATA + A_B) + (unsigned)(cb >> 1) * (unsigned)CG_B + (unsigned)(kt * 16384 + (cb & 1) * 8192);
    fl = (unsigned)OFF_FB + (unsigned)((cb >> 1) * 1024 + kt * 8 + (cb & 1) * 4);
}
__device__ __forceinline__ int panel_row(int pub, int panel, int blk) {
    const int gs = pub & 4, s = panel / 3, k = panel - 3 * s;
    const int r = panel == 16 ? pub : panel >= 12 ? (gs ^ 4) + panel - 12 : (k == 2 && blk == 0 ? gs ^ 4 : gs) + s;
    return (int)((RMAP >> (4 * r)) & 7u) * 1024 + (panel >= 12 ? 768 : k * 256) + blk * 32;
}
__device__ __forceinline__ size_t pk_off(int pk, unsigned call) { return OFF_PK + (call & 1) * PK_B + (size_t)(pk + pk / 127 + 1) * 16384; }
__device__ __forceinline__ unsigned pk_half(char* myws, int pk, unsigned call) {
    return __hip_atomic_load((GAS unsigned*)(myws + OFF_MAP) + ((call & 1) * PK_PG + pk / 127) * (MAPS / 4), __ATOMIC_RELAXED, __HIP_MEMORY_SCOPE_SYSTEM);
}

#define PERM(hi, lo, sel) __builtin_amdgcn_perm(hi, lo, sel)
#define ABIT(hi, lo, s) __builtin_amdgcn_alignbit(hi, lo, s)
#define HW(k) h[(k) >> 2][(k) & 3]
#define FENCE() asm volatile("" ::: "memory")
__device__ __forceinline__ unsigned bfi(unsigned m, unsigned x, unsigned y) { return (x & m) | (y & ~m); }
__device__ __forceinline__ unsigned wave_scan(unsigned v) {
    v += __builtin_amdgcn_update_dpp(0u, v, 0x111, 0xF, 0xF, true);
    v += __builtin_amdgcn_update_dpp(0u, v, 0x112, 0xF, 0xF, true);
    v += __builtin_amdgcn_update_dpp(0u, v, 0x114, 0xF, 0xF, true);
    v += __builtin_amdgcn_update_dpp(0u, v, 0x118, 0xF, 0xF, true);
    v += __builtin_amdgcn_update_dpp(0u, v, 0x142, 0xA, 0xF, false);
    v += __builtin_amdgcn_update_dpp(0u, v, 0x143, 0xC, 0xF, false);
    return v;
}
__device__ __forceinline__ int wave_sum(unsigned v) { return __builtin_amdgcn_readlane(wave_scan(v), 63); }
__device__ __forceinline__ unsigned escapes(unsigned c, unsigned acc) { return (unsigned)__builtin_popcount(c & (c >> 1) & 0x55555555u) + acc; }
__device__ __forceinline__ void make_lut(LAS char* lut, bool pack, int lane) {
#pragma unroll
    for (int j = 0; j < 4; ++j) {
        unsigned sel = pack ? 0x0C0C0C0Cu : 0u, k = 0;
#pragma unroll
        for (int i = 0; i < 4; ++i) {
            const unsigned c = i < 3 ? ((unsigned)lane >> (2 * i)) & 3u : (unsigned)j;
            if (pack) sel ^= c == 3u ? (0x0Cu ^ (unsigned)i) << (8 * k) : 0u; else sel |= (c == 3u ? k : 4u + c) << (8 * i);
            k += c == 3u;
        }
        *(LAS u2*)(lut + (j * 64 + lane) * 8) = (u2){sel, pack ? 8u * k : k};
    }
}
__device__ __forceinline__ unsigned code4(unsigned thi, unsigned tlo, unsigned s) { return __builtin_amdgcn_udot4(PERM(thi, tlo, s), 0x80200802u, 0u - 0xF3u * 170u, false); }
template <int N> __device__ __forceinline__ void append(const unsigned* s, const u2* e, unsigned pos) {
#pragma unroll
    for (int g = 0; g < N; ++g) {
        const unsigned long long c = (unsigned long long)PERM(0u, s[g], e[g][0]) << (pos & 31u);
        asm volatile("v_cmpx_ne_u32_e32 vcc, 0, %1\n\tds_or_b32 %0, %1\n\tv_cmpx_ne_u32_e32 vcc, 0, %2\n\tds_or_b32 %0, %2 offset:4\n\ts_mov_b64 exec, -1"
                     :: "v"((pos >> 3) & ~3u), "v"((unsigned)c), "v"((unsigned)(c >> 32)) : "vcc", "memory");
        pos += e[g][1];
    }
}
constexpr int SCR_B = 3328;
constexpr int SCR_3 = 2064, SCR_T = 2592, PBUF = 16384 + SCR_B;
template <bool F32>
__device__ __forceinline__ unsigned pack_unit(u4 (&h)[8], LAS char* scr, const LAS char* lut, int lane, unsigned& r0, unsigned& r2) {
    constexpr int N1 = F32 ? 8 : 16, N2 = F32 ? 4 : 8;
    const u4 z = {0u, 0u, 0u, 0u};
    *(LAS u4*)(scr + lane * 16) = z; *(LAS u4*)(scr + 1024 + lane * 16) = z; *(LAS u4*)(scr + 2048 + lane * 16) = z;
    FENCE();
    unsigned G[16], s[16], a[16], cw[4], b3, type;
    if (F32) {
#pragma unroll
        for (int g = 0; g < 8; ++g)
            G[g] = PERM(PERM(HW(4 * g + 3), HW(4 * g + 2), 0x07060302u) >> 7, PERM(HW(4 * g + 1), HW(4 * g), 0x07060302u) >> 7, 0x06040200u);
        unsigned es = 0;
#pragma unroll
        for (int g = 0; g < 4; ++g) es = __builtin_amdgcn_udot4(G[2 * g + (g & 1)], 0x02020202u, es, false);
        const int m = wave_sum(es) + 205;
        b3 = (unsigned)(m >> 11) + 3u; b3 = b3 < 27u ? 27u : b3 > 251u ? 251u : b3;
        type = (m & 2047) > 600;
        unsigned out = 0;
#pragma unroll
        for (int g = 0; g < 8; ++g) { const unsigned d = (b3 + 4u) * 0x01010101u - G[g]; out |= d; s[g] = (2u * d + 0x39393939u) & 0x3F3F3F3Fu; }
        if (__builtin_amdgcn_ballot_w64((out & 0xE0E0E0E0u) != 0) != 0) return M_RAW;
    } else {
#pragma unroll
        for (int g = 0; g < 16; ++g) { G[g] = PERM(HW(2 * g + 1), HW(2 * g), 0x07050301u); s[g] = (G[g] >> 1) & 0x3E3E3E3Eu; }
        const int m = wave_sum(__builtin_amdgcn_udot4(s[0] + s[5] + s[10] + s[15], 0x01010101u, 0u, false)) + 205;
        b3 = ((unsigned)(m >> 11) + 3u) & 31u; type = (m & 2047) > 1120;
#pragma unroll
        for (int g = 0; g < 16; ++g) s[g] = ((2u * b3 + 65u) * 0x01010101u - s[g]) & 0x3F3F3F3Fu;
    }
    u2 e[16];
#pragma unroll
    for (int g = 0; g < N1; ++g) {
        a[g] = code4(0xF300F700u, 0xFB00FF00u, s[g]);
        e[g] = *(const LAS u2*)(lut + a[g]);
    }
    unsigned n = 0;
#pragma unroll
    for (int j = 0; j < N1 / 4; ++j) { cw[j] = (a[4 * j] >> 3) | (a[4 * j + 1] << 5) | (a[4 * j + 2] << 13) | (a[4 * j + 3] << 21); n = escapes(cw[j], n); }
    unsigned sc = wave_scan(n);
    r2 = (__builtin_amdgcn_readlane(sc, 63) + (4u * N2 - 1u)) / (4u * N2);
    if (r2 > (F32 ? 64u : 62u)) return M_RAW;
    append<N1>(s, e, (sc - n + (unsigned)(size_t)scr) * 8u);
    FENCE();
    u4 q[N2 / 4];
#pragma unroll
    for (int j = 0; j < N2 / 4; ++j) q[j] = *(const LAS u4*)(scr + lane * (4 * N2) + 16 * j);
#pragma unroll
    for (int g = 0; g < N2; ++g) {
        const unsigned x = q[g >> 2][g & 3];
        s[g] = (x | ~PERM(0xFF000000u, 0xFF00F9F6u, x)) - 0x06060606u;
        a[g] = code4(type ? 0xFF00F700u : 0xFB00F700u, type ? 0xF300FB00u : 0xF300FF00u, s[g]);
        e[g] = *(const LAS u2*)(lut + a[g]);
    }
    const unsigned c0 = (a[0] >> 3) | (a[1] << 5) | (a[2] << 13) | (a[3] << 21), c1 = F32 ? 0u : (a[4] >> 3) | (a[5] << 5) | (a[6] << 13) | (a[7] << 21);
    n = escapes(c1, escapes(c0, 0u));
    sc = wave_scan(n);
    const unsigned r3 = (__builtin_amdgcn_readlane(sc, 63) + 7u) >> 3;
    if (r3 > 64u) return M_RAW;
    append<N2>(s, e, (sc - n + (unsigned)(size_t)(scr + SCR_3)) * 8u);
    FENCE();
    u2 y = *(const LAS u2*)(scr + SCR_3 + lane * 8), c;
#pragma unroll
    for (int j = 0; j < 2; ++j) {
        y[j] = (y[j] | ~PERM(0xFF000000u, 0xFF00F9F8u, y[j])) - 0x07070707u;
        const unsigned sw = F32 ? ~(((y[j] ^ 0x0E0E0E0Eu) + 0x3F3F3F3Fu) & ((y[j] ^ 0x32323232u) + 0x3F3F3F3Fu)) & 0x40404040u : 0u;
        c[j] = y[j] ^ (sw - (sw >> 4));
    }
    unsigned w = ((c[0] >> 1) & 0x07070707u) | ((c[1] << 2) & 0x38383838u);
    w |= ((w >> 24) * 0x41040u) & 0xC0C0C0u;
    LAS char* t = scr + SCR_T;
    t[4 + lane * 3] = (char)w; t[5 + lane * 3] = (char)(w >> 8); t[6 + lane * 3] = (char)(w >> 16);
    FENCE();
    const unsigned l0 = (3u * r3 + 5u) >> 1;
    unsigned bad = ((c[0] >> 4) & 0x03030303u) | (c[1] & 0x30303030u), nl = 0;
    for (;;) {
        const unsigned long long mk = __builtin_amdgcn_ballot_w64(bad != 0);
        if (mk == 0 || nl > 64u) break;
        if (bad != 0) {
            const unsigned p = (unsigned)__builtin_ctz(bad), k = l0 + nl + __builtin_amdgcn_mbcnt_hi((unsigned)(mk >> 32), __builtin_amdgcn_mbcnt_lo((unsigned)mk, 0u));
            ((LAS unsigned short*)t)[k] = (unsigned short)((((unsigned)lane * 8u + (p & 4u) + (p >> 3)) << 7) | ((4u * b3 - 24u - 2u * (((p & 4u) ? y[1] : y[0]) >> (p & 24u))) & 0x7Cu));
            bad &= ~(15u << (p & 28u));
        }
        nl += (unsigned)__builtin_popcountll(mk);
    }
    r0 = (l0 + nl + 3u) >> 2;
    const unsigned rs = r0 + (F32 ? (r2 + 1u) >> 1 : r2);
    if (nl > 64u || rs > 64u) return M_RAW;
    FENCE();
    u2 f = *(const LAS u2*)(t + lane * 8);
    FENCE();
    if (lane == 0) f[0] = b3 | (r0 << 8) | (type << 15) | (nl << 16) | (l0 << 24);
    if (F32) {
#pragma unroll
        for (int g = 0; g < 8; ++g) {
            const unsigned x = ABIT(HW(4 * g + 3), HW(4 * g + 3), 31);
#pragma unroll
            for (int j = 0; j < 3; ++j) HW(3 * g + j) = PERM(x, ABIT(HW(4 * g + j), HW(4 * g + j), 31), 0x04020100u + ((unsigned)j << 24));
        }
        h[6] = (u4){cw[0], cw[1], f[0], f[1]};
    } else {
        unsigned L[16];
#pragma unroll
        for (int g = 0; g < 16; ++g) L[g] = PERM(HW(2 * g + 1), HW(2 * g), 0x06040200u);
#pragma unroll
        for (int g = 0; g < 16; ++g) HW(g) = L[g];
#pragma unroll
        for (int j = 0; j < 6; ++j) {
            const unsigned x = bfi(0x83838383u, G[2 * j], ABIT(G[2 * j + 1], G[2 * j + 1], 29));
            HW(16 + j) = j < 4 ? bfi(0x60606060u, G[12 + j] << 5, x) : bfi(0x20202020u, G[2 * j + 4] >> 2, bfi(0x40404040u, G[2 * j + 5] >> 1, x));
        }
        h[5][2] = cw[0]; h[5][3] = cw[1]; h[6] = (u4){cw[2], cw[3], f[0], f[1]};
    }
    h[7][0] = c0; h[7][1] = c1;
    return (rs + 7u) >> 3;
}
template <bool F32>
__device__ __forceinline__ void store_unit(const u4 (&h)[8], char* d0, unsigned mode, unsigned r0, unsigned r2, unsigned vl16) {
    char* d1 = d0 + 4096;
    GST4(vl16, h[0], d0, 0); GST4(vl16, h[1], d0, 1024); GST4(vl16, h[2], d0, 2048); GST4(vl16, h[3], d0, 3072);
    GST4(vl16, h[4], d1, 0); GST4(vl16, h[5], d1, 1024);
    if (mode < M_RAW) {
        const u2 p0 = {h[6][0], h[6][1]}, p1 = {h[6][2], h[6][3]}, p2 = {h[7][0], h[7][1]};
        asm volatile("global_store_dwordx2 %0, %1, %3 offset:2048 sc0 sc1\n\t"
                     "s_mov_b64 exec, %4\n\tglobal_store_dwordx2 %0, %2, %3 offset:2560 sc0 sc1\n\ts_mov_b64 exec, -1"
                     :: "v"(vl16 >> 1), "v"(p0), "v"(p1), "s"(d1), "s"(lane_mask(r0)) : "memory");
        if (F32) asm volatile("s_mov_b64 exec, %3\n\tglobal_store_dword %0, %1, %2 offset:2560 sc0 sc1\n\ts_mov_b64 exec, -1"
                              :: "v"(vl16 >> 2), "v"(p2[0]), "s"(d1 + r0 * 8u), "s"(lane_mask(r2)) : "memory");
        else asm volatile("s_mov_b64 exec, %3\n\tglobal_store_dwordx2 %0, %1, %2 offset:2560 sc0 sc1\n\ts_mov_b64 exec, -1"
                          :: "v"(vl16 >> 1), "v"(p2), "s"(d1 + r0 * 8u), "s"(lane_mask(r2)) : "memory");
    } else { GST4(vl16, h[6], d1, 2048); GST4(vl16, h[7], d1, 3072); }
}
template <int N> __device__ __forceinline__ void deposit(unsigned* o, const u2* e, unsigned tab, unsigned pos) {
    u2 x[N]; unsigned p[N];
#pragma unroll
    for (int g = 0; g < N; ++g) {
        const LAS unsigned* d = (const LAS unsigned*)(size_t)(pos & ~3u);
        x[g] = (u2){d[0], d[1]}; p[g] = pos;
        pos += e[g][1];
    }
#pragma unroll
    for (int g = 0; g < N; ++g) o[g] = PERM(tab, __builtin_amdgcn_alignbyte(x[g][1], x[g][0], p[g]), e[g][0]);
}
template <bool F32>
__device__ __forceinline__ void unpack_unit(u4 (&h)[8], LAS char* scr, const LAS char* lut, int lane) {
    constexpr int N1 = F32 ? 8 : 16, N2 = F32 ? 4 : 8;
    LAS char* t = scr + SCR_T;
    *(LAS u2*)(t + lane * 8) = (u2){h[6][2], h[6][3]};
    FENCE();
    const unsigned hd = __builtin_amdgcn_readfirstlane(h[6][2]), r0 = (hd >> 8) & 63u, type = hd & 32768u;
    const unsigned kb = (F32 ? hd & 255u : ((hd & 31u) << 2) | 0x80u) * 0x01010101u;
    auto tab = [&](unsigned t4) { return F32 ? kb - t4 : (kb - 4u * t4) & 0x7C7C7C7Cu; };
    const LAS unsigned* d3 = (const LAS unsigned*)(t + ((4 + lane * 3) & ~3));
    const unsigned w0 = d3[0], w1 = d3[1];
    unsigned le = 0;
    const bool ll = (unsigned)lane < ((hd >> 16) & 127u);
    if (ll) le = ((const LAS unsigned short*)t)[(hd >> 24) + lane];
    const unsigned c0 = F32 ? *(const LAS unsigned*)(t + r0 * 8u + lane * 4) : (unsigned)lane < r0 ? 0u : h[6][2], c1 = F32 || (unsigned)lane < r0 ? 0u : h[6][3];
    const unsigned cw[4] = {F32 ? h[6][0] : h[5][2], F32 ? h[6][1] : h[5][3], F32 ? 0u : h[6][0], F32 ? 0u : h[6][1]};
    u2 e2[N2], e1[N1];
#pragma unroll
    for (int g = 0; g < N2; ++g) e2[g] = *(const LAS u2*)(lut + (((g < 4 ? c0 : c1) >> (8 * (g & 3))) & 255u) * 8u);
#pragma unroll
    for (int g = 0; g < N1; ++g) e1[g] = *(const LAS u2*)(lut + ((cw[g >> 2] >> (8 * (g & 3))) & 255u) * 8u);
    const unsigned n2 = escapes(c1, escapes(c0, 0u)), n1 = escapes(cw[3], escapes(cw[2], escapes(cw[1], escapes(cw[0], 0u))));
    const unsigned n = n1 | (n2 << 16), sc = wave_scan(n) - n;
    unsigned w = __builtin_amdgcn_alignbyte(w1, w0, (unsigned)lane * 3u);
    w = bfi(0x003F3F3Fu, w, __builtin_amdgcn_udot4(w & 0xC0C0C0u, 0x100401u, 0u, false) << 18);
    const unsigned t3l = tab(type ? 0x09080706u : 0x09080700u), t3h = tab(F32 ? 0xFF0C0B0Au : 0x0D0C0B0Au);
    *(LAS u2*)(scr + SCR_3 + lane * 8) = (u2){PERM(t3h, t3l, w & 0x07070707u), PERM(t3h, t3l, (w >> 3) & 0x07070707u)};
    FENCE();
    if (ll) scr[SCR_3 + (le >> 7)] = (char)(F32 ? hd - 27u + (((le >> 2) - hd + 27u) & 31u) : le & 0x7Cu);
    FENCE();
    unsigned E[16];
    deposit<N2>(E, e2, tab(type ? 0x000504u : 0x060504u), (sc >> 16) + (unsigned)(size_t)(scr + SCR_3));
    FENCE();
#pragma unroll
    for (int j = 0; j < N2 / 4; ++j) *(LAS u4*)(scr + lane * (4 * N2) + 16 * j) = (u4){E[4 * j], E[4 * j + 1], E[4 * j + 2], E[4 * j + 3]};
    FENCE();
    deposit<N1>(E, e1, tab(0x010203u), (sc & 0xFFFFu) + (F32 ? 0u : r0 * 32u) + (unsigned)(size_t)scr);
    FENCE();
    if (F32) {
#pragma unroll
        for (int g = 7; g >= 0; --g) {
            const unsigned x = PERM(PERM(E[g], HW(3 * g + 2), 0x07030000u), PERM(HW(3 * g + 1), HW(3 * g), 0x00000703u), 0x07060100u);
            HW(4 * g + 3) = ABIT(x, x, 1);
#pragma unroll
            for (int j = 2; j >= 0; --j) { const unsigned v = PERM(E[g], HW(3 * g + j), 0x04020100u + ((unsigned)j << 24)); HW(4 * g + j) = ABIT(v, v, 1); }
        }
    } else {
        unsigned L[16];
#pragma unroll
        for (int g = 0; g < 16; ++g) {
            L[g] = HW(g);
            const unsigned x = HW(16 + (g < 12 ? g >> 1 : g - 12));
            E[g] = bfi(0x7C7C7C7Cu, E[g], g >= 12 ? bfi(0x80808080u, HW(14 + (g >> 1)) << (2 - (g & 1)), x >> 5) : (g & 1) ? ABIT(x, x, 3) : x);
        }
#pragma unroll
        for (int g = 0; g < 16; ++g) { HW(2 * g) = PERM(E[g], L[g], 0x05010400u); HW(2 * g + 1) = PERM(E[g], L[g], 0x07030602u); }
    }
}

__device__ __forceinline__ void unit_decode(int u, int& kind, int& ch, int& g) {
    constexpr int U0 = 56 * SKC, U1 = U0 + 64 * (64 - SKC);
    if (u < U0) { const int s = u / 7; kind = u - 7 * s; ch = s >> 3; g = s & 7; if (ch < 8 && kind < 5) ch = (CHMAP >> (4 * ch)) & 7; }
    else if (u < U1) { const int v = u - U0; kind = v & 7; g = (v >> 3) & 7; ch = (v >> 6) + (kind == 7 ? 0 : SKC); }
    else { const int v = u - U1; kind = 7; g = v & 7; ch = 64 - SKC + (v >> 3); }
}
__device__ __forceinline__ void fetch_input(const CAS Params& P, LAS char* buf, int u, int lane) {
    int kind, ch, g;
    unit_decode(u, kind, ch, g);
    const bool isx = kind == 0 || kind > 4;
    const char* src = isx ? (const char*)P.x + (size_t)((unsigned)((kind ? kind - 4 : 0) * 8 + g) * (32 * KDIM * 4) + (unsigned)ch * 512)
                          : (const char*)P.w + (size_t)((unsigned)(2 * ch + (g >> 2)) * (64 * NLOC * 4) + (unsigned)((kind - 1) * 4 + (g & 3)) * 256);
    const unsigned s0 = isx ? 64 : NLOC * 4, s1 = isx ? 16 * KDIM * 4 : 16 * NLOC * 4, s2 = isx ? 256 : 32 * NLOC * 4;
    const unsigned voff = isx ? (unsigned)(lane & 15) * (KDIM * 4) + (unsigned)(lane >> 4) * 16 : (unsigned)(lane >> 4) * (4 * NLOC * 4) + (unsigned)(lane & 15) * 16;
#pragma unroll
    for (int i = 0; i < 16; ++i)
        __builtin_amdgcn_global_load_lds((const GAS void*)(src + ((i & 3) * s0 + ((i >> 2) & 1) * s1 + (i >> 3) * s2) + voff), (LAS void*)(buf + i * 1024), 16, 0, 0);
}
#define TOUCHU 448
__device__ __forceinline__ void touch(const CAS Params& P, char* myws, LAS char* scr, int u, int lane) {
    int ch, g, kind;
    unit_decode(u, kind, ch, g);
    const bool isx = kind == 0 || kind > 4;
    const int rb = (kind ? kind - 4 : 0) * 8 + g, kt = 2 * ch + (g >> 2), cb = (kind - 1) * 4 + (g & 3);
    const int pk = isx ? ch * 32 + rb : PK_W + (kind - 1) * 512 + ch * 8 + g;
    unsigned off, fl;
    if (isx) place_x(rb < 24 ? 3 * (P.rank & 3) + (rb >> 3) : 16, rb, ch, lane & 1, off, fl); else place_w(cb, kt, off, fl);
    const unsigned a = lane & 2 ? off : (unsigned)pk_off(pk, lane & 1);
    if (lane < 4) __builtin_amdgcn_global_load_lds((const GAS void*)(myws + a), (LAS void*)scr, 4, 0, 17);
}
__device__ __forceinline__ void pace(unsigned long long t0, int u) {
    if (u < UG) while (__builtin_amdgcn_s_memrealtime() - t0 < (unsigned)(u < U1ST ? u * PACE1 : (u - U1ST) * PACE + U1ST * PACE2) >> 4) __builtin_amdgcn_s_sleep(2);
}
__device__ __forceinline__ int next_unit(int u, int wid) {
    if (u >= UG) return u + 8 * NPROD;
    u = u < 8 * NGEMM && blockIdx.x < NB2 ? 8 * NGEMM + wid * NB2 + (int)blockIdx.x : UG;
    return u < UG ? u : PK_O;
}
__device__ __forceinline__ void producer(const CAS Params& P, char* myws, LAS char* buf, LAS char* lut, int u, unsigned long long t0, unsigned call, int wid, int lane) {
    const int q = P.rank & 3;
    const unsigned vl16 = (unsigned)lane * 16;
    LAS char* scr = buf + 16384;
#define PT(i) do { } while (0)
    for (int un; u < PK_O; u = un) {
        un = next_unit(u, wid);
        int ch, g, kind;
        unit_decode(u, kind, ch, g);
        const bool isx = kind == 0 || kind > 4;
        const int rb = (kind ? kind - 4 : 0) * 8 + g, kt = 2 * ch + (g >> 2), cb = (kind - 1) * 4 + (g & 3);
        const int idx = isx ? pos_x(rb, ch) : pos_w(kt, g & 3), pk = isx ? ch * 32 + rb : PK_W + (kind - 1) * 512 + ch * 8 + g;
        const int rd = !isx ? kind - 1 : rb == 16 || rb >= 24 ? 5 : 4;
        f4 v[16]; u4 h[8];
        unsigned hv;
        asm volatile("global_load_dword %0, %1, %2 sc0 sc1" : "=&v"(hv) : "v"((unsigned)OFF_MAP + ((call & 1) * PK_PG + pk / 127) * MAPS), "s"(myws) : "memory");
#pragma unroll
        for (int i = 0; i < 16; ++i) v[i] = *(const LAS f4*)(buf + i * 1024 + lane * 16);
        __builtin_amdgcn_sched_barrier(0);
        WAIT_L0();
        const bool ahead = u >= AHEAD && un < PK_O;
        PT(0);
        if (ahead) { pace(t0, un); fetch_input(P, buf, un, lane); }
        PT(1);
        __builtin_amdgcn_sched_barrier(0);
        if (isx) {
#pragma unroll
            for (int i = 0; i < 8; ++i)
                h[i] = pack8(v[2 * i][0], v[2 * i][1], v[2 * i][2], v[2 * i][3], v[2 * i + 1][0], v[2 * i + 1][1], v[2 * i + 1][2], v[2 * i + 1][3]);
        } else {
#pragma unroll
            for (int i = 0; i < 8; ++i) {
                const int f = i >> 1, k = (i & 1) * 8;
                h[i] = pack8(v[k][f], v[k + 1][f], v[k + 2][f], v[k + 3][f], v[k + 4][f], v[k + 5][f], v[k + 6][f], v[k + 7][f]);
            }
        }
        unsigned off, fl;
        if (isx) place_x(rb < 24 ? 3 * q + (rb >> 3) : 16, rb, ch, call, off, fl); else place_w(cb, kt, off, fl);
        const bool own = !isx || rb != 16;
        const unsigned long long h11 = NOW();
        if (idx < NPUSH) {
            unsigned off2 = off, fl2 = fl;
            if (!isx) place_w(16 + 4 * q + (g & 3), kt, off2, fl2);
            while (__builtin_amdgcn_s_memrealtime() - t0 < (unsigned)(PT0 + idx * PDT)) __builtin_amdgcn_s_sleep(1);
            char* fp = myws + fl;
            const unsigned long long h14 = NOW();
            for (int i = 1; i < (isx ? 4 : 2); ++i) {
                char* pw = P.ws[isx ? (P.rank & 4) + ((q + i) & 3) : ((P.rank & 4) ^ 4) + rd];
                store8(h, pw + off2, isx ? 32768u : 4096u, vl16);
                fp = lane >> 3 == i ? pw + fl2 : fp;
            }
            store8(h, myws + off, isx ? 32768u : 4096u, vl16);
            asm volatile("s_waitcnt vmcnt(0)" : "+v"(hv) :: "memory");
            if (lane == (int)xcc_id()) flag_local(myws, fl, call);
            else if (lane < (isx ? 32 : 16)) flag_store(fp + (lane & 7) * FSTR, call);
            HST(12, NOW()); HST(11, h11); HST(14, h14); HST(13, (unsigned long long)(rd * 4096 + idx + 1));
            if (un < PK_O) { pace(t0, un); fetch_input(P, buf, un, lane); WAIT_V(0); }
            continue;
        }
        if (own) store8(h, myws + off, isx ? 32768u : 4096u, vl16);
        unsigned mode = M_RAW, r0 = 0, r2 = 0;
        PT(2);
        if (idx >= RAW0 && (rd != 4 || idx < NI - RAW1)) mode = pack_unit<false>(h, scr, lut, lane, r0, r2);
        PT(3);
        const unsigned long long h14 = NOW();
        if (own) asm volatile("s_waitcnt vmcnt(8)" : "+v"(hv) :: "memory"); else asm volatile("s_waitcnt vmcnt(0)" : "+v"(hv) :: "memory");
        PT(4);
        const unsigned half = SMODE == 0 ? idx & 1 : (__builtin_amdgcn_readfirstlane(hv) ^ (rd != 4) ^ (SMODE == 2)) & 1u;
        store_unit<false>(h, myws + (pk_off(pk, call) + half * 8192), mode, r0, r2, vl16);
        const unsigned pfo = lane < 5 ? (unsigned)OFF_FP + (call & 1) * (unsigned)FP_B + (unsigned)(lane * NSEQ + idx) * 4u : fl + (lane & 7) * FSTR;
        WAIT_V(0);
        if (own && lane == 8 + (int)xcc_id()) flag_local(myws, fl, call);
        else if ((rd == 5 ? lane < 4 : lane == rd) || (own && lane >> 3 == 1)) flag_store(myws + pfo, lane < 5 ? call * 32u + half * 16u + mode : call);
        HST(12, NOW()); HST(11, h11); HST(14, h14); HST(13, (unsigned long long)(rd * 4096 + idx + 1));
        if (!ahead && un < PK_O) { pace(t0, un); fetch_input(P, buf, un, lane); WAIT_V(0); }
        PT(5);
    }
}

__device__ __forceinline__ void flag_mirror(char* myws, const char* fp, int nseq, LAS unsigned* info, LAS unsigned* prog, unsigned call, int lane) {
    GAS unsigned* tmo = (GAS unsigned*)(myws + OFF_TMO);
    int pos = NPUSH;
    bool all = false;
    unsigned long long t0 = __builtin_amdgcn_s_memrealtime();
    while (pos < nseq) {
        const int pr = (int)__builtin_amdgcn_readfirstlane(__hip_atomic_load(prog, __ATOMIC_RELAXED, __HIP_MEMORY_SCOPE_WORKGROUP));
        if (pos <= pr + (all ? LOOK : LOOK2)) {
            const int lo = pos & ~31, hi = all ? lo + 32 * MALL : ((pr + LOOK2) | 31) + 1, i = lo + 4 * lane;
            bool bad = i >= hi;
            if (!bad && i < nseq) {
                u4 f = *(const LAS u4*)(info + i);
                if (i >= pos && (f[0] < call * 32u || f[1] < call * 32u || f[2] < call * 32u || f[3] < call * 32u)) {
                    asm volatile("global_load_dwordx4 %0, %1, %2 sc0 sc1\n\ts_waitcnt vmcnt(0)" : "=&v"(f) : "v"(i * 4), "s"(fp) : "memory");
                    *(LAS u4*)(info + i) = f;
                    bad = f[0] < call * 32u || f[1] < call * 32u || f[2] < call * 32u || f[3] < call * 32u;
                }
            }
            const unsigned long long m = __builtin_amdgcn_ballot_w64(bad);
            const int n = lo + 4 * (m ? __builtin_ctzll(m) : 64);
            all = n >= hi;
            if (n > pos) { pos = n; t0 = __builtin_amdgcn_s_memrealtime(); if (all) continue; }
        }
        if (
            __builtin_amdgcn_s_memrealtime() - t0 > DEADLINE) {
            __hip_atomic_store(tmo, call, __ATOMIC_RELAXED, __HIP_MEMORY_SCOPE_AGENT);
            asm volatile("s_waitcnt vmcnt(0)" ::: "memory");
            for (int i = pos + lane; i < nseq; i += 64) info[i] = call * 32u + M_DEAD;
            return;
        }
        if (pos > pr) __builtin_amdgcn_s_sleep(MSL);
    }
}

#define STR2(x) #x
#define STR(x) STR2(x)
#define SP ""
#define RD ""
#define PULL6 "global_load_dwordx4 %[h0], %[v16], %[src] offset:-4096 sc0 sc1\n\t" SP "global_load_dwordx4 %[h1], %[v16], %[src] offset:-3072 sc0 sc1\n\t" SP \
              "global_load_dwordx4 %[h2], %[v16], %[src] offset:-2048 sc0 sc1\n\t" SP "global_load_dwordx4 %[h3], %[v16], %[src] offset:-1024 sc0 sc1\n\t" SP \
              "global_load_dwordx4 %[h4], %[v16], %[src] sc0 sc1\n\t" SP "global_load_dwordx4 %[h5], %[v16], %[src] offset:1024 sc0 sc1\n\t" SP
#define PULLT "s_mov_b64 exec, 1\n\tglobal_atomic_add %[tk], %[ao], %[one], %[ws] sc0\n\t" \
              "s_waitcnt vmcnt(%[rn] + 9 - " STR(RELAT) ")\n\ts_mov_b32 exec_lo, %[m1]\n\tglobal_store_dword %[pf], %[cl], %[ws] sc0 sc1\n\t" \
              "s_mov_b32 exec_lo, %[m2]\n\tglobal_store_dword %[pf], %[cl], %[ws]\n\ts_waitcnt vmcnt(%[rn] + 2)\n\t" RD \
              "ds_add_u32 %[rel], %[one]\n\ts_mov_b64 exec, -1\n\ts_waitcnt vmcnt(2)"
#define PULLO [h0] "=&v"(h[0]), [h1] "=&v"(h[1]), [h2] "=&v"(h[2]), [h3] "=&v"(h[3]), [h4] "=&v"(h[4]), [h5] "=&v"(h[5]), [tk] "=&v"(tk)
#define PULLI [v16] "v"(vl16), [src] "s"(src), [ao] "v"(aoff), [one] "v"(one), [ws] "s"(myws), [rel] "v"(rel), [pf] "v"(pfl + fcop), [cl] "v"(call), [m1] "s"(m1), [m2] "s"(m2)
#undef VBLK
#define VBLK (NGEMM + vg)
__device__ __forceinline__ void puller(int vg, const CAS Params& P, char* myws, LAS unsigned* info, LAS unsigned* prog, const LAS unsigned short* tab,
                                       LAS char* scr, const LAS char* lut, int link, int peer, int cur, int nw, unsigned win, unsigned call, int wid, int lane) {
    const int q = P.rank & 3, qs = peer & 3;
    const bool cross = link < 4;
    const int nseq = cross ? NSEQ : NI;
    const char* pk = P.ws[peer] + 4096;
    const unsigned vl16 = (unsigned)lane * 16, one = 1,
        aoff = (unsigned)OFF_CTR + (unsigned)link * 128u;
    const unsigned ovoff = (unsigned)(lane >> 4) * (4 * NLOC * 4) + (unsigned)(lane & 15) * 16;
    const unsigned base = (call - 1u) * (unsigned)(nseq - NPUSH) - (unsigned)(nw + NPUSH);
    const unsigned rel = (unsigned)(size_t)(prog + 2);
    const unsigned nofl = (unsigned)OFF_FN + (unsigned)(vg * 128 + wid * 4);
    const unsigned m2 = 1u << xcc_id(), fcop = ((lane & 7) + (lane == (int)xcc_id() ? 8u : 0u)) * FSTR;
    auto flags = [&](unsigned fl) {
        asm volatile("s_mov_b64 exec, %3\n\tglobal_store_dword %0, %1, %2 sc0 sc1\n\ts_mov_b32 exec_lo, %4\n\tglobal_store_dword %0, %1, %2\n\ts_mov_b64 exec, -1"
                     :: "v"(fl + fcop), "v"(call), "s"(myws), "s"((unsigned long long)(m2 ^ 255u)), "s"(m2) : "memory");
    };
    unsigned pfl = nofl;
    const unsigned tz = __builtin_amdgcn_readfirstlane(prog[4]);
    if (TOUCH & 2) {
        const int id = __builtin_amdgcn_readfirstlane((int)tab[cur]), rb = id & 31;
        unsigned off, fl, d;
        if (id < PK_W) place_x(rb < 24 ? 3 * qs + (rb >> 3) : 12 + qs, rb, id >> 5, call, off, fl);
        else place_w(16 + 4 * qs + (id & 3), (id - PK_W) >> 2, off, fl);
        const char* a = lane ? myws + off : pk + pk_off(id < PK_W ? id : id + q * 512, call);
        if (lane < 2) asm volatile("global_load_dword %0, %1, off sc0 sc1\n\ts_waitcnt vmcnt(0)" : "=&v"(d) : "v"(a) : "memory");
    }
    for (;;) {
        const bool isout = cross && cur >= NCI;
        const int id = isout ? 0 : __builtin_amdgcn_readfirstlane((int)tab[cur]);
        const int slot = isout ? PK_O + q * NO + cur - NCI : id < PK_W ? id : id + q * 512;
        unsigned mode;
        if (pfl != nofl) {
            const unsigned i0 = __hip_atomic_load(info + cur, __ATOMIC_RELAXED, __HIP_MEMORY_SCOPE_WORKGROUP);
            const unsigned i1 = __hip_atomic_load(prog + 1, __ATOMIC_RELAXED, __HIP_MEMORY_SCOPE_WORKGROUP), i2 = __hip_atomic_load(prog + 2, __ATOMIC_RELAXED, __HIP_MEMORY_SCOPE_WORKGROUP);
            if (__builtin_amdgcn_readfirstlane(!DEFER || i0 - call * 32u > 31u || i1 - i2 >= win)) {
                WAIT_V(0);
                flags(pfl);
                pfl = nofl;
            }
        }
        for (;;) {
            mode = __builtin_amdgcn_readfirstlane(__hip_atomic_load(info + cur, __ATOMIC_ACQUIRE, __HIP_MEMORY_SCOPE_WORKGROUP)) - call * 32u;
            if (mode <= 31u) break;
            __builtin_amdgcn_s_sleep(1);
        }
        const char* src = pk + (pk_off(slot, call) + (mode & 16u) * 512u);
        mode &= 15u;
        const unsigned long long mask = lane_mask(mode * 8u);
        unsigned my = 0;
        if (lane == 0) my = __hip_atomic_fetch_add(prog + 1, 1u, __ATOMIC_RELAXED, __HIP_MEMORY_SCOPE_WORKGROUP);
        my = __builtin_amdgcn_readfirstlane(my) - win - (cur < HWIN && cur >= HWIN0);
        while ((int)(my - __builtin_amdgcn_readfirstlane(__hip_atomic_load(prog + 2, __ATOMIC_RELAXED, __HIP_MEMORY_SCOPE_WORKGROUP))) >= 0) { }
        {
            unsigned ts = 0;
            if (lane == 0) {
                __hip_atomic_fetch_max(prog + 3, ((unsigned)__builtin_amdgcn_s_memrealtime() - tz) << 8, __ATOMIC_RELAXED, __HIP_MEMORY_SCOPE_WORKGROUP);
                ts = __hip_atomic_fetch_add(prog + 3, (mode >= M_RAW ? 128u : 104u + ((mode + 1u) & ~1u)) * (cross ? PRATE : PRATEI), __ATOMIC_RELAXED, __HIP_MEMORY_SCOPE_WORKGROUP);
            }
            ts = __builtin_amdgcn_readfirstlane(ts);
            ts = (ts >> 8) + tz - PSLACK;
            while ((int)((unsigned)__builtin_amdgcn_s_memrealtime() - ts) < 0) { }
        }
        u4 h[8]; unsigned tk;
        const unsigned m1 = pfl != nofl ? m2 ^ 255u : 256u;
        const unsigned tis = (unsigned)__builtin_amdgcn_s_memrealtime();
        if (mode >= M_RAW) {
            asm volatile(PULL6 "global_load_dwordx4 %[h6], %[v16], %[src] offset:2048 sc0 sc1\n\tglobal_load_dwordx4 %[h7], %[v16], %[src] offset:3072 sc0 sc1\n\t" PULLT
                         : PULLO, [h6] "=&v"(h[6]), [h7] "=&v"(h[7]) : PULLI, [rn] "n"(RELAT) : "memory");
        } else {
            u2 p0, p1;
            asm volatile(PULL6 "global_load_dwordx2 %[p0], %[v8], %[src] offset:2048 sc0 sc1\n\t"
                         "s_mov_b64 exec, %[mask]\n\tglobal_load_dwordx2 %[p1], %[v8], %[src] offset:2560 sc0 sc1\n\t" PULLT
                         : PULLO, [p0] "=&v"(p0), [p1] "=&v"(p1)
                         : PULLI, [v8] "v"(vl16 >> 1), [mask] "s"(mask), [rn] "n"(RELAT) : "memory");
            h[6] = (u4){p0[0], p0[1], p1[0], p1[1]};
        }
        const unsigned lat = (unsigned)__builtin_amdgcn_s_memrealtime() - tis;
        const unsigned nxt = __builtin_amdgcn_readfirstlane(tk) - base;
        if (lane == 0) __hip_atomic_fetch_max(prog, nxt, __ATOMIC_RELAXED, __HIP_MEMORY_SCOPE_WORKGROUP);
        if (isout) {
            const int o = cur - NCI;
            int panel, blk, col;
            if (o < O_B) {
                const int r = o < O_A0 ? o : o < O_C ? (o - O_A0) & 127 : o - O_C, hn = o < O_A0 ? 16 : o < O_C ? 64 : 48;
                const int hf = r >= hn, t = (r - hf * hn) >> 3;
                panel = o < O_A0 ? 3 * qs + 2 : o < O_C ? 3 * (t >> 1) + (o >= O_A1) : 12 + ((q + 1 + (t >> 1)) & 3);
                blk = (r & 6) + hf; col = (t & 1) * 128 + (r & 1) * 64;
            } else {
                const int j = (o - O_B) / 24, r = o - O_B - j * 24;
                panel = 3 * ((qs + 1 + (r >> 3)) & 3) + 2; blk = 2 * j + ((r >> 1) & 1); col = ((r >> 2) & 1) * 128 + (r & 1) * 64;
            }
            char* ob = (char*)P.out + (size_t)(panel_row(peer, panel, blk) * NLOC + qs * 256 + col) * 4;
            if (mode < M_RAW) unpack_unit<true>(h, scr, lut, lane);
            if (mode == M_DEAD) h[0] = (u4){0x7FC00000u, 0x7FC00000u, 0x7FC00000u, 0x7FC00000u};
#pragma unroll
            for (int k = 0; k < 8; ++k) GST4P(ovoff, h[k], ob + ((k >> 2) * 16 + (k & 3)) * (NLOC * 4));
            if (!DEFER) WAIT_V(0);
            pfl = nofl;
        } else {
            if (mode < M_RAW) unpack_unit<false>(h, scr, lut, lane);
            unsigned off, fl;
            const int rb = id & 31;
            if (id < PK_W) place_x(rb < 24 ? 3 * qs + (rb >> 3) : 12 + qs, rb, id >> 5, call, off, fl);
            else place_w(16 + 4 * qs + (id & 3), (id - PK_W) >> 2, off, fl);
            store8(h, myws + off, id < PK_W ? 32768u : 4096u, vl16);
            pfl = fl;
        }
        if (nxt >= (unsigned)nseq) break;
        cur = (int)nxt;
    }
    WAIT_V(0);
    if (pfl != nofl) flags(pfl);
}

#undef VBLK
#define VBLK (blockIdx.x < NGEMM ? blockIdx.x : blockIdx.x + NPC)
__device__ __forceinline__ float silu(float y) {
    const float c = __builtin_amdgcn_fmed3f(y, -60.0f, 60.0f);
    const float e = __builtin_amdgcn_exp2f(c * -1.4426950408889634f);
    return y * __builtin_amdgcn_rcpf(1.0f + e);
}
__device__ __forceinline__ u4 silu4(const f4 (&a)[4], int r, bool dead) {
    f4 o = {silu(a[0][r]), silu(a[1][r]), silu(a[2][r]), silu(a[3][r])};
    const float nanv = __builtin_nanf("");
    if (dead) o = (f4){nanv, nanv, nanv, nanv};
    return __builtin_bit_cast(u4, o);
}

#define BAR() do { __builtin_amdgcn_sched_barrier(0); __builtin_amdgcn_s_barrier(); asm volatile("" ::: "memory"); __builtin_amdgcn_sched_barrier(0); } while (0)
#define GLDS(g, l, imm) __builtin_amdgcn_global_load_lds((const GAS void*)(g), (LAS void*)(l), 16, imm, AUX)
template <int V> struct IC { static constexpr int value = V; };

__device__ __forceinline__ void gemm_panel(const CAS Params& P, LAS char* lds, char* myws, int panel, int cg, int cg2, int o0, int ohalf,
                                           unsigned call, int wid, int lane) {
    constexpr int ABYTES = 32768, STAGE = ABYTES + 16384;
    const char* ga = myws + panel_off(panel, call) + wid * 4096 + lane * 16;
    const char* gb = myws + OFF_DATA + A_B + (size_t)cg * CG_B + wid * 2048 + lane * 16;
    const int wm = wid >> 1, wn = wid & 1;
    const unsigned frow = flag_copy(lane) + ((lane & 2) ? (unsigned)OFF_FB + (unsigned)cg * 1024u : (unsigned)OFF_FA + (unsigned)panel * 2048u + (unsigned)(lane & 1) * 16u);
    const int fch = lane >> 2 & 7;
    const unsigned fstep = (lane & 2) ? 16u : 32u;
    GAS unsigned* tmo = (GAS unsigned*)(myws + OFF_TMO);
    char* oflag = myws + (OFF_FP + (call & 1) * FP_B) + (((cg - 8) >> 1) * NSEQ + NCI + o0 + wid) * 4;
    int ready = 0;
    unsigned tv = 0;

    auto early = [&](int c) {
        if (c < ready || c >= NCH) return;
        const int cc = c + fch;
        __builtin_amdgcn_global_load_lds((const GAS void*)(myws + (frow + (unsigned)(cc < NCH ? cc : NCH - 1) * fstep)), (LAS void*)(lds + 3 * STAGE), 16, 0, AUX);
    };
    auto ensure = [&](int c, bool pend) {
        if (c < ready) return;
        const unsigned long long t0 = __builtin_amdgcn_s_memrealtime();
        int lim = 7;
        for (;;) {
            int cc = c + (fch < lim ? fch : lim); cc = cc < NCH ? cc : NCH - 1;
            u4 f;
            if (pend) asm volatile("s_waitcnt vmcnt(%2)\n\tds_read_b128 %0, %1\n\ts_waitcnt lgkmcnt(0)"
                                   : "=&v"(f) : "v"((unsigned)(size_t)(lds + 3 * STAGE) + (unsigned)lane * 16u), "n"(3 * PD) : "memory");
            else asm volatile("global_load_dwordx4 %0, %1, %2 sc0 sc1\n\ts_waitcnt vmcnt(0)"
                              : "=&v"(f) : "v"(frow + (unsigned)cc * fstep), "s"(myws) : "memory");
            const unsigned m = unset(f[0], call) | unset(f[1], call) | unset(f[2], call) | unset(f[3], call);
            const int n = m ? (__builtin_ctz(m) >> 2) : lim + 1;
            if (n > 0) { ready = c + n; return; }
            if (pend) { pend = false; continue; }
            if (
                __builtin_amdgcn_s_memrealtime() - t0 > DEADLINE) {
                __hip_atomic_store(tmo, call, __ATOMIC_RELAXED, __HIP_MEMORY_SCOPE_AGENT);
                ready = NCH; return;
            }
            lim = 0;
            __builtin_amdgcn_s_sleep(6);
        }
    };
    auto stage = [&](int t, int kf, int bufoff) {
        const char* a = ga + (size_t)t * 32768 + kf * 1024; const char* b = gb + (size_t)t * 16384 + kf * 1024;
        LAS char* la = lds + bufoff + wid * 4096 + kf * 1024; LAS char* lb = lds + bufoff + ABYTES + wid * 2048 + kf * 1024;
        GLDS(a, la, 0); GLDS(a, la, 2048); GLDS(b, lb, 0);
    };

    f4 acc[4][4];
#pragma unroll
    for (int m = 0; m < 4; ++m)
#pragma unroll
        for (int n = 0; n < 4; ++n) acc[m][n] = (f4){0.f, 0.f, 0.f, 0.f};

    const int aoff = wm * 8192 + lane * 16, boff = ABYTES + wn * 8192 + lane * 16;
    auto phase = [&](auto waitn, int rbuf, int kf, int lt, int lkf, int lbuf, int pc, int pe = NCH) {
        const LAS char* pa = lds + rbuf + aoff + kf * 1024; const LAS char* pb = lds + rbuf + boff + kf * 1024;
        h8 a[4], b[4];
#pragma unroll
        for (int n = 0; n < 4; ++n) b[n] = *(const LAS h8*)(pb + n * 2048);
#pragma unroll
        for (int m = 0; m < 4; ++m) a[m] = *(const LAS h8*)(pa + m * 2048);
        __builtin_amdgcn_sched_barrier(0);
        if (lt >= 0) stage(lt, lkf, lbuf);
        if (decltype(waitn)::value >= 0) WAIT_V(decltype(waitn)::value < 0 ? 0 : decltype(waitn)::value);
        WAIT_L0();
        BAR();
        __builtin_amdgcn_s_setprio(1);
#pragma unroll
        for (int m = 0; m < 4; ++m)
#pragma unroll
            for (int n = 0; n < 4; ++n)
                asm volatile("v_mfma_f32_16x16x32_f16 %0, %1, %2, %0" : "+v"(acc[m][n]) : "v"(a[m]), "v"(b[n]));
        __builtin_amdgcn_s_setprio(0);
        __builtin_amdgcn_sched_barrier(0);
        if (wid == 0) { if (pc >= 0) { ensure(pc, PD > 0); TREC(pc); } early(pe); }
        BAR();
    };

    if (wid == 0) { ensure(0, false); ensure(1, false); }
    BAR();
    STAMP(1);
    stage(0, 0, 0); stage(0, 1, 0); stage(1, 0, STAGE); stage(1, 1, STAGE); stage(2, 0, 2 * STAGE);
    WAIT_V(12);
    BAR();
    if (wid >= 4) BAR();
    int rb = 0, pb = 2 * STAGE;
    for (int t = 0; t < NKT - 3; ++t) {
        phase(IC<12>{}, rb, 0, t + 2, 1, pb, (t & 1) ? (t + 3) >> 1 : -1, PD == 2 && !(t & 1) ? (t + 4) >> 1 : NCH);
        phase(IC<12>{}, rb, 1, t + 3, 0, rb, -1, PD == 1 && !(t & 1) ? (t + 4) >> 1 : NCH);
        pb = rb; rb = rb == 2 * STAGE ? 0 : rb + STAGE;
    }
    static_assert((NKT - 3) % 3 == 2, "tail buffers");
    phase(IC<12>{}, 2 * STAGE, 0, NKT - 1, 1, STAGE, -1);
    if (wid == 4) { const char* f = myws + b_flags(cg2, lane); GLDS(f, lds + BFLG, 0); GLDS(f + 512, lds + BFLG + 1024, 0); }
    FENCE();
    const int pk0 = PK_O + ((cg < 8 ? 0 : cg - 8) >> 1) * NO + o0 + wid;
    const unsigned hv[2] = {pk_half(myws, pk0, call), pk_half(myws, pk0 + ohalf, call)};
    const unsigned tm = __hip_atomic_load(tmo, __ATOMIC_RELAXED, __HIP_MEMORY_SCOPE_AGENT);
    FENCE();
    phase(IC<9 + 3>{}, 2 * STAGE, 1, -1, 0, 0, -1);
    phase(IC<6 + 3>{}, 0, 0, -1, 0, 0, -1);
    phase(IC<3 + 3>{}, 0, 1, -1, 0, 0, -1);
    phase(IC<0 + 3>{}, STAGE, 0, -1, 0, 0, -1);
    phase(IC<-1>{}, STAGE, 1, -1, 0, 0, -1);
#pragma unroll
    for (int m = 0; m < 4; ++m) asm volatile("s_nop 7" : "+v"(acc[m][0]), "+v"(acc[m][1]), "+v"(acc[m][2]), "+v"(acc[m][3]));
    if (wid < 4) BAR();
    STAMP(2);
    TDUMP(0);

    const bool dead = __builtin_amdgcn_readfirstlane(tm) == call;
    const unsigned vl16 = (unsigned)lane * 16, ovoff = (unsigned)(lane >> 4) * (4 * NLOC * 4) + (unsigned)(lane & 15) * 16;
    unsigned mode = 0;
#pragma unroll
    for (int u = 0; u < 2; ++u) {
        u4 h[8];
#pragma unroll
        for (int k = 0; k < 8; ++k) h[k] = silu4(acc[2 * u + (k >> 2)], k & 3, dead);
        if (cg < 8) {
            char* ob = (char*)P.out + (size_t)(panel_row(P.rank, panel, 2 * wm + u) * NLOC + cg * 128 + wn * 64) * 4;
#pragma unroll
            for (int k = 0; k < 8; ++k) GST4P(ovoff, h[k], ob + ((k >> 2) * 16 + (k & 3)) * (NLOC * 4));
        } else {
            const unsigned prev = mode;
            unsigned r0 = 0, r2 = 0;
            mode = pack_unit<true>(h, lds + wid * 4096, lds + (8 * PBUF + 16), lane, r0, r2);
            if (u) { WAIT_V(0); if (lane == 0) flag_store(oflag, call * 32u + prev); }
            const unsigned half = SMODE == 0 ? wid & 1 : (~__builtin_amdgcn_readfirstlane(hv[u]) ^ (SMODE == 2)) & 1u;
            store_unit<true>(h, myws + (pk_off(pk0 + u * ohalf, call) + half * 8192), mode, r0, r2, vl16);
            mode += half * 16u;
        }
    }
    if (cg >= 8) {
        WAIT_V(0);
        if (lane == 0) flag_store(oflag + ohalf * 4, call * 32u + mode);
    }
    STAMP(3);
}

__device__ __forceinline__ bool all_set(u4 f, u4 e, unsigned call) {
    unsigned m = 0;
#pragma unroll
    for (int i = 0; i < 4; ++i) m |= unset(f[i], call) | unset(e[i], call);
    return m == 0;
}
__device__ __forceinline__ bool all_b(const char* myws, int cg, unsigned call, int lane) {
    u4 f, e;
    asm volatile("global_load_dwordx4 %0, %2, %3 sc0 sc1\n\tglobal_load_dwordx4 %1, %2, %3 offset:512 sc0 sc1\n\ts_waitcnt vmcnt(0)"
                 : "=&v"(f), "=&v"(e) : "v"(b_flags(cg, lane)), "s"(myws) : "memory");
    return all_set(f, e, call);
}

__device__ __forceinline__ void gemm_quarter(const CAS Params& P, LAS char* lds, char* myws, int panel, int qt, int cg, int o0,
                                             unsigned call, int wid, int lane) {
    constexpr int STAGE = 49152;
    const int kp = wid >> 2, lw = wid & 3, wm = lw >> 1, wn = lw & 1;
    const char* gsrc = myws + OFF_DATA + lane * 16 + (lw < 2 ? (size_t)panel * PANEL_B + kp * 32768 + qt * 8192
                                                             : A_B + (size_t)cg * CG_B + kp * 16384 + (lw - 2) * 8192);
    const unsigned gstep = lw < 2 ? 65536u : 32768u;
    LAS char* ring = lds + kp * 24576;
    LAS unsigned* pl = (LAS unsigned*)(lds + 160768);
    const unsigned frow = flag_copy(lane) + (unsigned)OFF_FA + (unsigned)(panel * 2048 + qt * 8 + (lane & 1) * 4);
    GAS unsigned* tmo = (GAS unsigned*)(myws + OFF_TMO);
    char* oflag = myws + (OFF_FP + (call & 1) * FP_B) + (((cg - 8) >> 1) * NSEQ + NCI + o0) * 4;
    unsigned tv = 0;
    int ready = 0; bool pend = false;
#define QPOLL 8
    auto paddr = [&]() { const int cc = ready + (lane >> 1 & 15); return (lane & 31) < 2 * QPOLL ? frow + (unsigned)(cc < NCH ? cc : NCH - 1) * 32u : (unsigned)OFF_TMO; };
    auto late = [&](unsigned tm, unsigned long long t0) {
        if (tm != call && __builtin_amdgcn_s_memrealtime() - t0 <= DEADLINE) return false;
        __hip_atomic_store(tmo, call, __ATOMIC_RELAXED, __HIP_MEMORY_SCOPE_AGENT);
        WAIT_V(0);
        ready = NCH;
        return true;
    };
    auto ensure = [&](int c) {
        if (c >= ready) {
            const unsigned long long t0 = __builtin_amdgcn_s_memrealtime();
            for (;;) {
                unsigned f;
                if (pend) { WAIT_V(0); f = pl[lane]; pend = false; }
                else asm volatile("global_load_dword %0, %1, %2 sc0 sc1\n\ts_waitcnt vmcnt(0)" : "=&v"(f) : "v"(paddr()), "s"(myws) : "memory");
                ready += __builtin_ctz(unset(f, call) | (1u << (2 * QPOLL))) >> 1;
                if (c < ready || late(__builtin_amdgcn_readlane(f, 31), t0)) break;
                __builtin_amdgcn_s_sleep(QSLEEP);
            }
        }
        if (!pend && ready < NCH && ready <= c + 2) {
            __builtin_amdgcn_global_load_lds((const GAS void*)(myws + paddr()), (LAS void*)pl, 4, 0, AUX);
            pend = true;
        }
    };
    auto stage = [&](int c, int buf) {
        const char* g = gsrc + (size_t)((unsigned)c * gstep);
        LAS char* l = ring + buf + (lw - 1) * 8192;
        GLDS(g, l, 0); GLDS(g, l, 1024); GLDS(g, l, 2048); GLDS(g, l, 3072);
        GLDS(g + 4096, l + 4096, 0); GLDS(g + 4096, l + 4096, 1024); GLDS(g + 4096, l + 4096, 2048); GLDS(g + 4096, l + 4096, 3072);
    };

    const int pk0 = PK_O + ((cg < 8 ? 0 : cg - 8) >> 1) * NO + o0;
    if (wid == 0) pl[64] = pk_half(myws, pk0, call);
    f4 acc[2][4];
#pragma unroll
    for (int m = 0; m < 2; ++m)
#pragma unroll
        for (int n = 0; n < 4; ++n) acc[m][n] = (f4){0.f, 0.f, 0.f, 0.f};
    const int aoff = wm * 4096 + lane * 16, boff = 8192 + wn * 8192 + lane * 16;
    auto phase = [&](auto waitn, auto role, int buf, int lc) {
        h8 a[4], b[8];
#pragma unroll
        for (int n = 0; n < 8; ++n) b[n] = *(const LAS h8*)(ring + buf + boff + n * 1024);
#pragma unroll
        for (int m = 0; m < 4; ++m) a[m] = *(const LAS h8*)(ring + buf + aoff + m * 1024);
        __builtin_amdgcn_sched_barrier(0);
        if (decltype(role)::value == 2 && wid == 0) {
            ensure(lc); TREC(lc);
        }
        WAIT_L0();
        BAR();
        if (decltype(role)::value == 1) stage(lc, buf);
#pragma unroll
        for (int k = 0; k < 2; ++k)
#pragma unroll
            for (int m = 0; m < 2; ++m)
#pragma unroll
                for (int n = 0; n < 4; ++n) acc[m][n] = __builtin_amdgcn_mfma_f32_16x16x32_f16(a[m * 2 + k], b[n * 2 + k], acc[m][n], 0, 0, 0);
        if (decltype(role)::value == 1) {
#pragma unroll
            for (int i = 0; i < 8; ++i) { __builtin_amdgcn_sched_group_barrier(0x010, 1, 0); __builtin_amdgcn_sched_group_barrier(0x008, 2, 0); }
        }
        __builtin_amdgcn_sched_barrier(0);
        if (decltype(waitn)::value >= 0) { WAIT_V(decltype(waitn)::value < 0 ? 0 : decltype(waitn)::value); BAR(); }
    };

    if (wid == 0 && ready < NCH) {
        const unsigned long long t0 = __builtin_amdgcn_s_memrealtime();
        for (;;) {
            if (all_b(myws, cg, call, lane) ||
                late(__builtin_amdgcn_readfirstlane(__hip_atomic_load(tmo, __ATOMIC_RELAXED, __HIP_MEMORY_SCOPE_AGENT)), t0)) break;
            __builtin_amdgcn_s_sleep(6);
        }
        ensure(2);
    }
    WAIT_L0();
    BAR();
    STAMP(4);
    const unsigned half = SMODE == 0 ? cg & 1 : (~__builtin_amdgcn_readfirstlane(pl[64]) ^ (SMODE == 2)) & 1u;
    if (lw != 0) { stage(0, 0); stage(1, STAGE); stage(2, 2 * STAGE); }
    WAIT_V(16);
    BAR();
    if (kp) BAR();
    if (lw != 0) for (int c = 3, buf = 0; c < NCH; ++c, buf = buf == 2 * STAGE ? 0 : buf + STAGE) phase(IC<16>{}, IC<1>{}, buf, c);
    else for (int c = 3, buf = 0; c < NCH; ++c, buf = buf == 2 * STAGE ? 0 : buf + STAGE) phase(IC<16>{}, IC<2>{}, buf, c);
    static_assert((NCH - 3) % 3 == 1, "tail buffers");
    const unsigned tm = __hip_atomic_load(tmo, __ATOMIC_RELAXED, __HIP_MEMORY_SCOPE_AGENT);
    phase(IC<8>{}, IC<0>{}, STAGE, -1);
    phase(IC<0>{}, IC<0>{}, 2 * STAGE, -1);
    phase(IC<-1>{}, IC<0>{}, 0, -1);
    f4 r[4];
#pragma unroll
    for (int n = 0; n < 4; ++n) {
        const f4 a0 = acc[0][n], a1 = acc[1][n];
        r[n] = kp ? a1 : a0;
        *(LAS f4*)(ring + lw * 4096 + n * 1024 + lane * 16) = kp ? a0 : a1;
    }
    const bool dead = __builtin_amdgcn_readfirstlane(tm) == call;
    WAIT_L0();
    BAR();
    if (!kp) BAR();
    STAMP(5);
#pragma unroll
    for (int n = 0; n < 4; ++n) r[n] += *(const LAS f4*)(lds + (kp ^ 1) * 24576 + lw * 4096 + n * 1024 + lane * 16);
    if (cg < 8) {
        const unsigned ovoff = (unsigned)(lane >> 4) * (4 * NLOC * 4) + (unsigned)(lane & 15) * 16;
        char* ob = (char*)P.out + (size_t)((panel_row(P.rank, panel, 2 * qt + wm) + kp * 16) * NLOC + cg * 128 + wn * 64) * 4;
#pragma unroll
        for (int k = 0; k < 4; ++k) { const u4 o = silu4(r, k, dead); GST4P(ovoff, o, ob + k * (NLOC * 4)); }
    } else {
        char* d0 = myws + (pk_off(pk0 + lw, call) + half * 8192 + kp * 4096);
#pragma unroll
        for (int k = 0; k < 4; ++k) { const u4 o = silu4(r, k, dead); GST4((unsigned)lane * 16, o, d0 + k * 1024, 0); }
        WAIT_V(0);
        BAR();
        if (wid == 0 && lane < 4) flag_store(oflag + lane * 4, call * 32u + half * 16u + M_RAW);
    }
    STAMP(6);
    TDUMP(1);
}

#define QPK 2
#define QSL1 1
#define HALF 1
#define NPOLL 2
#define QAHEAD 12
template <int NB>
__device__ __forceinline__ void gemm_ring(const CAS Params& P, LAS char* lds, char* myws, int panel, int qt, int cg, int hb, int o0, bool pre,
                                          unsigned call, int wid, int lane) {
    volatile LAS unsigned* ctl = (volatile LAS unsigned*)(lds + 160768);
    auto put = [&](int i, unsigned v) {
        asm volatile("s_mov_b64 exec, 1\n\tds_write_b32 %0, %1\n\ts_mov_b64 exec, -1" :: "v"((unsigned)(size_t)(ctl + i)), "v"(v) : "memory");
    };
    GAS unsigned* tmo = (GAS unsigned*)(myws + OFF_TMO);
    auto late = [&](unsigned long long t0) {
        if (__builtin_amdgcn_s_memrealtime() - t0 <= DEADLINE) return false;
        __hip_atomic_store(tmo, call, __ATOMIC_RELAXED, __HIP_MEMORY_SCOPE_AGENT);
        WAIT_V(0);
        return true;
    };
    const int pk0 = PK_O + ((cg < 8 ? 0 : cg - 8) >> 1) * NO;
    if (wid == 4) {
        if (lane < 8) ctl[lane] = 0;
        const unsigned long long t0 = __builtin_amdgcn_s_memrealtime();
        if (!pre || !all_set(*(const LAS u4*)(lds + BFLG + lane * 16), *(const LAS u4*)(lds + BFLG + 1024 + lane * 16), call)) for (;;) {
            if (all_b(myws, cg, call, lane) || late(t0)) break;
            __builtin_amdgcn_s_sleep(6);
        }
    }
    WAIT_L0();
    BAR();
    STAMP(4);
    const unsigned vl16 = (unsigned)lane * 16;
    if (wid >= 6) {
        const unsigned frow = flag_copy(lane) + (unsigned)OFF_FA + (unsigned)(panel * 2048 + qt * 8 + (lane & 1) * 4);
        if (wid == 7) put(10, pk_half(myws, pk0 + o0, call));
        unsigned long long t0 = __builtin_amdgcn_s_memrealtime();
        unsigned f;
        for (;;) {
            const u4 st = *(const volatile LAS u4*)ctl;
            const int c = __builtin_amdgcn_readfirstlane(st[2]);
            if (c >= NCH || wid >= 6 + NPOLL) break;
            if (c > (int)__builtin_amdgcn_readfirstlane(st[0]) + QAHEAD) { __builtin_amdgcn_s_sleep(4); t0 = __builtin_amdgcn_s_memrealtime(); continue; }
            asm volatile("global_load_dword %0, %1, %2 sc0 sc1\n\ts_waitcnt vmcnt(0)" : "=&v"(f) : "v"(frow + (unsigned)min(c + (lane >> 1 & 7), NCH - 1) * 32u), "s"(myws) : "memory");
            int n = __builtin_ctz(unset(f, call) | 65536u) >> 1;
            if (n) t0 = __builtin_amdgcn_s_memrealtime();
            else if (late(t0)) n = NCH;
            if (n) asm volatile("s_mov_b64 exec, 1\n\tds_max_u32 %0, %1\n\ts_mov_b64 exec, -1" :: "v"((unsigned)(size_t)(ctl + 2)), "v"(c + n) : "memory");
            else __builtin_amdgcn_s_sleep(QSLEEP);
        }
        asm volatile("global_load_dword %0, %1, %2 sc0 sc1\n\ts_waitcnt vmcnt(0)" : "=&v"(f) : "v"((unsigned)OFF_TMO), "s"(myws) : "memory");
        put(2 + wid, f);
        WAIT_L0();
        BAR();
    } else if (wid >= 4) {
        const char* g = myws + OFF_DATA + ((size_t)panel * PANEL_B + qt * 8192 + (wid - 4) * 32768) + vl16;
        int c = 0, pub = 0;
        while (pub < NCH) {
            const u4 fr = *(const volatile LAS u4*)(ctl + 4);
            const int lim = min((int)__builtin_amdgcn_readfirstlane(min(min(min(fr[0], fr[1]), min(fr[2], fr[3])) + 8u, ctl[2])), min(pub + 7, NCH));
            const bool go = c < lim;
            for (; c < lim; ++c, g += 65536) {
                LAS char* l = lds + (c & 7) * 16384 + (wid - 4) * 8192;
                GLDS(g, l, 0); GLDS(g, l, 1024); GLDS(g, l, 2048); GLDS(g, l, 3072);
                GLDS(g + 4096, l + 4096, 0); GLDS(g + 4096, l + 4096, 1024); GLDS(g + 4096, l + 4096, 2048); GLDS(g + 4096, l + 4096, 3072);
            }
            FENCE();
            const unsigned st = __builtin_amdgcn_s_getreg(7 | (31 << 11));
            const int k = (int)(((st & 15u) | (st >> 18 & 48u)) + 7u) >> 3;
            if (c - k > pub) {
                if (k < 4) { if (k < 2) { if (k == 0) WAIT_V(0); else WAIT_V(8); } else { if (k == 2) WAIT_V(16); else WAIT_V(24); } }
                else if (k == 4) WAIT_V(32); else if (k == 5) WAIT_V(40); else WAIT_V(48);
                pub = c - k;
                put(wid - 4, (unsigned)pub);
            } else if (!go) __builtin_amdgcn_s_sleep(QSL1);
        }
        WAIT_L0();
        BAR();
    } else {
        constexpr int R = 16 / NB;
        const char* gb = myws + OFF_DATA + (A_B + (size_t)cg * CG_B + (NB == 2 ? wid * 4096 : hb * 8192 + wid * 2048));
        f4 acc[4][NB];
#pragma unroll
        for (int i = 0; i < 4 * NB; ++i) acc[i / NB][i % NB] = (f4){0.f, 0.f, 0.f, 0.f};
        asm volatile("" ::: "a0", "a127");
        unsigned long long em = ~0ull;
        auto loadb = [&](auto sl, auto kk) {
            constexpr int r = decltype(sl)::value * 8 * NB + decltype(kk)::value * 4, o = decltype(kk)::value * 1024;
            if (NB == 2) asm volatile("s_mov_b64 exec, %2\n\tglobal_load_dwordx4 a[%3:%4], %0, %1 offset:%7 sc0 sc1\n\t"
                                      "global_load_dwordx4 a[%5:%6], %0, %1 offset:%8 sc0 sc1\n\ts_mov_b64 exec, -1"
                                      :: "v"(vl16), "s"(gb), "s"(em), "n"(r), "n"(r + 3), "n"(r + 8), "n"(r + 11), "n"(o), "n"(o + 2048) : "memory");
            else asm volatile("s_mov_b64 exec, %2\n\tglobal_load_dwordx4 a[%3:%4], %0, %1 offset:%5 sc0 sc1\n\ts_mov_b64 exec, -1"
                              :: "v"(vl16), "s"(gb), "s"(em), "n"(r), "n"(r + 3), "n"(o) : "memory");
            if (decltype(kk)::value) gb += 16384;
        };
        unsigned la = vl16 + (unsigned)(size_t)lds;
        auto fetch = [&](h8 (&a)[8], int i, int t) { a[i] = *(const LAS h8*)(size_t)(la + (unsigned)(t * 8192 + i * 1024)); };
        auto step = [&](auto sl, const h8 (&a)[8], h8 (&a2)[8], int c0) {
            constexpr int S = decltype(sl)::value, T = (S + 1) & 7;
            if (T == 0) la ^= 65536u;
            WAIT_V(32 - 2 * NB);
            u2 x = {0, 0};
            if (S & 1) x = *(const volatile LAS u2*)ctl;
#pragma unroll
            for (int i = 0; i < 8 * NB; ++i) {
                const int j = i / NB, k = j >> 2, m = j & 3, n = i % NB, r = S * 8 * NB + n * 8 + k * 4;
                asm volatile("v_mfma_f32_16x16x32_f16 %0, %1, a[%2:%3], %0" : "+v"(acc[m][n]) : "v"(a[m * 2 + k]), "n"(r), "n"(r + 3) : "memory");
                if (n == 0) fetch(a2, m * 2 + k, T);
                if (i == 4 * NB) loadb(sl, IC<0>{});
            }
            if (!(S & 1)) put(4 + wid, (unsigned)(c0 + S / 2 + 1));
            loadb(sl, IC<1>{});
            if (S & 1) {
                const int need = min(c0 + (S + 1) / 2, NCH - 1);
                if ((int)__builtin_amdgcn_readfirstlane(min(x[0], x[1])) <= need) {
                    do x = *(const volatile LAS u2*)ctl; while ((int)__builtin_amdgcn_readfirstlane(min(x[0], x[1])) <= need);
                    FENCE();
#pragma unroll
                    for (int i = 0; i < 8; ++i) fetch(a2, (i & 3) * 2 + (i >> 2), T);
                }
            }
        };
        auto fill = [&](auto sl) { loadb(sl, IC<0>{}); loadb(sl, IC<1>{}); };
        fill(IC<0>{}); fill(IC<1>{}); fill(IC<2>{}); fill(IC<3>{}); fill(IC<4>{}); fill(IC<5>{}); fill(IC<6>{}); fill(IC<7>{});
        if constexpr (NB == 1) { fill(IC<8>{}); fill(IC<9>{}); fill(IC<10>{}); fill(IC<11>{}); fill(IC<12>{}); fill(IC<13>{}); fill(IC<14>{}); fill(IC<15>{}); }
        h8 a0[8], a1[8];
        while (__builtin_amdgcn_readfirstlane(min(ctl[0], ctl[1])) == 0) { }
        FENCE();
#pragma unroll
        for (int i = 0; i < 8; ++i) fetch(a0, (i & 3) * 2 + (i >> 2), 0);
#pragma unroll 1
        for (int c = 0; c < NCH; c += R / 2) {
            if (c == NCH - R / 2) em = 1;
            step(IC<0>{}, a0, a1, c); step(IC<1>{}, a1, a0, c); step(IC<2>{}, a0, a1, c); step(IC<3>{}, a1, a0, c);
            step(IC<4>{}, a0, a1, c); step(IC<5>{}, a1, a0, c); step(IC<6>{}, a0, a1, c); step(IC<7>{}, a1, a0, c);
            if constexpr (NB == 1) {
                step(IC<8>{}, a0, a1, c); step(IC<9>{}, a1, a0, c); step(IC<10>{}, a0, a1, c); step(IC<11>{}, a1, a0, c);
                step(IC<12>{}, a0, a1, c); step(IC<13>{}, a1, a0, c); step(IC<14>{}, a0, a1, c); step(IC<15>{}, a1, a0, c);
            }
        }
#pragma unroll
        for (int n = 0; n < NB; ++n) asm volatile("s_nop 15" : "+v"(acc[0][n]), "+v"(acc[1][n]), "+v"(acc[2][n]), "+v"(acc[3][n]) :: "memory");
        const int wm = NB == 2 ? wid & 1 : wid >> 1, wn = NB == 2 ? wid >> 1 : hb, un = o0 + wm * 2 + wn;
        LAS char* ex = lds + 131072 + vl16;
        f4 o[NB][4];
#pragma unroll
        for (int i = 0; i < 4; ++i) {
            if constexpr (NB == 2) {
                const f4 lo = acc[i >> 1][i & 1], hi = acc[2 + (i >> 1)][i & 1];
                *(LAS f4*)(ex + wid * 4096 + i * 1024) = wm ? lo : hi;
                o[i >> 1][i & 1] = wm ? hi : lo;
            } else *(LAS f4*)(ex + i * 4096 + wid * 1024) = acc[i][0];
        }
        WAIT_L0();
        BAR();
        STAMP(5);
        const u4 tw = *(const volatile LAS u4*)(ctl + 8);
#pragma unroll
        for (int i = 0; i < 4; ++i) {
            if constexpr (NB == 2) {
                const f4 y = *(const LAS f4*)(ex + (wid ^ 1) * 4096 + i * 1024), z = o[i >> 1][i & 1];
                o[i >> 1][i & 1] = wm ? y : z; o[i >> 1][2 + (i & 1)] = wm ? z : y;
            } else o[0][i] = *(const LAS f4*)(ex + wid * 4096 + i * 1024);
        }
        const bool dead = __builtin_amdgcn_readfirstlane(tw[0] == call || tw[1] == call);
        const unsigned half = ~__builtin_amdgcn_readfirstlane(tw[2]) & 1u;
        char* dst = myws + (pk_off(pk0 + un, call) + half * 8192);
        char* oflag = myws + (OFF_FP + (call & 1) * FP_B) + (((cg - 8) >> 1) * NSEQ + NCI + un) * 4;
        u4 h[4 * NB];
#pragma unroll
        for (int k = 0; k < 4 * NB; ++k) h[k] = silu4(o[k >> 2], k & 3, dead);
        if constexpr (NB == 1) {
            dst += (wid & 1) * 4096;
            GST4(vl16, h[0], dst, 0); GST4(vl16, h[1], dst, 1024); GST4(vl16, h[2], dst, 2048); GST4(vl16, h[3], dst, 3072);
            WAIT_V(0);
            BAR();
            if (lane == 0 && !(wid & 1)) flag_store(oflag, call * 32u + half * 16u + M_RAW);
        } else if (cg < 8) {
            const unsigned ovoff = (unsigned)(lane >> 4) * (4 * NLOC * 4) + (unsigned)(lane & 15) * 16;
            char* ob = (char*)P.out + (size_t)(panel_row(P.rank, panel, 2 * qt + wm) * NLOC + cg * 128 + wn * 64) * 4;
#pragma unroll
            for (int k = 0; k < 8; ++k) GST4P(ovoff, h[k], ob + ((k >> 2) * 16 + (k & 3)) * (NLOC * 4));
        } else {
            unsigned mode = M_RAW, r0 = 0, r2 = 0;
            if (qt < QPK) mode = pack_unit<true>(h, lds + wid * 4096, lds + (8 * PBUF + 16), lane, r0, r2);
            store_unit<true>(h, dst, mode, r0, r2, vl16);
            WAIT_V(0);
            if (lane == 0) flag_store(oflag, call * 32u + half * 16u + mode);
        }
        STAMP(6);
    }
    if (NB == 1 && wid >= 4) BAR();
}

__host__ __device__ __forceinline__ int tile2(int slot, int sown) {
    return slot == 26 ? 0 : slot == 25 ? 7 : slot == 24 ? 2 : slot == sown ? 1 : slot < 4 ? slot + 3 : slot == 4 ? 8 : slot < 17 ? slot + 7
         : slot < 20 ? slot - 8 : 33 + ((slot - sown - 1) & 3);
}

__global__ void __launch_bounds__(512, 2) ag_gemm_kernel(const Params Parg) {
    const CAS Params& P = *(const CAS Params*)__builtin_amdgcn_kernarg_segment_ptr();
    (void)Parg;
    __shared__ __attribute__((aligned(16))) char lds_raw[5 * 32768];
    LAS char* lds = (LAS char*)lds_raw;
    const int lane = threadIdx.x & 63, wid = __builtin_amdgcn_readfirstlane(threadIdx.x >> 6);
    const int xcd = blockIdx.x & 7, slot = blockIdx.x >> 3;
    char* myws = P.ws[P.rank];
    const int q = P.rank & 3;
    const unsigned long long t0 = __builtin_amdgcn_s_memrealtime();
    const unsigned side = (__builtin_amdgcn_s_getreg(20 | (3 << 11)) >> 2) & 1u;
    int g = (int)blockIdx.x - NGEMM;
    if (slot == NGEMM / 8) g = NPC + xcd;
    else if (g >= 0) {
        if (threadIdx.x == 0) {
            GAS unsigned* rt = (GAS unsigned*)(myws + OFF_ROLE);
            const unsigned t = __hip_atomic_fetch_add(rt, side ? 65536u : 1u, __ATOMIC_RELAXED, __HIP_MEMORY_SCOPE_AGENT);
            if ((t & 65535u) + (t >> 16) >= 247u - NGEMM) __hip_atomic_store(rt, 0u, __ATOMIC_RELAXED, __HIP_MEMORY_SCOPE_AGENT);
            *(LAS unsigned*)(lds + 8 * PBUF) = t;
        }
        __syncthreads();
        const unsigned t = __builtin_amdgcn_readfirstlane(*(const LAS unsigned*)(lds + 8 * PBUF));
        constexpr unsigned N0 = NPC / 7 * 3, NL = 247 - NGEMM;
        const unsigned e = (unsigned)(slot - 27) * 4u + ((__builtin_amdgcn_s_getreg(20 | (3 << 11)) + (XMODE == 3 ? slot : 0)) & 3u);
        const unsigned r = XMODE >= 2 ? (side ? NL - e : e) : (side ? NL - (t >> 16) : t & 65535u) % (NL + 1), b = NL - r;
        g = r < N0 ? r / 3 * 7 + 4 + r % 3 : r < N0 + NPROD - 8 ? NPC + 8 + r - N0 : b / 4 * 7 + b % 4;
        if (XMODE == 0) { g = (int)blockIdx.x - NGEMM; g = g < NPC + 8 ? g - 8 : g; }
    }
#undef VBLK
#define VBLK (NGEMM + g)
    if (PROBET >= 0 && wid == 7 && blockIdx.x < 2 * PK_PG) {
        char* a = myws + (OFF_PK + ((size_t)blockIdx.x << 21) + PROBEX * (blockIdx.x & 31) * 256);
        unsigned t[2] = {~0u, ~0u};
        while (__builtin_amdgcn_s_memrealtime() - t0 < PROBET) __builtin_amdgcn_s_sleep(4);
#pragma unroll
        for (int i = 0; i < 6; ++i) {
            const unsigned long long c0 = __builtin_amdgcn_s_memtime();
            if (PROBE1) asm volatile("s_waitcnt lgkmcnt(0)\n\ts_mov_b64 exec, 1\n\tglobal_store_dword %0, %0, %1 sc0 sc1\n\ts_mov_b64 exec, -1\n\ts_waitcnt vmcnt(0)" :: "v"((i & 1) * 8192), "s"(a), "s"(c0) : "memory");
            else asm volatile("s_waitcnt lgkmcnt(0)\n\tglobal_store_dword %0, %0, %1 sc0 sc1\n\ts_waitcnt vmcnt(0)" :: "v"((i & 1) * 8192), "s"(a), "s"(c0) : "memory");
            const unsigned d = (unsigned)(__builtin_amdgcn_s_memtime() - c0);
            t[i & 1] = d < t[i & 1] ? d : t[i & 1];
        }
        if (lane == 0) flag_store(myws + OFF_MAP + blockIdx.x * MAPS, (t[1] < t[0]) ^ side);
        STAMP(10);
    }
    const int u0 = g < 0 ? wid * NGEMM + (int)blockIdx.x : UG + wid * NPROD + g - NPC;
    if (g < 0 || g >= NPC) {
        pace(t0, u0); fetch_input(P, lds + wid * PBUF, u0, lane);
        if ((TOUCH & 1) && u0 < TOUCHU) touch(P, myws, lds + wid * PBUF + 16384, u0, lane);
        make_lut(lds + (8 * PBUF + 16), true, lane);
    }
    GAS unsigned* seq = (GAS unsigned*)(myws + OFF_SEQ) + (blockIdx.x * 8 + wid) * 4;
    const unsigned call = __builtin_amdgcn_readfirstlane(__hip_atomic_load(seq, __ATOMIC_RELAXED, __HIP_MEMORY_SCOPE_AGENT)) + 1u;
    WAIT_V(0);
    if (lane == 0) __hip_atomic_store(seq, call, __ATOMIC_RELAXED, __HIP_MEMORY_SCOPE_AGENT);

    if (g < 0 || g >= NPC) {
        producer(P, myws, lds + wid * PBUF, lds + (8 * PBUF + 16), u0, t0, call, wid, lane);
        STAMP(7);
    }
    if (slot <= NGEMM / 8) {
        const int pi = slot >> 3, ps = xcd >> 1, h = xcd & 1, sown = 20 + ps;
        const int ci = pi == 3 ? (slot & 1) * 4 + ps : slot == sown ? ps : slot & 7;
        const int panel = pi < 2 ? 3 * ps + pi : pi == 3 ? 3 * q + 2 : slot == sown ? 16 : 12 + ps;
        const int k = tile2(slot, sown), kk = k % 12, qt = kk / 3, si = kk - qt * 3, cg2 = k < 12 || k >= 24 ? 8 + xcd : xcd;
        const int bp = 3 * ((q + 1 + si) & 3) + 2, o0 = O_B + qt * 24 + (si * 2 + h) * 4;
        if (g < 0) gemm_panel(P, lds, myws, panel, ci < 4 ? 4 * h + ci : 2 * ci + h, cg2,
                   pi == 3 ? h * 8 : pi == 2 ? O_C + (((ps - ci - 1) & 3) * 2 + h) * 8 : O_A0 + pi * 128 + xcd * 8, pi == 3 ? 16 : pi == 2 ? 48 : 64, call, wid, lane);
        if (HALF && qt == 3 && cg2 >= 8) gemm_ring<1>(P, lds, myws, bp, qt, cg2, k >= 24, o0, g < 0, call, wid, lane);
        else if (k < 24) gemm_ring<2>(P, lds, myws, bp, qt, cg2, 0, o0, g < 0, call, wid, lane);
    } else if (g < NPC) {
        LAS unsigned* info = (LAS unsigned*)lds;
        LAS unsigned* prog = info + NSEQ;
        LAS unsigned short* tab = (LAS unsigned short*)(lds + 8192);
        const int j = g / 7, link = g - j * 7, ncu = (NPC + 6 - link) / 7;
        for (int i = threadIdx.x; i < NSEQ + 4; i += 512) info[i] = 0;
        if (threadIdx.x == 0) { prog[4] = (unsigned)t0; prog[5] = 0; }
        for (int i = threadIdx.x; i < PK_W; i += 512)
            if (((i & 31) == 16 || (i & 31) >= 24) == (link < 4)) tab[pos_x(i & 31, i >> 5)] = (unsigned short)i;
        if (link < 4) tab[pos_w(threadIdx.x >> 2, threadIdx.x & 3)] = (unsigned short)(PK_W + threadIdx.x);
        if (wid == 0) make_lut(lds + 14336, false, lane);
        __syncthreads();
        const int peer = link < 4 ? ((P.rank & 4) ^ 4) + ((q + link) & 3) : (P.rank & 4) + ((q + link - 3) & 3);
        if (wid < WPC) {
            if (lane == 0) __hip_atomic_fetch_max(prog, (unsigned)(NPUSH + wid * ncu + j), __ATOMIC_RELAXED, __HIP_MEMORY_SCOPE_WORKGROUP);
            puller(g, P, myws, info, prog, tab, lds + 16384 + wid * 8192, lds + 14336, link, peer, NPUSH + wid * ncu + j, ncu * WPC,
                   (unsigned)(((link < 4 ? NWIN : NWINI) + ncu - 1 - j) / ncu), call, wid, lane);
        } else if (wid == 7) {
            flag_mirror(myws, P.ws[peer] + (OFF_FP + (call & 1) * FP_B + (link < 4 ? q : 4) * (NSEQ * 4)), link < 4 ? NSEQ : NI, info, prog, call, lane);
        }
        STAMP(3);
    }
}

}

extern "C" void kernel_launch(int rank, int nranks, ncclComm_t comm, hipStream_t stream,
                              void* const* d_in, const int* in_sizes, int n_in,
                              void* d_out, int out_size, const int* in_dtypes, int out_dtype,
                              void* const* d_out_all, void* const* d_ws_all, size_t ws_bytes) {
    (void)comm; (void)in_dtypes; (void)out_dtype; (void)d_out_all;
    if (nranks != 8 || (unsigned)rank >= 8u || n_in < 2 || ws_bytes < WS_NEED || out_size != KDIM * NLOC ||
        in_sizes[0] != 1024 * KDIM || in_sizes[1] != KDIM * NLOC) {
        fprintf(stderr, "ag_gemm: unexpected problem shape\n");
        return;
    }
    Params P;
    P.x = (const float*)d_in[0]; P.w = (const float*)d_in[1];
    P.out = (float*)d_out;
    P.rank = 0; P.pad = 0;
    for (int p = 0; p < 8; ++p) {
        const int r = (int)((RMAP >> (4 * p)) & 7u);
        P.ws[p] = (char*)d_ws_all[r];
        if (r == rank) P.rank = p;
    }
    hipLaunchKernelGGL(ag_gemm_kernel, dim3(256), dim3(512), 0, stream, P);
    HIP_CHECK(hipGetLastError());
}
```

```cpp
#include <hip/hip_runtime.h>
#include <rccl/rccl.h>
#include <cstdio>

#define HIP_CHECK(expr) do { hipError_t _e = (expr); if (_e != hipSuccess) { \
    fprintf(stderr, "HIP error %s at %s:%d\n", hipGetErrorString(_e), __FILE__, __LINE__); \
    return; } } while (0)

namespace {

typedef _Float16 h8 __attribute__((ext_vector_type(8)));
typedef float f4 __attribute__((ext_vector_type(4)));
typedef unsigned u4 __attribute__((ext_vector_type(4)));
typedef unsigned u2 __attribute__((ext_vector_type(2)));
#define LAS __attribute__((address_space(3)))
#define GAS __attribute__((address_space(1)))
#define CAS __attribute__((address_space(4)))

constexpr int KDIM = 8192, NLOC = 1024;
constexpr int NKT = KDIM / 64;
constexpr int NCH = NKT / 2;
constexpr int NI = 23 * 64;
constexpr int NCI = 17 * 64, NO = 480;
constexpr int NSEQ = NCI + NO;
constexpr int O_A0 = 32, O_A1 = 160, O_C = 288, O_B = 384;
constexpr size_t PANEL_B = 4u << 20;
constexpr size_t CG_B = 2u << 20;
constexpr size_t OFF_SEQ = 0;
constexpr size_t OFF_TMO = 64u << 10;
constexpr size_t OFF_CTR = OFF_TMO + 256;
constexpr size_t OFF_ROLE = OFF_TMO + 1280;
constexpr size_t OFF_MAP = OFF_TMO + 2048;
#define MAPS 256u
constexpr size_t OFF_FP = 256u << 10;
constexpr size_t FP_B = 5 * NSEQ * 4;
constexpr size_t OFF_DBG = 640u << 10;
constexpr size_t OFF_DATA = 1u << 20;
constexpr size_t A_B = 21 * PANEL_B, B_B = 16 * CG_B;
constexpr size_t OFF_FA = OFF_DATA + A_B + B_B;
constexpr size_t OFF_FB = OFF_FA + 17 * 64 * 32;
constexpr size_t OFF_FN = OFF_FB + 16 * 64 * 16;
constexpr unsigned FSTR = 219 * 256;
constexpr size_t OFF_PK = OFF_DATA + A_B + B_B + (1u << 20);
constexpr int PK_W = 2048, PK_O = 4096, PK_N = PK_O + 4 * NO;
constexpr int PK_PG = (PK_N + 126) / 127;
constexpr size_t PK_B = (size_t)PK_PG << 21;
constexpr size_t WS_NEED = OFF_PK + 2 * PK_B;
static_assert(OFF_MAP + 2 * PK_PG * MAPS <= OFF_FP && OFF_FN + 35 * 128 <= OFF_FA + FSTR && 16 * FSTR <= (1u << 20) && OFF_FP + 2 * FP_B <= OFF_DBG, "flag area");
constexpr unsigned long long DEADLINE = 30000000ull;
constexpr unsigned M_RAW = 9, M_DEAD = 10;

#define NOTMO
#define NOTMO1
#define NOTMO2
#define NPC 35
#define WPC 7
#define SKC 14
#define SKA 5
#define SQ1 5
#define SQ2 15
#define SQ3 28
#define PACE 12
#define U1ST 448
#define PACE1 PACE
#define PACE2 PACE1
#define PROBE1 1
#define LOOK 64
#define PCH 2
#define PT0 180
#define PDT 13
#define CHMAP 0x76543210u
#define PROBET 0
#define PROBEX 0
#define TOUCH 0
#define RAW0 32
#define RAW1 0
#define DEFER 1
#define PD 2
#define NWIN 10
#define NWINI NWIN
#define PRATE 134
#define PRATEI PRATE
#define LGAIN 64
#define LCAP 30
#define HWIN 280
#define HWIN0 0
#define GLINK 1
#define PSLACK 0
#define RELAT 8
#define UG 3000
#define EXN 32
#define NB2 (200 - EXN)
#define AHEAD UG
#define PDEF 0
#define LOOK2 4
#define MALL 4
#define MSL 8
#define QSLEEP 0
#define AUX 17
#define RMAP 0x75416320u
#define EARLY 0
#define ORAW 0
#define SMODE 1
#define XMODE 1
constexpr int NGEMM = 208;
constexpr int NPROD = 256 - NGEMM - NPC;

struct Params {
    const float* x;
    const float* w;
    float* out;
    char* ws[8];
    int rank;
    int pad;
};

#define VBLK blockIdx.x
#define STAMP(i) do { } while (0)

#define TREC(c) do { } while (0)
#define TDUMP(k) do { (void)tv; } while (0)
#define HST(i, v) do { } while (0)
#define NOW() __builtin_amdgcn_s_memrealtime()
#define GST4(voff, data, sbase, imm) \
    asm volatile("global_store_dwordx4 %0, %1, %2 offset:" #imm " sc0 sc1\n\ts_nop 1" :: "v"(voff), "v"(data), "s"(sbase) : "memory")
#define OSUF " sc0 sc1"
#define GST4P(voff, data, sbase) \
    asm volatile("global_store_dwordx4 %0, %1, %2" OSUF "\n\ts_nop 1" :: "v"(voff), "v"(data), "s"(sbase) : "memory")
#define WAIT_V(n) asm volatile("s_waitcnt vmcnt(%0)" :: "n"(n) : "memory")
#define WAIT_L0() asm volatile("s_waitcnt lgkmcnt(0)" ::: "memory")

__device__ __forceinline__ u4 pack8(float a0, float a1, float a2, float a3, float a4, float a5, float a6, float a7) {
    h8 h = {(_Float16)a0, (_Float16)a1, (_Float16)a2, (_Float16)a3, (_Float16)a4, (_Float16)a5, (_Float16)a6, (_Float16)a7};
    return __builtin_bit_cast(u4, h);
}
__device__ __forceinline__ void flag_store(char* p, unsigned v) {
    __hip_atomic_store((GAS unsigned*)p, v, __ATOMIC_RELAXED, __HIP_MEMORY_SCOPE_SYSTEM);
}
__device__ __forceinline__ unsigned xcc_id() { return __builtin_amdgcn_s_getreg(20 | (3 << 11)) & 7u; }
__device__ __forceinline__ void flag_local(char* base, unsigned off, unsigned v) {
    asm volatile("global_store_dword %0, %1, %2" :: "v"(off + (8u + xcc_id()) * FSTR), "v"(v), "s"(base) : "memory");
}
__device__ __forceinline__ unsigned flag_copy(int lane) { return (xcc_id() + (unsigned)(lane >> 5) * 8u) * FSTR; }
__device__ __forceinline__ unsigned b_flags(int cg, int lane) { return flag_copy(lane) + (unsigned)OFF_FB + (unsigned)(cg * 1024 + (lane & 31) * 16); }
constexpr int BFLG = 148480;
__device__ __forceinline__ unsigned unset(unsigned f, unsigned call) {
    const unsigned long long m = __builtin_amdgcn_ballot_w64(f < call);
    return (unsigned)m & (unsigned)(m >> 32);
}
__device__ __forceinline__ void store8(const u4 (&h)[8], char* d0, unsigned half, unsigned vl16) {
    char* d1 = d0 + half;
    GST4(vl16, h[0], d0, 0); GST4(vl16, h[1], d0, 1024); GST4(vl16, h[2], d0, 2048); GST4(vl16, h[3], d0, 3072);
    GST4(vl16, h[4], d1, 0); GST4(vl16, h[5], d1, 1024); GST4(vl16, h[6], d1, 2048); GST4(vl16, h[7], d1, 3072);
}
__device__ __forceinline__ unsigned long long lane_mask(unsigned n) { return n >= 64u ? ~0ull : (1ull << n) - 1ull; }

__device__ __forceinline__ int clamp64(int v) { return v < 0 ? 0 : v > 64 ? 64 : v; }
__device__ __forceinline__ int pos_w(int kt, int g) { const int ch = kt >> 1; return (ch < PCH ? 8 : 9) * ch + 8 * clamp64(ch - SKC) + (kt & 1) * 4 + g; }
__device__ __forceinline__ int pos_x(int rb, int ch) {
    if (rb < 8) return 8 * ch + 8 * clamp64(ch - SKA) + rb;
    if (rb < 16) return 8 * ch + 8 * clamp64(ch + SKA + 1) + rb - 8;
    if (rb == 16) return ch < PCH ? 8 * PCH + ch : 9 * ch + 8 * clamp64(ch - SKC) + 8;
    if (rb >= 24) return 8 * ch + 9 * clamp64(ch + SKC + 1) + rb - 24;
    const int j = (rb - 16) >> 1, r = ch + (j == 0 ? 0 : j == 1 ? SQ1 : j == 2 ? SQ2 : SQ3);
    return 1024 + clamp64(r + (j > 0)) + 2 * (clamp64(r - SQ1 + (j > 1)) + clamp64(r - SQ2 + (j > 2)) + clamp64(r - SQ3)) + (j ? rb & 1 : 0);
}
static_assert(NPC % 7 == 0 && 2 * PK_PG <= NGEMM && UG >= 8 * NGEMM && UG <= 8 * (NGEMM + NB2) && UG + 8 * NPROD <= PK_O && 0 <= SQ1 && SQ1 <= SQ2 && SQ2 <= SQ3 && SKC < 64 && SKA < 64 && PCH <= SKA && PCH <= SKC && SKC >= 8, "skews");
constexpr int NPUSH = 8 * PCH;
__device__ __forceinline__ unsigned panel_off(int panel, unsigned call) {
    return (unsigned)OFF_DATA + (unsigned)((call & 1) && panel < 12 && panel % 3 == 0 ? 17 + panel / 3 : panel) * (unsigned)PANEL_B;
}
__device__ __forceinline__ void place_x(int panel, int rb, int ch, unsigned call, unsigned& off, unsigned& fl) {
    off = panel_off(panel, call) + (unsigned)(ch * 65536 + (rb & 7) * 4096);
    fl = (unsigned)OFF_FA + (unsigned)(panel * 2048 + ch * 32 + (rb & 7) * 4);
}
__device__ __forceinline__ void place_w(int cb, int kt, unsigned& off, unsigned& fl) {
    off = (unsigned)(OFF_DATA + A_B) + (unsigned)(cb >> 1) * (unsigned)CG_B + (unsigned)(kt * 16384 + (cb & 1) * 8192);
    fl = (unsigned)OFF_FB + (unsigned)((cb >> 1) * 1024 + kt * 8 + (cb & 1) * 4);
}
__device__ __forceinline__ int panel_row(int pub, int panel, int blk) {
    const int gs = pub & 4, s = panel / 3, k = panel - 3 * s;
    const int r = panel == 16 ? pub : panel >= 12 ? (gs ^ 4) + panel - 12 : (k == 2 && blk == 0 ? gs ^ 4 : gs) + s;
    return (int)((RMAP >> (4 * r)) & 7u) * 1024 + (panel >= 12 ? 768 : k * 256) + blk * 32;
}
__device__ __forceinline__ size_t pk_off(int pk, unsigned call) { return OFF_PK + (call & 1) * PK_B + (size_t)(pk + pk / 127 + 1) * 16384; }
__device__ __forceinline__ unsigned pk_half(char* myws, int pk, unsigned call) {
    return __hip_atomic_load((GAS unsigned*)(myws + OFF_MAP) + ((call & 1) * PK_PG + pk / 127) * (MAPS / 4), __ATOMIC_RELAXED, __HIP_MEMORY_SCOPE_SYSTEM);
}

#define PERM(hi, lo, sel) __builtin_amdgcn_perm(hi, lo, sel)
#define ABIT(hi, lo, s) __builtin_amdgcn_alignbit(hi, lo, s)
#define HW(k) h[(k) >> 2][(k) & 3]
#define FENCE() asm volatile("" ::: "memory")
__device__ __forceinline__ unsigned bfi(unsigned m, unsigned x, unsigned y) { return (x & m) | (y & ~m); }
__device__ __forceinline__ unsigned wave_scan(unsigned v) {
    v += __builtin_amdgcn_update_dpp(0u, v, 0x111, 0xF, 0xF, true);
    v += __builtin_amdgcn_update_dpp(0u, v, 0x112, 0xF, 0xF, true);
    v += __builtin_amdgcn_update_dpp(0u, v, 0x114, 0xF, 0xF, true);
    v += __builtin_amdgcn_update_dpp(0u, v, 0x118, 0xF, 0xF, true);
    v += __builtin_amdgcn_update_dpp(0u, v, 0x142, 0xA, 0xF, false);
    v += __builtin_amdgcn_update_dpp(0u, v, 0x143, 0xC, 0xF, false);
    return v;
}
__device__ __forceinline__ int wave_sum(unsigned v) { return __builtin_amdgcn_readlane(wave_scan(v), 63); }
__device__ __forceinline__ unsigned escapes(unsigned c, unsigned acc) { return (unsigned)__builtin_popcount(c & (c >> 1) & 0x55555555u) + acc; }
__device__ __forceinline__ void make_lut(LAS char* lut, bool pack, int lane) {
#pragma unroll
    for (int j = 0; j < 4; ++j) {
        unsigned sel = pack ? 0x0C0C0C0Cu : 0u, k = 0;
#pragma unroll
        for (int i = 0; i < 4; ++i) {
            const unsigned c = i < 3 ? ((unsigned)lane >> (2 * i)) & 3u : (unsigned)j;
            if (pack) sel ^= c == 3u ? (0x0Cu ^ (unsigned)i) << (8 * k) : 0u; else sel |= (c == 3u ? k : 4u + c) << (8 * i);
            k += c == 3u;
        }
        *(LAS u2*)(lut + (j * 64 + lane) * 8) = (u2){sel, pack ? 8u * k : k};
    }
}
__device__ __forceinline__ unsigned code4(unsigned thi, unsigned tlo, unsigned s) { return __builtin_amdgcn_udot4(PERM(thi, tlo, s), 0x80200802u, 0u - 0xF3u * 170u, false); }
template <int N> __device__ __forceinline__ void append(const unsigned* s, const u2* e, unsigned pos) {
#pragma unroll
    for (int g = 0; g < N; ++g) {
        const unsigned long long c = (unsigned long long)PERM(0u, s[g], e[g][0]) << (pos & 31u);
        asm volatile("v_cmpx_ne_u32_e32 vcc, 0, %1\n\tds_or_b32 %0, %1\n\tv_cmpx_ne_u32_e32 vcc, 0, %2\n\tds_or_b32 %0, %2 offset:4\n\ts_mov_b64 exec, -1"
                     :: "v"((pos >> 3) & ~3u), "v"((unsigned)c), "v"((unsigned)(c >> 32)) : "vcc", "memory");
        pos += e[g][1];
    }
}
constexpr int SCR_B = 3328;
constexpr int SCR_3 = 2064, SCR_T = 2592, PBUF = 16384 + SCR_B;
template <bool F32>
__device__ __forceinline__ unsigned pack_unit(u4 (&h)[8], LAS char* scr, const LAS char* lut, int lane, unsigned& r0, unsigned& r2) {
    constexpr int N1 = F32 ? 8 : 16, N2 = F32 ? 4 : 8;
    const u4 z = {0u, 0u, 0u, 0u};
    *(LAS u4*)(scr + lane * 16) = z; *(LAS u4*)(scr + 1024 + lane * 16) = z; *(LAS u4*)(scr + 2048 + lane * 16) = z;
    FENCE();
    unsigned G[16], s[16], a[16], cw[4], b3, type;
    if (F32) {
#pragma unroll
        for (int g = 0; g < 8; ++g)
            G[g] = PERM(PERM(HW(4 * g + 3), HW(4 * g + 2), 0x07060302u) >> 7, PERM(HW(4 * g + 1), HW(4 * g), 0x07060302u) >> 7, 0x06040200u);
        unsigned es = 0;
#pragma unroll
        for (int g = 0; g < 4; ++g) es = __builtin_amdgcn_udot4(G[2 * g + (g & 1)], 0x02020202u, es, false);
        const int m = wave_sum(es) + 205;
        b3 = (unsigned)(m >> 11) + 3u; b3 = b3 < 27u ? 27u : b3 > 251u ? 251u : b3;
        type = (m & 2047) > 600;
        unsigned out = 0;
#pragma unroll
        for (int g = 0; g < 8; ++g) { const unsigned d = (b3 + 4u) * 0x01010101u - G[g]; out |= d; s[g] = (2u * d + 0x39393939u) & 0x3F3F3F3Fu; }
        if (__builtin_amdgcn_ballot_w64((out & 0xE0E0E0E0u) != 0) != 0) return M_RAW;
    } else {
#pragma unroll
        for (int g = 0; g < 16; ++g) { G[g] = PERM(HW(2 * g + 1), HW(2 * g), 0x07050301u); s[g] = (G[g] >> 1) & 0x3E3E3E3Eu; }
        const int m = wave_sum(__builtin_amdgcn_udot4(s[0] + s[5] + s[10] + s[15], 0x01010101u, 0u, false)) + 205;
        b3 = ((unsigned)(m >> 11) + 3u) & 31u; type = (m & 2047) > 1120;
#pragma unroll
        for (int g = 0; g < 16; ++g) s[g] = ((2u * b3 + 65u) * 0x01010101u - s[g]) & 0x3F3F3F3Fu;
    }
    u2 e[16];
#pragma unroll
    for (int g = 0; g < N1; ++g) {
        a[g] = code4(0xF300F700u, 0xFB00FF00u, s[g]);
        e[g] = *(const LAS u2*)(lut + a[g]);
    }
    unsigned n = 0;
#pragma unroll
    for (int j = 0; j < N1 / 4; ++j) { cw[j] = (a[4 * j] >> 3) | (a[4 * j + 1] << 5) | (a[4 * j + 2] << 13) | (a[4 * j + 3] << 21); n = escapes(cw[j], n); }
    unsigned sc = wave_scan(n);
    r2 = (__builtin_amdgcn_readlane(sc, 63) + (4u * N2 - 1u)) / (4u * N2);
    if (r2 > (F32 ? 64u : 62u)) return M_RAW;
    append<N1>(s, e, (sc - n + (unsigned)(size_t)scr) * 8u);
    FENCE();
    u4 q[N2 / 4];
#pragma unroll
    for (int j = 0; j < N2 / 4; ++j) q[j] = *(const LAS u4*)(scr + lane * (4 * N2) + 16 * j);
#pragma unroll
    for (int g = 0; g < N2; ++g) {
        const unsigned x = q[g >> 2][g & 3];
        s[g] = (x | ~PERM(0xFF000000u, 0xFF00F9F6u, x)) - 0x06060606u;
        a[g] = code4(type ? 0xFF00F700u : 0xFB00F700u, type ? 0xF300FB00u : 0xF300FF00u, s[g]);
        e[g] = *(const LAS u2*)(lut + a[g]);
    }
    const unsigned c0 = (a[0] >> 3) | (a[1] << 5) | (a[2] << 13) | (a[3] << 21), c1 = F32 ? 0u : (a[4] >> 3) | (a[5] << 5) | (a[6] << 13) | (a[7] << 21);
    n = escapes(c1, escapes(c0, 0u));
    sc = wave_scan(n);
    const unsigned r3 = (__builtin_amdgcn_readlane(sc, 63) + 7u) >> 3;
    if (r3 > 64u) return M_RAW;
    append<N2>(s, e, (sc - n + (unsigned)(size_t)(scr + SCR_3)) * 8u);
    FENCE();
    u2 y = *(const LAS u2*)(scr + SCR_3 + lane * 8), c;
#pragma unroll
    for (int j = 0; j < 2; ++j) {
        y[j] = (y[j] | ~PERM(0xFF000000u, 0xFF00F9F8u, y[j])) - 0x07070707u;
        const unsigned sw = F32 ? ~(((y[j] ^ 0x0E0E0E0Eu) + 0x3F3F3F3Fu) & ((y[j] ^ 0x32323232u) + 0x3F3F3F3Fu)) & 0x40404040u : 0u;
        c[j] = y[j] ^ (sw - (sw >> 4));
    }
    unsigned w = ((c[0] >> 1) & 0x07070707u) | ((c[1] << 2) & 0x38383838u);
    w |= ((w >> 24) * 0x41040u) & 0xC0C0C0u;
    LAS char* t = scr + SCR_T;
    t[4 + lane * 3] = (char)w; t[5 + lane * 3] = (char)(w >> 8); t[6 + lane * 3] = (char)(w >> 16);
    FENCE();
    const unsigned l0 = (3u * r3 + 5u) >> 1;
    unsigned bad = ((c[0] >> 4) & 0x03030303u) | (c[1] & 0x30303030u), nl = 0;
    for (;;) {
        const unsigned long long mk = __builtin_amdgcn_ballot_w64(bad != 0);
        if (mk == 0 || nl > 64u) break;
        if (bad != 0) {
            const unsigned p = (unsigned)__builtin_ctz(bad), k = l0 + nl + __builtin_amdgcn_mbcnt_hi((unsigned)(mk >> 32), __builtin_amdgcn_mbcnt_lo((unsigned)mk, 0u));
            ((LAS unsigned short*)t)[k] = (unsigned short)((((unsigned)lane * 8u + (p & 4u) + (p >> 3)) << 7) | ((4u * b3 - 24u - 2u * (((p & 4u) ? y[1] : y[0]) >> (p & 24u))) & 0x7Cu));
            bad &= ~(15u << (p & 28u));
        }
        nl += (unsigned)__builtin_popcountll(mk);
    }
    r0 = (l0 + nl + 3u) >> 2;
    const unsigned rs = r0 + (F32 ? (r2 + 1u) >> 1 : r2);
    if (nl > 64u || rs > 64u) return M_RAW;
    FENCE();
    u2 f = *(const LAS u2*)(t + lane * 8);
    FENCE();
    if (lane == 0) f[0] = b3 | (r0 << 8) | (type << 15) | (nl << 16) | (l0 << 24);
    if (F32) {
#pragma unroll
        for (int g = 0; g < 8; ++g) {
            const unsigned x = ABIT(HW(4 * g + 3), HW(4 * g + 3), 31);
#pragma unroll
            for (int j = 0; j < 3; ++j) HW(3 * g + j) = PERM(x, ABIT(HW(4 * g + j), HW(4 * g + j), 31), 0x04020100u + ((unsigned)j << 24));
        }
        h[6] = (u4){cw[0], cw[1], f[0], f[1]};
    } else {
        unsigned L[16];
#pragma unroll
        for (int g = 0; g < 16; ++g) L[g] = PERM(HW(2 * g + 1), HW(2 * g), 0x06040200u);
#pragma unroll
        for (int g = 0; g < 16; ++g) HW(g) = L[g];
#pragma unroll
        for (int j = 0; j < 6; ++j) {
            const unsigned x = bfi(0x83838383u, G[2 * j], ABIT(G[2 * j + 1], G[2 * j + 1], 29));
            HW(16 + j) = j < 4 ? bfi(0x60606060u, G[12 + j] << 5, x) : bfi(0x20202020u, G[2 * j + 4] >> 2, bfi(0x40404040u, G[2 * j + 5] >> 1, x));
        }
        h[5][2] = cw[0]; h[5][3] = cw[1]; h[6] = (u4){cw[2], cw[3], f[0], f[1]};
    }
    h[7][0] = c0; h[7][1] = c1;
    return (rs + 7u) >> 3;
}
template <bool F32>
__device__ __forceinline__ void store_unit(const u4 (&h)[8], char* d0, unsigned mode, unsigned r0, unsigned r2, unsigned vl16) {
    char* d1 = d0 + 4096;
    GST4(vl16, h[0], d0, 0); GST4(vl16, h[1], d0, 1024); GST4(vl16, h[2], d0, 2048); GST4(vl16, h[3], d0, 3072);
    GST4(vl16, h[4], d1, 0); GST4(vl16, h[5], d1, 1024);
    if (mode < M_RAW) {
        const u2 p0 = {h[6][0], h[6][1]}, p1 = {h[6][2], h[6][3]}, p2 = {h[7][0], h[7][1]};
        asm volatile("global_store_dwordx2 %0, %1, %3 offset:2048 sc0 sc1\n\t"
                     "s_mov_b64 exec, %4\n\tglobal_store_dwordx2 %0, %2, %3 offset:2560 sc0 sc1\n\ts_mov_b64 exec, -1"
                     :: "v"(vl16 >> 1), "v"(p0), "v"(p1), "s"(d1), "s"(lane_mask(r0)) : "memory");
        if (F32) asm volatile("s_mov_b64 exec, %3\n\tglobal_store_dword %0, %1, %2 offset:2560 sc0 sc1\n\ts_mov_b64 exec, -1"
                              :: "v"(vl16 >> 2), "v"(p2[0]), "s"(d1 + r0 * 8u), "s"(lane_mask(r2)) : "memory");
        else asm volatile("s_mov_b64 exec, %3\n\tglobal_store_dwordx2 %0, %1, %2 offset:2560 sc0 sc1\n\ts_mov_b64 exec, -1"
                          :: "v"(vl16 >> 1), "v"(p2), "s"(d1 + r0 * 8u), "s"(lane_mask(r2)) : "memory");
    } else { GST4(vl16, h[6], d1, 2048); GST4(vl16, h[7], d1, 3072); }
}
template <int N> __device__ __forceinline__ void deposit(unsigned* o, const u2* e, unsigned tab, unsigned pos) {
    u2 x[N]; unsigned p[N];
#pragma unroll
    for (int g = 0; g < N; ++g) {
        const LAS unsigned* d = (const LAS unsigned*)(size_t)(pos & ~3u);
        x[g] = (u2){d[0], d[1]}; p[g] = pos;
        pos += e[g][1];
    }
#pragma unroll
    for (int g = 0; g < N; ++g) o[g] = PERM(tab, __builtin_amdgcn_alignbyte(x[g][1], x[g][0], p[g]), e[g][0]);
}
template <bool F32>
__device__ __forceinline__ void unpack_unit(u4 (&h)[8], LAS char* scr, const LAS char* lut, int lane) {
    constexpr int N1 = F32 ? 8 : 16, N2 = F32 ? 4 : 8;
    LAS char* t = scr + SCR_T;
    *(LAS u2*)(t + lane * 8) = (u2){h[6][2], h[6][3]};
    FENCE();
    const unsigned hd = __builtin_amdgcn_readfirstlane(h[6][2]), r0 = (hd >> 8) & 63u, type = hd & 32768u;
    const unsigned kb = (F32 ? hd & 255u : ((hd & 31u) << 2) | 0x80u) * 0x01010101u;
    auto tab = [&](unsigned t4) { return F32 ? kb - t4 : (kb - 4u * t4) & 0x7C7C7C7Cu; };
    const LAS unsigned* d3 = (const LAS unsigned*)(t + ((4 + lane * 3) & ~3));
    const unsigned w0 = d3[0], w1 = d3[1];
    unsigned le = 0;
    const bool ll = (unsigned)lane < ((hd >> 16) & 127u);
    if (ll) le = ((const LAS unsigned short*)t)[(hd >> 24) + lane];
    const unsigned c0 = F32 ? *(const LAS unsigned*)(t + r0 * 8u + lane * 4) : (unsigned)lane < r0 ? 0u : h[6][2], c1 = F32 || (unsigned)lane < r0 ? 0u : h[6][3];
    const unsigned cw[4] = {F32 ? h[6][0] : h[5][2], F32 ? h[6][1] : h[5][3], F32 ? 0u : h[6][0], F32 ? 0u : h[6][1]};
    u2 e2[N2], e1[N1];
#pragma unroll
    for (int g = 0; g < N2; ++g) e2[g] = *(const LAS u2*)(lut + (((g < 4 ? c0 : c1) >> (8 * (g & 3))) & 255u) * 8u);
#pragma unroll
    for (int g = 0; g < N1; ++g) e1[g] = *(const LAS u2*)(lut + ((cw[g >> 2] >> (8 * (g & 3))) & 255u) * 8u);
    const unsigned n2 = escapes(c1, escapes(c0, 0u)), n1 = escapes(cw[3], escapes(cw[2], escapes(cw[1], escapes(cw[0], 0u))));
    const unsigned n = n1 | (n2 << 16), sc = wave_scan(n) - n;
    unsigned w = __builtin_amdgcn_alignbyte(w1, w0, (unsigned)lane * 3u);
    w = bfi(0x003F3F3Fu, w, __builtin_amdgcn_udot4(w & 0xC0C0C0u, 0x100401u, 0u, false) << 18);
    const unsigned t3l = tab(type ? 0x09080706u : 0x09080700u), t3h = tab(F32 ? 0xFF0C0B0Au : 0x0D0C0B0Au);
    *(LAS u2*)(scr + SCR_3 + lane * 8) = (u2){PERM(t3h, t3l, w & 0x07070707u), PERM(t3h, t3l, (w >> 3) & 0x07070707u)};
    FENCE();
    if (ll) scr[SCR_3 + (le >> 7)] = (char)(F32 ? hd - 27u + (((le >> 2) - hd + 27u) & 31u) : le & 0x7Cu);
    FENCE();
    unsigned E[16];
    deposit<N2>(E, e2, tab(type ? 0x000504u : 0x060504u), (sc >> 16) + (unsigned)(size_t)(scr + SCR_3));
    FENCE();
#pragma unroll
    for (int j = 0; j < N2 / 4; ++j) *(LAS u4*)(scr + lane * (4 * N2) + 16 * j) = (u4){E[4 * j], E[4 * j + 1], E[4 * j + 2], E[4 * j + 3]};
    FENCE();
    deposit<N1>(E, e1, tab(0x010203u), (sc & 0xFFFFu) + (F32 ? 0u : r0 * 32u) + (unsigned)(size_t)scr);
    FENCE();
    if (F32) {
#pragma unroll
        for (int g = 7; g >= 0; --g) {
            const unsigned x = PERM(PERM(E[g], HW(3 * g + 2), 0x07030000u), PERM(HW(3 * g + 1), HW(3 * g), 0x00000703u), 0x07060100u);
            HW(4 * g + 3) = ABIT(x, x, 1);
#pragma unroll
            for (int j = 2; j >= 0; --j) { const unsigned v = PERM(E[g], HW(3 * g + j), 0x04020100u + ((unsigned)j << 24)); HW(4 * g + j) = ABIT(v, v, 1); }
        }
    } else {
        unsigned L[16];
#pragma unroll
        for (int g = 0; g < 16; ++g) {
            L[g] = HW(g);
            const unsigned x = HW(16 + (g < 12 ? g >> 1 : g - 12));
            E[g] = bfi(0x7C7C7C7Cu, E[g], g >= 12 ? bfi(0x80808080u, HW(14 + (g >> 1)) << (2 - (g & 1)), x >> 5) : (g & 1) ? ABIT(x, x, 3) : x);
        }
#pragma unroll
        for (int g = 0; g < 16; ++g) { HW(2 * g) = PERM(E[g], L[g], 0x05010400u); HW(2 * g + 1) = PERM(E[g], L[g], 0x07030602u); }
    }
}

__device__ __forceinline__ void unit_decode(int u, int& kind, int& ch, int& g) {
    constexpr int U0 = 56 * SKC, U1 = U0 + 64 * (64 - SKC);
    if (u < U0) { const int s = u / 7; kind = u - 7 * s; ch = s >> 3; g = s & 7; if (ch < 8 && kind < 5) ch = (CHMAP >> (4 * ch)) & 7; }
    else if (u < U1) { const int v = u - U0; kind = v & 7; g = (v >> 3) & 7; ch = (v >> 6) + (kind == 7 ? 0 : SKC); }
    else { const int v = u - U1; kind = 7; g = v & 7; ch = 64 - SKC + (v >> 3); }
}
__device__ __forceinline__ void fetch_input(const CAS Params& P, LAS char* buf, int u, int lane) {
    int kind, ch, g;
    unit_decode(u, kind, ch, g);
    const bool isx = kind == 0 || kind > 4;
    const char* src = isx ? (const char*)P.x + (size_t)((unsigned)((kind ? kind - 4 : 0) * 8 + g) * (32 * KDIM * 4) + (unsigned)ch * 512)
                          : (const char*)P.w + (size_t)((unsigned)(2 * ch + (g >> 2)) * (64 * NLOC * 4) + (unsigned)((kind - 1) * 4 + (g & 3)) * 256);
    const unsigned s0 = isx ? 64 : NLOC * 4, s1 = isx ? 16 * KDIM * 4 : 16 * NLOC * 4, s2 = isx ? 256 : 32 * NLOC * 4;
    const unsigned voff = isx ? (unsigned)(lane & 15) * (KDIM * 4) + (unsigned)(lane >> 4) * 16 : (unsigned)(lane >> 4) * (4 * NLOC * 4) + (unsigned)(lane & 15) * 16;
#pragma unroll
    for (int i = 0; i < 16; ++i)
        __builtin_amdgcn_global_load_lds((const GAS void*)(src + ((i & 3) * s0 + ((i >> 2) & 1) * s1 + (i >> 3) * s2) + voff), (LAS void*)(buf + i * 1024), 16, 0, 0);
}
#define TOUCHU 448
__device__ __forceinline__ void touch(const CAS Params& P, char* myws, LAS char* scr, int u, int lane) {
    int ch, g, kind;
    unit_decode(u, kind, ch, g);
    const bool isx = kind == 0 || kind > 4;
    const int rb = (kind ? kind - 4 : 0) * 8 + g, kt = 2 * ch + (g >> 2), cb = (kind - 1) * 4 + (g & 3);
    const int pk = isx ? ch * 32 + rb : PK_W + (kind - 1) * 512 + ch * 8 + g;
    unsigned off, fl;
    if (isx) place_x(rb < 24 ? 3 * (P.rank & 3) + (rb >> 3) : 16, rb, ch, lane & 1, off, fl); else place_w(cb, kt, off, fl);
    const unsigned a = lane & 2 ? off : (unsigned)pk_off(pk, lane & 1);
    if (lane < 4) __builtin_amdgcn_global_load_lds((const GAS void*)(myws + a), (LAS void*)scr, 4, 0, 17);
}
__device__ __forceinline__ void pace(unsigned long long t0, int u) {
    if (u < UG) while (__builtin_amdgcn_s_memrealtime() - t0 < (unsigned)(u < U1ST ? u * PACE1 : (u - U1ST) * PACE + U1ST * PACE2) >> 4) __builtin_amdgcn_s_sleep(2);
}
__device__ __forceinline__ int next_unit(int u, int wid) {
    if (u >= UG) return u + 8 * NPROD;
    const int b = blockIdx.x < 32 ? (int)blockIdx.x : blockIdx.x < 32 + EXN ? NB2 : (int)blockIdx.x - EXN;
    u = u < 8 * NGEMM && b < NB2 ? 8 * NGEMM + wid * NB2 + b : UG;
    return u < UG ? u : PK_O;
}
__device__ __forceinline__ void producer(const CAS Params& P, char* myws, LAS char* buf, LAS char* lut, int u, unsigned long long t0, unsigned call, int wid, int lane) {
    const int q = P.rank & 3;
    const unsigned vl16 = (unsigned)lane * 16;
    LAS char* scr = buf + 16384;
#define PT(i) do { } while (0)
    for (int un; u < PK_O; u = un) {
        un = next_unit(u, wid);
        int ch, g, kind;
        unit_decode(u, kind, ch, g);
        const bool isx = kind == 0 || kind > 4;
        const int rb = (kind ? kind - 4 : 0) * 8 + g, kt = 2 * ch + (g >> 2), cb = (kind - 1) * 4 + (g & 3);
        const int idx = isx ? pos_x(rb, ch) : pos_w(kt, g & 3), pk = isx ? ch * 32 + rb : PK_W + (kind - 1) * 512 + ch * 8 + g;
        const int rd = !isx ? kind - 1 : rb == 16 || rb >= 24 ? 5 : 4;
        f4 v[16]; u4 h[8];
        unsigned hv;
        asm volatile("global_load_dword %0, %1, %2 sc0 sc1" : "=&v"(hv) : "v"((unsigned)OFF_MAP + ((call & 1) * PK_PG + pk / 127) * MAPS), "s"(myws) : "memory");
#pragma unroll
        for (int i = 0; i < 16; ++i) v[i] = *(const LAS f4*)(buf + i * 1024 + lane * 16);
        __builtin_amdgcn_sched_barrier(0);
        WAIT_L0();
        const bool ahead = u >= AHEAD && un < PK_O;
        PT(0);
        if (ahead) { pace(t0, un); fetch_input(P, buf, un, lane); }
        PT(1);
        __builtin_amdgcn_sched_barrier(0);
        if (isx) {
#pragma unroll
            for (int i = 0; i < 8; ++i)
                h[i] = pack8(v[2 * i][0], v[2 * i][1], v[2 * i][2], v[2 * i][3], v[2 * i + 1][0], v[2 * i + 1][1], v[2 * i + 1][2], v[2 * i + 1][3]);
        } else {
#pragma unroll
            for (int i = 0; i < 8; ++i) {
                const int f = i >> 1, k = (i & 1) * 8;
                h[i] = pack8(v[k][f], v[k + 1][f], v[k + 2][f], v[k + 3][f], v[k + 4][f], v[k + 5][f], v[k + 6][f], v[k + 7][f]);
            }
        }
        unsigned off, fl;
        if (isx) place_x(rb < 24 ? 3 * q + (rb >> 3) : 16, rb, ch, call, off, fl); else place_w(cb, kt, off, fl);
        const bool own = !isx || rb != 16;
        const unsigned long long h11 = NOW();
        if (idx < NPUSH) {
            unsigned off2 = off, fl2 = fl;
            if (!isx) place_w(16 + 4 * q + (g & 3), kt, off2, fl2);
            while (__builtin_amdgcn_s_memrealtime() - t0 < (unsigned)(PT0 + idx * PDT)) __builtin_amdgcn_s_sleep(1);
            char* fp = myws + fl;
            const unsigned long long h14 = NOW();
            for (int i = 1; i < (isx ? 4 : 2); ++i) {
                char* pw = P.ws[isx ? (P.rank & 4) + ((q + i) & 3) : ((P.rank & 4) ^ 4) + rd];
                store8(h, pw + off2, isx ? 32768u : 4096u, vl16);
                fp = lane >> 3 == i ? pw + fl2 : fp;
            }
            store8(h, myws + off, isx ? 32768u : 4096u, vl16);
            asm volatile("s_waitcnt vmcnt(0)" : "+v"(hv) :: "memory");
            if (lane == (int)xcc_id()) flag_local(myws, fl, call);
            else if (lane < (isx ? 32 : 16)) flag_store(fp + (lane & 7) * FSTR, call);
            HST(12, NOW()); HST(11, h11); HST(14, h14); HST(13, (unsigned long long)(rd * 4096 + idx + 1));
            if (un < PK_O) { pace(t0, un); fetch_input(P, buf, un, lane); WAIT_V(0); }
            continue;
        }
        if (own) store8(h, myws + off, isx ? 32768u : 4096u, vl16);
        unsigned mode = M_RAW, r0 = 0, r2 = 0;
        PT(2);
        if (idx >= RAW0 && (rd != 4 || idx < NI - RAW1)) mode = pack_unit<false>(h, scr, lut, lane, r0, r2);
        PT(3);
        const unsigned long long h14 = NOW();
        if (own) asm volatile("s_waitcnt vmcnt(8)" : "+v"(hv) :: "memory"); else asm volatile("s_waitcnt vmcnt(0)" : "+v"(hv) :: "memory");
        PT(4);
        const unsigned half = SMODE == 0 ? idx & 1 : (__builtin_amdgcn_readfirstlane(hv) ^ (rd != 4) ^ (SMODE == 2)) & 1u;
        store_unit<false>(h, myws + (pk_off(pk, call) + half * 8192), mode, r0, r2, vl16);
        const unsigned pfo = lane < 5 ? (unsigned)OFF_FP + (call & 1) * (unsigned)FP_B + (unsigned)(lane * NSEQ + idx) * 4u : fl + (lane & 7) * FSTR;
        WAIT_V(0);
        if (own && lane == 8 + (int)xcc_id()) flag_local(myws, fl, call);
        else if ((rd == 5 ? lane < 4 : lane == rd) || (own && lane >> 3 == 1)) flag_store(myws + pfo, lane < 5 ? call * 32u + half * 16u + mode : call);
        HST(12, NOW()); HST(11, h11); HST(14, h14); HST(13, (unsigned long long)(rd * 4096 + idx + 1));
        if (!ahead && un < PK_O) { pace(t0, un); fetch_input(P, buf, un, lane); WAIT_V(0); }
        PT(5);
    }
}

__device__ __forceinline__ void flag_mirror(char* myws, const char* fp, int nseq, LAS unsigned* info, LAS unsigned* prog, unsigned call, int lane) {
    GAS unsigned* tmo = (GAS unsigned*)(myws + OFF_TMO);
    int pos = NPUSH;
    bool all = false;
    unsigned long long t0 = __builtin_amdgcn_s_memrealtime();
    while (pos < nseq) {
        const int pr = (int)__builtin_amdgcn_readfirstlane(__hip_atomic_load(prog, __ATOMIC_RELAXED, __HIP_MEMORY_SCOPE_WORKGROUP));
        if (pos <= pr + (all ? LOOK : LOOK2)) {
            const int lo = pos & ~31, hi = all ? lo + 32 * MALL : ((pr + LOOK2) | 31) + 1, i = lo + 4 * lane;
            bool bad = i >= hi;
            if (!bad && i < nseq) {
                u4 f = *(const LAS u4*)(info + i);
                if (i >= pos && (f[0] < call * 32u || f[1] < call * 32u || f[2] < call * 32u || f[3] < call * 32u)) {
                    asm volatile("global_load_dwordx4 %0, %1, %2 sc0 sc1\n\ts_waitcnt vmcnt(0)" : "=&v"(f) : "v"(i * 4), "s"(fp) : "memory");
                    *(LAS u4*)(info + i) = f;
                    bad = f[0] < call * 32u || f[1] < call * 32u || f[2] < call * 32u || f[3] < call * 32u;
                }
            }
            const unsigned long long m = __builtin_amdgcn_ballot_w64(bad);
            const int n = lo + 4 * (m ? __builtin_ctzll(m) : 64);
            all = n >= hi;
            if (n > pos) { pos = n; t0 = __builtin_amdgcn_s_memrealtime(); if (all) continue; }
        }
        if (
            __builtin_amdgcn_s_memrealtime() - t0 > DEADLINE) {
            __hip_atomic_store(tmo, call, __ATOMIC_RELAXED, __HIP_MEMORY_SCOPE_AGENT);
            asm volatile("s_waitcnt vmcnt(0)" ::: "memory");
            for (int i = pos + lane; i < nseq; i += 64) info[i] = call * 32u + M_DEAD;
            return;
        }
        if (pos > pr) __builtin_amdgcn_s_sleep(MSL);
    }
}

#define STR2(x) #x
#define STR(x) STR2(x)
#define SP ""
#define RD ""
#define PULL6 "global_load_dwordx4 %[h0], %[v16], %[src] offset:-4096 sc0 sc1\n\t" SP "global_load_dwordx4 %[h1], %[v16], %[src] offset:-3072 sc0 sc1\n\t" SP \
              "global_load_dwordx4 %[h2], %[v16], %[src] offset:-2048 sc0 sc1\n\t" SP "global_load_dwordx4 %[h3], %[v16], %[src] offset:-1024 sc0 sc1\n\t" SP \
              "global_load_dwordx4 %[h4], %[v16], %[src] sc0 sc1\n\t" SP "global_load_dwordx4 %[h5], %[v16], %[src] offset:1024 sc0 sc1\n\t" SP
#define PULLT "s_mov_b64 exec, 1\n\tglobal_atomic_add %[tk], %[ao], %[one], %[ws] sc0\n\t" \
              "s_waitcnt vmcnt(%[rn] + 9 - " STR(RELAT) ")\n\ts_mov_b32 exec_lo, %[m1]\n\tglobal_store_dword %[pf], %[cl], %[ws] sc0 sc1\n\t" \
              "s_mov_b32 exec_lo, %[m2]\n\tglobal_store_dword %[pf], %[cl], %[ws]\n\ts_waitcnt vmcnt(%[rn] + 2)\n\t" RD \
              "ds_add_u32 %[rel], %[one]\n\ts_mov_b64 exec, -1\n\ts_waitcnt vmcnt(2)"
#define PULLO [h0] "=&v"(h[0]), [h1] "=&v"(h[1]), [h2] "=&v"(h[2]), [h3] "=&v"(h[3]), [h4] "=&v"(h[4]), [h5] "=&v"(h[5]), [tk] "=&v"(tk)
#define PULLI [v16] "v"(vl16), [src] "s"(src), [ao] "v"(aoff), [one] "v"(one), [ws] "s"(myws), [rel] "v"(rel), [pf] "v"(pfl + fcop), [cl] "v"(call), [m1] "s"(m1), [m2] "s"(m2)
#undef VBLK
#define VBLK (NGEMM + vg)
__device__ __forceinline__ void puller(int vg, const CAS Params& P, char* myws, LAS unsigned* info, LAS unsigned* prog, const LAS unsigned short* tab,
                                       LAS char* scr, const LAS char* lut, int link, int peer, int cur, int nw, unsigned win, unsigned call, int wid, int lane) {
    const int q = P.rank & 3, qs = peer & 3;
    const bool cross = link < 4;
    const int nseq = cross ? NSEQ : NI;
    const char* pk = P.ws[peer] + 4096;
    const unsigned vl16 = (unsigned)lane * 16, one = 1,
        aoff = (unsigned)OFF_CTR + (unsigned)link * 128u;
    const unsigned ovoff = (unsigned)(lane >> 4) * (4 * NLOC * 4) + (unsigned)(lane & 15) * 16;
    const unsigned base = (call - 1u) * (unsigned)(nseq - NPUSH) - (unsigned)(nw + NPUSH);
    const unsigned rel = (unsigned)(size_t)(prog + 2);
    const unsigned nofl = (unsigned)OFF_FN + (unsigned)(vg * 128 + wid * 4);
    const unsigned m2 = 1u << xcc_id(), fcop = ((lane & 7) + (lane == (int)xcc_id() ? 8u : 0u)) * FSTR;
    auto flags = [&](unsigned fl) {
        asm volatile("s_mov_b64 exec, %3\n\tglobal_store_dword %0, %1, %2 sc0 sc1\n\ts_mov_b32 exec_lo, %4\n\tglobal_store_dword %0, %1, %2\n\ts_mov_b64 exec, -1"
                     :: "v"(fl + fcop), "v"(call), "s"(myws), "s"((unsigned long long)(m2 ^ 255u)), "s"(m2) : "memory");
    };
    unsigned pfl = nofl;
    const unsigned tz = __builtin_amdgcn_readfirstlane(prog[4]);
    if (TOUCH & 2) {
        const int id = __builtin_amdgcn_readfirstlane((int)tab[cur]), rb = id & 31;
        unsigned off, fl, d;
        if (id < PK_W) place_x(rb < 24 ? 3 * qs + (rb >> 3) : 12 + qs, rb, id >> 5, call, off, fl);
        else place_w(16 + 4 * qs + (id & 3), (id - PK_W) >> 2, off, fl);
        const char* a = lane ? myws + off : pk + pk_off(id < PK_W ? id : id + q * 512, call);
        if (lane < 2) asm volatile("global_load_dword %0, %1, off sc0 sc1\n\ts_waitcnt vmcnt(0)" : "=&v"(d) : "v"(a) : "memory");
    }
    for (;;) {
        const bool isout = cross && cur >= NCI;
        const int id = isout ? 0 : __builtin_amdgcn_readfirstlane((int)tab[cur]);
        const int slot = isout ? PK_O + q * NO + cur - NCI : id < PK_W ? id : id + q * 512;
        unsigned mode;
        if (pfl != nofl) {
            const unsigned i0 = __hip_atomic_load(info + cur, __ATOMIC_RELAXED, __HIP_MEMORY_SCOPE_WORKGROUP);
            const unsigned i1 = __hip_atomic_load(prog + 1, __ATOMIC_RELAXED, __HIP_MEMORY_SCOPE_WORKGROUP), i2 = __hip_atomic_load(prog + 2, __ATOMIC_RELAXED, __HIP_MEMORY_SCOPE_WORKGROUP);
            if (__builtin_amdgcn_readfirstlane(!DEFER || i0 - call * 32u > 31u || i1 - i2 >= win)) {
                WAIT_V(0);
                flags(pfl);
                pfl = nofl;
            }
        }
        for (;;) {
            mode = __builtin_amdgcn_readfirstlane(__hip_atomic_load(info + cur, __ATOMIC_ACQUIRE, __HIP_MEMORY_SCOPE_WORKGROUP)) - call * 32u;
            if (mode <= 31u) break;
            __builtin_amdgcn_s_sleep(1);
        }
        const char* src = pk + (pk_off(slot, call) + (mode & 16u) * 512u);
        mode &= 15u;
        const unsigned long long mask = lane_mask(mode * 8u);
        unsigned my = 0;
        if (lane == 0) my = __hip_atomic_fetch_add(prog + 1, 1u, __ATOMIC_RELAXED, __HIP_MEMORY_SCOPE_WORKGROUP);
        my = __builtin_amdgcn_readfirstlane(my) - win - (cur < HWIN && cur >= HWIN0);
        while ((int)(my - __builtin_amdgcn_readfirstlane(__hip_atomic_load(prog + 2, __ATOMIC_RELAXED, __HIP_MEMORY_SCOPE_WORKGROUP))) >= 0) { }
        {
            unsigned ts = 0;
            if (lane == 0) {
                __hip_atomic_fetch_max(prog + 3, ((unsigned)__builtin_amdgcn_s_memrealtime() - tz) << 8, __ATOMIC_RELAXED, __HIP_MEMORY_SCOPE_WORKGROUP);
                ts = __hip_atomic_fetch_add(prog + 3, (mode >= M_RAW ? 128u : 104u + ((mode + 1u) & ~1u)) * (cross ? PRATE : PRATEI), __ATOMIC_RELAXED, __HIP_MEMORY_SCOPE_WORKGROUP);
            }
            ts = __builtin_amdgcn_readfirstlane(ts);
            ts = (ts >> 8) + tz - PSLACK;
            while ((int)((unsigned)__builtin_amdgcn_s_memrealtime() - ts) < 0) { }
        }
        u4 h[8]; unsigned tk;
        const unsigned m1 = pfl != nofl ? m2 ^ 255u : 256u;
        const unsigned tis = (unsigned)__builtin_amdgcn_s_memrealtime();
        if (mode >= M_RAW) {
            asm volatile(PULL6 "global_load_dwordx4 %[h6], %[v16], %[src] offset:2048 sc0 sc1\n\tglobal_load_dwordx4 %[h7], %[v16], %[src] offset:3072 sc0 sc1\n\t" PULLT
                         : PULLO, [h6] "=&v"(h[6]), [h7] "=&v"(h[7]) : PULLI, [rn] "n"(RELAT) : "memory");
        } else {
            u2 p0, p1;
            asm volatile(PULL6 "global_load_dwordx2 %[p0], %[v8], %[src] offset:2048 sc0 sc1\n\t"
                         "s_mov_b64 exec, %[mask]\n\tglobal_load_dwordx2 %[p1], %[v8], %[src] offset:2560 sc0 sc1\n\t" PULLT
                         : PULLO, [p0] "=&v"(p0), [p1] "=&v"(p1)
                         : PULLI, [v8] "v"(vl16 >> 1), [mask] "s"(mask), [rn] "n"(RELAT) : "memory");
            h[6] = (u4){p0[0], p0[1], p1[0], p1[1]};
        }
        const unsigned lat = (unsigned)__builtin_amdgcn_s_memrealtime() - tis;
        const unsigned nxt = __builtin_amdgcn_readfirstlane(tk) - base;
        if (lane == 0) __hip_atomic_fetch_max(prog, nxt, __ATOMIC_RELAXED, __HIP_MEMORY_SCOPE_WORKGROUP);
        if (isout) {
            const int o = cur - NCI;
            int panel, blk, col;
            if (o < O_B) {
                const int r = o < O_A0 ? o : o < O_C ? (o - O_A0) & 127 : o - O_C, hn = o < O_A0 ? 16 : o < O_C ? 64 : 48;
                const int hf = r >= hn, t = (r - hf * hn) >> 3;
                panel = o < O_A0 ? 3 * qs + 2 : o < O_C ? 3 * (t >> 1) + (o >= O_A1) : 12 + ((q + 1 + (t >> 1)) & 3);
                blk = (r & 6) + hf; col = (t & 1) * 128 + (r & 1) * 64;
            } else {
                const int j = (o - O_B) / 24, r = o - O_B - j * 24;
                panel = 3 * ((qs + 1 + (r >> 3)) & 3) + 2; blk = 2 * j + ((r >> 1) & 1); col = ((r >> 2) & 1) * 128 + (r & 1) * 64;
            }
            char* ob = (char*)P.out + (size_t)(panel_row(peer, panel, blk) * NLOC + qs * 256 + col) * 4;
            if (mode < M_RAW) unpack_unit<true>(h, scr, lut, lane);
            if (mode == M_DEAD) h[0] = (u4){0x7FC00000u, 0x7FC00000u, 0x7FC00000u, 0x7FC00000u};
#pragma unroll
            for (int k = 0; k < 8; ++k) GST4P(ovoff, h[k], ob + ((k >> 2) * 16 + (k & 3)) * (NLOC * 4));
            if (!DEFER) WAIT_V(0);
            pfl = nofl;
        } else {
            if (mode < M_RAW) unpack_unit<false>(h, scr, lut, lane);
            unsigned off, fl;
            const int rb = id & 31;
            if (id < PK_W) place_x(rb < 24 ? 3 * qs + (rb >> 3) : 12 + qs, rb, id >> 5, call, off, fl);
            else place_w(16 + 4 * qs + (id & 3), (id - PK_W) >> 2, off, fl);
            store8(h, myws + off, id < PK_W ? 32768u : 4096u, vl16);
            pfl = fl;
        }
        if (nxt >= (unsigned)nseq) break;
        cur = (int)nxt;
    }
    WAIT_V(0);
    if (pfl != nofl) flags(pfl);
}

#undef VBLK
#define VBLK (blockIdx.x < NGEMM ? blockIdx.x : blockIdx.x + NPC)
__device__ __forceinline__ float silu(float y) {
    const float c = __builtin_amdgcn_fmed3f(y, -60.0f, 60.0f);
    const float e = __builtin_amdgcn_exp2f(c * -1.4426950408889634f);
    return y * __builtin_amdgcn_rcpf(1.0f + e);
}
__device__ __forceinline__ u4 silu4(const f4 (&a)[4], int r, bool dead) {
    f4 o = {silu(a[0][r]), silu(a[1][r]), silu(a[2][r]), silu(a[3][r])};
    const float nanv = __builtin_nanf("");
    if (dead) o = (f4){nanv, nanv, nanv, nanv};
    return __builtin_bit_cast(u4, o);
}

#define BAR() do { __builtin_amdgcn_sched_barrier(0); __builtin_amdgcn_s_barrier(); asm volatile("" ::: "memory"); __builtin_amdgcn_sched_barrier(0); } while (0)
#define GLDS(g, l, imm) __builtin_amdgcn_global_load_lds((const GAS void*)(g), (LAS void*)(l), 16, imm, AUX)
template <int V> struct IC { static constexpr int value = V; };

__device__ __forceinline__ void gemm_panel(const CAS Params& P, LAS char* lds, char* myws, int panel, int cg, int cg2, int o0, int ohalf,
                                           unsigned call, int wid, int lane) {
    constexpr int ABYTES = 32768, STAGE = ABYTES + 16384;
    const char* ga = myws + panel_off(panel, call) + wid * 4096 + lane * 16;
    const char* gb = myws + OFF_DATA + A_B + (size_t)cg * CG_B + wid * 2048 + lane * 16;
    const int wm = wid >> 1, wn = wid & 1;
    const unsigned frow = flag_copy(lane) + ((lane & 2) ? (unsigned)OFF_FB + (unsigned)cg * 1024u : (unsigned)OFF_FA + (unsigned)panel * 2048u + (unsigned)(lane & 1) * 16u);
    const int fch = lane >> 2 & 7;
    const unsigned fstep = (lane & 2) ? 16u : 32u;
    GAS unsigned* tmo = (GAS unsigned*)(myws + OFF_TMO);
    char* oflag = myws + (OFF_FP + (call & 1) * FP_B) + (((cg - 8) >> 1) * NSEQ + NCI + o0 + wid) * 4;
    int ready = 0;
    unsigned tv = 0;

    auto early = [&](int c) {
        if (c < ready || c >= NCH) return;
        const int cc = c + fch;
        __builtin_amdgcn_global_load_lds((const GAS void*)(myws + (frow + (unsigned)(cc < NCH ? cc : NCH - 1) * fstep)), (LAS void*)(lds + 3 * STAGE), 16, 0, AUX);
    };
    auto ensure = [&](int c, bool pend) {
        if (c < ready) return;
        const unsigned long long t0 = __builtin_amdgcn_s_memrealtime();
        int lim = 7;
        for (;;) {
            int cc = c + (fch < lim ? fch : lim); cc = cc < NCH ? cc : NCH - 1;
            u4 f;
            if (pend) asm volatile("s_waitcnt vmcnt(%2)\n\tds_read_b128 %0, %1\n\ts_waitcnt lgkmcnt(0)"
                                   : "=&v"(f) : "v"((unsigned)(size_t)(lds + 3 * STAGE) + (unsigned)lane * 16u), "n"(3 * PD) : "memory");
            else asm volatile("global_load_dwordx4 %0, %1, %2 sc0 sc1\n\ts_waitcnt vmcnt(0)"
                              : "=&v"(f) : "v"(frow + (unsigned)cc * fstep), "s"(myws) : "memory");
            const unsigned m = unset(f[0], call) | unset(f[1], call) | unset(f[2], call) | unset(f[3], call);
            const int n = m ? (__builtin_ctz(m) >> 2) : lim + 1;
            if (n > 0) { ready = c + n; return; }
            if (pend) { pend = false; continue; }
            if (
                __builtin_amdgcn_s_memrealtime() - t0 > DEADLINE) {
                __hip_atomic_store(tmo, call, __ATOMIC_RELAXED, __HIP_MEMORY_SCOPE_AGENT);
                ready = NCH; return;
            }
            lim = 0;
            __builtin_amdgcn_s_sleep(6);
        }
    };
    auto stage = [&](int t, int kf, int bufoff) {
        const char* a = ga + (size_t)t * 32768 + kf * 1024; const char* b = gb + (size_t)t * 16384 + kf * 1024;
        LAS char* la = lds + bufoff + wid * 4096 + kf * 1024; LAS char* lb = lds + bufoff + ABYTES + wid * 2048 + kf * 1024;
        GLDS(a, la, 0); GLDS(a, la, 2048); GLDS(b, lb, 0);
    };

    f4 acc[4][4];
#pragma unroll
    for (int m = 0; m < 4; ++m)
#pragma unroll
        for (int n = 0; n < 4; ++n) acc[m][n] = (f4){0.f, 0.f, 0.f, 0.f};

    const int aoff = wm * 8192 + lane * 16, boff = ABYTES + wn * 8192 + lane * 16;
    auto phase = [&](auto waitn, int rbuf, int kf, int lt, int lkf, int lbuf, int pc, int pe = NCH) {
        const LAS char* pa = lds + rbuf + aoff + kf * 1024; const LAS char* pb = lds + rbuf + boff + kf * 1024;
        h8 a[4], b[4];
#pragma unroll
        for (int n = 0; n < 4; ++n) b[n] = *(const LAS h8*)(pb + n * 2048);
#pragma unroll
        for (int m = 0; m < 4; ++m) a[m] = *(const LAS h8*)(pa + m * 2048);
        __builtin_amdgcn_sched_barrier(0);
        if (lt >= 0) stage(lt, lkf, lbuf);
        if (decltype(waitn)::value >= 0) WAIT_V(decltype(waitn)::value < 0 ? 0 : decltype(waitn)::value);
        WAIT_L0();
        BAR();
        __builtin_amdgcn_s_setprio(1);
#pragma unroll
        for (int m = 0; m < 4; ++m)
#pragma unroll
            for (int n = 0; n < 4; ++n)
                asm volatile("v_mfma_f32_16x16x32_f16 %0, %1, %2, %0" : "+v"(acc[m][n]) : "v"(a[m]), "v"(b[n]));
        __builtin_amdgcn_s_setprio(0);
        __builtin_amdgcn_sched_barrier(0);
        if (wid == 0) { if (pc >= 0) { ensure(pc, PD > 0); TREC(pc); } early(pe); }
        BAR();
    };

    if (wid == 0) { ensure(0, false); ensure(1, false); }
    BAR();
    STAMP(1);
    stage(0, 0, 0); stage(0, 1, 0); stage(1, 0, STAGE); stage(1, 1, STAGE); stage(2, 0, 2 * STAGE);
    WAIT_V(12);
    BAR();
    if (wid >= 4) BAR();
    int rb = 0, pb = 2 * STAGE;
    for (int t = 0; t < NKT - 3; ++t) {
        phase(IC<12>{}, rb, 0, t + 2, 1, pb, (t & 1) ? (t + 3) >> 1 : -1, PD == 2 && !(t & 1) ? (t + 4) >> 1 : NCH);
        phase(IC<12>{}, rb, 1, t + 3, 0, rb, -1, PD == 1 && !(t & 1) ? (t + 4) >> 1 : NCH);
        pb = rb; rb = rb == 2 * STAGE ? 0 : rb + STAGE;
    }
    static_assert((NKT - 3) % 3 == 2, "tail buffers");
    phase(IC<12>{}, 2 * STAGE, 0, NKT - 1, 1, STAGE, -1);
    if (wid == 4) { const char* f = myws + b_flags(cg2, lane); GLDS(f, lds + BFLG, 0); GLDS(f + 512, lds + BFLG + 1024, 0); }
    FENCE();
    const int pk0 = PK_O + ((cg < 8 ? 0 : cg - 8) >> 1) * NO + o0 + wid;
    const unsigned hv[2] = {pk_half(myws, pk0, call), pk_half(myws, pk0 + ohalf, call)};
    const unsigned tm = __hip_atomic_load(tmo, __ATOMIC_RELAXED, __HIP_MEMORY_SCOPE_AGENT);
    FENCE();
    phase(IC<9 + 3>{}, 2 * STAGE, 1, -1, 0, 0, -1);
    phase(IC<6 + 3>{}, 0, 0, -1, 0, 0, -1);
    phase(IC<3 + 3>{}, 0, 1, -1, 0, 0, -1);
    phase(IC<0 + 3>{}, STAGE, 0, -1, 0, 0, -1);
    phase(IC<-1>{}, STAGE, 1, -1, 0, 0, -1);
#pragma unroll
    for (int m = 0; m < 4; ++m) asm volatile("s_nop 7" : "+v"(acc[m][0]), "+v"(acc[m][1]), "+v"(acc[m][2]), "+v"(acc[m][3]));
    if (wid < 4) BAR();
    STAMP(2);
    TDUMP(0);

    const bool dead = __builtin_amdgcn_readfirstlane(tm) == call;
    const unsigned vl16 = (unsigned)lane * 16, ovoff = (unsigned)(lane >> 4) * (4 * NLOC * 4) + (unsigned)(lane & 15) * 16;
    unsigned mode = 0;
#pragma unroll
    for (int u = 0; u < 2; ++u) {
        u4 h[8];
#pragma unroll
        for (int k = 0; k < 8; ++k) h[k] = silu4(acc[2 * u + (k >> 2)], k & 3, dead);
        if (cg < 8) {
            char* ob = (char*)P.out + (size_t)(panel_row(P.rank, panel, 2 * wm + u) * NLOC + cg * 128 + wn * 64) * 4;
#pragma unroll
            for (int k = 0; k < 8; ++k) GST4P(ovoff, h[k], ob + ((k >> 2) * 16 + (k & 3)) * (NLOC * 4));
        } else {
            const unsigned prev = mode;
            unsigned r0 = 0, r2 = 0;
            mode = pack_unit<true>(h, lds + wid * 4096, lds + (8 * PBUF + 16), lane, r0, r2);
            if (u) { WAIT_V(0); if (lane == 0) flag_store(oflag, call * 32u + prev); }
            const unsigned half = SMODE == 0 ? wid & 1 : (~__builtin_amdgcn_readfirstlane(hv[u]) ^ (SMODE == 2)) & 1u;
            store_unit<true>(h, myws + (pk_off(pk0 + u * ohalf, call) + half * 8192), mode, r0, r2, vl16);
            mode += half * 16u;
        }
    }
    if (cg >= 8) {
        WAIT_V(0);
        if (lane == 0) flag_store(oflag + ohalf * 4, call * 32u + mode);
    }
    STAMP(3);
}

__device__ __forceinline__ bool all_set(u4 f, u4 e, unsigned call) {
    unsigned m = 0;
#pragma unroll
    for (int i = 0; i < 4; ++i) m |= unset(f[i], call) | unset(e[i], call);
    return m == 0;
}
__device__ __forceinline__ bool all_b(const char* myws, int cg, unsigned call, int lane) {
    u4 f, e;
    asm volatile("global_load_dwordx4 %0, %2, %3 sc0 sc1\n\tglobal_load_dwordx4 %1, %2, %3 offset:512 sc0 sc1\n\ts_waitcnt vmcnt(0)"
                 : "=&v"(f), "=&v"(e) : "v"(b_flags(cg, lane)), "s"(myws) : "memory");
    return all_set(f, e, call);
}

__device__ __forceinline__ void gemm_quarter(const CAS Params& P, LAS char* lds, char* myws, int panel, int qt, int cg, int o0,
                                             unsigned call, int wid, int lane) {
    constexpr int STAGE = 49152;
    const int kp = wid >> 2, lw = wid & 3, wm = lw >> 1, wn = lw & 1;
    const char* gsrc = myws + OFF_DATA + lane * 16 + (lw < 2 ? (size_t)panel * PANEL_B + kp * 32768 + qt * 8192
                                                             : A_B + (size_t)cg * CG_B + kp * 16384 + (lw - 2) * 8192);
    const unsigned gstep = lw < 2 ? 65536u : 32768u;
    LAS char* ring = lds + kp * 24576;
    LAS unsigned* pl = (LAS unsigned*)(lds + 160768);
    const unsigned frow = flag_copy(lane) + (unsigned)OFF_FA + (unsigned)(panel * 2048 + qt * 8 + (lane & 1) * 4);
    GAS unsigned* tmo = (GAS unsigned*)(myws + OFF_TMO);
    char* oflag = myws + (OFF_FP + (call & 1) * FP_B) + (((cg - 8) >> 1) * NSEQ + NCI + o0) * 4;
    unsigned tv = 0;
    int ready = 0; bool pend = false;
#define QPOLL 8
    auto paddr = [&]() { const int cc = ready + (lane >> 1 & 15); return (lane & 31) < 2 * QPOLL ? frow + (unsigned)(cc < NCH ? cc : NCH - 1) * 32u : (unsigned)OFF_TMO; };
    auto late = [&](unsigned tm, unsigned long long t0) {
        if (tm != call && __builtin_amdgcn_s_memrealtime() - t0 <= DEADLINE) return false;
        __hip_atomic_store(tmo, call, __ATOMIC_RELAXED, __HIP_MEMORY_SCOPE_AGENT);
        WAIT_V(0);
        ready = NCH;
        return true;
    };
    auto ensure = [&](int c) {
        if (c >= ready) {
            const unsigned long long t0 = __builtin_amdgcn_s_memrealtime();
            for (;;) {
                unsigned f;
                if (pend) { WAIT_V(0); f = pl[lane]; pend = false; }
                else asm volatile("global_load_dword %0, %1, %2 sc0 sc1\n\ts_waitcnt vmcnt(0)" : "=&v"(f) : "v"(paddr()), "s"(myws) : "memory");
                ready += __builtin_ctz(unset(f, call) | (1u << (2 * QPOLL))) >> 1;
                if (c < ready || late(__builtin_amdgcn_readlane(f, 31), t0)) break;
                __builtin_amdgcn_s_sleep(QSLEEP);
            }
        }
        if (!pend && ready < NCH && ready <= c + 2) {
            __builtin_amdgcn_global_load_lds((const GAS void*)(myws + paddr()), (LAS void*)pl, 4, 0, AUX);
            pend = true;
        }
    };
    auto stage = [&](int c, int buf) {
        const char* g = gsrc + (size_t)((unsigned)c * gstep);
        LAS char* l = ring + buf + (lw - 1) * 8192;
        GLDS(g, l, 0); GLDS(g, l, 1024); GLDS(g, l, 2048); GLDS(g, l, 3072);
        GLDS(g + 4096, l + 4096, 0); GLDS(g + 4096, l + 4096, 1024); GLDS(g + 4096, l + 4096, 2048); GLDS(g + 4096, l + 4096, 3072);
    };

    const int pk0 = PK_O + ((cg < 8 ? 0 : cg - 8) >> 1) * NO + o0;
    if (wid == 0) pl[64] = pk_half(myws, pk0, call);
    f4 acc[2][4];
#pragma unroll
    for (int m = 0; m < 2; ++m)
#pragma unroll
        for (int n = 0; n < 4; ++n) acc[m][n] = (f4){0.f, 0.f, 0.f, 0.f};
    const int aoff = wm * 4096 + lane * 16, boff = 8192 + wn * 8192 + lane * 16;
    auto phase = [&](auto waitn, auto role, int buf, int lc) {
        h8 a[4], b[8];
#pragma unroll
        for (int n = 0; n < 8; ++n) b[n] = *(const LAS h8*)(ring + buf + boff + n * 1024);
#pragma unroll
        for (int m = 0; m < 4; ++m) a[m] = *(const LAS h8*)(ring + buf + aoff + m * 1024);
        __builtin_amdgcn_sched_barrier(0);
        if (decltype(role)::value == 2 && wid == 0) {
            ensure(lc); TREC(lc);
        }
        WAIT_L0();
        BAR();
        if (decltype(role)::value == 1) stage(lc, buf);
#pragma unroll
        for (int k = 0; k < 2; ++k)
#pragma unroll
            for (int m = 0; m < 2; ++m)
#pragma unroll
                for (int n = 0; n < 4; ++n) acc[m][n] = __builtin_amdgcn_mfma_f32_16x16x32_f16(a[m * 2 + k], b[n * 2 + k], acc[m][n], 0, 0, 0);
        if (decltype(role)::value == 1) {
#pragma unroll
            for (int i = 0; i < 8; ++i) { __builtin_amdgcn_sched_group_barrier(0x010, 1, 0); __builtin_amdgcn_sched_group_barrier(0x008, 2, 0); }
        }
        __builtin_amdgcn_sched_barrier(0);
        if (decltype(waitn)::value >= 0) { WAIT_V(decltype(waitn)::value < 0 ? 0 : decltype(waitn)::value); BAR(); }
    };

    if (wid == 0 && ready < NCH) {
        const unsigned long long t0 = __builtin_amdgcn_s_memrealtime();
        for (;;) {
            if (all_b(myws, cg, call, lane) ||
                late(__builtin_amdgcn_readfirstlane(__hip_atomic_load(tmo, __ATOMIC_RELAXED, __HIP_MEMORY_SCOPE_AGENT)), t0)) break;
            __builtin_amdgcn_s_sleep(6);
        }
        ensure(2);
    }
    WAIT_L0();
    BAR();
    STAMP(4);
    const unsigned half = SMODE == 0 ? cg & 1 : (~__builtin_amdgcn_readfirstlane(pl[64]) ^ (SMODE == 2)) & 1u;
    if (lw != 0) { stage(0, 0); stage(1, STAGE); stage(2, 2 * STAGE); }
    WAIT_V(16);
    BAR();
    if (kp) BAR();
    if (lw != 0) for (int c = 3, buf = 0; c < NCH; ++c, buf = buf == 2 * STAGE ? 0 : buf + STAGE) phase(IC<16>{}, IC<1>{}, buf, c);
    else for (int c = 3, buf = 0; c < NCH; ++c, buf = buf == 2 * STAGE ? 0 : buf + STAGE) phase(IC<16>{}, IC<2>{}, buf, c);
    static_assert((NCH - 3) % 3 == 1, "tail buffers");
    const unsigned tm = __hip_atomic_load(tmo, __ATOMIC_RELAXED, __HIP_MEMORY_SCOPE_AGENT);
    phase(IC<8>{}, IC<0>{}, STAGE, -1);
    phase(IC<0>{}, IC<0>{}, 2 * STAGE, -1);
    phase(IC<-1>{}, IC<0>{}, 0, -1);
    f4 r[4];
#pragma unroll
    for (int n = 0; n < 4; ++n) {
        const f4 a0 = acc[0][n], a1 = acc[1][n];
        r[n] = kp ? a1 : a0;
        *(LAS f4*)(ring + lw * 4096 + n * 1024 + lane * 16) = kp ? a0 : a1;
    }
    const bool dead = __builtin_amdgcn_readfirstlane(tm) == call;
    WAIT_L0();
    BAR();
    if (!kp) BAR();
    STAMP(5);
#pragma unroll
    for (int n = 0; n < 4; ++n) r[n] += *(const LAS f4*)(lds + (kp ^ 1) * 24576 + lw * 4096 + n * 1024 + lane * 16);
    if (cg < 8) {
        const unsigned ovoff = (unsigned)(lane >> 4) * (4 * NLOC * 4) + (unsigned)(lane & 15) * 16;
        char* ob = (char*)P.out + (size_t)((panel_row(P.rank, panel, 2 * qt + wm) + kp * 16) * NLOC + cg * 128 + wn * 64) * 4;
#pragma unroll
        for (int k = 0; k < 4; ++k) { const u4 o = silu4(r, k, dead); GST4P(ovoff, o, ob + k * (NLOC * 4)); }
    } else {
        char* d0 = myws + (pk_off(pk0 + lw, call) + half * 8192 + kp * 4096);
#pragma unroll
        for (int k = 0; k < 4; ++k) { const u4 o = silu4(r, k, dead); GST4((unsigned)lane * 16, o, d0 + k * 1024, 0); }
        WAIT_V(0);
        BAR();
        if (wid == 0 && lane < 4) flag_store(oflag + lane * 4, call * 32u + half * 16u + M_RAW);
    }
    STAMP(6);
    TDUMP(1);
}

#define QPK 2
#define QSL1 1
#define HALF 1
#define NPOLL 2
#define QAHEAD 12
template <int KS>
__device__ __forceinline__ void gemm_ring(const CAS Params& P, LAS char* lds, char* myws, int panel, int qt, int cg, int hb, int o0, bool pre,
                                          unsigned call, int wid, int lane) {
    volatile LAS unsigned* ctl = (volatile LAS unsigned*)(lds + 160768);
    auto put = [&](int i, unsigned v) {
        asm volatile("s_mov_b64 exec, 1\n\tds_write_b32 %0, %1\n\ts_mov_b64 exec, -1" :: "v"((unsigned)(size_t)(ctl + i)), "v"(v) : "memory");
    };
    GAS unsigned* tmo = (GAS unsigned*)(myws + OFF_TMO);
    auto late = [&](unsigned long long t0) {
        if (__builtin_amdgcn_s_memrealtime() - t0 <= DEADLINE) return false;
        __hip_atomic_store(tmo, call, __ATOMIC_RELAXED, __HIP_MEMORY_SCOPE_AGENT);
        WAIT_V(0);
        return true;
    };
    const int pk0 = PK_O + ((cg < 8 ? 0 : cg - 8) >> 1) * NO;
    if (wid == 4) {
        if (lane < 8) ctl[lane] = 0;
        const unsigned long long t0 = __builtin_amdgcn_s_memrealtime();
        if (!pre || !all_set(*(const LAS u4*)(lds + BFLG + lane * 16), *(const LAS u4*)(lds + BFLG + 1024 + lane * 16), call)) for (;;) {
            if (all_b(myws, cg, call, lane) || late(t0)) break;
            __builtin_amdgcn_s_sleep(6);
        }
    }
    WAIT_L0();
    BAR();
    STAMP(4);
    const unsigned vl16 = (unsigned)lane * 16;
    if (wid >= 6) {
        const unsigned frow = flag_copy(lane) + (unsigned)OFF_FA + (unsigned)(panel * 2048 + qt * 8 + (lane & 1) * 4);
        if (wid == 7) put(10, pk_half(myws, pk0 + o0, call));
        unsigned long long t0 = __builtin_amdgcn_s_memrealtime();
        unsigned f;
        for (;;) {
            const u4 st = *(const volatile LAS u4*)ctl;
            const int c = __builtin_amdgcn_readfirstlane(st[2]);
            if (c >= NCH || wid >= 6 + NPOLL) break;
            if (c > (int)__builtin_amdgcn_readfirstlane(st[0]) + QAHEAD) { __builtin_amdgcn_s_sleep(4); t0 = __builtin_amdgcn_s_memrealtime(); continue; }
            asm volatile("global_load_dword %0, %1, %2 sc0 sc1\n\ts_waitcnt vmcnt(0)" : "=&v"(f) : "v"(frow + (unsigned)min(c + (lane >> 1 & 7), NCH - 1) * 32u), "s"(myws) : "memory");
            int n = __builtin_ctz(unset(f, call) | 65536u) >> 1;
            if (n) t0 = __builtin_amdgcn_s_memrealtime();
            else if (late(t0)) n = NCH;
            if (n) asm volatile("s_mov_b64 exec, 1\n\tds_max_u32 %0, %1\n\ts_mov_b64 exec, -1" :: "v"((unsigned)(size_t)(ctl + 2)), "v"(c + n) : "memory");
            else __builtin_amdgcn_s_sleep(QSLEEP);
        }
        asm volatile("global_load_dword %0, %1, %2 sc0 sc1\n\ts_waitcnt vmcnt(0)" : "=&v"(f) : "v"((unsigned)OFF_TMO), "s"(myws) : "memory");
        put(2 + wid, f);
        WAIT_L0();
        BAR();
    } else if (wid >= 4) {
        const char* g = myws + OFF_DATA + ((size_t)panel * PANEL_B + qt * 8192 + (wid - 4) * 32768) + vl16;
        int c = 0, pub = 0;
        while (pub < NCH) {
            const u4 fr = *(const volatile LAS u4*)(ctl + 4);
            const int lim = min((int)__builtin_amdgcn_readfirstlane(min(min(min(fr[0], fr[1]), min(fr[2], fr[3])) + 8u, ctl[2])), min(pub + 7, NCH));
            const bool go = c < lim;
            for (; c < lim; ++c, g += 65536) {
                LAS char* l = lds + (c & 7) * 16384 + (wid - 4) * 8192;
                GLDS(g, l, 0); GLDS(g, l, 1024); GLDS(g, l, 2048); GLDS(g, l, 3072);
                GLDS(g + 4096, l + 4096, 0); GLDS(g + 4096, l + 4096, 1024); GLDS(g + 4096, l + 4096, 2048); GLDS(g + 4096, l + 4096, 3072);
            }
            FENCE();
            const unsigned st = __builtin_amdgcn_s_getreg(7 | (31 << 11));
            const int k = (int)(((st & 15u) | (st >> 18 & 48u)) + 7u) >> 3;
            if (c - k > pub) {
                if (k < 4) { if (k < 2) { if (k == 0) WAIT_V(0); else WAIT_V(8); } else { if (k == 2) WAIT_V(16); else WAIT_V(24); } }
                else if (k == 4) WAIT_V(32); else if (k == 5) WAIT_V(40); else WAIT_V(48);
                pub = c - k;
                put(wid - 4, (unsigned)pub);
            } else if (!go) __builtin_amdgcn_s_sleep(QSL1);
        }
        WAIT_L0();
        BAR();
    } else {
        const char* gb = myws + OFF_DATA + (A_B + (size_t)cg * CG_B + (KS ? (wid >> 1) * 16384 + hb * 8192 + (wid & 1) * 4096 : wid * 4096));
        f4 acc[4][2];
#pragma unroll
        for (int i = 0; i < 8; ++i) acc[i >> 1][i & 1] = (f4){0.f, 0.f, 0.f, 0.f};
        asm volatile("" ::: "a0", "a127");
        unsigned long long em = ~0ull;
        auto loadb = [&](auto sl, auto kk) {
            constexpr int r = decltype(sl)::value * 16 + decltype(kk)::value * 4, o = decltype(kk)::value * 1024;
            asm volatile("s_mov_b64 exec, %2\n\tglobal_load_dwordx4 a[%3:%4], %0, %1 offset:%7 sc0 sc1\n\t"
                         "global_load_dwordx4 a[%5:%6], %0, %1 offset:%8 sc0 sc1\n\ts_mov_b64 exec, -1"
                         :: "v"(vl16), "s"(gb), "s"(em), "n"(r), "n"(r + 3), "n"(r + 8), "n"(r + 11), "n"(o), "n"(o + 2048) : "memory");
            if (decltype(kk)::value) gb += KS ? 32768 : 16384;
        };
        unsigned la = vl16 + (unsigned)(size_t)lds + (KS ? (wid >> 1) * 8192 : 0);
        auto fetch = [&](h8 (&a)[8], int i, int t) { a[i] = *(const LAS h8*)(size_t)(la + (unsigned)(t * 8192 + i * 1024)); };
        auto there = [&]() -> unsigned {
            if (KS) return ctl[wid >> 1];
            const u2 x = *(const volatile LAS u2*)ctl;
            return min(x[0], x[1]);
        };
        auto step = [&](auto sl, const h8 (&a)[8], h8 (&a2)[8], int c0) {
            constexpr int S = decltype(sl)::value, T = ((S + 1) << KS) & 7;
            if (T == 0) la ^= 65536u;
            WAIT_V(28);
            if (KS) put(4 + wid, (unsigned)(c0 + S + 1));
            u2 x = {0, 0};
            if (KS) x[0] = x[1] = ctl[wid >> 1]; else if (S & 1) x = *(const volatile LAS u2*)ctl;
#pragma unroll
            for (int i = 0; i < 16; ++i) {
                const int k = i >> 3, m = i >> 1 & 3, n = i & 1, r = S * 16 + n * 8 + k * 4;
                asm volatile("v_mfma_f32_16x16x32_f16 %0, %1, a[%2:%3], %0" : "+v"(acc[m][n]) : "v"(a[m * 2 + k]), "n"(r), "n"(r + 3) : "memory");
                if (n == 0) fetch(a2, m * 2 + k, T);
                if (i == 8) loadb(sl, IC<0>{});
            }
            if (!KS && !(S & 1)) put(4 + wid, (unsigned)(c0 + S / 2 + 1));
            loadb(sl, IC<1>{});
            if (KS || (S & 1)) {
                const int need = min(c0 + (KS ? S + 1 : (S + 1) / 2), NCH - 1);
                if ((int)__builtin_amdgcn_readfirstlane(min(x[0], x[1])) <= need) {
                    while ((int)__builtin_amdgcn_readfirstlane(there()) <= need) { }
                    FENCE();
#pragma unroll
                    for (int i = 0; i < 8; ++i) fetch(a2, (i & 3) * 2 + (i >> 2), T);
                }
            }
        };
        auto fill = [&](auto sl) { loadb(sl, IC<0>{}); loadb(sl, IC<1>{}); };
        fill(IC<0>{}); fill(IC<1>{}); fill(IC<2>{}); fill(IC<3>{}); fill(IC<4>{}); fill(IC<5>{}); fill(IC<6>{}); fill(IC<7>{});
        h8 a0[8], a1[8];
        while (__builtin_amdgcn_readfirstlane(there()) == 0) { }
        FENCE();
#pragma unroll
        for (int i = 0; i < 8; ++i) fetch(a0, (i & 3) * 2 + (i >> 2), 0);
#pragma unroll 1
        for (int c = 0; c < NCH; c += 4 << KS) {
            if (c == NCH - (4 << KS)) em = 1;
            step(IC<0>{}, a0, a1, c); step(IC<1>{}, a1, a0, c); step(IC<2>{}, a0, a1, c); step(IC<3>{}, a1, a0, c);
            step(IC<4>{}, a0, a1, c); step(IC<5>{}, a1, a0, c); step(IC<6>{}, a0, a1, c); step(IC<7>{}, a1, a0, c);
        }
        asm volatile("s_nop 15" : "+v"(acc[0][0]), "+v"(acc[0][1]), "+v"(acc[1][0]), "+v"(acc[1][1]), "+v"(acc[2][0]), "+v"(acc[2][1]), "+v"(acc[3][0]), "+v"(acc[3][1]) :: "memory");
        const int wm = KS ? wid >> 1 : wid & 1, wn = KS ? hb : wid >> 1, un = o0 + wm * 2 + wn;
        LAS char* ex = lds + 131072 + vl16;
        f4 o[2][4];
        const bool odd = wid & 1;
        if (KS) {
#pragma unroll
            for (int n = 0; n < 2; ++n) {
                const f4 x0 = odd ? acc[1][n] : acc[0][n], x1 = odd ? acc[0][n] : acc[1][n], x2 = odd ? acc[3][n] : acc[2][n], x3 = odd ? acc[2][n] : acc[3][n];
                o[1][n] = wm ? x2 : x0;
                *(LAS f4*)(ex + wid * 6144 + n * 1024) = wm ? x3 : x1;
                *(LAS f4*)(ex + wid * 6144 + 2048 + n * 1024) = wm ? x0 : x2;
                *(LAS f4*)(ex + wid * 6144 + 4096 + n * 1024) = wm ? x1 : x3;
            }
        } else {
#pragma unroll
            for (int i = 0; i < 4; ++i) {
                const f4 lo = acc[i >> 1][i & 1], hi = acc[2 + (i >> 1)][i & 1];
                *(LAS f4*)(ex + wid * 4096 + i * 1024) = wm ? lo : hi;
                o[i >> 1][i & 1] = wm ? hi : lo;
            }
        }
        WAIT_L0();
        BAR();
        STAMP(5);
        const u4 tw = *(const volatile LAS u4*)(ctl + 8);
        if (KS) {
#pragma unroll
            for (int n = 0; n < 2; ++n) {
                const f4 z = o[1][n] + *(const LAS f4*)(ex + (wid ^ 2) * 6144 + 2048 + n * 1024);
                const f4 y = *(const LAS f4*)(ex + (wid ^ 1) * 6144 + n * 1024) + *(const LAS f4*)(ex + (wid ^ 3) * 6144 + 4096 + n * 1024);
                o[0][n] = odd ? y : z; o[0][2 + n] = odd ? z : y;
            }
        } else {
#pragma unroll
            for (int i = 0; i < 4; ++i) {
                const f4 y = *(const LAS f4*)(ex + (wid ^ 1) * 4096 + i * 1024), z = o[i >> 1][i & 1];
                o[i >> 1][i & 1] = wm ? y : z; o[i >> 1][2 + (i & 1)] = wm ? z : y;
            }
        }
        const bool dead = __builtin_amdgcn_readfirstlane(tw[0] == call || tw[1] == call);
        const unsigned half = ~__builtin_amdgcn_readfirstlane(tw[2]) & 1u;
        char* dst = myws + (pk_off(pk0 + un, call) + half * 8192);
        char* oflag = myws + (OFF_FP + (call & 1) * FP_B) + (((cg - 8) >> 1) * NSEQ + NCI + un) * 4;
        u4 h[8];
#pragma unroll
        for (int k = 0; k < (KS ? 4 : 8); ++k) h[k] = silu4(o[k >> 2], k & 3, dead);
        if (KS) {
            dst += (wid & 1) * 4096;
            GST4(vl16, h[0], dst, 0); GST4(vl16, h[1], dst, 1024); GST4(vl16, h[2], dst, 2048); GST4(vl16, h[3], dst, 3072);
            WAIT_V(0);
            BAR();
            if (lane == 0 && !odd) flag_store(oflag, call * 32u + half * 16u + M_RAW);
        } else if (cg < 8) {
            const unsigned ovoff = (unsigned)(lane >> 4) * (4 * NLOC * 4) + (unsigned)(lane & 15) * 16;
            char* ob = (char*)P.out + (size_t)(panel_row(P.rank, panel, 2 * qt + wm) * NLOC + cg * 128 + wn * 64) * 4;
#pragma unroll
            for (int k = 0; k < 8; ++k) GST4P(ovoff, h[k], ob + ((k >> 2) * 16 + (k & 3)) * (NLOC * 4));
        } else {
            unsigned mode = M_RAW, r0 = 0, r2 = 0;
            if (qt < QPK) mode = pack_unit<true>(h, lds + wid * 4096, lds + (8 * PBUF + 16), lane, r0, r2);
            store_unit<true>(h, dst, mode, r0, r2, vl16);
            WAIT_V(0);
            if (lane == 0) flag_store(oflag, call * 32u + half * 16u + mode);
        }
        STAMP(6);
    }
    if (KS && wid >= 4) BAR();
}

__host__ __device__ __forceinline__ int tile2(int slot, int sown) {
    return slot == 26 ? 0 : slot == 25 ? 7 : slot == 24 ? 2 : slot == sown ? 1 : slot < 4 ? slot + 3 : slot == 4 ? 8 : slot < 17 ? slot + 7
         : slot < 20 ? slot - 8 : 33 + ((slot - sown - 1) & 3);
}

__global__ void __launch_bounds__(512, 2) ag_gemm_kernel(const Params Parg) {
    const CAS Params& P = *(const CAS Params*)__builtin_amdgcn_kernarg_segment_ptr();
    (void)Parg;
    __shared__ __attribute__((aligned(16))) char lds_raw[5 * 32768];
    LAS char* lds = (LAS char*)lds_raw;
    const int lane = threadIdx.x & 63, wid = __builtin_amdgcn_readfirstlane(threadIdx.x >> 6);
    const int xcd = blockIdx.x & 7, slot = blockIdx.x >> 3;
    char* myws = P.ws[P.rank];
    const int q = P.rank & 3;
    const unsigned long long t0 = __builtin_amdgcn_s_memrealtime();
    const unsigned side = (__builtin_amdgcn_s_getreg(20 | (3 << 11)) >> 2) & 1u;
    int g = (int)blockIdx.x - NGEMM;
    if (slot == NGEMM / 8) g = NPC + xcd;
    else if (g >= 0) {
        if (threadIdx.x == 0 && XMODE < 2) {
            GAS unsigned* rt = (GAS unsigned*)(myws + OFF_ROLE);
            const unsigned t = __hip_atomic_fetch_add(rt, side ? 65536u : 1u, __ATOMIC_RELAXED, __HIP_MEMORY_SCOPE_AGENT);
            if ((t & 65535u) + (t >> 16) >= 247u - NGEMM) __hip_atomic_store(rt, 0u, __ATOMIC_RELAXED, __HIP_MEMORY_SCOPE_AGENT);
            *(LAS unsigned*)(lds + 8 * PBUF) = t;
        }
        __syncthreads();
        const unsigned t = __builtin_amdgcn_readfirstlane(*(const LAS unsigned*)(lds + 8 * PBUF));
        constexpr unsigned N0 = NPC / 7 * 3, NL = 247 - NGEMM;
        const unsigned e = (unsigned)(slot - 27) * 4u + ((__builtin_amdgcn_s_getreg(20 | (3 << 11)) + (XMODE == 3 ? slot : 0)) & 3u);
        const unsigned r = XMODE >= 2 ? (side ? NL - e : e) : (side ? NL - (t >> 16) : t & 65535u) % (NL + 1), b = NL - r;
        g = r < N0 ? r / 3 * 7 + 4 + r % 3 : r < N0 + NPROD - 8 ? NPC + 8 + r - N0 : b / 4 * 7 + b % 4;
        if (XMODE == 0) { g = (int)blockIdx.x - NGEMM; g = g < NPC + 8 ? g - 8 : g; }
    }
#undef VBLK
#define VBLK (NGEMM + g)
    if (PROBET >= 0 && wid == 7 && blockIdx.x < 2 * PK_PG) {
        char* a = myws + (OFF_PK + ((size_t)blockIdx.x << 21) + PROBEX * (blockIdx.x & 31) * 256);
        unsigned t[2] = {~0u, ~0u};
        while (__builtin_amdgcn_s_memrealtime() - t0 < PROBET) __builtin_amdgcn_s_sleep(4);
#pragma unroll
        for (int i = 0; i < 6; ++i) {
            const unsigned long long c0 = __builtin_amdgcn_s_memtime();
            if (PROBE1) asm volatile("s_waitcnt lgkmcnt(0)\n\ts_mov_b64 exec, 1\n\tglobal_store_dword %0, %0, %1 sc0 sc1\n\ts_mov_b64 exec, -1\n\ts_waitcnt vmcnt(0)" :: "v"((i & 1) * 8192), "s"(a), "s"(c0) : "memory");
            else asm volatile("s_waitcnt lgkmcnt(0)\n\tglobal_store_dword %0, %0, %1 sc0 sc1\n\ts_waitcnt vmcnt(0)" :: "v"((i & 1) * 8192), "s"(a), "s"(c0) : "memory");
            const unsigned d = (unsigned)(__builtin_amdgcn_s_memtime() - c0);
            t[i & 1] = d < t[i & 1] ? d : t[i & 1];
        }
        if (lane == 0) flag_store(myws + OFF_MAP + blockIdx.x * MAPS, (t[1] < t[0]) ^ side);
        STAMP(10);
    }
    const int u0 = g < 0 ? wid * NGEMM + (int)blockIdx.x : UG + wid * NPROD + g - NPC;
    if (g < 0 || g >= NPC) {
        pace(t0, u0); fetch_input(P, lds + wid * PBUF, u0, lane);
        if ((TOUCH & 1) && u0 < TOUCHU) touch(P, myws, lds + wid * PBUF + 16384, u0, lane);
        make_lut(lds + (8 * PBUF + 16), true, lane);
    }
    GAS unsigned* seq = (GAS unsigned*)(myws + OFF_SEQ) + (blockIdx.x * 8 + wid) * 4;
    const unsigned call = __builtin_amdgcn_readfirstlane(__hip_atomic_load(seq, __ATOMIC_RELAXED, __HIP_MEMORY_SCOPE_AGENT)) + 1u;
    WAIT_V(0);

    if (g < 0 || g >= NPC) {
        producer(P, myws, lds + wid * PBUF, lds + (8 * PBUF + 16), u0, t0, call, wid, lane);
        STAMP(7);
        if (lane == 0) __hip_atomic_store(seq, call, __ATOMIC_RELAXED, __HIP_MEMORY_SCOPE_AGENT);
    }
    if (slot <= NGEMM / 8) {
        const int pi = slot >> 3, ps = xcd >> 1, h = xcd & 1, sown = 20 + ps;
        const int ci = pi == 3 ? (slot & 1) * 4 + ps : slot == sown ? ps : slot & 7;
        const int panel = pi < 2 ? 3 * ps + pi : pi == 3 ? 3 * q + 2 : slot == sown ? 16 : 12 + ps;
        const int k = tile2(slot, sown), kk = k % 12, qt = kk / 3, si = kk - qt * 3, cg2 = k < 12 || k >= 24 ? 8 + xcd : xcd;
        const int bp = 3 * ((q + 1 + si) & 3) + 2, o0 = O_B + qt * 24 + (si * 2 + h) * 4;
        if (g < 0) gemm_panel(P, lds, myws, panel, ci < 4 ? 4 * h + ci : 2 * ci + h, cg2,
                   pi == 3 ? h * 8 : pi == 2 ? O_C + (((ps - ci - 1) & 3) * 2 + h) * 8 : O_A0 + pi * 128 + xcd * 8, pi == 3 ? 16 : pi == 2 ? 48 : 64, call, wid, lane);
        if (HALF && qt == 3 && cg2 >= 8) gemm_ring<1>(P, lds, myws, bp, qt, cg2, k >= 24, o0, g < 0, call, wid, lane);
        else if (k < 24) gemm_ring<0>(P, lds, myws, bp, qt, cg2, 0, o0, g < 0, call, wid, lane);
    } else if (g < NPC) {
        LAS unsigned* info = (LAS unsigned*)lds;
        LAS unsigned* prog = info + NSEQ;
        LAS unsigned short* tab = (LAS unsigned short*)(lds + 8192);
        const int j = g / 7, link = g - j * 7, ncu = (NPC + 6 - link) / 7;
        for (int i = threadIdx.x; i < NSEQ + 4; i += 512) info[i] = 0;
        if (threadIdx.x == 0) { prog[4] = (unsigned)t0; prog[5] = 0; }
        for (int i = threadIdx.x; i < PK_W; i += 512)
            if (((i & 31) == 16 || (i & 31) >= 24) == (link < 4)) tab[pos_x(i & 31, i >> 5)] = (unsigned short)i;
        if (link < 4) tab[pos_w(threadIdx.x >> 2, threadIdx.x & 3)] = (unsigned short)(PK_W + threadIdx.x);
        if (wid == 0) make_lut(lds + 14336, false, lane);
        __syncthreads();
        if (lane == 0) __hip_atomic_store(seq, call, __ATOMIC_RELAXED, __HIP_MEMORY_SCOPE_AGENT);
        const int peer = link < 4 ? ((P.rank & 4) ^ 4) + ((q + link) & 3) : (P.rank & 4) + ((q + link - 3) & 3);
        if (wid < WPC) {
            if (lane == 0) __hip_atomic_fetch_max(prog, (unsigned)(NPUSH + wid * ncu + j), __ATOMIC_RELAXED, __HIP_MEMORY_SCOPE_WORKGROUP);
            puller(g, P, myws, info, prog, tab, lds + 16384 + wid * 8192, lds + 14336, link, peer, NPUSH + wid * ncu + j, ncu * WPC,
                   (unsigned)(((link < 4 ? NWIN : NWINI) + ncu - 1 - j) / ncu), call, wid, lane);
        } else if (wid == 7) {
            flag_mirror(myws, P.ws[peer] + (OFF_FP + (call & 1) * FP_B + (link < 4 ? q : 4) * (NSEQ * 4)), link < 4 ? NSEQ : NI, info, prog, call, lane);
        }
        STAMP(3);
    }
}

}

extern "C" void kernel_launch(int rank, int nranks, ncclComm_t comm, hipStream_t stream,
                              void* const* d_in, const int* in_sizes, int n_in,
                              void* d_out, int out_size, const int* in_dtypes, int out_dtype,
                              void* const* d_out_all, void* const* d_ws_all, size_t ws_bytes) {
    (void)comm; (void)in_dtypes; (void)out_dtype; (void)d_out_all;
    if (nranks != 8 || (unsigned)rank >= 8u || n_in < 2 || ws_bytes < WS_NEED || out_size != KDIM * NLOC ||
        in_sizes[0] != 1024 * KDIM || in_sizes[1] != KDIM * NLOC) {
        fprintf(stderr, "ag_gemm: unexpected problem shape\n");
        return;
    }
    Params P;
    P.x = (const float*)d_in[0]; P.w = (const float*)d_in[1];
    P.out = (float*)d_out;
    P.rank = 0; P.pad = 0;
    for (int p = 0; p < 8; ++p) {
        const int r = (int)((RMAP >> (4 * p)) & 7u);
        P.ws[p] = (char*)d_ws_all[r];
        if (r == rank) P.rank = p;
    }
    hipLaunchKernelGGL(ag_gemm_kernel, dim3(256), dim3(512), 0, stream, P);
    HIP_CHECK(hipGetLastError());
}
```

```cpp
#include <hip/hip_runtime.h>
#include <rccl/rccl.h>
#include <cstdio>

#define HIP_CHECK(expr) do { hipError_t _e = (expr); if (_e != hipSuccess) { \
    fprintf(stderr, "HIP error %s at %s:%d\n", hipGetErrorString(_e), __FILE__, __LINE__); \
    return; } } while (0)

namespace {

typedef _Float16 h8 __attribute__((ext_vector_type(8)));
typedef float f4 __attribute__((ext_vector_type(4)));
typedef unsigned u4 __attribute__((ext_vector_type(4)));
typedef unsigned u2 __attribute__((ext_vector_type(2)));
#define LAS __attribute__((address_space(3)))
#define GAS __attribute__((address_space(1)))
#define CAS __attribute__((address_space(4)))

constexpr int KDIM = 8192, NLOC = 1024;
constexpr int NKT = KDIM / 64;
constexpr int NCH = NKT / 2;
constexpr int NI = 23 * 64;
constexpr int NCI = 17 * 64, NO = 480;
constexpr int NSEQ = NCI + NO;
constexpr int O_A0 = 32, O_A1 = 160, O_C = 288, O_B = 384;
constexpr size_t PANEL_B = 4u << 20;
constexpr size_t CG_B = 2u << 20;
constexpr size_t OFF_SEQ = 0;
constexpr size_t OFF_TMO = 64u << 10;
constexpr size_t OFF_CTR = OFF_TMO + 256;
constexpr size_t OFF_ROLE = OFF_TMO + 1280;
constexpr size_t OFF_MAP = OFF_TMO + 2048;
#define MAPS 256u
constexpr size_t OFF_FP = 256u << 10;
constexpr size_t FP_B = 5 * NSEQ * 4;
constexpr size_t OFF_DATA = 1u << 20;
constexpr size_t A_B = 21 * PANEL_B, B_B = 16 * CG_B;
constexpr size_t OFF_FA = OFF_DATA + A_B + B_B;
constexpr size_t OFF_FB = OFF_FA + 17 * 64 * 32;
constexpr size_t OFF_FN = OFF_FB + 16 * 64 * 16;
constexpr unsigned FSTR = 219 * 256;
constexpr size_t OFF_PK = OFF_DATA + A_B + B_B + (1u << 20);
constexpr int PK_W = 2048, PK_O = 4096, PK_N = PK_O + 4 * NO;
constexpr int PK_PG = (PK_N + 126) / 127;
constexpr size_t PK_B = (size_t)PK_PG << 21;
constexpr size_t WS_NEED = OFF_PK + 2 * PK_B;
static_assert(OFF_MAP + 2 * PK_PG * MAPS <= OFF_FP && OFF_FN + 35 * 128 <= OFF_FA + FSTR && 16 * FSTR <= (1u << 20) && OFF_FP + 2 * FP_B <= OFF_DATA, "flag area");
constexpr unsigned long long DEADLINE = 30000000ull;
constexpr unsigned M_RAW = 9, M_DEAD = 10;

#define NPC 35
#define WPC 7
#define SKC 14
#define SKA 5
#define SQ1 5
#define SQ2 12
#define SQ3 30
#define PACE 12
#define LOOK 64
#define PCH 2
#define PT0 180
#define PDT 13
#define RAW0 32
#define PD 2
#define NWIN 10
#define NWINI NWIN
#define PRATE 134
#define HWIN 280
#define RELAT 8
#define UG 3000
#define EXN 32
#define NB2 (200 - EXN)
#define LOOK2 4
#define MALL 4
#define MSL 8
#define QSLEEP 0
#define AUX 17
#define RMAP 0x75416320u
constexpr int NGEMM = 208;
constexpr int NPROD = 256 - NGEMM - NPC;

struct Params {
    const float* x;
    const float* w;
    float* out;
    char* ws[8];
    int rank;
    int pad;
};

#define GST4(voff, data, sbase, imm) \
    asm volatile("global_store_dwordx4 %0, %1, %2 offset:" #imm " sc0 sc1\n\ts_nop 1" :: "v"(voff), "v"(data), "s"(sbase) : "memory")
#define OSUF " sc0 sc1"
#define GST4P(voff, data, sbase) \
    asm volatile("global_store_dwordx4 %0, %1, %2" OSUF "\n\ts_nop 1" :: "v"(voff), "v"(data), "s"(sbase) : "memory")
#define WAIT_V(n) asm volatile("s_waitcnt vmcnt(%0)" :: "n"(n) : "memory")
#define WAIT_L0() asm volatile("s_waitcnt lgkmcnt(0)" ::: "memory")

__device__ __forceinline__ u4 pack8(float a0, float a1, float a2, float a3, float a4, float a5, float a6, float a7) {
    h8 h = {(_Float16)a0, (_Float16)a1, (_Float16)a2, (_Float16)a3, (_Float16)a4, (_Float16)a5, (_Float16)a6, (_Float16)a7};
    return __builtin_bit_cast(u4, h);
}
__device__ __forceinline__ void flag_store(char* p, unsigned v) {
    __hip_atomic_store((GAS unsigned*)p, v, __ATOMIC_RELAXED, __HIP_MEMORY_SCOPE_SYSTEM);
}
__device__ __forceinline__ unsigned xcc_id() { return __builtin_amdgcn_s_getreg(20 | (3 << 11)) & 7u; }
__device__ __forceinline__ void flag_local(char* base, unsigned off, unsigned v) {
    asm volatile("global_store_dword %0, %1, %2" :: "v"(off + (8u + xcc_id()) * FSTR), "v"(v), "s"(base) : "memory");
}
__device__ __forceinline__ unsigned flag_copy(int lane) { return (xcc_id() + (unsigned)(lane >> 5) * 8u) * FSTR; }
__device__ __forceinline__ unsigned b_flags(int cg, int lane) { return flag_copy(lane) + (unsigned)OFF_FB + (unsigned)(cg * 1024 + (lane & 31) * 16); }
constexpr int BFLG = 148480;
__device__ __forceinline__ unsigned unset(unsigned f, unsigned call) {
    const unsigned long long m = __builtin_amdgcn_ballot_w64(f < call);
    return (unsigned)m & (unsigned)(m >> 32);
}
__device__ __forceinline__ void store8(const u4 (&h)[8], char* d0, unsigned half, unsigned vl16) {
    char* d1 = d0 + half;
    GST4(vl16, h[0], d0, 0); GST4(vl16, h[1], d0, 1024); GST4(vl16, h[2], d0, 2048); GST4(vl16, h[3], d0, 3072);
    GST4(vl16, h[4], d1, 0); GST4(vl16, h[5], d1, 1024); GST4(vl16, h[6], d1, 2048); GST4(vl16, h[7], d1, 3072);
}
__device__ __forceinline__ unsigned long long lane_mask(unsigned n) { return n >= 64u ? ~0ull : (1ull << n) - 1ull; }

__device__ __forceinline__ int clamp64(int v) { return v < 0 ? 0 : v > 64 ? 64 : v; }
__device__ __forceinline__ int pos_w(int kt, int g) { const int ch = kt >> 1; return (ch < PCH ? 8 : 9) * ch + 8 * clamp64(ch - SKC) + (kt & 1) * 4 + g; }
__device__ __forceinline__ int pos_x(int rb, int ch) {
    if (rb < 8) return 8 * ch + 8 * clamp64(ch - SKA) + rb;
    if (rb < 16) return 8 * ch + 8 * clamp64(ch + SKA + 1) + rb - 8;
    if (rb == 16) return ch < PCH ? 8 * PCH + ch : 9 * ch + 8 * clamp64(ch - SKC) + 8;
    if (rb >= 24) return 8 * ch + 9 * clamp64(ch + SKC + 1) + rb - 24;
    const int j = (rb - 16) >> 1, r = ch + (j == 0 ? 0 : j == 1 ? SQ1 : j == 2 ? SQ2 : SQ3);
    return 1024 + clamp64(r + (j > 0)) + 2 * (clamp64(r - SQ1 + (j > 1)) + clamp64(r - SQ2 + (j > 2)) + clamp64(r - SQ3)) + (j ? rb & 1 : 0);
}
static_assert(NPROD == 13 && NPC % 7 == 0 && 2 * PK_PG <= NGEMM && UG >= 8 * NGEMM && UG <= 8 * (NGEMM + NB2) && UG + 8 * NPROD <= PK_O && 0 <= SQ1 && SQ1 <= SQ2 && SQ2 <= SQ3 && SKC < 64 && SKA < 64 && PCH <= SKA && PCH <= SKC && SKC >= 8, "skews");
constexpr int NPUSH = 8 * PCH;
__device__ __forceinline__ unsigned panel_off(int panel, unsigned call) {
    return (unsigned)OFF_DATA + (unsigned)((call & 1) && panel < 12 && panel % 3 == 0 ? 17 + panel / 3 : panel) * (unsigned)PANEL_B;
}
__device__ __forceinline__ void place_x(int panel, int rb, int ch, unsigned call, unsigned& off, unsigned& fl) {
    off = panel_off(panel, call) + (unsigned)(ch * 65536 + (rb & 7) * 4096);
    fl = (unsigned)OFF_FA + (unsigned)(panel * 2048 + ch * 32 + (rb & 7) * 4);
}
__device__ __forceinline__ void place_w(int cb, int kt, unsigned& off, unsigned& fl) {
    off = (unsigned)(OFF_DATA + A_B) + (unsigned)(cb >> 1) * (unsigned)CG_B + (unsigned)(kt * 16384 + (cb & 1) * 8192);
    fl = (unsigned)OFF_FB + (unsigned)((cb >> 1) * 1024 + kt * 8 + (cb & 1) * 4);
}
__device__ __forceinline__ int panel_row(int pub, int panel, int blk) {
    const int gs = pub & 4, s = panel / 3, k = panel - 3 * s;
    const int r = panel == 16 ? pub : panel >= 12 ? (gs ^ 4) + panel - 12 : (k == 2 && blk == 0 ? gs ^ 4 : gs) + s;
    return (int)((RMAP >> (4 * r)) & 7u) * 1024 + (panel >= 12 ? 768 : k * 256) + blk * 32;
}
__device__ __forceinline__ size_t pk_off(int pk, unsigned call) { return OFF_PK + (call & 1) * PK_B + (size_t)(pk + pk / 127 + 1) * 16384; }
__device__ __forceinline__ unsigned pk_half(char* myws, int pk, unsigned call) {
    return __hip_atomic_load((GAS unsigned*)(myws + OFF_MAP) + ((call & 1) * PK_PG + pk / 127) * (MAPS / 4), __ATOMIC_RELAXED, __HIP_MEMORY_SCOPE_SYSTEM);
}

#define PERM(hi, lo, sel) __builtin_amdgcn_perm(hi, lo, sel)
#define ABIT(hi, lo, s) __builtin_amdgcn_alignbit(hi, lo, s)
#define HW(k) h[(k) >> 2][(k) & 3]
#define FENCE() asm volatile("" ::: "memory")
__device__ __forceinline__ unsigned bfi(unsigned m, unsigned x, unsigned y) { return (x & m) | (y & ~m); }
__device__ __forceinline__ unsigned wave_scan(unsigned v) {
    v += __builtin_amdgcn_update_dpp(0u, v, 0x111, 0xF, 0xF, true);
    v += __builtin_amdgcn_update_dpp(0u, v, 0x112, 0xF, 0xF, true);
    v += __builtin_amdgcn_update_dpp(0u, v, 0x114, 0xF, 0xF, true);
    v += __builtin_amdgcn_update_dpp(0u, v, 0x118, 0xF, 0xF, true);
    v += __builtin_amdgcn_update_dpp(0u, v, 0x142, 0xA, 0xF, false);
    v += __builtin_amdgcn_update_dpp(0u, v, 0x143, 0xC, 0xF, false);
    return v;
}
__device__ __forceinline__ int wave_sum(unsigned v) { return __builtin_amdgcn_readlane(wave_scan(v), 63); }
__device__ __forceinline__ unsigned escapes(unsigned c, unsigned acc) { return (unsigned)__builtin_popcount(c & (c >> 1) & 0x55555555u) + acc; }
__device__ __forceinline__ void make_lut(LAS char* lut, bool pack, int lane) {
#pragma unroll
    for (int j = 0; j < 4; ++j) {
        unsigned sel = pack ? 0x0C0C0C0Cu : 0u, k = 0;
#pragma unroll
        for (int i = 0; i < 4; ++i) {
            const unsigned c = i < 3 ? ((unsigned)lane >> (2 * i)) & 3u : (unsigned)j;
            if (pack) sel ^= c == 3u ? (0x0Cu ^ (unsigned)i) << (8 * k) : 0u; else sel |= (c == 3u ? k : 4u + c) << (8 * i);
            k += c == 3u;
        }
        *(LAS u2*)(lut + (j * 64 + lane) * 8) = (u2){sel, pack ? 8u * k : k};
    }
}
__device__ __forceinline__ unsigned code4(unsigned thi, unsigned tlo, unsigned s) { return __builtin_amdgcn_udot4(PERM(thi, tlo, s), 0x80200802u, 0u - 0xF3u * 170u, false); }
template <int N> __device__ __forceinline__ void append(const unsigned* s, const u2* e, unsigned pos) {
#pragma unroll
    for (int g = 0; g < N; ++g) {
        const unsigned long long c = (unsigned long long)PERM(0u, s[g], e[g][0]) << (pos & 31u);
        asm volatile("v_cmpx_ne_u32_e32 vcc, 0, %1\n\tds_or_b32 %0, %1\n\tv_cmpx_ne_u32_e32 vcc, 0, %2\n\tds_or_b32 %0, %2 offset:4\n\ts_mov_b64 exec, -1"
                     :: "v"((pos >> 3) & ~3u), "v"((unsigned)c), "v"((unsigned)(c >> 32)) : "vcc", "memory");
        pos += e[g][1];
    }
}
constexpr int SCR_B = 3328;
constexpr int SCR_3 = 2064, SCR_T = 2592, PBUF = 16384 + SCR_B;
template <bool F32>
__device__ __forceinline__ unsigned pack_unit(u4 (&h)[8], LAS char* scr, const LAS char* lut, int lane, unsigned& r0, unsigned& r2) {
    constexpr int N1 = F32 ? 8 : 16, N2 = F32 ? 4 : 8;
    const u4 z = {0u, 0u, 0u, 0u};
    *(LAS u4*)(scr + lane * 16) = z; *(LAS u4*)(scr + 1024 + lane * 16) = z; *(LAS u4*)(scr + 2048 + lane * 16) = z;
    FENCE();
    unsigned G[16], s[16], a[16], cw[4], b3, type;
    if (F32) {
#pragma unroll
        for (int g = 0; g < 8; ++g)
            G[g] = PERM(PERM(HW(4 * g + 3), HW(4 * g + 2), 0x07060302u) >> 7, PERM(HW(4 * g + 1), HW(4 * g), 0x07060302u) >> 7, 0x06040200u);
        unsigned es = 0;
#pragma unroll
        for (int g = 0; g < 4; ++g) es = __builtin_amdgcn_udot4(G[2 * g + (g & 1)], 0x02020202u, es, false);
        const int m = wave_sum(es) + 205;
        b3 = (unsigned)(m >> 11) + 3u; b3 = b3 < 27u ? 27u : b3 > 251u ? 251u : b3;
        type = (m & 2047) > 600;
        unsigned out = 0;
#pragma unroll
        for (int g = 0; g < 8; ++g) { const unsigned d = (b3 + 4u) * 0x01010101u - G[g]; out |= d; s[g] = (2u * d + 0x39393939u) & 0x3F3F3F3Fu; }
        if (__builtin_amdgcn_ballot_w64((out & 0xE0E0E0E0u) != 0) != 0) return M_RAW;
    } else {
#pragma unroll
        for (int g = 0; g < 16; ++g) { G[g] = PERM(HW(2 * g + 1), HW(2 * g), 0x07050301u); s[g] = (G[g] >> 1) & 0x3E3E3E3Eu; }
        const int m = wave_sum(__builtin_amdgcn_udot4(s[0] + s[5] + s[10] + s[15], 0x01010101u, 0u, false)) + 205;
        b3 = ((unsigned)(m >> 11) + 3u) & 31u; type = (m & 2047) > 1120;
#pragma unroll
        for (int g = 0; g < 16; ++g) s[g] = ((2u * b3 + 65u) * 0x01010101u - s[g]) & 0x3F3F3F3Fu;
    }
    u2 e[16];
#pragma unroll
    for (int g = 0; g < N1; ++g) {
        a[g] = code4(0xF300F700u, 0xFB00FF00u, s[g]);
        e[g] = *(const LAS u2*)(lut + a[g]);
    }
    unsigned n = 0;
#pragma unroll
    for (int j = 0; j < N1 / 4; ++j) { cw[j] = (a[4 * j] >> 3) | (a[4 * j + 1] << 5) | (a[4 * j + 2] << 13) | (a[4 * j + 3] << 21); n = escapes(cw[j], n); }
    unsigned sc = wave_scan(n);
    r2 = (__builtin_amdgcn_readlane(sc, 63) + (4u * N2 - 1u)) / (4u * N2);
    if (r2 > (F32 ? 64u : 62u)) return M_RAW;
    append<N1>(s, e, (sc - n + (unsigned)(size_t)scr) * 8u);
    FENCE();
    u4 q[N2 / 4];
#pragma unroll
    for (int j = 0; j < N2 / 4; ++j) q[j] = *(const LAS u4*)(scr + lane * (4 * N2) + 16 * j);
#pragma unroll
    for (int g = 0; g < N2; ++g) {
        const unsigned x = q[g >> 2][g & 3];
        s[g] = (x | ~PERM(0xFF000000u, 0xFF00F9F6u, x)) - 0x06060606u;
        a[g] = code4(type ? 0xFF00F700u : 0xFB00F700u, type ? 0xF300FB00u : 0xF300FF00u, s[g]);
        e[g] = *(const LAS u2*)(lut + a[g]);
    }
    const unsigned c0 = (a[0] >> 3) | (a[1] << 5) | (a[2] << 13) | (a[3] << 21), c1 = F32 ? 0u : (a[4] >> 3) | (a[5] << 5) | (a[6] << 13) | (a[7] << 21);
    n = escapes(c1, escapes(c0, 0u));
    sc = wave_scan(n);
    const unsigned r3 = (__builtin_amdgcn_readlane(sc, 63) + 7u) >> 3;
    if (r3 > 64u) return M_RAW;
    append<N2>(s, e, (sc - n + (unsigned)(size_t)(scr + SCR_3)) * 8u);
    FENCE();
    u2 y = *(const LAS u2*)(scr + SCR_3 + lane * 8), c;
#pragma unroll
    for (int j = 0; j < 2; ++j) {
        y[j] = (y[j] | ~PERM(0xFF000000u, 0xFF00F9F8u, y[j])) - 0x07070707u;
        const unsigned sw = F32 ? ~(((y[j] ^ 0x0E0E0E0Eu) + 0x3F3F3F3Fu) & ((y[j] ^ 0x32323232u) + 0x3F3F3F3Fu)) & 0x40404040u : 0u;
        c[j] = y[j] ^ (sw - (sw >> 4));
    }
    unsigned w = ((c[0] >> 1) & 0x07070707u) | ((c[1] << 2) & 0x38383838u);
    w |= ((w >> 24) * 0x41040u) & 0xC0C0C0u;
    LAS char* t = scr + SCR_T;
    t[4 + lane * 3] = (char)w; t[5 + lane * 3] = (char)(w >> 8); t[6 + lane * 3] = (char)(w >> 16);
    FENCE();
    const unsigned l0 = (3u * r3 + 5u) >> 1;
    unsigned bad = ((c[0] >> 4) & 0x03030303u) | (c[1] & 0x30303030u), nl = 0;
    for (;;) {
        const unsigned long long mk = __builtin_amdgcn_ballot_w64(bad != 0);
        if (mk == 0 || nl > 64u) break;
        if (bad != 0) {
            const unsigned p = (unsigned)__builtin_ctz(bad), k = l0 + nl + __builtin_amdgcn_mbcnt_hi((unsigned)(mk >> 32), __builtin_amdgcn_mbcnt_lo((unsigned)mk, 0u));
            ((LAS unsigned short*)t)[k] = (unsigned short)((((unsigned)lane * 8u + (p & 4u) + (p >> 3)) << 7) | ((4u * b3 - 24u - 2u * (((p & 4u) ? y[1] : y[0]) >> (p & 24u))) & 0x7Cu));
            bad &= ~(15u << (p & 28u));
        }
        nl += (unsigned)__builtin_popcountll(mk);
    }
    r0 = (l0 + nl + 3u) >> 2;
    const unsigned rs = r0 + (F32 ? (r2 + 1u) >> 1 : r2);
    if (nl > 64u || rs > 64u) return M_RAW;
    FENCE();
    u2 f = *(const LAS u2*)(t + lane * 8);
    FENCE();
    if (lane == 0) f[0] = b3 | (r0 << 8) | (type << 15) | (nl << 16) | (l0 << 24);
    if (F32) {
#pragma unroll
        for (int g = 0; g < 8; ++g) {
            const unsigned x = ABIT(HW(4 * g + 3), HW(4 * g + 3), 31);
#pragma unroll
            for (int j = 0; j < 3; ++j) HW(3 * g + j) = PERM(x, ABIT(HW(4 * g + j), HW(4 * g + j), 31), 0x04020100u + ((unsigned)j << 24));
        }
        h[6] = (u4){cw[0], cw[1], f[0], f[1]};
    } else {
        unsigned L[16];
#pragma unroll
        for (int g = 0; g < 16; ++g) L[g] = PERM(HW(2 * g + 1), HW(2 * g), 0x06040200u);
#pragma unroll
        for (int g = 0; g < 16; ++g) HW(g) = L[g];
#pragma unroll
        for (int j = 0; j < 6; ++j) {
            const unsigned x = bfi(0x83838383u, G[2 * j], ABIT(G[2 * j + 1], G[2 * j + 1], 29));
            HW(16 + j) = j < 4 ? bfi(0x60606060u, G[12 + j] << 5, x) : bfi(0x20202020u, G[2 * j + 4] >> 2, bfi(0x40404040u, G[2 * j + 5] >> 1, x));
        }
        h[5][2] = cw[0]; h[5][3] = cw[1]; h[6] = (u4){cw[2], cw[3], f[0], f[1]};
    }
    h[7][0] = c0; h[7][1] = c1;
    return (rs + 7u) >> 3;
}
template <bool F32>
__device__ __forceinline__ void store_unit(const u4 (&h)[8], char* d0, unsigned mode, unsigned r0, unsigned r2, unsigned vl16) {
    char* d1 = d0 + 4096;
    GST4(vl16, h[0], d0, 0); GST4(vl16, h[1], d0, 1024); GST4(vl16, h[2], d0, 2048); GST4(vl16, h[3], d0, 3072);
    GST4(vl16, h[4], d1, 0); GST4(vl16, h[5], d1, 1024);
    if (mode < M_RAW) {
        const u2 p0 = {h[6][0], h[6][1]}, p1 = {h[6][2], h[6][3]}, p2 = {h[7][0], h[7][1]};
        asm volatile("global_store_dwordx2 %0, %1, %3 offset:2048 sc0 sc1\n\t"
                     "s_mov_b64 exec, %4\n\tglobal_store_dwordx2 %0, %2, %3 offset:2560 sc0 sc1\n\ts_mov_b64 exec, -1"
                     :: "v"(vl16 >> 1), "v"(p0), "v"(p1), "s"(d1), "s"(lane_mask(r0)) : "memory");
        if (F32) asm volatile("s_mov_b64 exec, %3\n\tglobal_store_dword %0, %1, %2 offset:2560 sc0 sc1\n\ts_mov_b64 exec, -1"
                              :: "v"(vl16 >> 2), "v"(p2[0]), "s"(d1 + r0 * 8u), "s"(lane_mask(r2)) : "memory");
        else asm volatile("s_mov_b64 exec, %3\n\tglobal_store_dwordx2 %0, %1, %2 offset:2560 sc0 sc1\n\ts_mov_b64 exec, -1"
                          :: "v"(vl16 >> 1), "v"(p2), "s"(d1 + r0 * 8u), "s"(lane_mask(r2)) : "memory");
    } else { GST4(vl16, h[6], d1, 2048); GST4(vl16, h[7], d1, 3072); }
}
template <int N> __device__ __forceinline__ void deposit(unsigned* o, const u2* e, unsigned tab, unsigned pos) {
    u2 x[N]; unsigned p[N];
#pragma unroll
    for (int g = 0; g < N; ++g) {
        const LAS unsigned* d = (const LAS unsigned*)(size_t)(pos & ~3u);
        x[g] = (u2){d[0], d[1]}; p[g] = pos;
        pos += e[g][1];
    }
#pragma unroll
    for (int g = 0; g < N; ++g) o[g] = PERM(tab, __builtin_amdgcn_alignbyte(x[g][1], x[g][0], p[g]), e[g][0]);
}
template <bool F32>
__device__ __forceinline__ void unpack_unit(u4 (&h)[8], LAS char* scr, const LAS char* lut, int lane) {
    constexpr int N1 = F32 ? 8 : 16, N2 = F32 ? 4 : 8;
    LAS char* t = scr + SCR_T;
    *(LAS u2*)(t + lane * 8) = (u2){h[6][2], h[6][3]};
    FENCE();
    const unsigned hd = __builtin_amdgcn_readfirstlane(h[6][2]), r0 = (hd >> 8) & 63u, type = hd & 32768u;
    const unsigned kb = (F32 ? hd & 255u : ((hd & 31u) << 2) | 0x80u) * 0x01010101u;
    auto tab = [&](unsigned t4) { return F32 ? kb - t4 : (kb - 4u * t4) & 0x7C7C7C7Cu; };
    const LAS unsigned* d3 = (const LAS unsigned*)(t + ((4 + lane * 3) & ~3));
    const unsigned w0 = d3[0], w1 = d3[1];
    unsigned le = 0;
    const bool ll = (unsigned)lane < ((hd >> 16) & 127u);
    if (ll) le = ((const LAS unsigned short*)t)[(hd >> 24) + lane];
    const unsigned c0 = F32 ? *(const LAS unsigned*)(t + r0 * 8u + lane * 4) : (unsigned)lane < r0 ? 0u : h[6][2], c1 = F32 || (unsigned)lane < r0 ? 0u : h[6][3];
    const unsigned cw[4] = {F32 ? h[6][0] : h[5][2], F32 ? h[6][1] : h[5][3], F32 ? 0u : h[6][0], F32 ? 0u : h[6][1]};
    u2 e2[N2], e1[N1];
#pragma unroll
    for (int g = 0; g < N2; ++g) e2[g] = *(const LAS u2*)(lut + (((g < 4 ? c0 : c1) >> (8 * (g & 3))) & 255u) * 8u);
#pragma unroll
    for (int g = 0; g < N1; ++g) e1[g] = *(const LAS u2*)(lut + ((cw[g >> 2] >> (8 * (g & 3))) & 255u) * 8u);
    const unsigned n2 = escapes(c1, escapes(c0, 0u)), n1 = escapes(cw[3], escapes(cw[2], escapes(cw[1], escapes(cw[0], 0u))));
    const unsigned n = n1 | (n2 << 16), sc = wave_scan(n) - n;
    unsigned w = __builtin_amdgcn_alignbyte(w1, w0, (unsigned)lane * 3u);
    w = bfi(0x003F3F3Fu, w, __builtin_amdgcn_udot4(w & 0xC0C0C0u, 0x100401u, 0u, false) << 18);
    const unsigned t3l = tab(type ? 0x09080706u : 0x09080700u), t3h = tab(F32 ? 0xFF0C0B0Au : 0x0D0C0B0Au);
    *(LAS u2*)(scr + SCR_3 + lane * 8) = (u2){PERM(t3h, t3l, w & 0x07070707u), PERM(t3h, t3l, (w >> 3) & 0x07070707u)};
    FENCE();
    if (ll) scr[SCR_3 + (le >> 7)] = (char)(F32 ? hd - 27u + (((le >> 2) - hd + 27u) & 31u) : le & 0x7Cu);
    FENCE();
    unsigned E[16];
    deposit<N2>(E, e2, tab(type ? 0x000504u : 0x060504u), (sc >> 16) + (unsigned)(size_t)(scr + SCR_3));
    FENCE();
#pragma unroll
    for (int j = 0; j < N2 / 4; ++j) *(LAS u4*)(scr + lane * (4 * N2) + 16 * j) = (u4){E[4 * j], E[4 * j + 1], E[4 * j + 2], E[4 * j + 3]};
    FENCE();
    deposit<N1>(E, e1, tab(0x010203u), (sc & 0xFFFFu) + (F32 ? 0u : r0 * 32u) + (unsigned)(size_t)scr);
    FENCE();
    if (F32) {
#pragma unroll
        for (int g = 7; g >= 0; --g) {
            const unsigned x = PERM(PERM(E[g], HW(3 * g + 2), 0x07030000u), PERM(HW(3 * g + 1), HW(3 * g), 0x00000703u), 0x07060100u);
            HW(4 * g + 3) = ABIT(x, x, 1);
#pragma unroll
            for (int j = 2; j >= 0; --j) { const unsigned v = PERM(E[g], HW(3 * g + j), 0x04020100u + ((unsigned)j << 24)); HW(4 * g + j) = ABIT(v, v, 1); }
        }
    } else {
        unsigned L[16];
#pragma unroll
        for (int g = 0; g < 16; ++g) {
            L[g] = HW(g);
            const unsigned x = HW(16 + (g < 12 ? g >> 1 : g - 12));
            E[g] = bfi(0x7C7C7C7Cu, E[g], g >= 12 ? bfi(0x80808080u, HW(14 + (g >> 1)) << (2 - (g & 1)), x >> 5) : (g & 1) ? ABIT(x, x, 3) : x);
        }
#pragma unroll
        for (int g = 0; g < 16; ++g) { HW(2 * g) = PERM(E[g], L[g], 0x05010400u); HW(2 * g + 1) = PERM(E[g], L[g], 0x07030602u); }
    }
}

__device__ __forceinline__ void unit_decode(int u, int& kind, int& ch, int& g) {
    constexpr int U0 = 56 * SKC, U1 = U0 + 64 * (64 - SKC);
    if (u < U0) { const int s = u / 7; kind = u - 7 * s; ch = s >> 3; g = s & 7; }
    else if (u < U1) { const int v = u - U0; kind = v & 7; g = (v >> 3) & 7; ch = (v >> 6) + (kind == 7 ? 0 : SKC); }
    else { const int v = u - U1; kind = 7; g = v & 7; ch = 64 - SKC + (v >> 3); }
}
__device__ __forceinline__ void fetch_input(const CAS Params& P, LAS char* buf, int u, int lane) {
    int kind, ch, g;
    unit_decode(u, kind, ch, g);
    const bool isx = kind == 0 || kind > 4;
    const char* src = isx ? (const char*)P.x + (size_t)((unsigned)((kind ? kind - 4 : 0) * 8 + g) * (32 * KDIM * 4) + (unsigned)ch * 512)
                          : (const char*)P.w + (size_t)((unsigned)(2 * ch + (g >> 2)) * (64 * NLOC * 4) + (unsigned)((kind - 1) * 4 + (g & 3)) * 256);
    const unsigned s0 = isx ? 64 : NLOC * 4, s1 = isx ? 16 * KDIM * 4 : 16 * NLOC * 4, s2 = isx ? 256 : 32 * NLOC * 4;
    const unsigned voff = isx ? (unsigned)(lane & 15) * (KDIM * 4) + (unsigned)(lane >> 4) * 16 : (unsigned)(lane >> 4) * (4 * NLOC * 4) + (unsigned)(lane & 15) * 16;
#pragma unroll
    for (int i = 0; i < 16; ++i)
        __builtin_amdgcn_global_load_lds((const GAS void*)(src + ((i & 3) * s0 + ((i >> 2) & 1) * s1 + (i >> 3) * s2) + voff), (LAS void*)(buf + i * 1024), 16, 0, 0);
}
__device__ __forceinline__ void pace(unsigned long long t0, int u) {
    if (u < UG) while (__builtin_amdgcn_s_memrealtime() - t0 < (unsigned)(u * PACE) >> 4) __builtin_amdgcn_s_sleep(2);
}
__device__ __forceinline__ int next_unit(int u, int wid) {
    if (u >= UG) return u + 8 * NPROD;
    const int b = blockIdx.x < 32 ? (int)blockIdx.x : blockIdx.x < 32 + EXN ? NB2 : (int)blockIdx.x - EXN;
    u = u < 8 * NGEMM && b < NB2 ? 8 * NGEMM + wid * NB2 + b : UG;
    return u < UG ? u : PK_O;
}
__device__ __forceinline__ void producer(const CAS Params& P, char* myws, LAS char* buf, LAS char* lut, int u, unsigned long long t0, unsigned call, int wid, int lane) {
    const int q = P.rank & 3;
    const unsigned vl16 = (unsigned)lane * 16;
    LAS char* scr = buf + 16384;
#define PT(i) do { } while (0)
    for (int un; u < PK_O; u = un) {
        un = next_unit(u, wid);
        int ch, g, kind;
        unit_decode(u, kind, ch, g);
        const bool isx = kind == 0 || kind > 4;
        const int rb = (kind ? kind - 4 : 0) * 8 + g, kt = 2 * ch + (g >> 2), cb = (kind - 1) * 4 + (g & 3);
        const int idx = isx ? pos_x(rb, ch) : pos_w(kt, g & 3), pk = isx ? ch * 32 + rb : PK_W + (kind - 1) * 512 + ch * 8 + g;
        const int rd = !isx ? kind - 1 : rb == 16 || rb >= 24 ? 5 : 4;
        f4 v[16]; u4 h[8];
        unsigned hv;
        asm volatile("global_load_dword %0, %1, %2 sc0 sc1" : "=&v"(hv) : "v"((unsigned)OFF_MAP + ((call & 1) * PK_PG + pk / 127) * MAPS), "s"(myws) : "memory");
#pragma unroll
        for (int i = 0; i < 16; ++i) v[i] = *(const LAS f4*)(buf + i * 1024 + lane * 16);
        __builtin_amdgcn_sched_barrier(0);
        WAIT_L0();
        const bool ahead = u >= UG && un < PK_O;
        PT(0);
        if (ahead) { pace(t0, un); fetch_input(P, buf, un, lane); }
        PT(1);
        __builtin_amdgcn_sched_barrier(0);
        if (isx) {
#pragma unroll
            for (int i = 0; i < 8; ++i)
                h[i] = pack8(v[2 * i][0], v[2 * i][1], v[2 * i][2], v[2 * i][3], v[2 * i + 1][0], v[2 * i + 1][1], v[2 * i + 1][2], v[2 * i + 1][3]);
        } else {
#pragma unroll
            for (int i = 0; i < 8; ++i) {
                const int f = i >> 1, k = (i & 1) * 8;
                h[i] = pack8(v[k][f], v[k + 1][f], v[k + 2][f], v[k + 3][f], v[k + 4][f], v[k + 5][f], v[k + 6][f], v[k + 7][f]);
            }
        }
        unsigned off, fl;
        if (isx) place_x(rb < 24 ? 3 * q + (rb >> 3) : 16, rb, ch, call, off, fl); else place_w(cb, kt, off, fl);
        const bool own = !isx || rb != 16;
        if (idx < NPUSH) {
            unsigned off2 = off, fl2 = fl;
            if (!isx) place_w(16 + 4 * q + (g & 3), kt, off2, fl2);
            while (__builtin_amdgcn_s_memrealtime() - t0 < (unsigned)(PT0 + idx * PDT)) __builtin_amdgcn_s_sleep(1);
            char* fp = myws + fl;
            for (int i = 1; i < (isx ? 4 : 2); ++i) {
                char* pw = P.ws[isx ? (P.rank & 4) + ((q + i) & 3) : ((P.rank & 4) ^ 4) + rd];
                store8(h, pw + off2, isx ? 32768u : 4096u, vl16);
                fp = lane >> 3 == i ? pw + fl2 : fp;
            }
            store8(h, myws + off, isx ? 32768u : 4096u, vl16);
            asm volatile("s_waitcnt vmcnt(0)" : "+v"(hv) :: "memory");
            if (lane == (int)xcc_id()) flag_local(myws, fl, call);
            else if (lane < (isx ? 32 : 16)) flag_store(fp + (lane & 7) * FSTR, call);
            if (un < PK_O) { pace(t0, un); fetch_input(P, buf, un, lane); WAIT_V(0); }
            continue;
        }
        if (own) store8(h, myws + off, isx ? 32768u : 4096u, vl16);
        unsigned mode = M_RAW, r0 = 0, r2 = 0;
        PT(2);
        if (idx >= RAW0) mode = pack_unit<false>(h, scr, lut, lane, r0, r2);
        PT(3);
        if (own) asm volatile("s_waitcnt vmcnt(8)" : "+v"(hv) :: "memory"); else asm volatile("s_waitcnt vmcnt(0)" : "+v"(hv) :: "memory");
        PT(4);
        const unsigned half = (__builtin_amdgcn_readfirstlane(hv) ^ (rd != 4)) & 1u;
        store_unit<false>(h, myws + (pk_off(pk, call) + half * 8192), mode, r0, r2, vl16);
        const unsigned pfo = lane < 5 ? (unsigned)OFF_FP + (call & 1) * (unsigned)FP_B + (unsigned)(lane * NSEQ + idx) * 4u : fl + (lane & 7) * FSTR;
        WAIT_V(0);
        if (own && lane == 8 + (int)xcc_id()) flag_local(myws, fl, call);
        else if ((rd == 5 ? lane < 4 : lane == rd) || (own && lane >> 3 == 1)) flag_store(myws + pfo, lane < 5 ? call * 32u + half * 16u + mode : call);
        if (!ahead && un < PK_O) { pace(t0, un); fetch_input(P, buf, un, lane); WAIT_V(0); }
        PT(5);
    }
}

__device__ __forceinline__ void flag_mirror(char* myws, const char* fp, int nseq, LAS unsigned* info, LAS unsigned* prog, unsigned call, int lane) {
    GAS unsigned* tmo = (GAS unsigned*)(myws + OFF_TMO);
    int pos = NPUSH;
    bool all = false;
    unsigned long long t0 = __builtin_amdgcn_s_memrealtime();
    while (pos < nseq) {
        const int pr = (int)__builtin_amdgcn_readfirstlane(__hip_atomic_load(prog, __ATOMIC_RELAXED, __HIP_MEMORY_SCOPE_WORKGROUP));
        if (pos <= pr + (all ? LOOK : LOOK2)) {
            const int lo = pos & ~31, hi = all ? lo + 32 * MALL : ((pr + LOOK2) | 31) + 1, i = lo + 4 * lane;
            bool bad = i >= hi;
            if (!bad && i < nseq) {
                u4 f = *(const LAS u4*)(info + i);
                if (i >= pos && (f[0] < call * 32u || f[1] < call * 32u || f[2] < call * 32u || f[3] < call * 32u)) {
                    asm volatile("global_load_dwordx4 %0, %1, %2 sc0 sc1\n\ts_waitcnt vmcnt(0)" : "=&v"(f) : "v"(i * 4), "s"(fp) : "memory");
                    *(LAS u4*)(info + i) = f;
                    bad = f[0] < call * 32u || f[1] < call * 32u || f[2] < call * 32u || f[3] < call * 32u;
                }
            }
            const unsigned long long m = __builtin_amdgcn_ballot_w64(bad);
            const int n = lo + 4 * (m ? __builtin_ctzll(m) : 64);
            all = n >= hi;
            if (n > pos) { pos = n; t0 = __builtin_amdgcn_s_memrealtime(); if (all) continue; }
        }
        if (
            __builtin_amdgcn_s_memrealtime() - t0 > DEADLINE) {
            __hip_atomic_store(tmo, call, __ATOMIC_RELAXED, __HIP_MEMORY_SCOPE_AGENT);
            asm volatile("s_waitcnt vmcnt(0)" ::: "memory");
            for (int i = pos + lane; i < nseq; i += 64) info[i] = call * 32u + M_DEAD;
            return;
        }
        if (pos > pr) __builtin_amdgcn_s_sleep(MSL);
    }
}

#define STR2(x) #x
#define STR(x) STR2(x)
#define SP ""
#define RD ""
#define PULL6 "global_load_dwordx4 %[h0], %[v16], %[src] offset:-4096 sc0 sc1\n\t" SP "global_load_dwordx4 %[h1], %[v16], %[src] offset:-3072 sc0 sc1\n\t" SP \
              "global_load_dwordx4 %[h2], %[v16], %[src] offset:-2048 sc0 sc1\n\t" SP "global_load_dwordx4 %[h3], %[v16], %[src] offset:-1024 sc0 sc1\n\t" SP \
              "global_load_dwordx4 %[h4], %[v16], %[src] sc0 sc1\n\t" SP "global_load_dwordx4 %[h5], %[v16], %[src] offset:1024 sc0 sc1\n\t" SP
#define PULLT "s_mov_b64 exec, 1\n\tglobal_atomic_add %[tk], %[ao], %[one], %[ws] sc0\n\t" \
              "s_waitcnt vmcnt(%[rn] + 9 - " STR(RELAT) ")\n\ts_mov_b32 exec_lo, %[m1]\n\tglobal_store_dword %[pf], %[cl], %[ws] sc0 sc1\n\t" \
              "s_mov_b32 exec_lo, %[m2]\n\tglobal_store_dword %[pf], %[cl], %[ws]\n\ts_waitcnt vmcnt(%[rn] + 2)\n\t" RD \
              "ds_add_u32 %[rel], %[one]\n\ts_mov_b64 exec, -1\n\ts_waitcnt vmcnt(2)"
#define PULLO [h0] "=&v"(h[0]), [h1] "=&v"(h[1]), [h2] "=&v"(h[2]), [h3] "=&v"(h[3]), [h4] "=&v"(h[4]), [h5] "=&v"(h[5]), [tk] "=&v"(tk)
#define PULLI [v16] "v"(vl16), [src] "s"(src), [ao] "v"(aoff), [one] "v"(one), [ws] "s"(myws), [rel] "v"(rel), [pf] "v"(pfl + fcop), [cl] "v"(call), [m1] "s"(m1), [m2] "s"(m2)
__device__ __forceinline__ void puller(int vg, const CAS Params& P, char* myws, LAS unsigned* info, LAS unsigned* prog, const LAS unsigned short* tab,
                                       LAS char* scr, const LAS char* lut, int link, int peer, int cur, int nw, unsigned win, unsigned call, int wid, int lane) {
    const int q = P.rank & 3, qs = peer & 3;
    const bool cross = link < 4;
    const int nseq = cross ? NSEQ : NI;
    const char* pk = P.ws[peer] + 4096;
    const unsigned vl16 = (unsigned)lane * 16, one = 1,
        aoff = (unsigned)OFF_CTR + (unsigned)link * 128u;
    const unsigned ovoff = (unsigned)(lane >> 4) * (4 * NLOC * 4) + (unsigned)(lane & 15) * 16;
    const unsigned base = (call - 1u) * (unsigned)(nseq - NPUSH) - (unsigned)(nw + NPUSH);
    const unsigned rel = (unsigned)(size_t)(prog + 2);
    const unsigned nofl = (unsigned)OFF_FN + (unsigned)(vg * 128 + wid * 4);
    const unsigned m2 = 1u << xcc_id(), fcop = ((lane & 7) + (lane == (int)xcc_id() ? 8u : 0u)) * FSTR;
    auto flags = [&](unsigned fl) {
        asm volatile("s_mov_b64 exec, %3\n\tglobal_store_dword %0, %1, %2 sc0 sc1\n\ts_mov_b32 exec_lo, %4\n\tglobal_store_dword %0, %1, %2\n\ts_mov_b64 exec, -1"
                     :: "v"(fl + fcop), "v"(call), "s"(myws), "s"((unsigned long long)(m2 ^ 255u)), "s"(m2) : "memory");
    };
    unsigned pfl = nofl;
    const unsigned tz = __builtin_amdgcn_readfirstlane(prog[4]);
    for (;;) {
        const bool isout = cross && cur >= NCI;
        const int id = isout ? 0 : __builtin_amdgcn_readfirstlane((int)tab[cur]);
        const int slot = isout ? PK_O + q * NO + cur - NCI : id < PK_W ? id : id + q * 512;
        unsigned mode;
        if (pfl != nofl) {
            const unsigned i0 = __hip_atomic_load(info + cur, __ATOMIC_RELAXED, __HIP_MEMORY_SCOPE_WORKGROUP);
            const unsigned i1 = __hip_atomic_load(prog + 1, __ATOMIC_RELAXED, __HIP_MEMORY_SCOPE_WORKGROUP), i2 = __hip_atomic_load(prog + 2, __ATOMIC_RELAXED, __HIP_MEMORY_SCOPE_WORKGROUP);
            if (__builtin_amdgcn_readfirstlane(i0 - call * 32u > 31u || i1 - i2 >= win)) {
                WAIT_V(0);
                flags(pfl);
                pfl = nofl;
            }
        }
        for (;;) {
            mode = __builtin_amdgcn_readfirstlane(__hip_atomic_load(info + cur, __ATOMIC_ACQUIRE, __HIP_MEMORY_SCOPE_WORKGROUP)) - call * 32u;
            if (mode <= 31u) break;
            __builtin_amdgcn_s_sleep(1);
        }
        const char* src = pk + (pk_off(slot, call) + (mode & 16u) * 512u);
        mode &= 15u;
        const unsigned long long mask = lane_mask(mode * 8u);
        unsigned my = 0;
        if (lane == 0) my = __hip_atomic_fetch_add(prog + 1, 1u, __ATOMIC_RELAXED, __HIP_MEMORY_SCOPE_WORKGROUP);
        my = __builtin_amdgcn_readfirstlane(my) - win - (cur < HWIN);
        while ((int)(my - __builtin_amdgcn_readfirstlane(__hip_atomic_load(prog + 2, __ATOMIC_RELAXED, __HIP_MEMORY_SCOPE_WORKGROUP))) >= 0) { }
        {
            unsigned ts = 0;
            if (lane == 0) {
                __hip_atomic_fetch_max(prog + 3, ((unsigned)__builtin_amdgcn_s_memrealtime() - tz) << 8, __ATOMIC_RELAXED, __HIP_MEMORY_SCOPE_WORKGROUP);
                ts = __hip_atomic_fetch_add(prog + 3, (mode >= M_RAW ? 128u : 104u + ((mode + 1u) & ~1u)) * PRATE, __ATOMIC_RELAXED, __HIP_MEMORY_SCOPE_WORKGROUP);
            }
            ts = __builtin_amdgcn_readfirstlane(ts);
            ts = (ts >> 8) + tz;
            while ((int)((unsigned)__builtin_amdgcn_s_memrealtime() - ts) < 0) { }
        }
        u4 h[8]; unsigned tk;
        const unsigned m1 = pfl != nofl ? m2 ^ 255u : 256u;
        if (mode >= M_RAW) {
            asm volatile(PULL6 "global_load_dwordx4 %[h6], %[v16], %[src] offset:2048 sc0 sc1\n\tglobal_load_dwordx4 %[h7], %[v16], %[src] offset:3072 sc0 sc1\n\t" PULLT
                         : PULLO, [h6] "=&v"(h[6]), [h7] "=&v"(h[7]) : PULLI, [rn] "n"(RELAT) : "memory");
        } else {
            u2 p0, p1;
            asm volatile(PULL6 "global_load_dwordx2 %[p0], %[v8], %[src] offset:2048 sc0 sc1\n\t"
                         "s_mov_b64 exec, %[mask]\n\tglobal_load_dwordx2 %[p1], %[v8], %[src] offset:2560 sc0 sc1\n\t" PULLT
                         : PULLO, [p0] "=&v"(p0), [p1] "=&v"(p1)
                         : PULLI, [v8] "v"(vl16 >> 1), [mask] "s"(mask), [rn] "n"(RELAT) : "memory");
            h[6] = (u4){p0[0], p0[1], p1[0], p1[1]};
        }
        const unsigned nxt = __builtin_amdgcn_readfirstlane(tk) - base;
        if (lane == 0) __hip_atomic_fetch_max(prog, nxt, __ATOMIC_RELAXED, __HIP_MEMORY_SCOPE_WORKGROUP);
        if (isout) {
            const int o = cur - NCI;
            int panel, blk, col;
            if (o < O_B) {
                const int r = o < O_A0 ? o : o < O_C ? (o - O_A0) & 127 : o - O_C, hn = o < O_A0 ? 16 : o < O_C ? 64 : 48;
                const int hf = r >= hn, t = (r - hf * hn) >> 3;
                panel = o < O_A0 ? 3 * qs + 2 : o < O_C ? 3 * (t >> 1) + (o >= O_A1) : 12 + ((q + 1 + (t >> 1)) & 3);
                blk = (r & 6) + hf; col = (t & 1) * 128 + (r & 1) * 64;
            } else {
                const int j = (o - O_B) / 24, r = o - O_B - j * 24;
                panel = 3 * ((qs + 1 + (r >> 3)) & 3) + 2; blk = 2 * j + ((r >> 1) & 1); col = ((r >> 2) & 1) * 128 + (r & 1) * 64;
            }
            char* ob = (char*)P.out + (size_t)(panel_row(peer, panel, blk) * NLOC + qs * 256 + col) * 4;
            if (mode < M_RAW) unpack_unit<true>(h, scr, lut, lane);
            if (mode == M_DEAD) h[0] = (u4){0x7FC00000u, 0x7FC00000u, 0x7FC00000u, 0x7FC00000u};
#pragma unroll
            for (int k = 0; k < 8; ++k) GST4P(ovoff, h[k], ob + ((k >> 2) * 16 + (k & 3)) * (NLOC * 4));
            pfl = nofl;
        } else {
            if (mode < M_RAW) unpack_unit<false>(h, scr, lut, lane);
            unsigned off, fl;
            const int rb = id & 31;
            if (id < PK_W) place_x(rb < 24 ? 3 * qs + (rb >> 3) : 12 + qs, rb, id >> 5, call, off, fl);
            else place_w(16 + 4 * qs + (id & 3), (id - PK_W) >> 2, off, fl);
            store8(h, myws + off, id < PK_W ? 32768u : 4096u, vl16);
            pfl = fl;
        }
        if (nxt >= (unsigned)nseq) break;
        cur = (int)nxt;
    }
    WAIT_V(0);
    if (pfl != nofl) flags(pfl);
}

__device__ __forceinline__ float silu(float y) {
    const float c = __builtin_amdgcn_fmed3f(y, -60.0f, 60.0f);
    const float e = __builtin_amdgcn_exp2f(c * -1.4426950408889634f);
    return y * __builtin_amdgcn_rcpf(1.0f + e);
}
__device__ __forceinline__ u4 silu4(const f4 (&a)[4], int r, bool dead) {
    f4 o = {silu(a[0][r]), silu(a[1][r]), silu(a[2][r]), silu(a[3][r])};
    const float nanv = __builtin_nanf("");
    if (dead) o = (f4){nanv, nanv, nanv, nanv};
    return __builtin_bit_cast(u4, o);
}

#define BAR() do { __builtin_amdgcn_sched_barrier(0); __builtin_amdgcn_s_barrier(); asm volatile("" ::: "memory"); __builtin_amdgcn_sched_barrier(0); } while (0)
#define GLDS(g, l, imm) __builtin_amdgcn_global_load_lds((const GAS void*)(g), (LAS void*)(l), 16, imm, AUX)
template <int V> struct IC { static constexpr int value = V; };

__device__ __forceinline__ void gemm_panel(const CAS Params& P, LAS char* lds, char* myws, int panel, int cg, int cg2, int o0, int ohalf,
                                           unsigned call, int wid, int lane) {
    constexpr int ABYTES = 32768, STAGE = ABYTES + 16384;
    const char* ga = myws + panel_off(panel, call) + wid * 4096 + lane * 16;
    const char* gb = myws + OFF_DATA + A_B + (size_t)cg * CG_B + wid * 2048 + lane * 16;
    const int wm = wid >> 1, wn = wid & 1;
    const unsigned frow = flag_copy(lane) + ((lane & 2) ? (unsigned)OFF_FB + (unsigned)cg * 1024u : (unsigned)OFF_FA + (unsigned)panel * 2048u + (unsigned)(lane & 1) * 16u);
    const int fch = lane >> 2 & 7;
    const unsigned fstep = (lane & 2) ? 16u : 32u;
    GAS unsigned* tmo = (GAS unsigned*)(myws + OFF_TMO);
    char* oflag = myws + (OFF_FP + (call & 1) * FP_B) + (((cg - 8) >> 1) * NSEQ + NCI + o0 + wid) * 4;
    int ready = 0;

    auto early = [&](int c) {
        if (c < ready || c >= NCH) return;
        const int cc = c + fch;
        __builtin_amdgcn_global_load_lds((const GAS void*)(myws + (frow + (unsigned)(cc < NCH ? cc : NCH - 1) * fstep)), (LAS void*)(lds + 3 * STAGE), 16, 0, AUX);
    };
    auto ensure = [&](int c, bool pend) {
        if (c < ready) return;
        const unsigned long long t0 = __builtin_amdgcn_s_memrealtime();
        int lim = 7;
        for (;;) {
            int cc = c + (fch < lim ? fch : lim); cc = cc < NCH ? cc : NCH - 1;
            u4 f;
            if (pend) asm volatile("s_waitcnt vmcnt(%2)\n\tds_read_b128 %0, %1\n\ts_waitcnt lgkmcnt(0)"
                                   : "=&v"(f) : "v"((unsigned)(size_t)(lds + 3 * STAGE) + (unsigned)lane * 16u), "n"(3 * PD) : "memory");
            else asm volatile("global_load_dwordx4 %0, %1, %2 sc0 sc1\n\ts_waitcnt vmcnt(0)"
                              : "=&v"(f) : "v"(frow + (unsigned)cc * fstep), "s"(myws) : "memory");
            const unsigned m = unset(f[0], call) | unset(f[1], call) | unset(f[2], call) | unset(f[3], call);
            const int n = m ? (__builtin_ctz(m) >> 2) : lim + 1;
            if (n > 0) { ready = c + n; return; }
            if (pend) { pend = false; continue; }
            if (
                __builtin_amdgcn_s_memrealtime() - t0 > DEADLINE) {
                __hip_atomic_store(tmo, call, __ATOMIC_RELAXED, __HIP_MEMORY_SCOPE_AGENT);
                ready = NCH; return;
            }
            lim = 0;
            __builtin_amdgcn_s_sleep(6);
        }
    };
    auto stage = [&](int t, int kf, int bufoff) {
        const char* a = ga + (size_t)t * 32768 + kf * 1024; const char* b = gb + (size_t)t * 16384 + kf * 1024;
        LAS char* la = lds + bufoff + wid * 4096 + kf * 1024; LAS char* lb = lds + bufoff + ABYTES + wid * 2048 + kf * 1024;
        GLDS(a, la, 0); GLDS(a, la, 2048); GLDS(b, lb, 0);
    };

    f4 acc[4][4];
#pragma unroll
    for (int m = 0; m < 4; ++m)
#pragma unroll
        for (int n = 0; n < 4; ++n) acc[m][n] = (f4){0.f, 0.f, 0.f, 0.f};

    const int aoff = wm * 8192 + lane * 16, boff = ABYTES + wn * 8192 + lane * 16;
    auto phase = [&](auto waitn, int rbuf, int kf, int lt, int lkf, int lbuf, int pc, int pe = NCH) {
        const LAS char* pa = lds + rbuf + aoff + kf * 1024; const LAS char* pb = lds + rbuf + boff + kf * 1024;
        h8 a[4], b[4];
#pragma unroll
        for (int n = 0; n < 4; ++n) b[n] = *(const LAS h8*)(pb + n * 2048);
#pragma unroll
        for (int m = 0; m < 4; ++m) a[m] = *(const LAS h8*)(pa + m * 2048);
        __builtin_amdgcn_sched_barrier(0);
        if (lt >= 0) stage(lt, lkf, lbuf);
        if (decltype(waitn)::value >= 0) WAIT_V(decltype(waitn)::value < 0 ? 0 : decltype(waitn)::value);
        WAIT_L0();
        BAR();
        __builtin_amdgcn_s_setprio(1);
#pragma unroll
        for (int m = 0; m < 4; ++m)
#pragma unroll
            for (int n = 0; n < 4; ++n)
                asm volatile("v_mfma_f32_16x16x32_f16 %0, %1, %2, %0" : "+v"(acc[m][n]) : "v"(a[m]), "v"(b[n]));
        __builtin_amdgcn_s_setprio(0);
        __builtin_amdgcn_sched_barrier(0);
        if (wid == 0) { if (pc >= 0) ensure(pc, PD > 0); early(pe); }
        BAR();
    };

    if (wid == 0) { ensure(0, false); ensure(1, false); }
    BAR();
    stage(0, 0, 0); stage(0, 1, 0); stage(1, 0, STAGE); stage(1, 1, STAGE); stage(2, 0, 2 * STAGE);
    WAIT_V(12);
    BAR();
    if (wid >= 4) BAR();
    int rb = 0, pb = 2 * STAGE;
    for (int t = 0; t < NKT - 3; ++t) {
        phase(IC<12>{}, rb, 0, t + 2, 1, pb, (t & 1) ? (t + 3) >> 1 : -1, PD == 2 && !(t & 1) ? (t + 4) >> 1 : NCH);
        phase(IC<12>{}, rb, 1, t + 3, 0, rb, -1, PD == 1 && !(t & 1) ? (t + 4) >> 1 : NCH);
        pb = rb; rb = rb == 2 * STAGE ? 0 : rb + STAGE;
    }
    static_assert((NKT - 3) % 3 == 2, "tail buffers");
    phase(IC<12>{}, 2 * STAGE, 0, NKT - 1, 1, STAGE, -1);
    if (wid == 4) { const char* f = myws + b_flags(cg2, lane); GLDS(f, lds + BFLG, 0); GLDS(f + 512, lds + BFLG + 1024, 0); }
    FENCE();
    const int pk0 = PK_O + ((cg < 8 ? 0 : cg - 8) >> 1) * NO + o0 + wid;
    const unsigned hv[2] = {pk_half(myws, pk0, call), pk_half(myws, pk0 + ohalf, call)};
    const unsigned tm = __hip_atomic_load(tmo, __ATOMIC_RELAXED, __HIP_MEMORY_SCOPE_AGENT);
    FENCE();
    phase(IC<9 + 3>{}, 2 * STAGE, 1, -1, 0, 0, -1);
    phase(IC<6 + 3>{}, 0, 0, -1, 0, 0, -1);
    phase(IC<3 + 3>{}, 0, 1, -1, 0, 0, -1);
    phase(IC<0 + 3>{}, STAGE, 0, -1, 0, 0, -1);
    phase(IC<-1>{}, STAGE, 1, -1, 0, 0, -1);
#pragma unroll
    for (int m = 0; m < 4; ++m) asm volatile("s_nop 7" : "+v"(acc[m][0]), "+v"(acc[m][1]), "+v"(acc[m][2]), "+v"(acc[m][3]));
    if (wid < 4) BAR();

    const bool dead = __builtin_amdgcn_readfirstlane(tm) == call;
    const unsigned vl16 = (unsigned)lane * 16, ovoff = (unsigned)(lane >> 4) * (4 * NLOC * 4) + (unsigned)(lane & 15) * 16;
    unsigned mode = 0;
#pragma unroll
    for (int u = 0; u < 2; ++u) {
        u4 h[8];
#pragma unroll
        for (int k = 0; k < 8; ++k) h[k] = silu4(acc[2 * u + (k >> 2)], k & 3, dead);
        if (cg < 8) {
            char* ob = (char*)P.out + (size_t)(panel_row(P.rank, panel, 2 * wm + u) * NLOC + cg * 128 + wn * 64) * 4;
#pragma unroll
            for (int k = 0; k < 8; ++k) GST4P(ovoff, h[k], ob + ((k >> 2) * 16 + (k & 3)) * (NLOC * 4));
        } else {
            const unsigned prev = mode;
            unsigned r0 = 0, r2 = 0;
            mode = pack_unit<true>(h, lds + wid * 4096, lds + (8 * PBUF + 16), lane, r0, r2);
            if (u) { WAIT_V(0); if (lane == 0) flag_store(oflag, call * 32u + prev); }
            const unsigned half = ~__builtin_amdgcn_readfirstlane(hv[u]) & 1u;
            store_unit<true>(h, myws + (pk_off(pk0 + u * ohalf, call) + half * 8192), mode, r0, r2, vl16);
            mode += half * 16u;
        }
    }
    if (cg >= 8) {
        WAIT_V(0);
        if (lane == 0) flag_store(oflag + ohalf * 4, call * 32u + mode);
    }
}

__device__ __forceinline__ bool all_set(u4 f, u4 e, unsigned call) {
    unsigned m = 0;
#pragma unroll
    for (int i = 0; i < 4; ++i) m |= unset(f[i], call) | unset(e[i], call);
    return m == 0;
}
__device__ __forceinline__ bool all_b(const char* myws, int cg, unsigned call, int lane) {
    u4 f, e;
    asm volatile("global_load_dwordx4 %0, %2, %3 sc0 sc1\n\tglobal_load_dwordx4 %1, %2, %3 offset:512 sc0 sc1\n\ts_waitcnt vmcnt(0)"
                 : "=&v"(f), "=&v"(e) : "v"(b_flags(cg, lane)), "s"(myws) : "memory");
    return all_set(f, e, call);
}

#define QPK 3
#define QSL1 1
#define NPOLL 2
#define QAHEAD 12
template <int KS>
__device__ __forceinline__ void gemm_ring(const CAS Params& P, LAS char* lds, char* myws, int panel, int qt, int cg, int hb, int o0, bool pre,
                                          unsigned call, int wid, int lane) {
    volatile LAS unsigned* ctl = (volatile LAS unsigned*)(lds + 160768);
    auto put = [&](int i, unsigned v) {
        asm volatile("s_mov_b64 exec, 1\n\tds_write_b32 %0, %1\n\ts_mov_b64 exec, -1" :: "v"((unsigned)(size_t)(ctl + i)), "v"(v) : "memory");
    };
    GAS unsigned* tmo = (GAS unsigned*)(myws + OFF_TMO);
    auto late = [&](unsigned long long t0) {
        if (__builtin_amdgcn_s_memrealtime() - t0 <= DEADLINE) return false;
        __hip_atomic_store(tmo, call, __ATOMIC_RELAXED, __HIP_MEMORY_SCOPE_AGENT);
        WAIT_V(0);
        return true;
    };
    const int pk0 = PK_O + ((cg < 8 ? 0 : cg - 8) >> 1) * NO;
    if (wid == 4) {
        if (lane < 8) ctl[lane] = 0;
        const unsigned long long t0 = __builtin_amdgcn_s_memrealtime();
        if (!pre || !all_set(*(const LAS u4*)(lds + BFLG + lane * 16), *(const LAS u4*)(lds + BFLG + 1024 + lane * 16), call)) for (;;) {
            if (all_b(myws, cg, call, lane) || late(t0)) break;
            __builtin_amdgcn_s_sleep(6);
        }
    }
    WAIT_L0();
    BAR();
    const unsigned vl16 = (unsigned)lane * 16;
    if (wid >= 6) {
        const unsigned frow = flag_copy(lane) + (unsigned)OFF_FA + (unsigned)(panel * 2048 + qt * 8 + (lane & 1) * 4);
        if (wid == 7) put(10, pk_half(myws, pk0 + o0, call));
        unsigned long long t0 = __builtin_amdgcn_s_memrealtime();
        unsigned f;
        for (;;) {
            const u4 st = *(const volatile LAS u4*)ctl;
            const int c = __builtin_amdgcn_readfirstlane(st[2]);
            if (c >= NCH || wid >= 6 + NPOLL) break;
            if (c > (int)__builtin_amdgcn_readfirstlane(st[0]) + QAHEAD) { __builtin_amdgcn_s_sleep(4); t0 = __builtin_amdgcn_s_memrealtime(); continue; }
            asm volatile("global_load_dword %0, %1, %2 sc0 sc1\n\ts_waitcnt vmcnt(0)" : "=&v"(f) : "v"(frow + (unsigned)min(c + (lane >> 1 & 7), NCH - 1) * 32u), "s"(myws) : "memory");
            int n = __builtin_ctz(unset(f, call) | 65536u) >> 1;
            if (n) t0 = __builtin_amdgcn_s_memrealtime();
            else if (late(t0)) n = NCH;
            if (n) asm volatile("s_mov_b64 exec, 1\n\tds_max_u32 %0, %1\n\ts_mov_b64 exec, -1" :: "v"((unsigned)(size_t)(ctl + 2)), "v"(c + n) : "memory");
            else __builtin_amdgcn_s_sleep(QSLEEP);
        }
        asm volatile("global_load_dword %0, %1, %2 sc0 sc1\n\ts_waitcnt vmcnt(0)" : "=&v"(f) : "v"((unsigned)OFF_TMO), "s"(myws) : "memory");
        put(2 + wid, f);
        WAIT_L0();
        BAR();
    } else if (wid >= 4) {
        const char* g = myws + OFF_DATA + ((size_t)panel * PANEL_B + qt * 8192 + (wid - 4) * 32768) + vl16;
        int c = 0, pub = 0;
        while (pub < NCH) {
            const u4 fr = *(const volatile LAS u4*)(ctl + 4);
            const int lim = min((int)__builtin_amdgcn_readfirstlane(min(min(min(fr[0], fr[1]), min(fr[2], fr[3])) + 8u, ctl[2])), min(pub + 7, NCH));
            const bool go = c < lim;
            for (; c < lim; ++c, g += 65536) {
                LAS char* l = lds + (c & 7) * 16384 + (wid - 4) * 8192;
                GLDS(g, l, 0); GLDS(g, l, 1024); GLDS(g, l, 2048); GLDS(g, l, 3072);
                GLDS(g + 4096, l + 4096, 0); GLDS(g + 4096, l + 4096, 1024); GLDS(g + 4096, l + 4096, 2048); GLDS(g + 4096, l + 4096, 3072);
            }
            FENCE();
            const unsigned st = __builtin_amdgcn_s_getreg(7 | (31 << 11));
            const int k = (int)(((st & 15u) | (st >> 18 & 48u)) + 7u) >> 3;
            if (c - k > pub) {
                if (k < 4) { if (k < 2) { if (k == 0) WAIT_V(0); else WAIT_V(8); } else { if (k == 2) WAIT_V(16); else WAIT_V(24); } }
                else if (k == 4) WAIT_V(32); else if (k == 5) WAIT_V(40); else WAIT_V(48);
                pub = c - k;
                put(wid - 4, (unsigned)pub);
            } else if (!go) __builtin_amdgcn_s_sleep(QSL1);
        }
        WAIT_L0();
        BAR();
    } else {
        const char* gb = myws + OFF_DATA + (A_B + (size_t)cg * CG_B + (KS ? (wid >> 1) * 16384 + hb * 8192 + (wid & 1) * 4096 : wid * 4096));
        f4 acc[4][2];
#pragma unroll
        for (int i = 0; i < 8; ++i) acc[i >> 1][i & 1] = (f4){0.f, 0.f, 0.f, 0.f};
        asm volatile("" ::: "a0", "a127");
        unsigned long long em = ~0ull;
        auto loadb = [&](auto sl, auto kk) {
            constexpr int r = decltype(sl)::value * 16 + decltype(kk)::value * 4, o = decltype(kk)::value * 1024;
            asm volatile("s_mov_b64 exec, %2\n\tglobal_load_dwordx4 a[%3:%4], %0, %1 offset:%7 sc0 sc1\n\t"
                         "global_load_dwordx4 a[%5:%6], %0, %1 offset:%8 sc0 sc1\n\ts_mov_b64 exec, -1"
                         :: "v"(vl16), "s"(gb), "s"(em), "n"(r), "n"(r + 3), "n"(r + 8), "n"(r + 11), "n"(o), "n"(o + 2048) : "memory");
            if (decltype(kk)::value) gb += KS ? 32768 : 16384;
        };
        unsigned la = vl16 + (unsigned)(size_t)lds + (KS ? (wid >> 1) * 8192 : 0);
        auto fetch = [&](h8 (&a)[8], int i, int t) { a[i] = *(const LAS h8*)(size_t)(la + (unsigned)(t * 8192 + i * 1024)); };
        auto there = [&]() -> unsigned {
            if (KS) return ctl[wid >> 1];
            const u2 x = *(const volatile LAS u2*)ctl;
            return min(x[0], x[1]);
        };
        auto step = [&](auto sl, const h8 (&a)[8], h8 (&a2)[8], int c0) {
            constexpr int S = decltype(sl)::value, T = ((S + 1) << KS) & 7;
            if (T == 0) la ^= 65536u;
            WAIT_V(28);
            if (KS) put(4 + wid, (unsigned)(c0 + S + 1));
            u2 x = {0, 0};
            if (KS) x[0] = x[1] = ctl[wid >> 1]; else if (S & 1) x = *(const volatile LAS u2*)ctl;
#pragma unroll
            for (int i = 0; i < 16; ++i) {
                const int k = i >> 3, m = i >> 1 & 3, n = i & 1, r = S * 16 + n * 8 + k * 4;
                asm volatile("v_mfma_f32_16x16x32_f16 %0, %1, a[%2:%3], %0" : "+v"(acc[m][n]) : "v"(a[m * 2 + k]), "n"(r), "n"(r + 3) : "memory");
                if (n == 0) fetch(a2, m * 2 + k, T);
                if (i == 8) loadb(sl, IC<0>{});
            }
            if (!KS && !(S & 1)) put(4 + wid, (unsigned)(c0 + S / 2 + 1));
            loadb(sl, IC<1>{});
            if (KS || (S & 1)) {
                const int need = min(c0 + (KS ? S + 1 : (S + 1) / 2), NCH - 1);
                if ((int)__builtin_amdgcn_readfirstlane(min(x[0], x[1])) <= need) {
                    while ((int)__builtin_amdgcn_readfirstlane(there()) <= need) { }
                    FENCE();
#pragma unroll
                    for (int i = 0; i < 8; ++i) fetch(a2, (i & 3) * 2 + (i >> 2), T);
                }
            }
        };
        auto fill = [&](auto sl) { loadb(sl, IC<0>{}); loadb(sl, IC<1>{}); };
        fill(IC<0>{}); fill(IC<1>{}); fill(IC<2>{}); fill(IC<3>{}); fill(IC<4>{}); fill(IC<5>{}); fill(IC<6>{}); fill(IC<7>{});
        h8 a0[8], a1[8];
        while (__builtin_amdgcn_readfirstlane(there()) == 0) { }
        FENCE();
#pragma unroll
        for (int i = 0; i < 8; ++i) fetch(a0, (i & 3) * 2 + (i >> 2), 0);
#pragma unroll 1
        for (int c = 0; c < NCH; c += 4 << KS) {
            if (c == NCH - (4 << KS)) em = 1;
            step(IC<0>{}, a0, a1, c); step(IC<1>{}, a1, a0, c); step(IC<2>{}, a0, a1, c); step(IC<3>{}, a1, a0, c);
            step(IC<4>{}, a0, a1, c); step(IC<5>{}, a1, a0, c); step(IC<6>{}, a0, a1, c); step(IC<7>{}, a1, a0, c);
        }
        asm volatile("s_nop 15" : "+v"(acc[0][0]), "+v"(acc[0][1]), "+v"(acc[1][0]), "+v"(acc[1][1]), "+v"(acc[2][0]), "+v"(acc[2][1]), "+v"(acc[3][0]), "+v"(acc[3][1]) :: "memory");
        const int wm = KS ? wid >> 1 : wid & 1, wn = KS ? hb : wid >> 1, un = o0 + wm * 2 + wn;
        LAS char* ex = lds + 131072 + vl16;
        f4 o[2][4];
        const bool odd = wid & 1;
        if (KS) {
#pragma unroll
            for (int n = 0; n < 2; ++n) {
                const f4 x0 = odd ? acc[1][n] : acc[0][n], x1 = odd ? acc[0][n] : acc[1][n], x2 = odd ? acc[3][n] : acc[2][n], x3 = odd ? acc[2][n] : acc[3][n];
                o[1][n] = wm ? x2 : x0;
                *(LAS f4*)(ex + wid * 6144 + n * 1024) = wm ? x3 : x1;
                *(LAS f4*)(ex + wid * 6144 + 2048 + n * 1024) = wm ? x0 : x2;
                *(LAS f4*)(ex + wid * 6144 + 4096 + n * 1024) = wm ? x1 : x3;
            }
        } else {
#pragma unroll
            for (int i = 0; i < 4; ++i) {
                const f4 lo = acc[i >> 1][i & 1], hi = acc[2 + (i >> 1)][i & 1];
                *(LAS f4*)(ex + wid * 4096 + i * 1024) = wm ? lo : hi;
                o[i >> 1][i & 1] = wm ? hi : lo;
            }
        }
        WAIT_L0();
        BAR();
        const u4 tw = *(const volatile LAS u4*)(ctl + 8);
        if (KS) {
#pragma unroll
            for (int n = 0; n < 2; ++n) {
                const f4 z = o[1][n] + *(const LAS f4*)(ex + (wid ^ 2) * 6144 + 2048 + n * 1024);
                const f4 y = *(const LAS f4*)(ex + (wid ^ 1) * 6144 + n * 1024) + *(const LAS f4*)(ex + (wid ^ 3) * 6144 + 4096 + n * 1024);
                o[0][n] = odd ? y : z; o[0][2 + n] = odd ? z : y;
            }
        } else {
#pragma unroll
            for (int i = 0; i < 4; ++i) {
                const f4 y = *(const LAS f4*)(ex + (wid ^ 1) * 4096 + i * 1024), z = o[i >> 1][i & 1];
                o[i >> 1][i & 1] = wm ? y : z; o[i >> 1][2 + (i & 1)] = wm ? z : y;
            }
        }
        const bool dead = __builtin_amdgcn_readfirstlane(tw[0] == call || tw[1] == call);
        const unsigned half = ~__builtin_amdgcn_readfirstlane(tw[2]) & 1u;
        char* dst = myws + (pk_off(pk0 + un, call) + half * 8192);
        char* oflag = myws + (OFF_FP + (call & 1) * FP_B) + (((cg - 8) >> 1) * NSEQ + NCI + un) * 4;
        u4 h[8];
#pragma unroll
        for (int k = 0; k < (KS ? 4 : 8); ++k) h[k] = silu4(o[k >> 2], k & 3, dead);
        if (KS && cg >= 8) {
            dst += (wid & 1) * 4096;
            GST4(vl16, h[0], dst, 0); GST4(vl16, h[1], dst, 1024); GST4(vl16, h[2], dst, 2048); GST4(vl16, h[3], dst, 3072);
            WAIT_V(0);
            BAR();
            if (lane == 0 && !odd) flag_store(oflag, call * 32u + half * 16u + M_RAW);
        } else if (KS || cg < 8) {
            const unsigned ovoff = (unsigned)(lane >> 4) * (4 * NLOC * 4) + (unsigned)(lane & 15) * 16;
            char* ob = (char*)P.out + (size_t)((panel_row(P.rank, panel, 2 * qt + wm) + (KS ? (wid & 1) * 16 : 0)) * NLOC + cg * 128 + wn * 64) * 4;
#pragma unroll
            for (int k = 0; k < (KS ? 4 : 8); ++k) GST4P(ovoff, h[k], ob + ((k >> 2) * 16 + (k & 3)) * (NLOC * 4));
        } else {
            unsigned mode = M_RAW, r0 = 0, r2 = 0;
            if (qt < QPK) mode = pack_unit<true>(h, lds + wid * 4096, lds + (8 * PBUF + 16), lane, r0, r2);
            store_unit<true>(h, dst, mode, r0, r2, vl16);
            WAIT_V(0);
            if (lane == 0) flag_store(oflag, call * 32u + half * 16u + mode);
        }
    }
    if (KS && cg >= 8 && wid >= 4) BAR();
}

__host__ __device__ __forceinline__ int tile2(int slot, int sown) {
    return slot == 26 ? 0 : slot == 25 ? 7 : slot == 24 ? 2 : slot == sown ? 1 : slot < 4 ? slot + 3 : slot == 4 ? 8 : slot < 16 ? slot + 8 : slot == 16 ? 12
         : slot < 20 ? slot - 8 : 33 + ((slot - sown - 1) & 3);
}

__global__ void __launch_bounds__(512, 2) ag_gemm_kernel(const Params Parg) {
    const CAS Params& P = *(const CAS Params*)__builtin_amdgcn_kernarg_segment_ptr();
    (void)Parg;
    __shared__ __attribute__((aligned(16))) char lds_raw[5 * 32768];
    LAS char* lds = (LAS char*)lds_raw;
    const int lane = threadIdx.x & 63, wid = __builtin_amdgcn_readfirstlane(threadIdx.x >> 6);
    const int xcd = blockIdx.x & 7, slot = blockIdx.x >> 3;
    char* myws = P.ws[P.rank];
    const int q = P.rank & 3;
    const unsigned long long t0 = __builtin_amdgcn_s_memrealtime();
    const unsigned side = (__builtin_amdgcn_s_getreg(20 | (3 << 11)) >> 2) & 1u;
    int g = (int)blockIdx.x - NGEMM;
    if (slot == NGEMM / 8) g = NPC + xcd;
    else if (g >= 0) {
        if (threadIdx.x == 0) {
            GAS unsigned* rt = (GAS unsigned*)(myws + OFF_ROLE);
            const unsigned t = __hip_atomic_fetch_add(rt, side ? 65536u : 1u, __ATOMIC_RELAXED, __HIP_MEMORY_SCOPE_AGENT);
            if ((t & 65535u) + (t >> 16) >= 247u - NGEMM) __hip_atomic_store(rt, 0u, __ATOMIC_RELAXED, __HIP_MEMORY_SCOPE_AGENT);
            *(LAS unsigned*)(lds + 8 * PBUF) = t;
        }
        __syncthreads();
        const unsigned t = __builtin_amdgcn_readfirstlane(*(const LAS unsigned*)(lds + 8 * PBUF));
        constexpr unsigned N0 = NPC / 7 * 3, NL = 247 - NGEMM;
        const unsigned r = (side ? NL - (t >> 16) : t & 65535u) % (NL + 1), b = NL - r;
        g = r < N0 ? r / 3 * 7 + 4 + r % 3 : r < N0 + NPROD - 8 ? NPC + 8 + r - N0 : b / 4 * 7 + b % 4;
    }
    if (wid == 7 && blockIdx.x < 2 * PK_PG) {
        char* a = myws + (OFF_PK + ((size_t)blockIdx.x << 21));
        unsigned t[2] = {~0u, ~0u};
#pragma unroll
        for (int i = 0; i < 6; ++i) {
            const unsigned long long c0 = __builtin_amdgcn_s_memtime();
            asm volatile("s_waitcnt lgkmcnt(0)\n\ts_mov_b64 exec, 1\n\tglobal_store_dword %0, %0, %1 sc0 sc1\n\ts_mov_b64 exec, -1\n\ts_waitcnt vmcnt(0)" :: "v"((i & 1) * 8192), "s"(a), "s"(c0) : "memory");
            const unsigned d = (unsigned)(__builtin_amdgcn_s_memtime() - c0);
            t[i & 1] = d < t[i & 1] ? d : t[i & 1];
        }
        if (lane == 0) flag_store(myws + OFF_MAP + blockIdx.x * MAPS, (t[1] < t[0]) ^ side);
    }
    const int u0 = g < 0 ? wid * NGEMM + (int)blockIdx.x : UG + wid * NPROD + g - NPC;
    if (g < 0 || g >= NPC) {
        pace(t0, u0); fetch_input(P, lds + wid * PBUF, u0, lane);
        make_lut(lds + (8 * PBUF + 16), true, lane);
    }
    GAS unsigned* seq = (GAS unsigned*)(myws + OFF_SEQ) + (blockIdx.x * 8 + wid) * 4;
    const unsigned call = __builtin_amdgcn_readfirstlane(__hip_atomic_load(seq, __ATOMIC_RELAXED, __HIP_MEMORY_SCOPE_AGENT)) + 1u;
    WAIT_V(0);

    if (g < 0 || g >= NPC) {
        producer(P, myws, lds + wid * PBUF, lds + (8 * PBUF + 16), u0, t0, call, wid, lane);
        if (lane == 0) __hip_atomic_store(seq, call, __ATOMIC_RELAXED, __HIP_MEMORY_SCOPE_AGENT);
    }
    if (g < 0 || g >= NPC) {
        const int pi = slot >> 3, ps = xcd >> 1, h = xcd & 1, sown = 20 + ps;
        const int ci = pi == 3 ? (slot & 1) * 4 + ps : slot == sown ? ps : slot & 7;
        const int panel = pi < 2 ? 3 * ps + pi : pi == 3 ? 3 * q + 2 : slot == sown ? 16 : 12 + ps;
        const int k = g >= NPC + 8 ? 12 : tile2(slot, sown), kk = k % 12, qt = kk / 3, si = kk - qt * 3;
        const bool ho = slot == 16;
        const int cg2 = g >= NPC + 8 ? g - (NPC + 5) : ho ? ps : k < 12 || k >= 24 ? 8 + xcd : xcd;
        const int bp = 3 * ((q + 1 + si) & 3) + 2, o0 = O_B + qt * 24 + (si * 2 + h) * 4;
        if (g < 0) gemm_panel(P, lds, myws, panel, ci < 4 ? 4 * h + ci : 2 * ci + h, cg2,
                   pi == 3 ? h * 8 : pi == 2 ? O_C + (((ps - ci - 1) & 3) * 2 + h) * 8 : O_A0 + pi * 128 + xcd * 8, pi == 3 ? 16 : pi == 2 ? 48 : 64, call, wid, lane);
        if (ho ? ps < 3 : qt == 3 && cg2 >= 8) gemm_ring<1>(P, lds, myws, bp, qt, cg2, ho ? h : k >= 24, o0, g < 0, call, wid, lane);
        else if (!ho) gemm_ring<0>(P, lds, myws, bp, qt, cg2, 0, o0, g < 0, call, wid, lane);
    } else {
        LAS unsigned* info = (LAS unsigned*)lds;
        LAS unsigned* prog = info + NSEQ;
        LAS unsigned short* tab = (LAS unsigned short*)(lds + 8192);
        const int j = g / 7, link = g - j * 7, ncu = (NPC + 6 - link) / 7;
        for (int i = threadIdx.x; i < NSEQ + 4; i += 512) info[i] = 0;
        if (threadIdx.x == 0) { prog[4] = (unsigned)t0; prog[5] = 0; }
        for (int i = threadIdx.x; i < PK_W; i += 512)
            if (((i & 31) == 16 || (i & 31) >= 24) == (link < 4)) tab[pos_x(i & 31, i >> 5)] = (unsigned short)i;
        if (link < 4) tab[pos_w(threadIdx.x >> 2, threadIdx.x & 3)] = (unsigned short)(PK_W + threadIdx.x);
        if (wid == 0) make_lut(lds + 14336, false, lane);
        __syncthreads();
        if (lane == 0) __hip_atomic_store(seq, call, __ATOMIC_RELAXED, __HIP_MEMORY_SCOPE_AGENT);
        const int peer = link < 4 ? ((P.rank & 4) ^ 4) + ((q + link) & 3) : (P.rank & 4) + ((q + link - 3) & 3);
        if (wid < WPC) {
            if (lane == 0) __hip_atomic_fetch_max(prog, (unsigned)(NPUSH + wid * ncu + j), __ATOMIC_RELAXED, __HIP_MEMORY_SCOPE_WORKGROUP);
            puller(g, P, myws, info, prog, tab, lds + 16384 + wid * 8192, lds + 14336, link, peer, NPUSH + wid * ncu + j, ncu * WPC,
                   (unsigned)(((link < 4 ? NWIN : NWINI) + ncu - 1 - j) / ncu), call, wid, lane);
        } else if (wid == 7) {
            flag_mirror(myws, P.ws[peer] + (OFF_FP + (call & 1) * FP_B + (link < 4 ? q : 4) * (NSEQ * 4)), link < 4 ? NSEQ : NI, info, prog, call, lane);
        }
    }
}

}

extern "C" void kernel_launch(int rank, int nranks, ncclComm_t comm, hipStream_t stream,
                              void* const* d_in, const int* in_sizes, int n_in,
                              void* d_out, int out_size, const int* in_dtypes, int out_dtype,
                              void* const* d_out_all, void* const* d_ws_all, size_t ws_bytes) {
    (void)comm; (void)in_dtypes; (void)out_dtype; (void)d_out_all;
    if (nranks != 8 || (unsigned)rank >= 8u || n_in < 2 || ws_bytes < WS_NEED || out_size != KDIM * NLOC ||
        in_sizes[0] != 1024 * KDIM || in_sizes[1] != KDIM * NLOC) {
        fprintf(stderr, "ag_gemm: unexpected problem shape\n");
        return;
    }
    Params P;
    P.x = (const float*)d_in[0]; P.w = (const float*)d_in[1];
    P.out = (float*)d_out;
    P.rank = 0; P.pad = 0;
    for (int p = 0; p < 8; ++p) {
        const int r = (int)((RMAP >> (4 * p)) & 7u);
        P.ws[p] = (char*)d_ws_all[r];
        if (r == rank) P.rank = p;
    }
    hipLaunchKernelGGL(ag_gemm_kernel, dim3(256), dim3(512), 0, stream, P);
    HIP_CHECK(hipGetLastError());
}
```
